```python
import numpy as np

kernel_name = "gpu_a2a_gemm_m8192_k8192_n8192_bf16_gelu"


ABSMAX_RTOL = 0.008

COMPUTE_DTYPE = "bfloat16"
EPILOGUE = "gelu"
IN_QUANT = None
OUT_QUANT = None
IN_DTYPES = {"x": "bfloat16", "w_mat": "bfloat16"}
WORLD_SIZE = 8

def _to_bf16(x):
    u = np.ascontiguousarray(x, dtype=np.float32).view(np.uint32)
    return ((u + 0x7FFF + ((u >> 16) & 1)) & 0xFFFF0000).view(np.float32)

def setup_inputs():

    x = _to_bf16(np.random.randn(8192, 8192).astype(np.float32))
    w_mat = _to_bf16((np.random.randn(8192, 8192) / np.sqrt(8192)).astype(np.float32))
    return {"x": x, "w_mat": w_mat}

def _epilogue(y):
    c = 0.7978845608028654
    return (0.5 * y * (1.0 + np.tanh(c * (y + 0.044715 * y ** 3)))).astype(np.float32)

def reference(x, w_mat):
    y = (x @ w_mat).astype(np.float32)
    return _epilogue(y)
```

```cpp
#include <hip/hip_runtime.h>
#include <rccl/rccl.h>
#include <cstdio>
#include <cstdint>

#define HIP_CHECK(expr) do { hipError_t _e = (expr); if (_e != hipSuccess) { \
    fprintf(stderr, "HIP error %s at %s:%d\n", hipGetErrorString(_e), __FILE__, __LINE__); \
    return; } } while (0)

#define LAS __attribute__((address_space(3)))
#define GAS __attribute__((address_space(1)))
typedef short bf16x8 __attribute__((ext_vector_type(8)));
typedef short s16x4 __attribute__((ext_vector_type(4)));
typedef float f32x4 __attribute__((ext_vector_type(4)));
typedef unsigned u32x4 __attribute__((ext_vector_type(4)));
typedef unsigned u32x2 __attribute__((ext_vector_type(2)));

#define XCHG_BF16 0
#define MAP_OPT 1
#define PACE_NUM 9
#define PACE_DEN 448

namespace {
constexpr int NR = 8;
constexpr int MS = 1024;
constexpr int KB = 1024;
constexpr int NN = 8192;
constexpr int NT = 64;
constexpr int NWG = 256;
constexpr int HTB = 16384;
constexpr int L_SEND = 8 * HTB;
constexpr int L_FLAG = L_SEND + 8192;
constexpr int L_CTL = L_FLAG + 4096;
constexpr int LDS_BYTES = L_CTL + 64;
constexpr size_t W_TICKET = 0;
constexpr size_t W_TMO = 256;
constexpr size_t W_XFLAG = 4096;
constexpr size_t W_EFLAG = 65536;
constexpr size_t W_RECV = 1u << 20;
constexpr size_t RECV_BLK = (size_t)MS * KB * 2;
constexpr size_t RECV_PAR = RECV_BLK * NR;
constexpr size_t W_SLAB = W_RECV + 2 * RECV_PAR;
constexpr size_t SLAB_B = 131072;
constexpr size_t WS_NEED = W_SLAB + (size_t)NWG * SLAB_B;
constexpr unsigned long long DEADLINE_TICKS = 40000000ull;

__device__ __forceinline__ int lds_byteA(int r, int c) { const int st = (r >> 4) * 2 + (c >> 5), rr = r & 15, cc = c & 31, ob = rr * 64 + cc * 2; return st * 1024 + (ob ^ (((ob >> 9) & 1) << 5)); }
__device__ __forceinline__ void stage_rcA(int b, int& R, int& C) { const int st = b / 1024, sb = b % 1024, swz = sb ^ (((sb >> 9) & 1) << 5); R = (st >> 1) * 16 + swz / 64; C = (st & 1) * 32 + (swz % 64) / 2; }
__device__ __forceinline__ unsigned cvt_pk_bf16(float lo, float hi) { unsigned r; asm volatile("v_cvt_pk_bf16_f32 %0, %1, %2" : "=v"(r) : "v"(lo), "v"(hi)); return r; }
__device__ __forceinline__ float gelu_tanh(float y) {
    const float u = y * y * 0.044715f + 1.0f;
    const float e = __builtin_amdgcn_exp2f(y * u * (-2.0f * 0.7978845608028654f * 1.4426950408889634f));
    return y * __builtin_amdgcn_rcpf(1.0f + e);
}
}

__global__ __launch_bounds__(512, 2)
void a2a_gemm_gelu_kernel(int rank, const unsigned short* __restrict__ xloc, const unsigned short* __restrict__ wmat,
                          unsigned short* __restrict__ out, void* const* __restrict__ ws_all) {
    extern __shared__ __attribute__((aligned(16))) unsigned char shm[];
    LAS unsigned char* lds = (LAS unsigned char*)shm;
    const int tid = threadIdx.x, wid = __builtin_amdgcn_readfirstlane(tid >> 6), lane = tid & 63, wr = wid >> 2, wc = wid & 3, fr = lane & 15, fq = lane >> 4;
    const int bid = blockIdx.x, xcd = bid & 7, idx = bid >> 3;
    const int pn = xcd * 4 + (idx & 3), pm = (idx >> 2) & 3, h = idx >> 4;
    const int partner = bid ^ (16 << 3);
    char* const myws = (char*)ws_all[rank];

    unsigned voffA[2], voffB[2];
#pragma unroll
    for (int i = 0; i < 2; ++i) {
        int R, C; stage_rcA(tid * 16 + i * 8192, R, C);
        voffA[i] = (unsigned)(R * KB + C) * 2u;
        const int b = tid * 16 + i * 8192, k = b >> 8, cp = (b >> 4) & 15, f = (k & 3) | (((k >> 3) & 1) << 2), c = cp ^ (f << 1);
        voffB[i] = (unsigned)k * (NN * 2) + (unsigned)c * 16u;
    }
    const unsigned ldsw = (unsigned)wid * 1024u;
    const int aoff = lds_byteA(wr * 64 + fr, fq * 8);
    int boff[2];
    { const int g = lane >> 4, q = (lane & 15) >> 2, p = lane & 3, f = q | ((g & 1) << 2);
#pragma unroll
      for (int nf = 0; nf < 2; ++nf) boff[nf] = (int)(unsigned)(uintptr_t)lds + 4 * HTB + (8 * g + q) * 256 + (((wc * 4 + nf * 2 + (p >> 1)) ^ (f << 1)) * 16) + (p & 1) * 8; }

#define SA(b, hh) (((b) * 2 + (hh)) * HTB)
#define SB(b, hh) ((4 + (b) * 2 + (hh)) * HTB)
#define STAGE(bufoff, gbase, voff) do { _Pragma("unroll") for (int _i = 0; _i < 2; ++_i) \
        __builtin_amdgcn_global_load_lds((const unsigned*)((const char*)(gbase) + (voff)[_i]), (LAS unsigned*)(lds + (bufoff) + ldsw + _i * 8192), 16, 0, 0); } while (0)
#define LDA(dst, b, hh) do { _Pragma("unroll") for (int m = 0; m < 4; ++m) _Pragma("unroll") for (int k = 0; k < 2; ++k) dst[m][k] = *(const LAS bf16x8*)(lds + SA(b, hh) + aoff + m * 2048 + k * 1024); } while (0)
#define LDB(dst, b, hh) do { s16x4 _r0, _r1, _r2, _r3, _r4, _r5, _r6, _r7; \
        asm volatile("ds_read_b64_tr_b16 %0, %8 offset:%10\n\tds_read_b64_tr_b16 %1, %8 offset:%11\n\tds_read_b64_tr_b16 %2, %8 offset:%12\n\tds_read_b64_tr_b16 %3, %8 offset:%13\n\t" \
                     "ds_read_b64_tr_b16 %4, %9 offset:%10\n\tds_read_b64_tr_b16 %5, %9 offset:%11\n\tds_read_b64_tr_b16 %6, %9 offset:%12\n\tds_read_b64_tr_b16 %7, %9 offset:%13" \
                     : "=&v"(_r0), "=&v"(_r1), "=&v"(_r2), "=&v"(_r3), "=&v"(_r4), "=&v"(_r5), "=&v"(_r6), "=&v"(_r7) \
                     : "v"(boff[0]), "v"(boff[1]), "i"(((b) * 2 + (hh)) * HTB), "i"(((b) * 2 + (hh)) * HTB + 1024), "i"(((b) * 2 + (hh)) * HTB + 8192), "i"(((b) * 2 + (hh)) * HTB + 9216) : "memory"); \
        dst[0][0] = __builtin_shufflevector(_r0, _r1, 0, 1, 2, 3, 4, 5, 6, 7); dst[0][1] = __builtin_shufflevector(_r2, _r3, 0, 1, 2, 3, 4, 5, 6, 7); \
        dst[1][0] = __builtin_shufflevector(_r4, _r5, 0, 1, 2, 3, 4, 5, 6, 7); dst[1][1] = __builtin_shufflevector(_r6, _r7, 0, 1, 2, 3, 4, 5, 6, 7); } while (0)
#define MMA(ai, bj, At, Bt) do { __builtin_amdgcn_s_setprio(1); _Pragma("unroll") for (int m = 0; m < 4; ++m) _Pragma("unroll") for (int n = 0; n < 2; ++n) _Pragma("unroll") for (int k = 0; k < 2; ++k) \
        acc[ai][bj][m][n] = __builtin_amdgcn_mfma_f32_16x16x32_bf16(Bt[n][k], At[m][k], acc[ai][bj][m][n], 0, 0, 0); __builtin_amdgcn_s_setprio(0); } while (0)
#define WAIT_V(n) asm volatile("s_waitcnt vmcnt(" #n ")" ::: "memory")
#define WAIT_L(n) asm volatile("s_waitcnt lgkmcnt(" #n ")" ::: "memory")
#define BAR __builtin_amdgcn_s_barrier()
#define SCHED __builtin_amdgcn_sched_barrier(0)

    f32x4 acc[2][2][4][2];
#pragma unroll
    for (int a = 0; a < 2; ++a)
#pragma unroll
        for (int b = 0; b < 2; ++b)
#pragma unroll
            for (int m = 0; m < 4; ++m)
#pragma unroll
                for (int n = 0; n < 2; ++n) acc[a][b][m][n] = (f32x4){0.f, 0.f, 0.f, 0.f};
    bf16x8 At[4][2], B0[2][2], B1[2][2];

    constexpr size_t HSTEP_A = (size_t)128 * KB * 2;
    constexpr size_t HSTEP_B = 256;
    const char* const aLoc = (const char*)xloc + ((size_t)rank * MS + (size_t)pm * 256) * KB * 2 + (size_t)h * 8 * 128;
    const char* const wCol = (const char*)wmat + (size_t)pn * 512;
    const char* const bLoc = wCol + ((size_t)rank * KB + (size_t)h * 8 * 64) * (NN * 2);

    STAGE(SB(0, 0), bLoc, voffB); STAGE(SA(0, 0), aLoc, voffA); STAGE(SB(0, 1), bLoc + HSTEP_B, voffB); STAGE(SA(0, 1), aLoc + HSTEP_A, voffA);
    STAGE(SB(1, 0), bLoc + (size_t)64 * NN * 2, voffB); STAGE(SA(1, 0), aLoc + 128, voffA); STAGE(SB(1, 1), bLoc + (size_t)64 * NN * 2 + HSTEP_B, voffB);
    if (tid == 0) {
        const unsigned ticket = __hip_atomic_fetch_add((GAS unsigned*)(myws + W_TICKET), 1u, __ATOMIC_RELAXED, __HIP_MEMORY_SCOPE_AGENT);
        *(volatile LAS unsigned*)(lds + L_CTL) = ticket / NWG + 1u;
    }
    WAIT_V(6); WAIT_L(0); BAR; asm volatile("" ::: "memory");
    const unsigned seq = __builtin_amdgcn_readfirstlane(*(volatile LAS unsigned*)(lds + L_CTL));
    const size_t parOff = W_RECV + (size_t)(seq & 1u) * RECV_PAR;
    const char* const aRem = myws + parOff + (size_t)pm * 256 * KB * 2 + (size_t)h * 128;
    const char* const fRem = myws + W_XFLAG + (size_t)pm * 128 + (size_t)(lane & 7) * 4;
    GAS unsigned* const tmo = (GAS unsigned*)(myws + W_TMO);
    if (wr == 1) BAR;

    auto tileA = [&](int T) -> const char* {
        if (T >= NT) T = NT - 1;
        if (T < 8) return aLoc + (size_t)T * 128;
        const int u = T - 8, a = u / 7, j = u - a * 7, src = (rank + 1 + j) & 7;
        return aRem + (size_t)src * RECV_BLK + (size_t)a * 256;
    };
    auto tileB = [&](int T) -> const char* {
        if (T >= NT) T = NT - 1;
        if (T < 8) return bLoc + (size_t)T * 64 * (NN * 2);
        const int u = T - 8, a = u / 7, j = u - a * 7, src = (rank + 1 + j) & 7;
        return wCol + ((size_t)src * KB + (size_t)(2 * a + h) * 64) * (NN * 2);
    };
    auto tileF = [&](int T) -> const char* {
        const int u = T - 8, a = u / 7, j = u - a * 7, src = (rank + 1 + j) & 7;
        return fRem + (size_t)(src * 8 + a) * 512;
    };

    int sidx = 0;
    int sslot = (bid * PACE_NUM) / PACE_DEN;
    u32x4 pv = {0u, 0u, 0u, 0u};
    unsigned fv = 0xffffffffu;

#define K_TILE(b, t) do { \
        const char* a1 = tileA((t) + 1); const char* a2 = tileA((t) + 2); const char* b2 = tileB((t) + 2); \
        const int sd = (t) - sslot; \
          \
        if (sd == 1) pv = *(const volatile LAS u32x4*)(lds + L_SEND + ldsw + lane * 16); \
        if ((t) + 2 >= 8 && (t) + 2 < NT) fv = *(const volatile LAS unsigned*)(lds + L_FLAG + (b) * 2048 + wid * 256 + lane * 4); \
        if ((t) + 3 >= 8 && (t) + 3 < NT) __builtin_amdgcn_global_load_lds((const unsigned*)tileF((t) + 3), (LAS unsigned*)(lds + L_FLAG + ((b) ^ 1) * 2048 + wid * 256), 4, 0, 17); \
        if (sd == 0) { const int q = sidx * 256 + bid, a = q / 224, r = q - a * 224, rg = r / 7, dj = r - rg * 7, d = (rank + 1 + dj) & 7; \
            const char* src = (const char*)xloc + ((size_t)d * MS + (size_t)rg * 32 + wid * 4 + (lane >> 4)) * (KB * 2) + (size_t)a * 256 + (lane & 15) * 16; \
            __builtin_amdgcn_global_load_lds((const unsigned*)src, (LAS unsigned*)(lds + L_SEND + ldsw), 16, 0, 0); } \
        LDB(B0, b, 0); SCHED; LDA(At, b, 0); STAGE(SA((b) ^ 1, 1), a1 + HSTEP_A, voffA); \
        if (sd == 3) { if (wid == 0) { const int q = sidx * 256 + bid, a = q / 224, r = q - a * 224, rg = r / 7, dj = r - rg * 7, d = (rank + 1 + dj) & 7; \
            char* fp = (char*)ws_all[d] + W_XFLAG + (size_t)((rank * 8 + a) * 4 + (rg >> 3)) * 128 + (size_t)(rg & 7) * 4; \
            asm volatile("global_store_dword %0, %1, off sc0 sc1" :: "v"(fp), "v"(seq) : "memory"); } } \
        WAIT_L(8); BAR; WAIT_L(0); SCHED; MMA(0, 0, At, B0); BAR; SCHED; \
          \
        LDB(B1, b, 1); \
        if (sd == 1) { const int q = sidx * 256 + bid, a = q / 224, r = q - a * 224, rg = r / 7, dj = r - rg * 7, d = (rank + 1 + dj) & 7; \
            char* dp = (char*)ws_all[d] + parOff + (size_t)rank * RECV_BLK + ((size_t)rg * 32 + wid * 4 + (lane >> 4)) * (KB * 2) + (size_t)a * 256 + (lane & 15) * 16; \
            asm volatile("global_store_dwordx4 %0, %1, off sc0 sc1\n\ts_nop 1" :: "v"(dp), "v"(pv) : "memory"); } \
        STAGE(SB(b, 0), b2, voffB); \
        BAR; WAIT_L(0); SCHED; MMA(0, 1, At, B1); BAR; \
          \
        LDA(At, b, 1); \
        if ((t) + 2 >= 8 && (t) + 2 < NT) { if (!__all(fv >= seq)) { \
            const GAS unsigned* fp = (const GAS unsigned*)tileF((t) + 2); const unsigned long long t0 = __builtin_amdgcn_s_memrealtime(); \
            while (!__all(__hip_atomic_load(fp, __ATOMIC_RELAXED, __HIP_MEMORY_SCOPE_SYSTEM) >= seq)) { \
                if (__builtin_amdgcn_readfirstlane(__hip_atomic_load(tmo, __ATOMIC_RELAXED, __HIP_MEMORY_SCOPE_AGENT)) == seq || __builtin_amdgcn_s_memrealtime() - t0 > DEADLINE_TICKS) { \
                    __hip_atomic_store(tmo, seq, __ATOMIC_RELAXED, __HIP_MEMORY_SCOPE_AGENT); break; } \
                __builtin_amdgcn_s_sleep(2); } \
            asm volatile("" ::: "memory"); } } \
        STAGE(SA(b, 0), a2, voffA); \
        BAR; WAIT_L(0); MMA(1, 0, At, B0); BAR; SCHED; \
          \
        STAGE(SB(b, 1), b2 + HSTEP_B, voffB); \
        if (sd == 1 || (sd == 3 && wid == 0)) WAIT_V(7); else WAIT_V(6); \
        BAR; MMA(1, 1, At, B1); BAR; \
        if (sd == 3) { ++sidx; sslot = sidx < 7 ? ((sidx * 256 + bid) * PACE_NUM) / PACE_DEN : 1000; } \
    } while (0)

    for (int t = 0; t < NT; t += 2) {
        K_TILE(0, t);
        K_TILE(1, t + 1);
    }
    WAIT_V(0);
    if (wr == 0) BAR;

    {
        char* slabOut = myws + W_SLAB + (size_t)bid * SLAB_B + (size_t)wid * 16384 + lane * 16;
        const char* slabIn = myws + W_SLAB + (size_t)partner * SLAB_B + (size_t)wid * 16384 + lane * 16;
#pragma unroll
        for (int bj = 0; bj < 2; ++bj)
#pragma unroll
            for (int m = 0; m < 4; ++m) {
#pragma unroll
                for (int n = 0; n < 2; ++n) {
                    const f32x4 v = h ? acc[0][bj][m][n] : acc[1][bj][m][n];
                    asm volatile("global_store_dwordx4 %0, %1, off sc1\n\ts_nop 1" :: "v"(slabOut + ((bj * 4 + m) * 2 + n) * 1024), "v"(v) : "memory");
                }
            }
        WAIT_V(0);
        if (lane == 0) __hip_atomic_store((GAS unsigned*)(myws + W_EFLAG + (size_t)(bid * 8 + wid) * 128), seq, __ATOMIC_RELAXED, __HIP_MEMORY_SCOPE_AGENT);
        {
            const GAS unsigned* fp = (const GAS unsigned*)(myws + W_EFLAG + (size_t)(partner * 8 + wid) * 128);
            const unsigned long long t0 = __builtin_amdgcn_s_memrealtime();
            while ((unsigned)__builtin_amdgcn_readfirstlane(__hip_atomic_load(fp, __ATOMIC_RELAXED, __HIP_MEMORY_SCOPE_AGENT)) < seq) {
                if (__builtin_amdgcn_readfirstlane(__hip_atomic_load(tmo, __ATOMIC_RELAXED, __HIP_MEMORY_SCOPE_AGENT)) == seq || __builtin_amdgcn_s_memrealtime() - t0 > DEADLINE_TICKS) {
                    __hip_atomic_store(tmo, seq, __ATOMIC_RELAXED, __HIP_MEMORY_SCOPE_AGENT); break; }
                __builtin_amdgcn_s_sleep(1);
            }
            asm volatile("" ::: "memory");
        }
        const bool bad = __builtin_amdgcn_readfirstlane(__hip_atomic_load(tmo, __ATOMIC_RELAXED, __HIP_MEMORY_SCOPE_AGENT)) == seq;
        f32x4 in[16];
#pragma unroll
        for (int i = 0; i < 16; ++i) asm volatile("global_load_dwordx4 %0, %1, off sc1" : "=v"(in[i]) : "v"(slabIn + i * 1024) : "memory");
        asm volatile("s_waitcnt vmcnt(0)" : "+v"(in[0]), "+v"(in[1]), "+v"(in[2]), "+v"(in[3]), "+v"(in[4]), "+v"(in[5]), "+v"(in[6]), "+v"(in[7]),
                     "+v"(in[8]), "+v"(in[9]), "+v"(in[10]), "+v"(in[11]), "+v"(in[12]), "+v"(in[13]), "+v"(in[14]), "+v"(in[15]) :: "memory");
        unsigned short* orow = out + (size_t)(pm * 256 + h * 128 + wr * 64 + fr) * NN + pn * 256 + wc * 32 + (fq & 1) * 16 + (fq >> 1) * 8;
#pragma unroll
        for (int bj = 0; bj < 2; ++bj)
#pragma unroll
            for (int m = 0; m < 4; ++m) {
                f32x4 v0 = h ? acc[1][bj][m][0] : acc[0][bj][m][0], v1 = h ? acc[1][bj][m][1] : acc[0][bj][m][1];
                v0 += in[(bj * 4 + m) * 2]; v1 += in[(bj * 4 + m) * 2 + 1];
                unsigned a0 = cvt_pk_bf16(gelu_tanh(v0[0]), gelu_tanh(v0[1])), a1 = cvt_pk_bf16(gelu_tanh(v0[2]), gelu_tanh(v0[3]));
                unsigned c0 = cvt_pk_bf16(gelu_tanh(v1[0]), gelu_tanh(v1[1])), c1 = cvt_pk_bf16(gelu_tanh(v1[2]), gelu_tanh(v1[3]));
                { auto r = __builtin_amdgcn_permlane16_swap(a0, c0, false, false); a0 = r[0]; c0 = r[1]; }
                { auto r = __builtin_amdgcn_permlane16_swap(a1, c1, false, false); a1 = r[0]; c1 = r[1]; }
                u32x4 w = {a0, a1, c0, c1};
                if (bad) w = (u32x4){0x7fc07fc0u, 0x7fc07fc0u, 0x7fc07fc0u, 0x7fc07fc0u};
                *(u32x4*)(orow + (size_t)m * 16 * NN + bj * 128) = w;
            }
    }
}

extern "C" void kernel_launch(int rank, int nranks, ncclComm_t comm,
                              hipStream_t stream,
                              void* const* d_in, const int* in_sizes, int n_in,
                              void* d_out, int out_size,
                              const int* in_dtypes, int out_dtype,
                              void* const* d_out_all,
                              void* const* d_ws_all, size_t ws_bytes) {
    (void)comm; (void)in_sizes; (void)n_in; (void)out_size; (void)in_dtypes; (void)out_dtype; (void)d_out_all;
    if (nranks != NR || (unsigned)rank >= (unsigned)NR || ws_bytes < WS_NEED) { fprintf(stderr, "a2a_gemm: unsupported configuration\n"); return; }
    static bool attr_set = false;
    if (!attr_set) {
        HIP_CHECK(hipFuncSetAttribute((const void*)a2a_gemm_gelu_kernel, hipFuncAttributeMaxDynamicSharedMemorySize, LDS_BYTES));
        attr_set = true;
    }
    hipLaunchKernelGGL(a2a_gemm_gelu_kernel, dim3(NWG), dim3(512), LDS_BYTES, stream, rank,
                       (const unsigned short*)d_in[0], (const unsigned short*)d_in[1], (unsigned short*)d_out, d_ws_all);
    HIP_CHECK(hipGetLastError());
}
```

```cpp
#include <hip/hip_runtime.h>
#include <rccl/rccl.h>
#include <cstdio>
#include <cstdint>
#include <utility>

#define HIP_CHECK(expr) do { hipError_t _e = (expr); if (_e != hipSuccess) { \
    fprintf(stderr, "HIP error %s at %s:%d\n", hipGetErrorString(_e), __FILE__, __LINE__); \
    return; } } while (0)

#define LAS __attribute__((address_space(3)))
#define GAS __attribute__((address_space(1)))
typedef short bf16x8 __attribute__((ext_vector_type(8)));
typedef short s16x4 __attribute__((ext_vector_type(4)));
typedef float f32x4 __attribute__((ext_vector_type(4)));
typedef unsigned u32x4 __attribute__((ext_vector_type(4)));

#define MAP_OPT 1
#define PACE_NUM 9
#define PACE_DEN 448
#define EARLY_BAR 0
#define SHARED_KB 16
#define XS_PERSEG 1
#define XS_PS 2

namespace {
constexpr int NR = 8;
constexpr int MS = 1024;
constexpr int KB = 1024;
constexpr int NN = 8192;
constexpr int SH = SHARED_KB;
constexpr int NF = 64 - SH / 2;
constexpr int NWG = 256;
constexpr int HTB = 16384;
static_assert(SH % 4 == 0 && SH >= 4 && SH <= 16, "tail size");
static_assert((1791 * PACE_NUM) / PACE_DEN + 3 < NF - 2, "all sends must finish inside the generic K-loop");
constexpr int L_X = 8 * HTB;
constexpr int L_SEND = 9 * HTB;
constexpr int L_FLAG = L_SEND + 8192;
constexpr int LDS_BYTES = L_FLAG + 4096;
constexpr size_t W_TMO = 256;
constexpr size_t W_XFLAG = 4096;
constexpr size_t W_EFLAG = 65536;
constexpr size_t W_SEQ = W_EFLAG + 262144;
constexpr size_t W_RECV = 1u << 20;
constexpr size_t RECV_BLK = (size_t)MS * KB * 2;
constexpr size_t RECV_PAR = RECV_BLK * NR;
constexpr size_t W_SLAB = W_RECV + 2 * RECV_PAR;
constexpr size_t SLAB_B = 65536;
constexpr size_t WS_NEED = W_SLAB + (size_t)NWG * SLAB_B;
constexpr unsigned long long DEADLINE_TICKS = 40000000ull;

constexpr int XS_PER = XS_PERSEG;
constexpr int S_FLAG = 8 / XS_PER + 1;
constexpr int S_SAMP = S_FLAG + XS_PS;
constexpr int S_LOAD = S_SAMP + 2;
static_assert(S_LOAD < SH, "tail too short for the hand-over schedule");
constexpr int n_stage(int u) { return u + 2 < SH ? (u == 0 ? 4 : 6) : 0; }
constexpr int n_store(int u) { return u * XS_PER < 8 ? XS_PER : 0; }
constexpr int n_ops(int u) { return n_store(u) + (u == S_FLAG) + (u == S_SAMP) + n_stage(u) + (u == S_LOAD ? 8 : 0); }
constexpr int tail_wait(int u) { return n_ops(u) + (u - 1 == S_LOAD ? 8 : 0); }
constexpr int slotB0(int u) { return u % 3 == 0 ? 4 * HTB : (u % 3 == 1 ? 6 * HTB : L_X); }
constexpr int slotA(int u) { return u % 3 == 0 ? 0 : (u % 3 == 1 ? 2 * HTB : HTB); }
constexpr int slotB1(int u) { return u % 3 == 0 ? 5 * HTB : (u % 3 == 1 ? 7 * HTB : 3 * HTB); }
constexpr int e_of(int T) { return T < 8 ? T : ((T - 8) % 7 + 1) * 8 + (T - 8) / 7; }

__device__ __forceinline__ int lds_byteA(int r, int c) { const int st = (r >> 4) * 2 + (c >> 5), rr = r & 15, cc = c & 31, ob = rr * 64 + cc * 2; return st * 1024 + (ob ^ (((ob >> 9) & 1) << 5)); }
__device__ __forceinline__ void stage_rcA(int b, int& R, int& C) { const int st = b / 1024, sb = b % 1024, swz = sb ^ (((sb >> 9) & 1) << 5); R = (st >> 1) * 16 + swz / 64; C = (st & 1) * 32 + (swz % 64) / 2; }
__device__ __forceinline__ unsigned cvt_pk_bf16(float lo, float hi) { unsigned r; asm volatile("v_cvt_pk_bf16_f32 %0, %1, %2" : "=v"(r) : "v"(lo), "v"(hi)); return r; }
__device__ __forceinline__ float gelu_tanh(float y) {
    const float u = y * y * 0.044715f + 1.0f;
    const float e = __builtin_amdgcn_exp2f(y * u * (-2.0f * 0.7978845608028654f * 1.4426950408889634f));
    return y * __builtin_amdgcn_rcpf(1.0f + e);
}
template <int N> __device__ __forceinline__ void wait_vm() { if constexpr (N < 63) asm volatile("s_waitcnt vmcnt(%0)" :: "n"(N) : "memory"); }
template <class F, int... I> __device__ __forceinline__ void static_for(F&& f, std::integer_sequence<int, I...>) { (f(std::integral_constant<int, I>{}), ...); }
}

__global__ __launch_bounds__(512, 2)
void a2a_gemm_gelu_kernel(int rank, const unsigned short* __restrict__ xloc, const unsigned short* __restrict__ wmat,
                          unsigned short* __restrict__ out, void* const* __restrict__ ws_all) {
    extern __shared__ __attribute__((aligned(16))) unsigned char shm[];
    LAS unsigned char* lds = (LAS unsigned char*)shm;
    const int tid = threadIdx.x, wid = __builtin_amdgcn_readfirstlane(tid >> 6), lane = tid & 63, wr = wid >> 2, wc = wid & 3, fr = lane & 15, fq = lane >> 4;
    const int bid = blockIdx.x, xcd = bid & 7, idx = bid >> 3;
    const int pn = xcd * 4 + (idx & 3), pm = (idx >> 2) & 3, h = idx >> 4;
    const int partner = bid ^ (16 << 3);
    char* const myws = (char*)ws_all[rank];
#define STAMP(i)
    STAMP(0);
    unsigned seqv;
    asm volatile("global_load_dword %0, %1, off sc1" : "=v"(seqv) : "v"(myws + W_SEQ + (size_t)bid * 128) : "memory");

    unsigned voffA[2], voffB[2];
#pragma unroll
    for (int i = 0; i < 2; ++i) {
        int R, C; stage_rcA(tid * 16 + i * 8192, R, C);
        voffA[i] = (unsigned)(R * KB + C) * 2u;
        const int b = tid * 16 + i * 8192, k = b >> 8, cp = (b >> 4) & 15, f = (k & 3) | (((k >> 3) & 1) << 2), c = cp ^ (f << 1);
        voffB[i] = (unsigned)k * (NN * 2) + (unsigned)c * 16u;
    }
    const unsigned ldsw = (unsigned)wid * 1024u;
    const int aoff = lds_byteA(wr * 64 + fr, fq * 8);
    int boff[2];
    { const int g = lane >> 4, q = (lane & 15) >> 2, p = lane & 3, f = q | ((g & 1) << 2);
#pragma unroll
      for (int nf = 0; nf < 2; ++nf) boff[nf] = (int)(unsigned)(uintptr_t)lds + 4 * HTB + (8 * g + q) * 256 + (((wc * 4 + nf * 2 + (p >> 1)) ^ (f << 1)) * 16) + (p & 1) * 8; }

#define SA(b, hh) (((b) * 2 + (hh)) * HTB)
#define SB(b, hh) ((4 + (b) * 2 + (hh)) * HTB)
#define EXP_NODS 0
#define EXP_NGLDS 2
#define STAGE(bufoff, gbase, voff) do { _Pragma("unroll") for (int _i = 0; _i < EXP_NGLDS; ++_i) \
        __builtin_amdgcn_global_load_lds((const unsigned*)((const char*)(gbase) + (voff)[_i]), (LAS unsigned*)(lds + (bufoff) + ldsw + _i * 8192), 16, 0, 0); } while (0)
#define GLDS1(bufoff, gbase, voff, _i) __builtin_amdgcn_global_load_lds((const unsigned*)((const char*)(gbase) + (voff)[_i]), (LAS unsigned*)(lds + (bufoff) + ldsw + (_i) * 8192), 16, 0, 0)
#define LDAH(dst, slot, m0) do { _Pragma("unroll") for (int m = (m0); m < (m0) + 2; ++m) _Pragma("unroll") for (int k = 0; k < 2; ++k) dst[m][k] = *(const LAS bf16x8*)(lds + (slot) + aoff + m * 2048 + k * 1024); } while (0)
#define LDA(dst, slot) do { _Pragma("unroll") for (int m = 0; m < 4; ++m) _Pragma("unroll") for (int k = 0; k < 2; ++k) dst[m][k] = *(const LAS bf16x8*)(lds + (slot) + aoff + m * 2048 + k * 1024); } while (0)
#define LDB(dst, slot) LDB2(dst, ((slot) / (4 * HTB)) * (4 * HTB) - 4 * HTB, (slot) % (4 * HTB))
#define LDB2(dst, _hi, _lo) do { s16x4 _r0, _r1, _r2, _r3, _r4, _r5, _r6, _r7; \
        asm volatile("ds_read_b64_tr_b16 %0, %8 offset:%10\n\tds_read_b64_tr_b16 %1, %8 offset:%11\n\tds_read_b64_tr_b16 %2, %8 offset:%12\n\tds_read_b64_tr_b16 %3, %8 offset:%13\n\t" \
                     "ds_read_b64_tr_b16 %4, %9 offset:%10\n\tds_read_b64_tr_b16 %5, %9 offset:%11\n\tds_read_b64_tr_b16 %6, %9 offset:%12\n\tds_read_b64_tr_b16 %7, %9 offset:%13" \
                     : "=&v"(_r0), "=&v"(_r1), "=&v"(_r2), "=&v"(_r3), "=&v"(_r4), "=&v"(_r5), "=&v"(_r6), "=&v"(_r7) \
                     : "v"(boff[0] + (_hi)), "v"(boff[1] + (_hi)), "i"(_lo), "i"((_lo) + 1024), "i"((_lo) + 8192), "i"((_lo) + 9216) : "memory"); \
        dst[0][0] = __builtin_shufflevector(_r0, _r1, 0, 1, 2, 3, 4, 5, 6, 7); dst[0][1] = __builtin_shufflevector(_r2, _r3, 0, 1, 2, 3, 4, 5, 6, 7); \
        dst[1][0] = __builtin_shufflevector(_r4, _r5, 0, 1, 2, 3, 4, 5, 6, 7); dst[1][1] = __builtin_shufflevector(_r6, _r7, 0, 1, 2, 3, 4, 5, 6, 7); } while (0)
#define MMA16(ai, bj, At, Bt) do { _Pragma("unroll") for (int i = 0; i < 16; ++i) { const int k = i >> 3, m = (i >> 1) & 3, n = i & 1; \
        acc[ai][bj][m][n] = __builtin_amdgcn_mfma_f32_16x16x32_bf16(Bt[n][k], At[m][k], acc[ai][bj][m][n], 0, 0, 0); } } while (0)
#define MMA_BAR(ai, bj, At, Bt) do { __builtin_amdgcn_s_setprio(1); MMA16(ai, bj, At, Bt); __builtin_amdgcn_s_setprio(0); BAR; SCHED; } while (0)
#define MMA32_BAR(ai) do { __builtin_amdgcn_s_setprio(1); MMA16(ai, 0, At, B0); MMA16(ai, 1, At, B1); __builtin_amdgcn_s_setprio(0); BAR; SCHED; } while (0)
#define WAIT_V(n) asm volatile("s_waitcnt vmcnt(" #n ")" ::: "memory")
#define WAIT_L(n) asm volatile("s_waitcnt lgkmcnt(" #n ")" ::: "memory")
#define BAR __builtin_amdgcn_s_barrier()
#define SCHED __builtin_amdgcn_sched_barrier(0)

    f32x4 acc[2][2][4][2];
#pragma unroll
    for (int a = 0; a < 2; ++a)
#pragma unroll
        for (int b = 0; b < 2; ++b)
#pragma unroll
            for (int m = 0; m < 4; ++m)
#pragma unroll
                for (int n = 0; n < 2; ++n) acc[a][b][m][n] = (f32x4){0.f, 0.f, 0.f, 0.f};
    bf16x8 At[4][2], B0[2][2], B1[2][2];

    constexpr long long HSTEP_A = (long long)128 * KB * 2;
    constexpr size_t HSTEP_B = 256;
    const long long dA = h ? -HSTEP_A : HSTEP_A;
    const size_t rowOff = ((size_t)pm * 256 + (size_t)h * 128) * KB * 2;
    const char* const aLoc = (const char*)xloc + (size_t)rank * RECV_BLK + rowOff + (size_t)h * 128;
    const char* const wT = (const char*)wmat + (size_t)pn * 512;
    const char* const wCol = wT + ((size_t)h << 20);
    const char* const bLoc = wCol + ((size_t)rank << 24);

    STAGE(SB(0, 0), bLoc, voffB); STAGE(SA(0, 0), aLoc, voffA); STAGE(SB(0, 1), bLoc + HSTEP_B, voffB); STAGE(SA(0, 1), aLoc + dA, voffA);
    STAGE(SB(1, 0), bLoc + ((size_t)1 << 21), voffB); STAGE(SA(1, 0), aLoc + 256, voffA); STAGE(SB(1, 1), bLoc + ((size_t)1 << 21) + HSTEP_B, voffB);
    asm volatile("s_waitcnt vmcnt(6)" : "+v"(seqv) :: "memory");
    BAR; asm volatile("" ::: "memory");
    const unsigned seq = (unsigned)__builtin_amdgcn_readfirstlane(seqv) + 1u;
    if (tid == 0) asm volatile("global_store_dword %0, %1, off sc1" :: "v"(myws + W_SEQ + (size_t)bid * 128), "v"(seq) : "memory");
    const size_t parOff = W_RECV + (size_t)(seq & 1u) * RECV_PAR;
    const char* const aRemT = myws + parOff + rowOff;
    const char* const aRem = aRemT + (size_t)h * 128;
    const char* const aLocB = aLoc - ((size_t)rank << 21);
    const char* const fRem = myws + W_XFLAG + (size_t)pm * 128;
    const unsigned vfl = (unsigned)(lane & 7) * 4u;
    const unsigned r8 = (unsigned)rank * 8u;
    GAS unsigned* const tmo = (GAS unsigned*)(myws + W_TMO);

    const char* pA1 = aLoc + 256; const char* pA2 = aLoc + 512; const char* pB2 = bLoc + ((size_t)2 << 21);
    const char* pA3 = aLoc + 768; const char* pB3 = bLoc + ((size_t)3 << 21);
    const char* pF2 = fRem; const char* pF3 = fRem;
    unsigned e3 = 3;
    int sidx = 0;
    int sslot = (bid * PACE_NUM) / PACE_DEN;
    const unsigned vsend = (unsigned)(wid * 4 + (lane >> 4)) * (KB * 2) + (unsigned)(lane & 15) * 16u;
    const char* sSrc; char* sDst; char* sFlag;
#define SEND_SETUP() do { const int q = sidx * 256 + bid, a = q / 224, r = q - a * 224, rg = r / 7, dj = r - rg * 7, d = (rank + 1 + dj) & 7; \
        char* const pw = (char*)ws_all[d]; const size_t o = (size_t)rg * 32 * (KB * 2) + (size_t)a * 256; \
        sSrc = (const char*)xloc + (size_t)d * RECV_BLK + o; sDst = pw + parOff + (size_t)rank * RECV_BLK + o; \
        sFlag = pw + W_XFLAG + (size_t)((rank * 8 + a) * 4 + (rg >> 3)) * 128 + (size_t)(rg & 7) * 4; } while (0)
    SEND_SETUP();
    u32x4 pv = {0u, 0u, 0u, 0u};
    unsigned fv = 0xffffffffu;
    if (wr == 1) BAR;
    STAMP(1);

#define DIAG_SLOW
#define UNLIKELY(x) __builtin_expect(!!(x), 0)
#define SPIN_FLAGS(ptr) do { const GAS unsigned* fp = (const GAS unsigned*)(ptr); const unsigned long long t0 = __builtin_amdgcn_s_memrealtime(); \
            while (!__all(__hip_atomic_load(fp, __ATOMIC_RELAXED, __HIP_MEMORY_SCOPE_SYSTEM) >= seq)) { \
                if (__builtin_amdgcn_readfirstlane(__hip_atomic_load(tmo, __ATOMIC_RELAXED, __HIP_MEMORY_SCOPE_AGENT)) == seq || __builtin_amdgcn_s_memrealtime() - t0 > DEADLINE_TICKS) { \
                    __hip_atomic_store(tmo, seq, __ATOMIC_RELAXED, __HIP_MEMORY_SCOPE_AGENT); break; } \
                __builtin_amdgcn_s_sleep(2); } \
            asm volatile("" ::: "memory"); DIAG_SLOW; } while (0)
#define FSLOT(b) (L_FLAG + (b) * 2048 + wid * 256)
#define TS(x)
#define DIAG2_ACC
#define K_TILE(b, t) do { \
        const int sd = (t) - sslot; \
        TS(ts0); \
          \
        __builtin_amdgcn_global_load_lds((const unsigned*)(pF3 + vfl), (LAS unsigned*)(lds + FSLOT(b)), 4, 0, 17); \
        if (UNLIKELY(sd == 0)) __builtin_amdgcn_global_load_lds((const unsigned*)(sSrc + vsend), (LAS unsigned*)(lds + L_SEND + ldsw), 16, 0, 0); \
        if (UNLIKELY(sd == 3 && wid == 0)) asm volatile("global_store_dword %0, %1, off sc0 sc1" :: "v"(sFlag), "v"(seq) : "memory"); \
        LDB(B0, SB(b, 0)); SCHED; GLDS1(SA((b) ^ 1, 1), pA1 + dA, voffA, 0); SCHED; \
        LDAH(At, SA(b, 0), 0); SCHED; GLDS1(SA((b) ^ 1, 1), pA1 + dA, voffA, 1); SCHED; \
        LDAH(At, SA(b, 0), 2); SCHED; GLDS1(rgW, pB2 + HSTEP_B, voffB, 0); SCHED; \
        LDB2(B1, rgR - 4 * HTB, 0); SCHED; GLDS1(rgW, pB2 + HSTEP_B, voffB, 1); \
        TS(ts1); \
        if (UNLIKELY(sd == 0 || sd == 2 || (sd == 3 && wid == 0))) WAIT_V(12); else WAIT_V(11); \
        WAIT_L(0); TS(ts2); BAR; SCHED; TS(ts3); MMA32_BAR(0); \
        TS(ts4); \
          \
        asm volatile("ds_read_b32 %0, %1 offset:%2" : "=v"(fv) : "v"(fadr), "i"(((b) ^ 1) * 2048) : "memory"); \
        if (UNLIKELY(sd == 1)) pv = *(const volatile LAS u32x4*)(lds + L_SEND + ldsw + lane * 16); \
        GLDS1(SB(b, 0), pB2, voffB, 0); SCHED; \
        LDAH(At, SA(b, 1), 0); SCHED; GLDS1(SB(b, 0), pB2, voffB, 1); SCHED; \
        LDAH(At, SA(b, 1), 2); SCHED; \
        asm volatile("s_waitcnt lgkmcnt(8)" : "+v"(fv) :: "memory"); \
        if (UNLIKELY((t) >= 6 && !__all(fv >= seq))) SPIN_FLAGS(pF2 + vfl); \
        GLDS1(SA(b, 0), pA2, voffA, 0); \
        if (UNLIKELY(sd == 3)) { ++sidx; if (sidx < 7) { sslot = ((sidx * 256 + bid) * PACE_NUM) / PACE_DEN; SEND_SETUP(); } else sslot = 1000; } \
          \
        const char* const pA2o = pA2; \
        pA1 = pA2; pA2 = pA3; pB2 = pB3; pF2 = pF3; \
        e3 += (e3 < 8u) ? 1u : 8u; e3 -= (e3 >= 64u) ? 55u : 0u; \
        { const unsigned c = (r8 + e3) & 63u; \
          pA3 = (e3 < 8u ? aLocB : aRem) + (((c >> 3) << 21) | ((c & 7u) << 8)); \
          pB3 = wCol + ((size_t)c << 21); pF3 = fRem + (c << 9); } \
        { const int r = rgR; rgR = rgN; rgN = rgW; rgW = r; } \
        asm volatile("" : "+s"(pA3), "+s"(pB3), "+s"(pF3), "+s"(e3), "+s"(rgR), "+s"(rgN), "+s"(rgW)); \
        SCHED; GLDS1(SA(b, 0), pA2o, voffA, 1); \
        if (UNLIKELY(sd == 1)) asm volatile("global_store_dwordx4 %0, %1, off sc0 sc1\n\ts_nop 1" :: "v"(sDst + vsend), "v"(pv) : "memory"); \
        TS(ts5); \
        if (UNLIKELY(sd == 0 || sd == 1 || (sd == 3 && wid == 0))) WAIT_V(10); else WAIT_V(9); \
        WAIT_L(0); DIAG2_ACC; BAR; SCHED; MMA32_BAR(1); \
    } while (0)

    const unsigned fadr = (unsigned)(uintptr_t)lds + FSLOT(0) + lane * 4;
    static_assert(NF % 3 == 2, "the ring of second w halves must end in its home position");
    int rgR = SB(0, 1), rgN = SB(1, 1), rgW = L_X;
    for (int t = 0; t < NF - 2; t += 2) {
        K_TILE(0, t);
        K_TILE(1, t + 1);
    }

#define TAIL_C(u) ((r8 + (unsigned)e_of(NF + (u) / 2)) & 63u)
#define TAIL_A(u) (aRemT + ((((TAIL_C(u) >> 3) << 21) | ((TAIL_C(u) & 7u) << 8)) + ((u) & 1) * 128))
#define TAIL_B(u) (wT + (((size_t)TAIL_C(u) << 21) + ((size_t)((u) & 1) << 20)))
    const char* tfp;
    { int k = lane >> 3; k = k < SH / 2 ? k : SH / 2 - 1; const int u = NF + k - 8, a = u / 7, j = u - a * 7;
      tfp = fRem + (((r8 + (unsigned)((j + 1) * 8 + a)) & 63u) << 9) + vfl; }
    {
        LDB(B0, SB(0, 0)); LDA(At, SA(0, 0)); LDB(B1, SB(0, 1));
        __builtin_amdgcn_global_load_lds((const unsigned*)tfp, (LAS unsigned*)(lds + FSLOT(0)), 4, 0, 17);
        STAGE(SA(1, 1), pA1 + dA, voffA);
        WAIT_V(9); WAIT_L(0); BAR; SCHED; MMA32_BAR(0);
        fv = *(const volatile LAS unsigned*)(lds + FSLOT(1) + lane * 4);
        LDA(At, SA(0, 1));
        STAGE(slotB0(0), TAIL_B(0), voffB); STAGE(slotB1(0), TAIL_B(0) + HSTEP_B, voffB);
        if (UNLIKELY(!__all(fv >= seq))) SPIN_FLAGS(pF2 + vfl);
        STAGE(slotA(0), TAIL_A(0), voffA);
        WAIT_V(9); WAIT_L(0); BAR; SCHED; MMA32_BAR(1);
        LDB(B0, SB(1, 0)); LDA(At, SA(1, 0)); LDB(B1, SB(1, 1));
        STAGE(slotB0(2), TAIL_B(2), voffB);
        WAIT_V(8); WAIT_L(0); BAR; SCHED; MMA32_BAR(0);
        fv = *(const volatile LAS unsigned*)(lds + FSLOT(0) + lane * 4);
        LDA(At, SA(1, 1));
        STAGE(slotB0(1), TAIL_B(1), voffB);
        if (UNLIKELY(!__all(fv >= seq))) SPIN_FLAGS(tfp);
        STAGE(slotA(1), TAIL_A(1), voffA); STAGE(slotB1(1), TAIL_B(1) + HSTEP_B, voffB);
        WAIT_V(8); WAIT_L(0); BAR; SCHED; MMA32_BAR(1);
    }
    STAMP(2);
    char* const slabOut = myws + W_SLAB + (size_t)bid * SLAB_B + (size_t)wid * 8192 + lane * 16;
    const char* const slabIn = myws + W_SLAB + (size_t)partner * SLAB_B + (size_t)wid * 8192 + lane * 16;
    const char* const pflag = myws + W_EFLAG + (size_t)(partner * 8 + wid) * 128;
    u32x4 in[8];
    bool early = false;
    static_for([&](auto uc) {
        constexpr int u = decltype(uc)::value;
        constexpr bool st = u + 2 < SH;
#pragma unroll
        for (int jj = 0; jj < n_store(u); ++jj) {
            const int j = u * XS_PER + jj, bj = j >> 2, m = j & 3;
            const f32x4 v0 = acc[1][bj][m][0], v1 = acc[1][bj][m][1];
            u32x4 w; w.x = cvt_pk_bf16(v0[0], v0[1]); w.y = cvt_pk_bf16(v0[2], v0[3]); w.z = cvt_pk_bf16(v1[0], v1[1]); w.w = cvt_pk_bf16(v1[2], v1[3]);
            asm volatile("global_store_dwordx4 %0, %1, off sc1\n\ts_nop 1" :: "v"(slabOut + j * 1024), "v"(w) : "memory");
        }
        if constexpr (u == S_FLAG) asm volatile("global_store_dword %0, %1, off sc1" :: "v"(myws + W_EFLAG + (size_t)(bid * 8 + wid) * 128), "v"(seq) : "memory");
        if constexpr (u == S_SAMP) __builtin_amdgcn_global_load_lds((const unsigned*)pflag, (LAS unsigned*)(lds + FSLOT(0)), 4, 0, 17);
        if constexpr (u == S_LOAD) asm volatile("ds_read_b32 %0, %1" : "=v"(fv) : "v"(fadr) : "memory");
        SCHED;
        if constexpr (st && u > 0) { GLDS1(slotB0(u + 2), TAIL_B(u + 2), voffB, 0); SCHED; }
        LDB(B0, slotB0(u)); SCHED;
        if constexpr (st && u > 0) { GLDS1(slotB0(u + 2), TAIL_B(u + 2), voffB, 1); SCHED; }
        LDAH(At, slotA(u), 0); SCHED;
        if constexpr (st) { GLDS1(slotA(u + 2), TAIL_A(u + 2), voffA, 0); SCHED; }
        LDAH(At, slotA(u), 2); SCHED;
        if constexpr (st) { GLDS1(slotA(u + 2), TAIL_A(u + 2), voffA, 1); SCHED; }
        LDB(B1, slotB1(u)); SCHED;
        if constexpr (st) { GLDS1(slotB1(u + 2), TAIL_B(u + 2) + HSTEP_B, voffB, 0); GLDS1(slotB1(u + 2), TAIL_B(u + 2) + HSTEP_B, voffB, 1); }
        asm volatile("" ::: "memory");
        if constexpr (u == S_LOAD) {
            asm volatile("s_waitcnt lgkmcnt(15)" : "+v"(fv) :: "memory");
            early = (unsigned)__builtin_amdgcn_readfirstlane(fv) >= seq;
            asm volatile("" ::: "memory");
#pragma unroll
            for (int i = 0; i < 8; ++i) asm volatile("global_load_dwordx4 %0, %1, off offset:%2 sc1" : "=v"(in[i]) : "v"(slabIn + (i >> 2) * 4096), "i"((i & 3) * 1024) : "memory");
        }
        if constexpr (u + 1 < SH) wait_vm<tail_wait(u)>();
        WAIT_L(0); BAR; SCHED; MMA32_BAR(0);
    }, std::make_integer_sequence<int, SH>{});
    if (wr == 0) BAR;
    STAMP(3);

    {
        asm volatile("s_waitcnt vmcnt(0)" : "+v"(in[0]), "+v"(in[1]), "+v"(in[2]), "+v"(in[3]), "+v"(in[4]), "+v"(in[5]), "+v"(in[6]), "+v"(in[7]) :: "memory");
        if (UNLIKELY(!early)) {
            const GAS unsigned* fp = (const GAS unsigned*)pflag;
            const unsigned long long t0 = __builtin_amdgcn_s_memrealtime();
            while ((unsigned)__builtin_amdgcn_readfirstlane(__hip_atomic_load(fp, __ATOMIC_RELAXED, __HIP_MEMORY_SCOPE_AGENT)) < seq) {
                if (__builtin_amdgcn_readfirstlane(__hip_atomic_load(tmo, __ATOMIC_RELAXED, __HIP_MEMORY_SCOPE_AGENT)) == seq || __builtin_amdgcn_s_memrealtime() - t0 > DEADLINE_TICKS) {
                    __hip_atomic_store(tmo, seq, __ATOMIC_RELAXED, __HIP_MEMORY_SCOPE_AGENT); break; }
                __builtin_amdgcn_s_sleep(1);
            }
            asm volatile("" ::: "memory");
#pragma unroll
            for (int i = 0; i < 8; ++i) asm volatile("global_load_dwordx4 %0, %1, off offset:%2 sc1" : "=v"(in[i]) : "v"(slabIn + (i >> 2) * 4096), "i"((i & 3) * 1024) : "memory");
            asm volatile("s_waitcnt vmcnt(0)" : "+v"(in[0]), "+v"(in[1]), "+v"(in[2]), "+v"(in[3]), "+v"(in[4]), "+v"(in[5]), "+v"(in[6]), "+v"(in[7]) :: "memory");
            DIAG_SLOW;
        }
        const bool bad = __builtin_amdgcn_readfirstlane(__hip_atomic_load(tmo, __ATOMIC_RELAXED, __HIP_MEMORY_SCOPE_AGENT)) == seq;
        STAMP(4);
        unsigned short* orow = out + (size_t)(pm * 256 + h * 128 + wr * 64 + fr) * NN + pn * 256 + wc * 32 + (fq & 1) * 16 + (fq >> 1) * 8;
#pragma unroll
        for (int bj = 0; bj < 2; ++bj)
#pragma unroll
            for (int m = 0; m < 4; ++m) {
                f32x4 v0 = acc[0][bj][m][0], v1 = acc[0][bj][m][1];
                const u32x4 p = in[bj * 4 + m];
                v0[0] += __uint_as_float(p.x << 16); v0[1] += __uint_as_float(p.x & 0xffff0000u); v0[2] += __uint_as_float(p.y << 16); v0[3] += __uint_as_float(p.y & 0xffff0000u);
                v1[0] += __uint_as_float(p.z << 16); v1[1] += __uint_as_float(p.z & 0xffff0000u); v1[2] += __uint_as_float(p.w << 16); v1[3] += __uint_as_float(p.w & 0xffff0000u);
                unsigned a0 = cvt_pk_bf16(gelu_tanh(v0[0]), gelu_tanh(v0[1])), a1 = cvt_pk_bf16(gelu_tanh(v0[2]), gelu_tanh(v0[3]));
                unsigned c0 = cvt_pk_bf16(gelu_tanh(v1[0]), gelu_tanh(v1[1])), c1 = cvt_pk_bf16(gelu_tanh(v1[2]), gelu_tanh(v1[3]));
                { auto r = __builtin_amdgcn_permlane16_swap(a0, c0, false, false); a0 = r[0]; c0 = r[1]; }
                { auto r = __builtin_amdgcn_permlane16_swap(a1, c1, false, false); a1 = r[0]; c1 = r[1]; }
                u32x4 w = {a0, a1, c0, c1};
                if (bad) w = (u32x4){0x7fc07fc0u, 0x7fc07fc0u, 0x7fc07fc0u, 0x7fc07fc0u};
                *(u32x4*)(orow + (size_t)m * 16 * NN + bj * 128) = w;
            }
    }
}

extern "C" void kernel_launch(int rank, int nranks, ncclComm_t comm,
                              hipStream_t stream,
                              void* const* d_in, const int* in_sizes, int n_in,
                              void* d_out, int out_size,
                              const int* in_dtypes, int out_dtype,
                              void* const* d_out_all,
                              void* const* d_ws_all, size_t ws_bytes) {
    (void)comm; (void)in_sizes; (void)n_in; (void)out_size; (void)in_dtypes; (void)out_dtype; (void)d_out_all;
    if (nranks != NR || (unsigned)rank >= (unsigned)NR || ws_bytes < WS_NEED) { fprintf(stderr, "a2a_gemm: unsupported configuration\n"); return; }
    static bool attr_set = false;
    if (!attr_set) {
        HIP_CHECK(hipFuncSetAttribute((const void*)a2a_gemm_gelu_kernel, hipFuncAttributeMaxDynamicSharedMemorySize, LDS_BYTES));
        attr_set = true;
    }
    hipLaunchKernelGGL(a2a_gemm_gelu_kernel, dim3(NWG), dim3(512), LDS_BYTES, stream, rank,
                       (const unsigned short*)d_in[0], (const unsigned short*)d_in[1], (unsigned short*)d_out, d_ws_all);
    HIP_CHECK(hipGetLastError());
}
```

```cpp
#include <hip/hip_runtime.h>
#include <rccl/rccl.h>
#include <cstdio>
#include <cstdint>
#include <utility>

#define HIP_CHECK(expr) do { hipError_t _e = (expr); if (_e != hipSuccess) { \
    fprintf(stderr, "HIP error %s at %s:%d\n", hipGetErrorString(_e), __FILE__, __LINE__); \
    return; } } while (0)

#define LAS __attribute__((address_space(3)))
#define GAS __attribute__((address_space(1)))
typedef short bf16x8 __attribute__((ext_vector_type(8)));
typedef short s16x4 __attribute__((ext_vector_type(4)));
typedef float f32x4 __attribute__((ext_vector_type(4)));
typedef unsigned u32x4 __attribute__((ext_vector_type(4)));

#define MAP_OPT 1
#define PACE_NUM 9
#define PACE_DEN 448
#define EARLY_BAR 0
#define PF_DIST 0
#define TOUCH_T 0
#define TOUCH_M 3
#define SHARED_KB 16
#define XS_PERSEG 8
#define XS_PS 6

namespace {
constexpr int NR = 8;
constexpr int MS = 1024;
constexpr int KB = 1024;
constexpr int NN = 8192;
constexpr int SH = SHARED_KB;
constexpr int NF = 64 - SH / 2;
constexpr int NWG = 256;
constexpr int HTB = 16384;
static_assert(SH % 4 == 0 && SH >= 4 && SH <= 16, "tail size");
static_assert((1791 * PACE_NUM) / PACE_DEN + 3 < NF - 2, "all sends must finish inside the generic K-loop");
static_assert(TOUCH_T == 0 || ((1791 * PACE_NUM) / PACE_DEN + 3 < TOUCH_T && TOUCH_T + 4 < NF - 2), "touch window");
constexpr int L_X = 8 * HTB;
constexpr int L_SEND = 9 * HTB;
constexpr int L_FLAG = L_SEND + 8192;
constexpr int L_JUNK = L_FLAG + 4096;
constexpr int LDS_BYTES = L_JUNK + 256;
constexpr size_t W_TMO = 256;
constexpr size_t W_XFLAG = 4096;
constexpr size_t W_EFLAG = 65536;
constexpr size_t W_SEQ = W_EFLAG + 262144;
constexpr size_t W_RECV = 1u << 20;
constexpr size_t RECV_BLK = (size_t)MS * KB * 2;
constexpr size_t RECV_PAR = RECV_BLK * NR;
constexpr size_t W_SLAB = W_RECV + 2 * RECV_PAR;
constexpr size_t SLAB_B = 65536;
constexpr size_t WS_NEED = W_SLAB + (size_t)NWG * SLAB_B;
constexpr unsigned long long DEADLINE_TICKS = 40000000ull;

constexpr int XS_PER = XS_PERSEG;
constexpr int S_FLAG = 8 / XS_PER + 1;
constexpr int S_SAMP = S_FLAG + XS_PS;
constexpr int S_LOAD = S_SAMP + 2;
static_assert(S_LOAD < SH, "tail too short for the hand-over schedule");
constexpr int n_stage(int u) { return u + 2 < SH ? (u == 0 ? 4 : 6) : 0; }
constexpr int n_store(int u) { return u * XS_PER < 8 ? XS_PER : 0; }
constexpr int pf_at(int T) { return (PF_DIST && T + PF_DIST < 64) ? 1 : 0; }
constexpr int pf_is(int T) { return pf_at(T); }
constexpr int n_post(int u) { return u < 0 ? pf_at(NF - 1) : (u == S_LOAD ? 8 : 0) + ((u & 1) ? pf_at(NF + u / 2) : 0); }
constexpr int n_ops(int u) { return n_store(u) + (u == S_FLAG) + (u == S_SAMP) + n_stage(u) + n_post(u); }
constexpr int tail_wait(int u) { return n_ops(u) + n_post(u - 1); }
constexpr int slotB0(int u) { return u % 3 == 0 ? 4 * HTB : (u % 3 == 1 ? 6 * HTB : L_X); }
constexpr int slotA(int u) { return u % 3 == 0 ? 0 : (u % 3 == 1 ? 2 * HTB : HTB); }
constexpr int slotB1(int u) { return u % 3 == 0 ? 5 * HTB : (u % 3 == 1 ? 7 * HTB : 3 * HTB); }
constexpr int e_of(int T) { return T < 8 ? T : ((T - 8) % 7 + 1) * 8 + (T - 8) / 7; }

__device__ __forceinline__ int lds_byteA(int r, int c) { const int st = (r >> 4) * 2 + (c >> 5), rr = r & 15, cc = c & 31, ob = rr * 64 + cc * 2; return st * 1024 + (ob ^ (((ob >> 9) & 1) << 5)); }
__device__ __forceinline__ void stage_rcA(int b, int& R, int& C) { const int st = b / 1024, sb = b % 1024, swz = sb ^ (((sb >> 9) & 1) << 5); R = (st >> 1) * 16 + swz / 64; C = (st & 1) * 32 + (swz % 64) / 2; }
__device__ __forceinline__ unsigned cvt_pk_bf16(float lo, float hi) { unsigned r; asm volatile("v_cvt_pk_bf16_f32 %0, %1, %2" : "=v"(r) : "v"(lo), "v"(hi)); return r; }
__device__ __forceinline__ float gelu_tanh(float y) {
    const float u = y * y * 0.044715f + 1.0f;
    const float e = __builtin_amdgcn_exp2f(y * u * (-2.0f * 0.7978845608028654f * 1.4426950408889634f));
    return y * __builtin_amdgcn_rcpf(1.0f + e);
}
template <int N> __device__ __forceinline__ void wait_vm() { if constexpr (N < 63) asm volatile("s_waitcnt vmcnt(%0)" :: "n"(N) : "memory"); }
template <class F, int... I> __device__ __forceinline__ void static_for(F&& f, std::integer_sequence<int, I...>) { (f(std::integral_constant<int, I>{}), ...); }
}

__global__ __launch_bounds__(512, 2)
void a2a_gemm_gelu_kernel(int rank, const unsigned short* __restrict__ xloc, const unsigned short* __restrict__ wmat,
                          unsigned short* __restrict__ out, void* const* __restrict__ ws_all) {
    extern __shared__ __attribute__((aligned(16))) unsigned char shm[];
    LAS unsigned char* lds = (LAS unsigned char*)shm;
    const int tid = threadIdx.x, wid = __builtin_amdgcn_readfirstlane(tid >> 6), lane = tid & 63, wr = wid >> 2, wc = wid & 3, fr = lane & 15, fq = lane >> 4;
    const int bid = blockIdx.x, xcd = bid & 7, idx = bid >> 3;
    const int pn = xcd * 4 + (idx & 3), pm = (idx >> 2) & 3, h = idx >> 4;
    const int partner = bid ^ (16 << 3);
    char* const myws = (char*)ws_all[rank];
#define STAMP(i)
    STAMP(0);
    unsigned seqv;
    asm volatile("global_load_dword %0, %1, off sc1" : "=v"(seqv) : "v"(myws + W_SEQ + (size_t)bid * 128) : "memory");

    unsigned voffA[2], voffB[2];
#pragma unroll
    for (int i = 0; i < 2; ++i) {
        int R, C; stage_rcA(tid * 16 + i * 8192, R, C);
        voffA[i] = (unsigned)(R * KB + C) * 2u;
        const int b = tid * 16 + i * 8192, k = b >> 8, cp = (b >> 4) & 15, f = (k & 3) | (((k >> 3) & 1) << 2), c = cp ^ (f << 1);
        voffB[i] = (unsigned)k * (NN * 2) + (unsigned)c * 16u;
    }
    const unsigned ldsw = (unsigned)wid * 1024u;
    const int aoff = lds_byteA(wr * 64 + fr, fq * 8);
    int boff[2];
    { const int g = lane >> 4, q = (lane & 15) >> 2, p = lane & 3, f = q | ((g & 1) << 2);
#pragma unroll
      for (int nf = 0; nf < 2; ++nf) boff[nf] = (int)(unsigned)(uintptr_t)lds + 4 * HTB + (8 * g + q) * 256 + (((wc * 4 + nf * 2 + (p >> 1)) ^ (f << 1)) * 16) + (p & 1) * 8; }

#define SA(b, hh) (((b) * 2 + (hh)) * HTB)
#define SB(b, hh) ((4 + (b) * 2 + (hh)) * HTB)
#define EXP_NODS 0
#define EXP_NGLDS 2
#define EXP_SVC 0
    const bool svc = EXP_SVC && wid == 7;
#define STAGE(bufoff, gbase, voff) do { if (!svc) _Pragma("unroll") for (int _i = 0; _i < EXP_NGLDS; ++_i) \
        __builtin_amdgcn_global_load_lds((const unsigned*)((const char*)(gbase) + (voff)[_i]), (LAS unsigned*)(lds + (bufoff) + ldsw + _i * 8192), 16, 0, 0); } while (0)
#define GLDS1(bufoff, gbase, voff, _i) do { if (!svc) __builtin_amdgcn_global_load_lds((const unsigned*)((const char*)(gbase) + (voff)[_i]), (LAS unsigned*)(lds + (bufoff) + ldsw + (_i) * 8192), 16, 0, 0); } while (0)
#define LDAH(dst, slot, m0) do { _Pragma("unroll") for (int m = (m0); m < (m0) + 2; ++m) _Pragma("unroll") for (int k = 0; k < 2; ++k) dst[m][k] = *(const LAS bf16x8*)(lds + (slot) + aoff + m * 2048 + k * 1024); } while (0)
#define LDA(dst, slot) do { _Pragma("unroll") for (int m = 0; m < 4; ++m) _Pragma("unroll") for (int k = 0; k < 2; ++k) dst[m][k] = *(const LAS bf16x8*)(lds + (slot) + aoff + m * 2048 + k * 1024); } while (0)
#define LDB(dst, slot) LDB2(dst, ((slot) / (4 * HTB)) * (4 * HTB) - 4 * HTB, (slot) % (4 * HTB))
#define LDB2(dst, _hi, _lo) do { s16x4 _r0, _r1, _r2, _r3, _r4, _r5, _r6, _r7; \
        asm volatile("ds_read_b64_tr_b16 %0, %8 offset:%10\n\tds_read_b64_tr_b16 %1, %8 offset:%11\n\tds_read_b64_tr_b16 %2, %8 offset:%12\n\tds_read_b64_tr_b16 %3, %8 offset:%13\n\t" \
                     "ds_read_b64_tr_b16 %4, %9 offset:%10\n\tds_read_b64_tr_b16 %5, %9 offset:%11\n\tds_read_b64_tr_b16 %6, %9 offset:%12\n\tds_read_b64_tr_b16 %7, %9 offset:%13" \
                     : "=&v"(_r0), "=&v"(_r1), "=&v"(_r2), "=&v"(_r3), "=&v"(_r4), "=&v"(_r5), "=&v"(_r6), "=&v"(_r7) \
                     : "v"(boff[0] + (_hi)), "v"(boff[1] + (_hi)), "i"(_lo), "i"((_lo) + 1024), "i"((_lo) + 8192), "i"((_lo) + 9216) : "memory"); \
        dst[0][0] = __builtin_shufflevector(_r0, _r1, 0, 1, 2, 3, 4, 5, 6, 7); dst[0][1] = __builtin_shufflevector(_r2, _r3, 0, 1, 2, 3, 4, 5, 6, 7); \
        dst[1][0] = __builtin_shufflevector(_r4, _r5, 0, 1, 2, 3, 4, 5, 6, 7); dst[1][1] = __builtin_shufflevector(_r6, _r7, 0, 1, 2, 3, 4, 5, 6, 7); } while (0)
#define MMA16(ai, bj, At, Bt) do { _Pragma("unroll") for (int i = 0; i < 16; ++i) { const int k = i >> 3, m = (i >> 1) & 3, n = i & 1; \
        acc[ai][bj][m][n] = __builtin_amdgcn_mfma_f32_16x16x32_bf16(Bt[n][k], At[m][k], acc[ai][bj][m][n], 0, 0, 0); } } while (0)
#define MMA_BAR(ai, bj, At, Bt) do { __builtin_amdgcn_s_setprio(1); MMA16(ai, bj, At, Bt); __builtin_amdgcn_s_setprio(0); BAR; SCHED; } while (0)
#define MMA32_BAR(ai) do { __builtin_amdgcn_s_setprio(1); MMA16(ai, 0, At, B0); MMA16(ai, 1, At, B1); __builtin_amdgcn_s_setprio(0); BAR; SCHED; } while (0)
#define WAIT_V(n) asm volatile("s_waitcnt vmcnt(" #n ")" ::: "memory")
#define WAIT_L(n) asm volatile("s_waitcnt lgkmcnt(" #n ")" ::: "memory")
#define BAR __builtin_amdgcn_s_barrier()
#define SCHED __builtin_amdgcn_sched_barrier(0)

    f32x4 acc[2][2][4][2];
#pragma unroll
    for (int a = 0; a < 2; ++a)
#pragma unroll
        for (int b = 0; b < 2; ++b)
#pragma unroll
            for (int m = 0; m < 4; ++m)
#pragma unroll
                for (int n = 0; n < 2; ++n) acc[a][b][m][n] = (f32x4){0.f, 0.f, 0.f, 0.f};
    bf16x8 At[4][2], B0[2][2], B1[2][2];

    constexpr long long HSTEP_A = (long long)128 * KB * 2;
    constexpr size_t HSTEP_B = 256;
    const long long dA = h ? -HSTEP_A : HSTEP_A;
    const size_t rowOff = ((size_t)pm * 256 + (size_t)h * 128) * KB * 2;
    const char* const aLoc = (const char*)xloc + (size_t)rank * RECV_BLK + rowOff + (size_t)h * 128;
    const char* const wT = (const char*)wmat + (size_t)pn * 512;
    const char* const wCol = wT + ((size_t)h << 20);
    const char* const bLoc = wCol + ((size_t)rank << 24);

    const unsigned vpf = (unsigned)(pm * 16 + wid * 2 + ((lane >> 2) & 1)) * (NN * 2) + (unsigned)(lane & 3) * 128u;
    unsigned eP = (unsigned)e_of(PF_DIST);
#define PREFETCH(ptr) __builtin_amdgcn_global_load_lds((const unsigned*)((ptr) + vpf), (LAS unsigned*)(lds + L_JUNK), 4, 0, 0)
#define PREFETCHA(ptr)
    constexpr int PFN = PF_DIST ? 1 : 0, PFI = PFN;
    static_assert(PF_DIST <= 8, "the prologue prefetch covers local K-blocks only");
    STAGE(SB(0, 0), bLoc, voffB); STAGE(SA(0, 0), aLoc, voffA); STAGE(SB(0, 1), bLoc + HSTEP_B, voffB); STAGE(SA(0, 1), aLoc + dA, voffA);
    STAGE(SB(1, 0), bLoc + ((size_t)1 << 21), voffB); STAGE(SA(1, 0), aLoc + 256, voffA); STAGE(SB(1, 1), bLoc + ((size_t)1 << 21) + HSTEP_B, voffB);
    if (EXP_SVC) { for (int tk = 2; tk < PF_DIST; ++tk) { PREFETCH(bLoc + ((size_t)tk << 21)); PREFETCHA(aLoc - h * HSTEP_A + tk * 256); }
        if (svc) asm volatile("s_waitcnt vmcnt(%0)" :: "n"(EXP_SVC >= 2 ? (PF_DIST - 2) * (EXP_SVC - 1) : 0) : "memory");
        asm volatile("s_waitcnt vmcnt(6)" : "+v"(seqv) :: "memory");
    } else
    if (PFN) {
        const unsigned tk = 2u + (unsigned)(lane >> 3);
        PREFETCH(bLoc + ((size_t)(tk < (unsigned)PF_DIST ? tk : 2u) << 21));
        asm volatile("s_waitcnt vmcnt(7)" : "+v"(seqv) :: "memory");
    } else asm volatile("s_waitcnt vmcnt(6)" : "+v"(seqv) :: "memory");
    BAR; asm volatile("" ::: "memory");
    const unsigned seq = (unsigned)__builtin_amdgcn_readfirstlane(seqv) + 1u;
    if (tid == 0) asm volatile("global_store_dword %0, %1, off sc1" :: "v"(myws + W_SEQ + (size_t)bid * 128), "v"(seq) : "memory");
    const size_t parOff = W_RECV + (size_t)(seq & 1u) * RECV_PAR;
    const char* const aRemT = myws + parOff + rowOff;
    const char* const aRem = aRemT + (size_t)h * 128;
    const char* const aLocB = aLoc - ((size_t)rank << 21);
    const char* const fRem = myws + W_XFLAG + (size_t)pm * 128;
    const unsigned vfl = (unsigned)(lane & 7) * 4u;
    const unsigned r8 = (unsigned)rank * 8u;
    GAS unsigned* const tmo = (GAS unsigned*)(myws + W_TMO);

    const char* pA1 = aLoc + 256; const char* pA2 = aLoc + 512; const char* pB2 = bLoc + ((size_t)2 << 21);
    const char* pA3 = aLoc + 768; const char* pB3 = bLoc + ((size_t)3 << 21);
    const char* pF2 = fRem; const char* pF3 = fRem;
    unsigned e3 = 3;
    int sidx = 0;
    int sslot = (bid * PACE_NUM) / PACE_DEN;
    const unsigned vsend = (unsigned)(wid * 4 + (lane >> 4)) * (KB * 2) + (unsigned)(lane & 15) * 16u;
    const char* sSrc; char* sDst; char* sFlag;
#define SEND_SETUP() do { const int q = sidx * 256 + bid, a = q / 224, r = q - a * 224, rg = r / 7, dj = r - rg * 7, d = (rank + 1 + dj) & 7; \
        char* const pw = (char*)ws_all[d]; const size_t o = (size_t)rg * 32 * (KB * 2) + (size_t)a * 256; \
        sSrc = (const char*)xloc + (size_t)d * RECV_BLK + o; sDst = pw + parOff + (size_t)rank * RECV_BLK + o; \
        sFlag = pw + W_XFLAG + (size_t)((rank * 8 + a) * 4 + (rg >> 3)) * 128 + (size_t)(rg & 7) * 4; } while (0)
    SEND_SETUP();
    u32x4 pv = {0u, 0u, 0u, 0u};
    unsigned fv = 0xffffffffu;
    if (wr == 1) BAR;
    STAMP(1);

#define DIAG_SLOW
#define UNLIKELY(x) __builtin_expect(!!(x), 0)
#define SPIN_FLAGS(ptr) do { const GAS unsigned* fp = (const GAS unsigned*)(ptr); const unsigned long long t0 = __builtin_amdgcn_s_memrealtime(); \
            while (!__all(__hip_atomic_load(fp, __ATOMIC_RELAXED, __HIP_MEMORY_SCOPE_SYSTEM) >= seq)) { \
                if (__builtin_amdgcn_readfirstlane(__hip_atomic_load(tmo, __ATOMIC_RELAXED, __HIP_MEMORY_SCOPE_AGENT)) == seq || __builtin_amdgcn_s_memrealtime() - t0 > DEADLINE_TICKS) { \
                    __hip_atomic_store(tmo, seq, __ATOMIC_RELAXED, __HIP_MEMORY_SCOPE_AGENT); break; } \
                __builtin_amdgcn_s_sleep(2); } \
            asm volatile("" ::: "memory"); DIAG_SLOW; } while (0)
#define FSLOT(b) (L_FLAG + (b) * 2048 + wid * 256)
#define TS(x)
#define DIAG2_ACC
#define K_TILE(b, t) do { \
        const int sd = (t) - sslot, td = (t) - TOUCH_T; \
        TS(ts0); \
          \
        __builtin_amdgcn_global_load_lds((const unsigned*)(pF3 + vfl), (LAS unsigned*)(lds + FSLOT(b)), 4, 0, 17); \
        if (UNLIKELY(sd == 0)) __builtin_amdgcn_global_load_lds((const unsigned*)(sSrc + vsend), (LAS unsigned*)(lds + L_SEND + ldsw), 16, 0, 0); \
        if (UNLIKELY(sd == 3 && wid == 0)) asm volatile("global_store_dword %0, %1, off sc0 sc1" :: "v"(sFlag), "v"(seq) : "memory"); \
        LDB(B0, SB(b, 0)); SCHED; GLDS1(SA((b) ^ 1, 1), pA1 + dA, voffA, 0); SCHED; \
        LDAH(At, SA(b, 0), 0); SCHED; GLDS1(SA((b) ^ 1, 1), pA1 + dA, voffA, 1); SCHED; \
        LDAH(At, SA(b, 0), 2); SCHED; GLDS1(rgW, pB2 + HSTEP_B, voffB, 0); SCHED; \
        LDB2(B1, rgR - 4 * HTB, 0); SCHED; GLDS1(rgW, pB2 + HSTEP_B, voffB, 1); \
        TS(ts1); \
        if (UNLIKELY(sd == 0 || sd == 2 || (sd == 3 && wid == 0) || (TOUCH_T && (td == 1 || td == 3)))) wait_vm<12 + PFN>(); else wait_vm<11 + PFN>(); \
        WAIT_L(0); TS(ts2); BAR; SCHED; TS(ts3); MMA32_BAR(0); \
        TS(ts4); \
          \
        asm volatile("ds_read_b32 %0, %1 offset:%2" : "=v"(fv) : "v"(fadr), "i"(((b) ^ 1) * 2048) : "memory"); \
        if (UNLIKELY(sd == 1)) pv = *(const volatile LAS u32x4*)(lds + L_SEND + ldsw + lane * 16); \
        GLDS1(SB(b, 0), pB2, voffB, 0); SCHED; \
        LDAH(At, SA(b, 1), 0); SCHED; GLDS1(SB(b, 0), pB2, voffB, 1); SCHED; \
        LDAH(At, SA(b, 1), 2); SCHED; \
        asm volatile("s_waitcnt lgkmcnt(8)" : "+v"(fv) :: "memory"); \
        if (UNLIKELY((t) >= 6 && !__all(fv >= seq))) SPIN_FLAGS(pF2 + vfl); \
        GLDS1(SA(b, 0), pA2, voffA, 0); \
        if (UNLIKELY(sd == 3)) { ++sidx; if (sidx < 7) { sslot = ((sidx * 256 + bid) * PACE_NUM) / PACE_DEN; SEND_SETUP(); } else sslot = 1000; } \
          \
        const char* const pA2o = pA2; \
        pA1 = pA2; pA2 = pA3; pB2 = pB3; pF2 = pF3; \
        e3 += (e3 < 8u) ? 1u : 8u; e3 -= (e3 >= 64u) ? 55u : 0u; \
        { const unsigned c = (r8 + e3) & 63u; \
          pA3 = (e3 < 8u ? aLocB : aRem) + (((c >> 3) << 21) | ((c & 7u) << 8)); \
          pB3 = wCol + ((size_t)c << 21); pF3 = fRem + (c << 9); } \
        { const int r = rgR; rgR = rgN; rgN = rgW; rgW = r; } \
        asm volatile("" : "+s"(pA3), "+s"(pB3), "+s"(pF3), "+s"(e3), "+s"(rgR), "+s"(rgN), "+s"(rgW)); \
        SCHED; GLDS1(SA(b, 0), pA2o, voffA, 1); \
        if (UNLIKELY(sd == 1)) asm volatile("global_store_dwordx4 %0, %1, off sc0 sc1\n\ts_nop 1" :: "v"(sDst + vsend), "v"(pv) : "memory"); \
        if (PFI) { const unsigned c = (r8 + eP) & 63u; PREFETCH(wCol + ((size_t)c << 21)); \
            PREFETCHA((eP < 8u ? aLocB : aRem) - h * HSTEP_A + (((c >> 3) << 21) | ((c & 7u) << 8))); \
            eP += (eP < 8u) ? 1u : 8u; eP -= (eP >= 64u) ? 55u : 0u; } \
        TS(ts5); \
        if (TOUCH_T && UNLIKELY(td == 0)) __builtin_amdgcn_global_load_lds((const unsigned*)((TOUCH_M & 1) ? myws + W_SLAB + (size_t)bid * SLAB_B + (size_t)wid * 8192 + lane * 128 : pF3 + vfl), (LAS unsigned*)(lds + L_JUNK), 4, 0, 0); \
        if (TOUCH_T && UNLIKELY(td == 2)) __builtin_amdgcn_global_load_lds((const unsigned*)((TOUCH_M & 2) ? (const char*)out + (size_t)(pm * 256 + h * 128 + wid * 16 + (lane >> 2)) * (NN * 2) + pn * 512 + (lane & 3) * 128 : pF3 + vfl), (LAS unsigned*)(lds + L_JUNK), 4, 0, 0); \
        if (UNLIKELY(sd == 0 || sd == 1 || (sd == 3 && wid == 0) || (TOUCH_T && (unsigned)td < 4u))) wait_vm<10 + 2 * PFN>(); else wait_vm<9 + 2 * PFN>(); \
        WAIT_L(0); DIAG2_ACC; BAR; SCHED; MMA32_BAR(1); \
    } while (0)

    const unsigned fadr = (unsigned)(uintptr_t)lds + FSLOT(0) + lane * 4;
    static_assert(NF % 3 == 2, "the ring of second w halves must end in its home position");
    int rgR = SB(0, 1), rgN = SB(1, 1), rgW = L_X;
    for (int t = 0; t < NF - 2; t += 2) {
        K_TILE(0, t);
        K_TILE(1, t + 1);
    }

#define PF_PTR(T) (wCol + ((size_t)((r8 + (unsigned)e_of((T) + PF_DIST)) & 63u) << 21))
#define PFA_C(T) ((r8 + (unsigned)e_of((T) + PF_DIST)) & 63u)
#define PFA_PTR(T) (aRem - h * HSTEP_A + (((PFA_C(T) >> 3) << 21) | ((PFA_C(T) & 7u) << 8)))
#define TAIL_C(u) ((r8 + (unsigned)e_of(NF + (u) / 2)) & 63u)
#define TAIL_A(u) (aRemT + ((((TAIL_C(u) >> 3) << 21) | ((TAIL_C(u) & 7u) << 8)) + ((u) & 1) * 128))
#define TAIL_B(u) (wT + (((size_t)TAIL_C(u) << 21) + ((size_t)((u) & 1) << 20)))
    const char* tfp;
    { int k = lane >> 3; k = k < SH / 2 ? k : SH / 2 - 1; const int u = NF + k - 8, a = u / 7, j = u - a * 7;
      tfp = fRem + (((r8 + (unsigned)((j + 1) * 8 + a)) & 63u) << 9) + vfl; }
    {
        LDB(B0, SB(0, 0)); LDA(At, SA(0, 0)); LDB(B1, SB(0, 1));
        __builtin_amdgcn_global_load_lds((const unsigned*)tfp, (LAS unsigned*)(lds + FSLOT(0)), 4, 0, 17);
        STAGE(SA(1, 1), pA1 + dA, voffA);
        wait_vm<9 + PFN>(); WAIT_L(0); BAR; SCHED; MMA32_BAR(0);
        fv = *(const volatile LAS unsigned*)(lds + FSLOT(1) + lane * 4);
        LDA(At, SA(0, 1));
        STAGE(slotB0(0), TAIL_B(0), voffB); STAGE(slotB1(0), TAIL_B(0) + HSTEP_B, voffB);
        if (UNLIKELY(!__all(fv >= seq))) SPIN_FLAGS(pF2 + vfl);
        STAGE(slotA(0), TAIL_A(0), voffA);
        if constexpr (pf_is(NF - 2)) { PREFETCH(PF_PTR(NF - 2)); PREFETCHA(PFA_PTR(NF - 2)); }
        wait_vm<9 + PFN + pf_at(NF - 2)>(); WAIT_L(0); BAR; SCHED; MMA32_BAR(1);
        LDB(B0, SB(1, 0)); LDA(At, SA(1, 0)); LDB(B1, SB(1, 1));
        STAGE(slotB0(2), TAIL_B(2), voffB);
        wait_vm<8 + pf_at(NF - 2)>(); WAIT_L(0); BAR; SCHED; MMA32_BAR(0);
        fv = *(const volatile LAS unsigned*)(lds + FSLOT(0) + lane * 4);
        LDA(At, SA(1, 1));
        STAGE(slotB0(1), TAIL_B(1), voffB);
        if (UNLIKELY(!__all(fv >= seq))) SPIN_FLAGS(tfp);
        STAGE(slotA(1), TAIL_A(1), voffA); STAGE(slotB1(1), TAIL_B(1) + HSTEP_B, voffB);
        if constexpr (pf_is(NF - 1)) { PREFETCH(PF_PTR(NF - 1)); PREFETCHA(PFA_PTR(NF - 1)); }
        wait_vm<8 + pf_at(NF - 2) + pf_at(NF - 1)>(); WAIT_L(0); BAR; SCHED; MMA32_BAR(1);
    }
    STAMP(2);
    char* const slabOut = myws + W_SLAB + (size_t)bid * SLAB_B + (size_t)wid * 8192 + lane * 16;
    const char* const slabIn = myws + W_SLAB + (size_t)partner * SLAB_B + (size_t)wid * 8192 + lane * 16;
    const char* const pflag = myws + W_EFLAG + (size_t)(partner * 8 + wid) * 128;
    u32x4 in[8];
    bool early = false;
    static_for([&](auto uc) {
        constexpr int u = decltype(uc)::value;
        constexpr bool st = u + 2 < SH;
#pragma unroll
        for (int jj = 0; jj < n_store(u); ++jj) {
            const int j = u * XS_PER + jj, bj = j >> 2, m = j & 3;
            const f32x4 v0 = acc[1][bj][m][0], v1 = acc[1][bj][m][1];
            u32x4 w; w.x = cvt_pk_bf16(v0[0], v0[1]); w.y = cvt_pk_bf16(v0[2], v0[3]); w.z = cvt_pk_bf16(v1[0], v1[1]); w.w = cvt_pk_bf16(v1[2], v1[3]);
            asm volatile("global_store_dwordx4 %0, %1, off sc1\n\ts_nop 1" :: "v"(slabOut + j * 1024), "v"(w) : "memory");
        }
        if constexpr (u == S_FLAG) asm volatile("global_store_dword %0, %1, off sc1" :: "v"(myws + W_EFLAG + (size_t)(bid * 8 + wid) * 128), "v"(seq) : "memory");
        if constexpr (u == S_SAMP) __builtin_amdgcn_global_load_lds((const unsigned*)pflag, (LAS unsigned*)(lds + FSLOT(0)), 4, 0, 17);
        if constexpr (u == S_LOAD) asm volatile("ds_read_b32 %0, %1" : "=v"(fv) : "v"(fadr) : "memory");
        SCHED;
        if constexpr (st && u > 0) { GLDS1(slotB0(u + 2), TAIL_B(u + 2), voffB, 0); SCHED; }
        LDB(B0, slotB0(u)); SCHED;
        if constexpr (st && u > 0) { GLDS1(slotB0(u + 2), TAIL_B(u + 2), voffB, 1); SCHED; }
        LDAH(At, slotA(u), 0); SCHED;
        if constexpr (st) { GLDS1(slotA(u + 2), TAIL_A(u + 2), voffA, 0); SCHED; }
        LDAH(At, slotA(u), 2); SCHED;
        if constexpr (st) { GLDS1(slotA(u + 2), TAIL_A(u + 2), voffA, 1); SCHED; }
        LDB(B1, slotB1(u)); SCHED;
        if constexpr (st) { GLDS1(slotB1(u + 2), TAIL_B(u + 2) + HSTEP_B, voffB, 0); GLDS1(slotB1(u + 2), TAIL_B(u + 2) + HSTEP_B, voffB, 1); }
        asm volatile("" ::: "memory");
        if constexpr (u == S_LOAD) {
            asm volatile("s_waitcnt lgkmcnt(15)" : "+v"(fv) :: "memory");
            early = (unsigned)__builtin_amdgcn_readfirstlane(fv) >= seq;
            asm volatile("" ::: "memory");
#pragma unroll
            for (int i = 0; i < 8; ++i) asm volatile("global_load_dwordx4 %0, %1, off offset:%2 sc1" : "=v"(in[i]) : "v"(slabIn + (i >> 2) * 4096), "i"((i & 3) * 1024) : "memory");
        }
        if constexpr ((u & 1) && pf_is(NF + u / 2)) { PREFETCH(PF_PTR(NF + u / 2)); PREFETCHA(PFA_PTR(NF + u / 2)); }
        if constexpr (u + 1 < SH) wait_vm<tail_wait(u)>();
        WAIT_L(0); BAR; SCHED; MMA32_BAR(0);
    }, std::make_integer_sequence<int, SH>{});
    if (wr == 0) BAR;
    STAMP(3);

    {
        asm volatile("s_waitcnt vmcnt(0)" : "+v"(in[0]), "+v"(in[1]), "+v"(in[2]), "+v"(in[3]), "+v"(in[4]), "+v"(in[5]), "+v"(in[6]), "+v"(in[7]) :: "memory");
        if (UNLIKELY(!early)) {
            const GAS unsigned* fp = (const GAS unsigned*)pflag;
            const unsigned long long t0 = __builtin_amdgcn_s_memrealtime();
            while ((unsigned)__builtin_amdgcn_readfirstlane(__hip_atomic_load(fp, __ATOMIC_RELAXED, __HIP_MEMORY_SCOPE_AGENT)) < seq) {
                if (__builtin_amdgcn_readfirstlane(__hip_atomic_load(tmo, __ATOMIC_RELAXED, __HIP_MEMORY_SCOPE_AGENT)) == seq || __builtin_amdgcn_s_memrealtime() - t0 > DEADLINE_TICKS) {
                    __hip_atomic_store(tmo, seq, __ATOMIC_RELAXED, __HIP_MEMORY_SCOPE_AGENT); break; }
                __builtin_amdgcn_s_sleep(1);
            }
            asm volatile("" ::: "memory");
#pragma unroll
            for (int i = 0; i < 8; ++i) asm volatile("global_load_dwordx4 %0, %1, off offset:%2 sc1" : "=v"(in[i]) : "v"(slabIn + (i >> 2) * 4096), "i"((i & 3) * 1024) : "memory");
            asm volatile("s_waitcnt vmcnt(0)" : "+v"(in[0]), "+v"(in[1]), "+v"(in[2]), "+v"(in[3]), "+v"(in[4]), "+v"(in[5]), "+v"(in[6]), "+v"(in[7]) :: "memory");
            DIAG_SLOW;
        }
        const bool bad = __builtin_amdgcn_readfirstlane(__hip_atomic_load(tmo, __ATOMIC_RELAXED, __HIP_MEMORY_SCOPE_AGENT)) == seq;
        STAMP(4);
        unsigned short* orow = out + (size_t)(pm * 256 + h * 128 + wr * 64 + fr) * NN + pn * 256 + wc * 32 + (fq & 1) * 16 + (fq >> 1) * 8;
#pragma unroll
        for (int bj = 0; bj < 2; ++bj)
#pragma unroll
            for (int m = 0; m < 4; ++m) {
                f32x4 v0 = acc[0][bj][m][0], v1 = acc[0][bj][m][1];
                const u32x4 p = in[bj * 4 + m];
                v0[0] += __uint_as_float(p.x << 16); v0[1] += __uint_as_float(p.x & 0xffff0000u); v0[2] += __uint_as_float(p.y << 16); v0[3] += __uint_as_float(p.y & 0xffff0000u);
                v1[0] += __uint_as_float(p.z << 16); v1[1] += __uint_as_float(p.z & 0xffff0000u); v1[2] += __uint_as_float(p.w << 16); v1[3] += __uint_as_float(p.w & 0xffff0000u);
                unsigned a0 = cvt_pk_bf16(gelu_tanh(v0[0]), gelu_tanh(v0[1])), a1 = cvt_pk_bf16(gelu_tanh(v0[2]), gelu_tanh(v0[3]));
                unsigned c0 = cvt_pk_bf16(gelu_tanh(v1[0]), gelu_tanh(v1[1])), c1 = cvt_pk_bf16(gelu_tanh(v1[2]), gelu_tanh(v1[3]));
                { auto r = __builtin_amdgcn_permlane16_swap(a0, c0, false, false); a0 = r[0]; c0 = r[1]; }
                { auto r = __builtin_amdgcn_permlane16_swap(a1, c1, false, false); a1 = r[0]; c1 = r[1]; }
                u32x4 w = {a0, a1, c0, c1};
                if (bad) w = (u32x4){0x7fc07fc0u, 0x7fc07fc0u, 0x7fc07fc0u, 0x7fc07fc0u};
                *(u32x4*)(orow + (size_t)m * 16 * NN + bj * 128) = w;
            }
    }
}

extern "C" void kernel_launch(int rank, int nranks, ncclComm_t comm,
                              hipStream_t stream,
                              void* const* d_in, const int* in_sizes, int n_in,
                              void* d_out, int out_size,
                              const int* in_dtypes, int out_dtype,
                              void* const* d_out_all,
                              void* const* d_ws_all, size_t ws_bytes) {
    (void)comm; (void)in_sizes; (void)n_in; (void)out_size; (void)in_dtypes; (void)out_dtype; (void)d_out_all;
    if (nranks != NR || (unsigned)rank >= (unsigned)NR || ws_bytes < WS_NEED) { fprintf(stderr, "a2a_gemm: unsupported configuration\n"); return; }
    static bool attr_set = false;
    if (!attr_set) {
        HIP_CHECK(hipFuncSetAttribute((const void*)a2a_gemm_gelu_kernel, hipFuncAttributeMaxDynamicSharedMemorySize, LDS_BYTES));
        attr_set = true;
    }
    hipLaunchKernelGGL(a2a_gemm_gelu_kernel, dim3(NWG), dim3(512), LDS_BYTES, stream, rank,
                       (const unsigned short*)d_in[0], (const unsigned short*)d_in[1], (unsigned short*)d_out, d_ws_all);
    HIP_CHECK(hipGetLastError());
}
```

```cpp
#include <hip/hip_runtime.h>
#include <rccl/rccl.h>
#include <cstdio>
#include <cstdint>
#include <utility>

#define HIP_CHECK(expr) do { hipError_t _e = (expr); if (_e != hipSuccess) { \
    fprintf(stderr, "HIP error %s at %s:%d\n", hipGetErrorString(_e), __FILE__, __LINE__); \
    return; } } while (0)

#define LAS __attribute__((address_space(3)))
#define GAS __attribute__((address_space(1)))
typedef short bf16x8 __attribute__((ext_vector_type(8)));
typedef short s16x4 __attribute__((ext_vector_type(4)));
typedef float f32x4 __attribute__((ext_vector_type(4)));
typedef unsigned u32x4 __attribute__((ext_vector_type(4)));

#define MAP_OPT 2
#define PACE_NUM 9
#define PACE_DEN 448
#define SEND_SHIFT 1
#define SRC_PERM 0
#define EARLY_BAR 0
#define PF_DIST 0
#define TOUCH_T 0
#define TOUCH_M 3
#define SHARED_KB 16
#define XS_PERSEG 8
#define XS_PS 6

namespace {
constexpr int NR = 8;
constexpr int MS = 1024;
constexpr int KB = 1024;
constexpr int NN = 8192;
constexpr int SH = SHARED_KB;
constexpr int NF = 64 - SH / 2;
constexpr int NWG = 256;
constexpr int HTB = 16384;
static_assert(SH % 4 == 0 && SH >= 4 && SH <= 16, "tail size");
static_assert((1791 * PACE_NUM) / PACE_DEN + 3 < NF - 2, "all sends must finish inside the generic K-loop");
static_assert(TOUCH_T == 0 || ((1791 * PACE_NUM) / PACE_DEN + 3 < TOUCH_T && TOUCH_T + 4 < NF - 2), "touch window");
constexpr int L_X = 8 * HTB;
constexpr int L_SEND = 9 * HTB;
constexpr int L_FLAG = L_SEND + 8192;
constexpr int L_JUNK = L_FLAG + 4096;
constexpr int LDS_BYTES = L_JUNK + 256;
#define CTRL_MIB 128
#define SLAB_MIB 112
#define RECV0_MIB 80
#define RECV1_MIB 96
constexpr size_t W_CTRL = (size_t)CTRL_MIB << 20;
constexpr size_t W_TMO = W_CTRL + 256;
constexpr size_t W_XFLAG = W_CTRL + 4096;
constexpr size_t W_EFLAG = W_CTRL + 65536;
constexpr size_t W_SEQ = W_EFLAG + 262144;
constexpr size_t W_DIAG = W_CTRL + 524288;
constexpr size_t RECV_BLK = (size_t)MS * KB * 2;
constexpr size_t W_RECV0 = (size_t)RECV0_MIB << 20;
constexpr size_t W_RECV1 = (size_t)RECV1_MIB << 20;
constexpr size_t W_SLAB = (size_t)SLAB_MIB << 20;
constexpr size_t SLAB_B = 65536;
constexpr size_t WS_NEED = (size_t)129 << 20;
constexpr unsigned long long DEADLINE_TICKS = 40000000ull;

constexpr int XS_PER = XS_PERSEG;
constexpr int S_FLAG = 8 / XS_PER + 1;
constexpr int S_SAMP = S_FLAG + XS_PS;
constexpr int S_LOAD = S_SAMP + 2;
static_assert(S_LOAD < SH, "tail too short for the hand-over schedule");
constexpr int n_stage(int u) { return u + 2 < SH ? (u == 0 ? 4 : 6) : 0; }
constexpr int n_store(int u) { return u * XS_PER < 8 ? XS_PER : 0; }
constexpr int pf_at(int T) { return (PF_DIST && T + PF_DIST < 64) ? 1 : 0; }
constexpr int pf_is(int T) { return pf_at(T); }
constexpr int n_post(int u) { return u < 0 ? pf_at(NF - 1) : (u == S_LOAD ? 8 : 0) + ((u & 1) ? pf_at(NF + u / 2) : 0); }
constexpr int n_ops(int u) { return n_store(u) + (u == S_FLAG) + (u == S_SAMP) + n_stage(u) + n_post(u); }
constexpr int tail_wait(int u) { return n_ops(u) + n_post(u - 1); }
constexpr int slotB0(int u) { return u % 3 == 0 ? 4 * HTB : (u % 3 == 1 ? 6 * HTB : L_X); }
constexpr int slotA(int u) { return u % 3 == 0 ? 0 : (u % 3 == 1 ? 2 * HTB : HTB); }
constexpr int slotB1(int u) { return u % 3 == 0 ? 5 * HTB : (u % 3 == 1 ? 7 * HTB : 3 * HTB); }
constexpr int e_of(int T) { return T < 8 ? T : ((T - 8) % 7 + 1) * 8 + (T - 8) / 7; }

__device__ __forceinline__ int lds_byteA(int r, int c) { const int st = (r >> 4) * 2 + (c >> 5), rr = r & 15, cc = c & 31, ob = rr * 64 + cc * 2; return st * 1024 + (ob ^ (((ob >> 9) & 1) << 5)); }
__device__ __forceinline__ void stage_rcA(int b, int& R, int& C) { const int st = b / 1024, sb = b % 1024, swz = sb ^ (((sb >> 9) & 1) << 5); R = (st >> 1) * 16 + swz / 64; C = (st & 1) * 32 + (swz % 64) / 2; }
__device__ __forceinline__ unsigned cvt_pk_bf16(float lo, float hi) { unsigned r; asm volatile("v_cvt_pk_bf16_f32 %0, %1, %2" : "=v"(r) : "v"(lo), "v"(hi)); return r; }
__device__ __forceinline__ float gelu_tanh(float y) {
    const float u = y * y * 0.044715f + 1.0f;
    const float e = __builtin_amdgcn_exp2f(y * u * (-2.0f * 0.7978845608028654f * 1.4426950408889634f));
    return y * __builtin_amdgcn_rcpf(1.0f + e);
}
template <int N> __device__ __forceinline__ void wait_vm() { if constexpr (N < 63) asm volatile("s_waitcnt vmcnt(%0)" :: "n"(N) : "memory"); }
template <class F, int... I> __device__ __forceinline__ void static_for(F&& f, std::integer_sequence<int, I...>) { (f(std::integral_constant<int, I>{}), ...); }
}

__global__ __launch_bounds__(512, 2)
void a2a_gemm_gelu_kernel(int rank, const unsigned short* __restrict__ xloc, const unsigned short* __restrict__ wmat,
                          unsigned short* __restrict__ out, void* const* __restrict__ ws_all) {
    extern __shared__ __attribute__((aligned(16))) unsigned char shm[];
    LAS unsigned char* lds = (LAS unsigned char*)shm;
    const int tid = threadIdx.x, wid = __builtin_amdgcn_readfirstlane(tid >> 6), lane = tid & 63, wr = wid >> 2, wc = wid & 3, fr = lane & 15, fq = lane >> 4;
    const int bid = blockIdx.x, xcd = bid & 7, idx = bid >> 3;
    const int h = xcd & 1, pn = (xcd >> 1) * 8 + (idx & 7), pm = idx >> 3;
    const int partner = bid ^ 1;
    char* const myws = (char*)ws_all[rank];
#define STAMP(i)
    STAMP(0);
    unsigned seqv;
    asm volatile("global_load_dword %0, %1, off sc1" : "=v"(seqv) : "v"(myws + W_SEQ + (size_t)bid * 128) : "memory");

    unsigned voffA[2], voffB[2];
#pragma unroll
    for (int i = 0; i < 2; ++i) {
        int R, C; stage_rcA(tid * 16 + i * 8192, R, C);
        voffA[i] = (unsigned)(R * KB + C) * 2u;
        const int b = tid * 16 + i * 8192, k = b >> 8, cp = (b >> 4) & 15, f = (k & 3) | (((k >> 3) & 1) << 2), c = cp ^ (f << 1);
        voffB[i] = (unsigned)k * (NN * 2) + (unsigned)c * 16u;
    }
    const unsigned ldsw = (unsigned)wid * 1024u;
    const int aoff = lds_byteA(wr * 64 + fr, fq * 8);
    int boff[2];
    { const int g = lane >> 4, q = (lane & 15) >> 2, p = lane & 3, f = q | ((g & 1) << 2);
#pragma unroll
      for (int nf = 0; nf < 2; ++nf) boff[nf] = (int)(unsigned)(uintptr_t)lds + 4 * HTB + (8 * g + q) * 256 + (((wc * 4 + nf * 2 + (p >> 1)) ^ (f << 1)) * 16) + (p & 1) * 8; }

#define SA(b, hh) (((b) * 2 + (hh)) * HTB)
#define SB(b, hh) ((4 + (b) * 2 + (hh)) * HTB)
#define EXP_NODS 0
#define EXP_NGLDS 2
#define EXP_SVC 0
    const bool svc = EXP_SVC && wid == 7;
#define STAGE(bufoff, gbase, voff) do { if (!svc) _Pragma("unroll") for (int _i = 0; _i < EXP_NGLDS; ++_i) \
        __builtin_amdgcn_global_load_lds((const unsigned*)((const char*)(gbase) + (voff)[_i]), (LAS unsigned*)(lds + (bufoff) + ldsw + _i * 8192), 16, 0, 0); } while (0)
#define GLDS1(bufoff, gbase, voff, _i) do { if (!svc) __builtin_amdgcn_global_load_lds((const unsigned*)((const char*)(gbase) + (voff)[_i]), (LAS unsigned*)(lds + (bufoff) + ldsw + (_i) * 8192), 16, 0, 0); } while (0)
#define LDAH(dst, slot, m0) do { _Pragma("unroll") for (int m = (m0); m < (m0) + 2; ++m) _Pragma("unroll") for (int k = 0; k < 2; ++k) dst[m][k] = *(const LAS bf16x8*)(lds + (slot) + aoff + m * 2048 + k * 1024); } while (0)
#define LDA(dst, slot) do { _Pragma("unroll") for (int m = 0; m < 4; ++m) _Pragma("unroll") for (int k = 0; k < 2; ++k) dst[m][k] = *(const LAS bf16x8*)(lds + (slot) + aoff + m * 2048 + k * 1024); } while (0)
#define LDB(dst, slot) LDB2(dst, ((slot) / (4 * HTB)) * (4 * HTB) - 4 * HTB, (slot) % (4 * HTB))
#define LDB2(dst, _hi, _lo) do { s16x4 _r0, _r1, _r2, _r3, _r4, _r5, _r6, _r7; \
        asm volatile("ds_read_b64_tr_b16 %0, %8 offset:%10\n\tds_read_b64_tr_b16 %1, %8 offset:%11\n\tds_read_b64_tr_b16 %2, %8 offset:%12\n\tds_read_b64_tr_b16 %3, %8 offset:%13\n\t" \
                     "ds_read_b64_tr_b16 %4, %9 offset:%10\n\tds_read_b64_tr_b16 %5, %9 offset:%11\n\tds_read_b64_tr_b16 %6, %9 offset:%12\n\tds_read_b64_tr_b16 %7, %9 offset:%13" \
                     : "=&v"(_r0), "=&v"(_r1), "=&v"(_r2), "=&v"(_r3), "=&v"(_r4), "=&v"(_r5), "=&v"(_r6), "=&v"(_r7) \
                     : "v"(boff[0] + (_hi)), "v"(boff[1] + (_hi)), "i"(_lo), "i"((_lo) + 1024), "i"((_lo) + 8192), "i"((_lo) + 9216) : "memory"); \
        dst[0][0] = __builtin_shufflevector(_r0, _r1, 0, 1, 2, 3, 4, 5, 6, 7); dst[0][1] = __builtin_shufflevector(_r2, _r3, 0, 1, 2, 3, 4, 5, 6, 7); \
        dst[1][0] = __builtin_shufflevector(_r4, _r5, 0, 1, 2, 3, 4, 5, 6, 7); dst[1][1] = __builtin_shufflevector(_r6, _r7, 0, 1, 2, 3, 4, 5, 6, 7); } while (0)
#define MMA16(ai, bj, At, Bt) do { _Pragma("unroll") for (int i = 0; i < 16; ++i) { const int k = i >> 3, m = (i >> 1) & 3, n = i & 1; \
        acc[ai][bj][m][n] = __builtin_amdgcn_mfma_f32_16x16x32_bf16(Bt[n][k], At[m][k], acc[ai][bj][m][n], 0, 0, 0); } } while (0)
#define MMA_BAR(ai, bj, At, Bt) do { __builtin_amdgcn_s_setprio(1); MMA16(ai, bj, At, Bt); __builtin_amdgcn_s_setprio(0); BAR; SCHED; } while (0)
#define MMA32_BAR(ai) do { __builtin_amdgcn_s_setprio(1); MMA16(ai, 0, At, B0); MMA16(ai, 1, At, B1); __builtin_amdgcn_s_setprio(0); BAR; SCHED; } while (0)
#define WAIT_V(n) asm volatile("s_waitcnt vmcnt(" #n ")" ::: "memory")
#define WAIT_L(n) asm volatile("s_waitcnt lgkmcnt(" #n ")" ::: "memory")
#define BAR __builtin_amdgcn_s_barrier()
#define SCHED __builtin_amdgcn_sched_barrier(0)

    f32x4 acc[2][2][4][2];
#pragma unroll
    for (int a = 0; a < 2; ++a)
#pragma unroll
        for (int b = 0; b < 2; ++b)
#pragma unroll
            for (int m = 0; m < 4; ++m)
#pragma unroll
                for (int n = 0; n < 2; ++n) acc[a][b][m][n] = (f32x4){0.f, 0.f, 0.f, 0.f};
    bf16x8 At[4][2], B0[2][2], B1[2][2];

    constexpr long long HSTEP_A = (long long)128 * KB * 2;
    constexpr size_t HSTEP_B = 256;
    const long long dA = h ? -HSTEP_A : HSTEP_A;
    const size_t rowOff = ((size_t)pm * 256 + (size_t)h * 128) * KB * 2;
    const char* const aLoc = (const char*)xloc + (size_t)rank * RECV_BLK + rowOff + (size_t)h * 128;
    const char* const wT = (const char*)wmat + (size_t)pn * 512;
    const char* const wCol = wT + ((size_t)h << 20);
    const char* const bLoc = wCol + ((size_t)rank << 24);

    const unsigned vpf = (unsigned)(pm * 16 + wid * 2 + ((lane >> 2) & 1)) * (NN * 2) + (unsigned)(lane & 3) * 128u;
    unsigned eP = (unsigned)e_of(PF_DIST);
#define PREFETCH(ptr) __builtin_amdgcn_global_load_lds((const unsigned*)((ptr) + vpf), (LAS unsigned*)(lds + L_JUNK), 4, 0, 0)
#define PREFETCHA(ptr)
    constexpr int PFN = PF_DIST ? 1 : 0, PFI = PFN;
    static_assert(PF_DIST <= 8, "the prologue prefetch covers local K-blocks only");
    STAGE(SB(0, 0), bLoc, voffB); STAGE(SA(0, 0), aLoc, voffA); STAGE(SB(0, 1), bLoc + HSTEP_B, voffB); STAGE(SA(0, 1), aLoc + dA, voffA);
    const unsigned vsend = (unsigned)(wid * 4 + (lane >> 4)) * (KB * 2) + (unsigned)(lane & 15) * 16u;
#define SEND_RG(p, dj) (SRC_PERM ? (((p) + 8 * (7 - (dj))) & 31) : (p))
    int sslot = (bid * PACE_NUM) / PACE_DEN - SEND_SHIFT;
    if (sslot < 0) {
        const int dj = bid % 7, rg = SEND_RG(bid / 7, dj), d = (rank + 1 + dj) & 7;
        __builtin_amdgcn_global_load_lds((const unsigned*)((const char*)xloc + (size_t)d * RECV_BLK + (size_t)rg * 32 * (KB * 2) + vsend), (LAS unsigned*)(lds + L_SEND + ldsw), 16, 0, 0);
    }
    STAGE(SB(1, 0), bLoc + ((size_t)1 << 21), voffB); STAGE(SA(1, 0), aLoc + 256, voffA); STAGE(SB(1, 1), bLoc + ((size_t)1 << 21) + HSTEP_B, voffB);
    if (EXP_SVC) { for (int tk = 2; tk < PF_DIST; ++tk) { PREFETCH(bLoc + ((size_t)tk << 21)); PREFETCHA(aLoc - h * HSTEP_A + tk * 256); }
        if (svc) asm volatile("s_waitcnt vmcnt(%0)" :: "n"(EXP_SVC >= 2 ? (PF_DIST - 2) * (EXP_SVC - 1) : 0) : "memory");
        asm volatile("s_waitcnt vmcnt(6)" : "+v"(seqv) :: "memory");
    } else
    if (PFN) {
        const unsigned tk = 2u + (unsigned)(lane >> 3);
        PREFETCH(bLoc + ((size_t)(tk < (unsigned)PF_DIST ? tk : 2u) << 21));
        asm volatile("s_waitcnt vmcnt(7)" : "+v"(seqv) :: "memory");
    } else asm volatile("s_waitcnt vmcnt(6)" : "+v"(seqv) :: "memory");
    BAR; asm volatile("" ::: "memory");
    const unsigned seq = (unsigned)__builtin_amdgcn_readfirstlane(seqv) + 1u;
    if (tid == 0) asm volatile("global_store_dword %0, %1, off sc1" :: "v"(myws + W_SEQ + (size_t)bid * 128), "v"(seq) : "memory");
    const size_t parOff = (seq & 1u) ? W_RECV1 : W_RECV0;
    const char* const aRemT = myws + parOff + rowOff;
    const char* const aRem = aRemT + (size_t)h * 128;
    const char* const aLocB = aLoc - ((size_t)rank << 21);
    const char* const fRem = myws + W_XFLAG + (size_t)pm * 128;
    const unsigned vfl = (unsigned)(lane & 7) * 4u;
    const unsigned r8 = (unsigned)rank * 8u;
    GAS unsigned* const tmo = (GAS unsigned*)(myws + W_TMO);

    const char* pA1 = aLoc + 256; const char* pA2 = aLoc + 512; const char* pB2 = bLoc + ((size_t)2 << 21);
    const char* pA3 = aLoc + 768; const char* pB3 = bLoc + ((size_t)3 << 21);
    const char* pF2 = fRem; const char* pF3 = fRem;
    unsigned e3 = 3;
    unsigned sord = 0;
    if (SRC_PERM) { int k = 1; for (int i = 0; i < 8; ++i) { const int dd = ((pm - (i >> 1)) & 3) + 4 * (i & 1); if (dd) { sord |= (unsigned)dd << (4 * k); ++k; } } }
    int sidx = 0;
    const char* sSrc; char* sDst; char* sFlag;
#define SEND_SETUP() do { const int q = sidx * 256 + bid, a = q / 224, r = q - a * 224, pp = r / 7, dj = r - pp * 7, rg = SEND_RG(pp, dj), d = (rank + 1 + dj) & 7; \
        char* const pw = (char*)ws_all[d]; const size_t o = (size_t)rg * 32 * (KB * 2) + (size_t)a * 256; \
        sSrc = (const char*)xloc + (size_t)d * RECV_BLK + o; sDst = pw + parOff + (size_t)rank * RECV_BLK + o; \
        sFlag = pw + W_XFLAG + (size_t)((rank * 8 + a) * 4 + (rg >> 3)) * 128 + (size_t)(rg & 7) * 4; } while (0)
    SEND_SETUP();
    u32x4 pv = {0u, 0u, 0u, 0u};
    unsigned fv = 0xffffffffu;
    if (wr == 1) BAR;
    STAMP(1);

#define DIAG_SLOW
#define UNLIKELY(x) __builtin_expect(!!(x), 0)
#define SPIN_FLAGS(ptr) do { const GAS unsigned* fp = (const GAS unsigned*)(ptr); const unsigned long long t0 = __builtin_amdgcn_s_memrealtime(); \
            while (!__all(__hip_atomic_load(fp, __ATOMIC_RELAXED, __HIP_MEMORY_SCOPE_SYSTEM) >= seq)) { \
                if (__builtin_amdgcn_readfirstlane(__hip_atomic_load(tmo, __ATOMIC_RELAXED, __HIP_MEMORY_SCOPE_AGENT)) == seq || __builtin_amdgcn_s_memrealtime() - t0 > DEADLINE_TICKS) { \
                    __hip_atomic_store(tmo, seq, __ATOMIC_RELAXED, __HIP_MEMORY_SCOPE_AGENT); break; } \
                __builtin_amdgcn_s_sleep(2); } \
            asm volatile("" ::: "memory"); DIAG_SLOW; } while (0)
#define FSLOT(b) (L_FLAG + (b) * 2048 + wid * 256)
#define TS(x)
#define DIAG2_ACC
#define K_TILE(b, t) do { \
        const int sd = (t) - sslot, td = (t) - TOUCH_T; \
        TS(ts0); \
          \
        __builtin_amdgcn_global_load_lds((const unsigned*)(pF3 + vfl), (LAS unsigned*)(lds + FSLOT(b)), 4, 0, 17); \
        if (UNLIKELY(sd == 0)) __builtin_amdgcn_global_load_lds((const unsigned*)(sSrc + vsend), (LAS unsigned*)(lds + L_SEND + ldsw), 16, 0, 0); \
        if (UNLIKELY(sd == 3 && wid == 0)) asm volatile("global_store_dword %0, %1, off sc0 sc1" :: "v"(sFlag), "v"(seq) : "memory"); \
        LDB(B0, SB(b, 0)); SCHED; GLDS1(SA((b) ^ 1, 1), pA1 + dA, voffA, 0); SCHED; \
        LDAH(At, SA(b, 0), 0); SCHED; GLDS1(SA((b) ^ 1, 1), pA1 + dA, voffA, 1); SCHED; \
        LDAH(At, SA(b, 0), 2); SCHED; GLDS1(rgW, pB2 + HSTEP_B, voffB, 0); SCHED; \
        LDB2(B1, rgR - 4 * HTB, 0); SCHED; GLDS1(rgW, pB2 + HSTEP_B, voffB, 1); \
        TS(ts1); \
        if (UNLIKELY(sd == 0 || sd == 2 || (sd == 3 && wid == 0) || (TOUCH_T && (td == 1 || td == 3)))) wait_vm<12 + PFN>(); else wait_vm<11 + PFN>(); \
        WAIT_L(0); TS(ts2); BAR; SCHED; TS(ts3); MMA32_BAR(0); \
        TS(ts4); \
          \
        asm volatile("ds_read_b32 %0, %1 offset:%2" : "=v"(fv) : "v"(fadr), "i"(((b) ^ 1) * 2048) : "memory"); \
        if (UNLIKELY(sd == 1)) pv = *(const volatile LAS u32x4*)(lds + L_SEND + ldsw + lane * 16); \
        GLDS1(SB(b, 0), pB2, voffB, 0); SCHED; \
        LDAH(At, SA(b, 1), 0); SCHED; GLDS1(SB(b, 0), pB2, voffB, 1); SCHED; \
        LDAH(At, SA(b, 1), 2); SCHED; \
        asm volatile("s_waitcnt lgkmcnt(8)" : "+v"(fv) :: "memory"); \
        if (UNLIKELY((t) >= 6 && !__all(fv >= seq))) SPIN_FLAGS(pF2 + vfl); \
        GLDS1(SA(b, 0), pA2, voffA, 0); \
        if (UNLIKELY(sd == 3)) { ++sidx; if (sidx < 7) { sslot = ((sidx * 256 + bid) * PACE_NUM) / PACE_DEN - SEND_SHIFT; SEND_SETUP(); } else sslot = 1000; } \
          \
        const char* const pA2o = pA2; \
        pA1 = pA2; pA2 = pA3; pB2 = pB3; pF2 = pF3; \
        e3 += (e3 < 8u) ? 1u : 8u; e3 -= (e3 >= 64u) ? 55u : 0u; \
        { const unsigned c = (r8 + ((SRC_PERM && !(e3 & 7u)) ? ((sord >> (e3 >> 1)) & 7u) << 3 : e3)) & 63u; \
          pA3 = (e3 < 8u ? aLocB : aRem) + (((c >> 3) << 21) | ((c & 7u) << 8)); \
          pB3 = wCol + ((size_t)c << 21); pF3 = fRem + (c << 9); } \
        { const int r = rgR; rgR = rgN; rgN = rgW; rgW = r; } \
        asm volatile("" : "+s"(pA3), "+s"(pB3), "+s"(pF3), "+s"(e3), "+s"(rgR), "+s"(rgN), "+s"(rgW)); \
        SCHED; GLDS1(SA(b, 0), pA2o, voffA, 1); \
        if (UNLIKELY(sd == 1)) asm volatile("global_store_dwordx4 %0, %1, off sc0 sc1\n\ts_nop 1" :: "v"(sDst + vsend), "v"(pv) : "memory"); \
        if (PFI) { const unsigned c = (r8 + eP) & 63u; PREFETCH(wCol + ((size_t)c << 21)); \
            PREFETCHA((eP < 8u ? aLocB : aRem) - h * HSTEP_A + (((c >> 3) << 21) | ((c & 7u) << 8))); \
            eP += (eP < 8u) ? 1u : 8u; eP -= (eP >= 64u) ? 55u : 0u; } \
        TS(ts5); \
        if (TOUCH_T && UNLIKELY(td == 0)) __builtin_amdgcn_global_load_lds((const unsigned*)((TOUCH_M & 1) ? myws + W_SLAB + (size_t)bid * SLAB_B + (size_t)wid * 8192 + lane * 128 : pF3 + vfl), (LAS unsigned*)(lds + L_JUNK), 4, 0, 0); \
        if (TOUCH_T && UNLIKELY(td == 2)) __builtin_amdgcn_global_load_lds((const unsigned*)((TOUCH_M & 2) ? (const char*)out + (size_t)(pm * 256 + h * 128 + wid * 16 + (lane >> 2)) * (NN * 2) + pn * 512 + (lane & 3) * 128 : pF3 + vfl), (LAS unsigned*)(lds + L_JUNK), 4, 0, 0); \
        if (UNLIKELY(sd == 0 || sd == 1 || (sd == 3 && wid == 0) || (TOUCH_T && (unsigned)td < 4u))) wait_vm<10 + 2 * PFN>(); else wait_vm<9 + 2 * PFN>(); \
        WAIT_L(0); DIAG2_ACC; BAR; SCHED; MMA32_BAR(1); \
    } while (0)

    const unsigned fadr = (unsigned)(uintptr_t)lds + FSLOT(0) + lane * 4;
    static_assert(NF % 3 == 2, "the ring of second w halves must end in its home position");
    int rgR = SB(0, 1), rgN = SB(1, 1), rgW = L_X;
    for (int t = 0; t < NF - 2; t += 2) {
        K_TILE(0, t);
        K_TILE(1, t + 1);
    }

#define PF_PTR(T) (wCol + ((size_t)((r8 + (unsigned)e_of((T) + PF_DIST)) & 63u) << 21))
#define PFA_C(T) ((r8 + (unsigned)e_of((T) + PF_DIST)) & 63u)
#define PFA_PTR(T) (aRem - h * HSTEP_A + (((PFA_C(T) >> 3) << 21) | ((PFA_C(T) & 7u) << 8)))
#define TAIL_C(u) ((r8 + (unsigned)e_of(NF + (u) / 2)) & 63u)
#define TAIL_A(u) (aRemT + ((((TAIL_C(u) >> 3) << 21) | ((TAIL_C(u) & 7u) << 8)) + ((u) & 1) * 128))
#define TAIL_B(u) (wT + (((size_t)TAIL_C(u) << 21) + ((size_t)((u) & 1) << 20)))
    const char* tfp;
    { int k = lane >> 3; k = k < SH / 2 ? k : SH / 2 - 1; const int u = NF + k - 8, a = u / 7, j = u - a * 7;
      tfp = fRem + (((r8 + (unsigned)((j + 1) * 8 + a)) & 63u) << 9) + vfl; }
    {
        LDB(B0, SB(0, 0)); LDA(At, SA(0, 0)); LDB(B1, SB(0, 1));
        __builtin_amdgcn_global_load_lds((const unsigned*)tfp, (LAS unsigned*)(lds + FSLOT(0)), 4, 0, 17);
        STAGE(SA(1, 1), pA1 + dA, voffA);
        wait_vm<9 + PFN>(); WAIT_L(0); BAR; SCHED; MMA32_BAR(0);
        fv = *(const volatile LAS unsigned*)(lds + FSLOT(1) + lane * 4);
        LDA(At, SA(0, 1));
        STAGE(slotB0(0), TAIL_B(0), voffB); STAGE(slotB1(0), TAIL_B(0) + HSTEP_B, voffB);
        if (UNLIKELY(!__all(fv >= seq))) SPIN_FLAGS(pF2 + vfl);
        STAGE(slotA(0), TAIL_A(0), voffA);
        if constexpr (pf_is(NF - 2)) { PREFETCH(PF_PTR(NF - 2)); PREFETCHA(PFA_PTR(NF - 2)); }
        wait_vm<9 + PFN + pf_at(NF - 2)>(); WAIT_L(0); BAR; SCHED; MMA32_BAR(1);
        LDB(B0, SB(1, 0)); LDA(At, SA(1, 0)); LDB(B1, SB(1, 1));
        STAGE(slotB0(2), TAIL_B(2), voffB);
        wait_vm<8 + pf_at(NF - 2)>(); WAIT_L(0); BAR; SCHED; MMA32_BAR(0);
        fv = *(const volatile LAS unsigned*)(lds + FSLOT(0) + lane * 4);
        LDA(At, SA(1, 1));
        STAGE(slotB0(1), TAIL_B(1), voffB);
        if (UNLIKELY(!__all(fv >= seq))) SPIN_FLAGS(tfp);
        STAGE(slotA(1), TAIL_A(1), voffA); STAGE(slotB1(1), TAIL_B(1) + HSTEP_B, voffB);
        if constexpr (pf_is(NF - 1)) { PREFETCH(PF_PTR(NF - 1)); PREFETCHA(PFA_PTR(NF - 1)); }
        wait_vm<8 + pf_at(NF - 2) + pf_at(NF - 1)>(); WAIT_L(0); BAR; SCHED; MMA32_BAR(1);
    }
    STAMP(2);
    char* const slabOut = myws + W_SLAB + (size_t)bid * SLAB_B + (size_t)wid * 8192 + lane * 16;
    const char* const slabIn = myws + W_SLAB + (size_t)partner * SLAB_B + (size_t)wid * 8192 + lane * 16;
    const char* const pflag = myws + W_EFLAG + (size_t)(partner * 8 + wid) * 128;
    u32x4 in[8];
    bool early = false;
    static_for([&](auto uc) {
        constexpr int u = decltype(uc)::value;
        constexpr bool st = u + 2 < SH;
#pragma unroll
        for (int jj = 0; jj < n_store(u); ++jj) {
            const int j = u * XS_PER + jj, bj = j >> 2, m = j & 3;
            const f32x4 v0 = acc[1][bj][m][0], v1 = acc[1][bj][m][1];
            u32x4 w; w.x = cvt_pk_bf16(v0[0], v0[1]); w.y = cvt_pk_bf16(v0[2], v0[3]); w.z = cvt_pk_bf16(v1[0], v1[1]); w.w = cvt_pk_bf16(v1[2], v1[3]);
            asm volatile("global_store_dwordx4 %0, %1, off sc1\n\ts_nop 1" :: "v"(slabOut + j * 1024), "v"(w) : "memory");
        }
        if constexpr (u == S_FLAG) asm volatile("global_store_dword %0, %1, off sc1" :: "v"(myws + W_EFLAG + (size_t)(bid * 8 + wid) * 128), "v"(seq) : "memory");
        if constexpr (u == S_SAMP) __builtin_amdgcn_global_load_lds((const unsigned*)pflag, (LAS unsigned*)(lds + FSLOT(0)), 4, 0, 17);
        if constexpr (u == S_LOAD) asm volatile("ds_read_b32 %0, %1" : "=v"(fv) : "v"(fadr) : "memory");
        SCHED;
        if constexpr (st && u > 0) { GLDS1(slotB0(u + 2), TAIL_B(u + 2), voffB, 0); SCHED; }
        LDB(B0, slotB0(u)); SCHED;
        if constexpr (st && u > 0) { GLDS1(slotB0(u + 2), TAIL_B(u + 2), voffB, 1); SCHED; }
        LDAH(At, slotA(u), 0); SCHED;
        if constexpr (st) { GLDS1(slotA(u + 2), TAIL_A(u + 2), voffA, 0); SCHED; }
        LDAH(At, slotA(u), 2); SCHED;
        if constexpr (st) { GLDS1(slotA(u + 2), TAIL_A(u + 2), voffA, 1); SCHED; }
        LDB(B1, slotB1(u)); SCHED;
        if constexpr (st) { GLDS1(slotB1(u + 2), TAIL_B(u + 2) + HSTEP_B, voffB, 0); GLDS1(slotB1(u + 2), TAIL_B(u + 2) + HSTEP_B, voffB, 1); }
        asm volatile("" ::: "memory");
        if constexpr (u == S_LOAD) {
            asm volatile("s_waitcnt lgkmcnt(15)" : "+v"(fv) :: "memory");
            early = (unsigned)__builtin_amdgcn_readfirstlane(fv) >= seq;
            asm volatile("" ::: "memory");
#pragma unroll
            for (int i = 0; i < 8; ++i) asm volatile("global_load_dwordx4 %0, %1, off offset:%2 sc1" : "=v"(in[i]) : "v"(slabIn + (i >> 2) * 4096), "i"((i & 3) * 1024) : "memory");
        }
        if constexpr ((u & 1) && pf_is(NF + u / 2)) { PREFETCH(PF_PTR(NF + u / 2)); PREFETCHA(PFA_PTR(NF + u / 2)); }
        if constexpr (u + 1 < SH) wait_vm<tail_wait(u)>();
        WAIT_L(0); BAR; SCHED; MMA32_BAR(0);
    }, std::make_integer_sequence<int, SH>{});
    if (wr == 0) BAR;
    STAMP(3);

    {
        asm volatile("s_waitcnt vmcnt(0)" : "+v"(in[0]), "+v"(in[1]), "+v"(in[2]), "+v"(in[3]), "+v"(in[4]), "+v"(in[5]), "+v"(in[6]), "+v"(in[7]) :: "memory");
        if (UNLIKELY(!early)) {
            const GAS unsigned* fp = (const GAS unsigned*)pflag;
            const unsigned long long t0 = __builtin_amdgcn_s_memrealtime();
            while ((unsigned)__builtin_amdgcn_readfirstlane(__hip_atomic_load(fp, __ATOMIC_RELAXED, __HIP_MEMORY_SCOPE_AGENT)) < seq) {
                if (__builtin_amdgcn_readfirstlane(__hip_atomic_load(tmo, __ATOMIC_RELAXED, __HIP_MEMORY_SCOPE_AGENT)) == seq || __builtin_amdgcn_s_memrealtime() - t0 > DEADLINE_TICKS) {
                    __hip_atomic_store(tmo, seq, __ATOMIC_RELAXED, __HIP_MEMORY_SCOPE_AGENT); break; }
                __builtin_amdgcn_s_sleep(1);
            }
            asm volatile("" ::: "memory");
#pragma unroll
            for (int i = 0; i < 8; ++i) asm volatile("global_load_dwordx4 %0, %1, off offset:%2 sc1" : "=v"(in[i]) : "v"(slabIn + (i >> 2) * 4096), "i"((i & 3) * 1024) : "memory");
            asm volatile("s_waitcnt vmcnt(0)" : "+v"(in[0]), "+v"(in[1]), "+v"(in[2]), "+v"(in[3]), "+v"(in[4]), "+v"(in[5]), "+v"(in[6]), "+v"(in[7]) :: "memory");
            DIAG_SLOW;
        }
        const bool bad = __builtin_amdgcn_readfirstlane(__hip_atomic_load(tmo, __ATOMIC_RELAXED, __HIP_MEMORY_SCOPE_AGENT)) == seq;
        STAMP(4);
        unsigned short* orow = out + (size_t)(pm * 256 + h * 128 + wr * 64 + fr) * NN + pn * 256 + wc * 32 + (fq & 1) * 16 + (fq >> 1) * 8;
#pragma unroll
        for (int bj = 0; bj < 2; ++bj)
#pragma unroll
            for (int m = 0; m < 4; ++m) {
                f32x4 v0 = acc[0][bj][m][0], v1 = acc[0][bj][m][1];
                const u32x4 p = in[bj * 4 + m];
                v0[0] += __uint_as_float(p.x << 16); v0[1] += __uint_as_float(p.x & 0xffff0000u); v0[2] += __uint_as_float(p.y << 16); v0[3] += __uint_as_float(p.y & 0xffff0000u);
                v1[0] += __uint_as_float(p.z << 16); v1[1] += __uint_as_float(p.z & 0xffff0000u); v1[2] += __uint_as_float(p.w << 16); v1[3] += __uint_as_float(p.w & 0xffff0000u);
                unsigned a0 = cvt_pk_bf16(gelu_tanh(v0[0]), gelu_tanh(v0[1])), a1 = cvt_pk_bf16(gelu_tanh(v0[2]), gelu_tanh(v0[3]));
                unsigned c0 = cvt_pk_bf16(gelu_tanh(v1[0]), gelu_tanh(v1[1])), c1 = cvt_pk_bf16(gelu_tanh(v1[2]), gelu_tanh(v1[3]));
                { auto r = __builtin_amdgcn_permlane16_swap(a0, c0, false, false); a0 = r[0]; c0 = r[1]; }
                { auto r = __builtin_amdgcn_permlane16_swap(a1, c1, false, false); a1 = r[0]; c1 = r[1]; }
                u32x4 w = {a0, a1, c0, c1};
                if (bad) w = (u32x4){0x7fc07fc0u, 0x7fc07fc0u, 0x7fc07fc0u, 0x7fc07fc0u};
                *(u32x4*)(orow + (size_t)m * 16 * NN + bj * 128) = w;
            }
    }
}

extern "C" void kernel_launch(int rank, int nranks, ncclComm_t comm,
                              hipStream_t stream,
                              void* const* d_in, const int* in_sizes, int n_in,
                              void* d_out, int out_size,
                              const int* in_dtypes, int out_dtype,
                              void* const* d_out_all,
                              void* const* d_ws_all, size_t ws_bytes) {
    (void)comm; (void)in_sizes; (void)n_in; (void)out_size; (void)in_dtypes; (void)out_dtype; (void)d_out_all;
    if (nranks != NR || (unsigned)rank >= (unsigned)NR || ws_bytes < WS_NEED) { fprintf(stderr, "a2a_gemm: unsupported configuration\n"); return; }
    static bool attr_set = false;
    if (!attr_set) {
        HIP_CHECK(hipFuncSetAttribute((const void*)a2a_gemm_gelu_kernel, hipFuncAttributeMaxDynamicSharedMemorySize, LDS_BYTES));
        attr_set = true;
    }
    hipLaunchKernelGGL(a2a_gemm_gelu_kernel, dim3(NWG), dim3(512), LDS_BYTES, stream, rank,
                       (const unsigned short*)d_in[0], (const unsigned short*)d_in[1], (unsigned short*)d_out, d_ws_all);
    HIP_CHECK(hipGetLastError());
}
```

```cpp
#include <hip/hip_runtime.h>
#include <rccl/rccl.h>
#include <cstdio>
#include <cstdint>
#include <utility>

#define HIP_CHECK(expr) do { hipError_t _e = (expr); if (_e != hipSuccess) { \
    fprintf(stderr, "HIP error %s at %s:%d\n", hipGetErrorString(_e), __FILE__, __LINE__); \
    return; } } while (0)

#define LAS __attribute__((address_space(3)))
#define GAS __attribute__((address_space(1)))
typedef short bf16x8 __attribute__((ext_vector_type(8)));
typedef short s16x4 __attribute__((ext_vector_type(4)));
typedef float f32x4 __attribute__((ext_vector_type(4)));
typedef unsigned u32x4 __attribute__((ext_vector_type(4)));

#define MAP_OPT 2
#define PACE_NUM 9
#define PACE_DEN 448
#define SEND_SHIFT 1
#define TPF_T 38
#define TPF_W 8
#define SRC_PERM 0
#define EARLY_BAR 0
#define PF_DIST 0
#define TOUCH_T 0
#define TOUCH_M 3
#define SHARED_KB 16
#define XS_PERSEG 8
#define XS_PS 6

namespace {
constexpr int NR = 8;
constexpr int MS = 1024;
constexpr int KB = 1024;
constexpr int NN = 8192;
constexpr int SH = SHARED_KB;
constexpr int NF = 64 - SH / 2;
constexpr int NWG = 256;
constexpr int HTB = 16384;
static_assert(SH % 4 == 0 && SH >= 4 && SH <= 16, "tail size");
static_assert((1791 * PACE_NUM) / PACE_DEN + 3 < NF - 2, "all sends must finish inside the generic K-loop");
static_assert(TOUCH_T == 0 || ((1791 * PACE_NUM) / PACE_DEN + 3 < TOUCH_T && TOUCH_T + 4 < NF - 2), "touch window");
constexpr int L_X = 8 * HTB;
constexpr int L_SEND = 9 * HTB;
constexpr int L_FLAG = L_SEND + 8192;
constexpr int L_JUNK = L_FLAG + 4096;
constexpr int LDS_BYTES = L_JUNK + 256;
#define CTRL_MIB 128
#define SLAB_MIB 112
#define RECV0_MIB 80
#define RECV1_MIB 96
constexpr size_t W_CTRL = (size_t)CTRL_MIB << 20;
constexpr size_t W_TMO = W_CTRL + 256;
constexpr size_t W_XFLAG = W_CTRL + 4096;
constexpr size_t W_EFLAG = W_CTRL + 65536;
constexpr size_t W_SEQ = W_EFLAG + 262144;
constexpr size_t W_DIAG = W_CTRL + 524288;
constexpr size_t RECV_BLK = (size_t)MS * KB * 2;
constexpr size_t W_RECV0 = (size_t)RECV0_MIB << 20;
constexpr size_t W_RECV1 = (size_t)RECV1_MIB << 20;
constexpr size_t W_SLAB = (size_t)SLAB_MIB << 20;
constexpr size_t SLAB_B = 65536;
constexpr size_t WS_NEED = (size_t)129 << 20;
constexpr unsigned long long DEADLINE_TICKS = 40000000ull;

constexpr int XS_PER = XS_PERSEG;
constexpr int S_FLAG = 8 / XS_PER + 1;
constexpr int S_SAMP = S_FLAG + XS_PS;
constexpr int S_LOAD = S_SAMP + 2;
static_assert(S_LOAD < SH, "tail too short for the hand-over schedule");
constexpr int n_stage(int u) { return u + 2 < SH ? (u == 0 ? 4 : 6) : 0; }
constexpr int n_store(int u) { return u * XS_PER < 8 ? XS_PER : 0; }
constexpr int pf_at(int T) { return (PF_DIST && T + PF_DIST < 64) ? 1 : 0; }
constexpr int pf_is(int T) { return pf_at(T); }
constexpr int n_post(int u) { return u < 0 ? pf_at(NF - 1) : (u == S_LOAD ? 8 : 0) + ((u & 1) ? pf_at(NF + u / 2) : 0); }
constexpr int n_ops(int u) { return n_store(u) + (u == S_FLAG) + (u == S_SAMP) + n_stage(u) + n_post(u); }
constexpr int tail_wait(int u) { return n_ops(u) + n_post(u - 1); }
constexpr int slotB0(int u) { return u % 3 == 0 ? 4 * HTB : (u % 3 == 1 ? 6 * HTB : L_X); }
constexpr int slotA(int u) { return u % 3 == 0 ? 0 : (u % 3 == 1 ? 2 * HTB : HTB); }
constexpr int slotB1(int u) { return u % 3 == 0 ? 5 * HTB : (u % 3 == 1 ? 7 * HTB : 3 * HTB); }
constexpr int e_of(int T) { return T < 8 ? T : ((T - 8) % 7 + 1) * 8 + (T - 8) / 7; }

__device__ __forceinline__ int lds_byteA(int r, int c) { const int st = (r >> 4) * 2 + (c >> 5), rr = r & 15, cc = c & 31, ob = rr * 64 + cc * 2; return st * 1024 + (ob ^ (((ob >> 9) & 1) << 5)); }
__device__ __forceinline__ void stage_rcA(int b, int& R, int& C) { const int st = b / 1024, sb = b % 1024, swz = sb ^ (((sb >> 9) & 1) << 5); R = (st >> 1) * 16 + swz / 64; C = (st & 1) * 32 + (swz % 64) / 2; }
__device__ __forceinline__ unsigned cvt_pk_bf16(float lo, float hi) { unsigned r; asm volatile("v_cvt_pk_bf16_f32 %0, %1, %2" : "=v"(r) : "v"(lo), "v"(hi)); return r; }
__device__ __forceinline__ float gelu_tanh(float y) {
    const float u = y * y * 0.044715f + 1.0f;
    const float e = __builtin_amdgcn_exp2f(y * u * (-2.0f * 0.7978845608028654f * 1.4426950408889634f));
    return y * __builtin_amdgcn_rcpf(1.0f + e);
}
template <int N> __device__ __forceinline__ void wait_vm() { if constexpr (N < 63) asm volatile("s_waitcnt vmcnt(%0)" :: "n"(N) : "memory"); }
template <class F, int... I> __device__ __forceinline__ void static_for(F&& f, std::integer_sequence<int, I...>) { (f(std::integral_constant<int, I>{}), ...); }
}

__global__ __launch_bounds__(512, 2)
void a2a_gemm_gelu_kernel(int rank, const unsigned short* __restrict__ xloc, const unsigned short* __restrict__ wmat,
                          unsigned short* __restrict__ out, void* const* __restrict__ ws_all) {
    extern __shared__ __attribute__((aligned(16))) unsigned char shm[];
    LAS unsigned char* lds = (LAS unsigned char*)shm;
    const int tid = threadIdx.x, wid = __builtin_amdgcn_readfirstlane(tid >> 6), lane = tid & 63, wr = wid >> 2, wc = wid & 3, fr = lane & 15, fq = lane >> 4;
    const int bid = blockIdx.x, xcd = bid & 7, idx = bid >> 3;
    const int h = xcd & 1, pn = (xcd >> 1) * 8 + (idx & 7), pm = idx >> 3;
    const int partner = bid ^ 1;
    char* const myws = (char*)ws_all[rank];
#define STAMP(i)
    STAMP(0);
    unsigned seqv;
    asm volatile("global_load_dword %0, %1, off sc1" : "=v"(seqv) : "v"(myws + W_SEQ + (size_t)bid * 128) : "memory");

    unsigned voffA[2], voffB[2];
#pragma unroll
    for (int i = 0; i < 2; ++i) {
        int R, C; stage_rcA(tid * 16 + i * 8192, R, C);
        voffA[i] = (unsigned)(R * KB + C) * 2u;
        const int b = tid * 16 + i * 8192, k = b >> 8, cp = (b >> 4) & 15, f = (k & 3) | (((k >> 3) & 1) << 2), c = cp ^ (f << 1);
        voffB[i] = (unsigned)k * (NN * 2) + (unsigned)c * 16u;
    }
    const unsigned ldsw = (unsigned)wid * 1024u;
    const int aoff = lds_byteA(wr * 64 + fr, fq * 8);
    int boff[2];
    { const int g = lane >> 4, q = (lane & 15) >> 2, p = lane & 3, f = q | ((g & 1) << 2);
#pragma unroll
      for (int nf = 0; nf < 2; ++nf) boff[nf] = (int)(unsigned)(uintptr_t)lds + 4 * HTB + (8 * g + q) * 256 + (((wc * 4 + nf * 2 + (p >> 1)) ^ (f << 1)) * 16) + (p & 1) * 8; }

#define SA(b, hh) (((b) * 2 + (hh)) * HTB)
#define SB(b, hh) ((4 + (b) * 2 + (hh)) * HTB)
#define EXP_NODS 0
#define EXP_NGLDS 2
#define EXP_SVC 0
    const bool svc = EXP_SVC && wid == 7;
#define STAGE(bufoff, gbase, voff) do { if (!svc) _Pragma("unroll") for (int _i = 0; _i < EXP_NGLDS; ++_i) \
        __builtin_amdgcn_global_load_lds((const unsigned*)((const char*)(gbase) + (voff)[_i]), (LAS unsigned*)(lds + (bufoff) + ldsw + _i * 8192), 16, 0, 0); } while (0)
#define GLDS1(bufoff, gbase, voff, _i) do { if (!svc) __builtin_amdgcn_global_load_lds((const unsigned*)((const char*)(gbase) + (voff)[_i]), (LAS unsigned*)(lds + (bufoff) + ldsw + (_i) * 8192), 16, 0, 0); } while (0)
#define LDAH(dst, slot, m0) do { _Pragma("unroll") for (int m = (m0); m < (m0) + 2; ++m) _Pragma("unroll") for (int k = 0; k < 2; ++k) dst[m][k] = *(const LAS bf16x8*)(lds + (slot) + aoff + m * 2048 + k * 1024); } while (0)
#define LDA(dst, slot) do { _Pragma("unroll") for (int m = 0; m < 4; ++m) _Pragma("unroll") for (int k = 0; k < 2; ++k) dst[m][k] = *(const LAS bf16x8*)(lds + (slot) + aoff + m * 2048 + k * 1024); } while (0)
#define LDB(dst, slot) LDB2(dst, ((slot) / (4 * HTB)) * (4 * HTB) - 4 * HTB, (slot) % (4 * HTB))
#define LDB2(dst, _hi, _lo) do { s16x4 _r0, _r1, _r2, _r3, _r4, _r5, _r6, _r7; \
        asm volatile("ds_read_b64_tr_b16 %0, %8 offset:%10\n\tds_read_b64_tr_b16 %1, %8 offset:%11\n\tds_read_b64_tr_b16 %2, %8 offset:%12\n\tds_read_b64_tr_b16 %3, %8 offset:%13\n\t" \
                     "ds_read_b64_tr_b16 %4, %9 offset:%10\n\tds_read_b64_tr_b16 %5, %9 offset:%11\n\tds_read_b64_tr_b16 %6, %9 offset:%12\n\tds_read_b64_tr_b16 %7, %9 offset:%13" \
                     : "=&v"(_r0), "=&v"(_r1), "=&v"(_r2), "=&v"(_r3), "=&v"(_r4), "=&v"(_r5), "=&v"(_r6), "=&v"(_r7) \
                     : "v"(boff[0] + (_hi)), "v"(boff[1] + (_hi)), "i"(_lo), "i"((_lo) + 1024), "i"((_lo) + 8192), "i"((_lo) + 9216) : "memory"); \
        dst[0][0] = __builtin_shufflevector(_r0, _r1, 0, 1, 2, 3, 4, 5, 6, 7); dst[0][1] = __builtin_shufflevector(_r2, _r3, 0, 1, 2, 3, 4, 5, 6, 7); \
        dst[1][0] = __builtin_shufflevector(_r4, _r5, 0, 1, 2, 3, 4, 5, 6, 7); dst[1][1] = __builtin_shufflevector(_r6, _r7, 0, 1, 2, 3, 4, 5, 6, 7); } while (0)
#define MMA16(ai, bj, At, Bt) do { _Pragma("unroll") for (int i = 0; i < 16; ++i) { const int k = i >> 3, m = (i >> 1) & 3, n = i & 1; \
        acc[ai][bj][m][n] = __builtin_amdgcn_mfma_f32_16x16x32_bf16(Bt[n][k], At[m][k], acc[ai][bj][m][n], 0, 0, 0); } } while (0)
#define MMA_BAR(ai, bj, At, Bt) do { __builtin_amdgcn_s_setprio(1); MMA16(ai, bj, At, Bt); __builtin_amdgcn_s_setprio(0); BAR; SCHED; } while (0)
#define MMA32_BAR(ai) do { __builtin_amdgcn_s_setprio(1); MMA16(ai, 0, At, B0); MMA16(ai, 1, At, B1); __builtin_amdgcn_s_setprio(0); BAR; SCHED; } while (0)
#define WAIT_V(n) asm volatile("s_waitcnt vmcnt(" #n ")" ::: "memory")
#define WAIT_L(n) asm volatile("s_waitcnt lgkmcnt(" #n ")" ::: "memory")
#define BAR __builtin_amdgcn_s_barrier()
#define SCHED __builtin_amdgcn_sched_barrier(0)

    f32x4 acc[2][2][4][2];
#pragma unroll
    for (int a = 0; a < 2; ++a)
#pragma unroll
        for (int b = 0; b < 2; ++b)
#pragma unroll
            for (int m = 0; m < 4; ++m)
#pragma unroll
                for (int n = 0; n < 2; ++n) acc[a][b][m][n] = (f32x4){0.f, 0.f, 0.f, 0.f};
    bf16x8 At[4][2], B0[2][2], B1[2][2];

    constexpr long long HSTEP_A = (long long)128 * KB * 2;
    constexpr size_t HSTEP_B = 256;
    const long long dA = h ? -HSTEP_A : HSTEP_A;
    const size_t rowOff = ((size_t)pm * 256 + (size_t)h * 128) * KB * 2;
    const char* const aLoc = (const char*)xloc + (size_t)rank * RECV_BLK + rowOff + (size_t)h * 128;
    const char* const wT = (const char*)wmat + (size_t)pn * 512;
    const char* const wCol = wT + ((size_t)h << 20);
    const char* const bLoc = wCol + ((size_t)rank << 24);

    const unsigned vpf = (unsigned)(pm * 16 + wid * 2 + ((lane >> 2) & 1)) * (NN * 2) + (unsigned)(lane & 3) * 128u;
    unsigned eP = (unsigned)e_of(PF_DIST);
#define PREFETCH(ptr) __builtin_amdgcn_global_load_lds((const unsigned*)((ptr) + vpf), (LAS unsigned*)(lds + L_JUNK), 4, 0, 0)
#define PREFETCHA(ptr)
    constexpr int PFN = PF_DIST ? 1 : 0, PFI = PFN;
    static_assert(PF_DIST <= 8, "the prologue prefetch covers local K-blocks only");
    STAGE(SB(0, 0), bLoc, voffB); STAGE(SA(0, 0), aLoc, voffA); STAGE(SB(0, 1), bLoc + HSTEP_B, voffB); STAGE(SA(0, 1), aLoc + dA, voffA);
    unsigned vsend = (unsigned)(wid * 4 + (lane >> 4)) * (KB * 2) + (unsigned)(lane & 15) * 16u;
#define SEND_RG(p, dj) (SRC_PERM ? (((p) + 8 * (7 - (dj))) & 31) : (p))
    int sslot = (bid * PACE_NUM) / PACE_DEN - SEND_SHIFT;
    if (sslot < 0) {
        const int dj = bid % 7, rg = SEND_RG(bid / 7, dj), d = (rank + 1 + dj) & 7;
        __builtin_amdgcn_global_load_lds((const unsigned*)((const char*)xloc + (size_t)d * RECV_BLK + (size_t)rg * 32 * (KB * 2) + vsend), (LAS unsigned*)(lds + L_SEND + ldsw), 16, 0, 0);
    }
    STAGE(SB(1, 0), bLoc + ((size_t)1 << 21), voffB); STAGE(SA(1, 0), aLoc + 256, voffA); STAGE(SB(1, 1), bLoc + ((size_t)1 << 21) + HSTEP_B, voffB);
    if (EXP_SVC) { for (int tk = 2; tk < PF_DIST; ++tk) { PREFETCH(bLoc + ((size_t)tk << 21)); PREFETCHA(aLoc - h * HSTEP_A + tk * 256); }
        if (svc) asm volatile("s_waitcnt vmcnt(%0)" :: "n"(EXP_SVC >= 2 ? (PF_DIST - 2) * (EXP_SVC - 1) : 0) : "memory");
        asm volatile("s_waitcnt vmcnt(6)" : "+v"(seqv) :: "memory");
    } else
    if (PFN) {
        const unsigned tk = 2u + (unsigned)(lane >> 3);
        PREFETCH(bLoc + ((size_t)(tk < (unsigned)PF_DIST ? tk : 2u) << 21));
        asm volatile("s_waitcnt vmcnt(7)" : "+v"(seqv) :: "memory");
    } else asm volatile("s_waitcnt vmcnt(6)" : "+v"(seqv) :: "memory");
    BAR; asm volatile("" ::: "memory");
    const unsigned seq = (unsigned)__builtin_amdgcn_readfirstlane(seqv) + 1u;
    if (tid == 0) asm volatile("global_store_dword %0, %1, off sc1" :: "v"(myws + W_SEQ + (size_t)bid * 128), "v"(seq) : "memory");
    const size_t parOff = (seq & 1u) ? W_RECV1 : W_RECV0;
    const char* const aRemT = myws + parOff + rowOff;
    const char* const aRem = aRemT + (size_t)h * 128;
    const char* const aLocB = aLoc - ((size_t)rank << 21);
    const char* const fRem = myws + W_XFLAG + (size_t)pm * 128;
    const unsigned vfl = (unsigned)(lane & 7) * 4u;
    const unsigned r8 = (unsigned)rank * 8u;
    GAS unsigned* const tmo = (GAS unsigned*)(myws + W_TMO);

    const char* pA1 = aLoc + 256; const char* pA2 = aLoc + 512; const char* pB2 = bLoc + ((size_t)2 << 21);
    const char* pA3 = aLoc + 768; const char* pB3 = bLoc + ((size_t)3 << 21);
    const char* pF2 = fRem; const char* pF3 = fRem;
    unsigned e3 = 3;
    unsigned sord = 0;
    if (SRC_PERM) { int k = 1; for (int i = 0; i < 8; ++i) { const int dd = ((pm - (i >> 1)) & 3) + 4 * (i & 1); if (dd) { sord |= (unsigned)dd << (4 * k); ++k; } } }
    int sidx = 0;
    const char* sSrc; char* sDst; char* sFlag;
#define SEND_SETUP() do { const int q = sidx * 256 + bid, a = q / 224, r = q - a * 224, pp = r / 7, dj = r - pp * 7, rg = SEND_RG(pp, dj), d = (rank + 1 + dj) & 7; \
        char* const pw = (char*)ws_all[d]; const size_t o = (size_t)rg * 32 * (KB * 2) + (size_t)a * 256; \
        sSrc = (const char*)xloc + (size_t)d * RECV_BLK + o; sDst = pw + parOff + (size_t)rank * RECV_BLK + o; \
        sFlag = pw + W_XFLAG + (size_t)((rank * 8 + a) * 4 + (rg >> 3)) * 128 + (size_t)(rg & 7) * 4; } while (0)
#define TPF_SETUP() do { if (TPF_T) { const int k = pm * 2 + h, u = NF + k - 8, a = u / 7, j = u - a * 7; \
        sSrc = wT + ((size_t)((r8 + (unsigned)((j + 1) * 8 + a)) & 63u) << 21); vsend = (unsigned)(wid * 16 + (lane >> 2)) * (NN * 2) + (unsigned)(lane & 3) * 128u; \
        sslot = TPF_T + ((2 * k + (pn & 1)) * TPF_W) / 16; } else sslot = 1000; } while (0)
    static_assert(TPF_T == 0 || (SH == 16 && (1791 * PACE_NUM) / PACE_DEN - SEND_SHIFT + 3 < TPF_T && TPF_T + TPF_W < NF - 2), "prefetch window");
    SEND_SETUP();
    u32x4 pv = {0u, 0u, 0u, 0u};
    unsigned fv = 0xffffffffu;
    if (wr == 1) BAR;
    STAMP(1);

#define DIAG_SLOW
#define UNLIKELY(x) __builtin_expect(!!(x), 0)
#define SPIN_FLAGS(ptr) do { const GAS unsigned* fp = (const GAS unsigned*)(ptr); const unsigned long long t0 = __builtin_amdgcn_s_memrealtime(); \
            while (!__all(__hip_atomic_load(fp, __ATOMIC_RELAXED, __HIP_MEMORY_SCOPE_SYSTEM) >= seq)) { \
                if (__builtin_amdgcn_readfirstlane(__hip_atomic_load(tmo, __ATOMIC_RELAXED, __HIP_MEMORY_SCOPE_AGENT)) == seq || __builtin_amdgcn_s_memrealtime() - t0 > DEADLINE_TICKS) { \
                    __hip_atomic_store(tmo, seq, __ATOMIC_RELAXED, __HIP_MEMORY_SCOPE_AGENT); break; } \
                __builtin_amdgcn_s_sleep(2); } \
            asm volatile("" ::: "memory"); DIAG_SLOW; } while (0)
#define FSLOT(b) (L_FLAG + (b) * 2048 + wid * 256)
#define TS(x)
#define DIAG2_ACC
#define K_TILE(b, t) do { \
        const int sd = (t) - sslot, td = (t) - TOUCH_T; \
        TS(ts0); \
          \
        __builtin_amdgcn_global_load_lds((const unsigned*)(pF3 + vfl), (LAS unsigned*)(lds + FSLOT(b)), 4, 0, 17); \
        if (UNLIKELY(sd == 0)) { __builtin_amdgcn_global_load_lds((const unsigned*)(sSrc + vsend), (LAS unsigned*)(lds + L_SEND + ldsw), 16, 0, 0); if (sidx >= 7) sslot = 1000; } \
        if (UNLIKELY(sd == 3 && wid == 0)) asm volatile("global_store_dword %0, %1, off sc0 sc1" :: "v"(sFlag), "v"(seq) : "memory"); \
        LDB(B0, SB(b, 0)); SCHED; GLDS1(SA((b) ^ 1, 1), pA1 + dA, voffA, 0); SCHED; \
        LDAH(At, SA(b, 0), 0); SCHED; GLDS1(SA((b) ^ 1, 1), pA1 + dA, voffA, 1); SCHED; \
        LDAH(At, SA(b, 0), 2); SCHED; GLDS1(rgW, pB2 + HSTEP_B, voffB, 0); SCHED; \
        LDB2(B1, rgR - 4 * HTB, 0); SCHED; GLDS1(rgW, pB2 + HSTEP_B, voffB, 1); \
        TS(ts1); \
        if (UNLIKELY(sd == 0 || sd == 2 || (sd == 3 && wid == 0) || (TOUCH_T && (td == 1 || td == 3)))) wait_vm<12 + PFN>(); else wait_vm<11 + PFN>(); \
        WAIT_L(0); TS(ts2); BAR; SCHED; TS(ts3); MMA32_BAR(0); \
        TS(ts4); \
          \
        asm volatile("ds_read_b32 %0, %1 offset:%2" : "=v"(fv) : "v"(fadr), "i"(((b) ^ 1) * 2048) : "memory"); \
        if (UNLIKELY(sd == 1)) pv = *(const volatile LAS u32x4*)(lds + L_SEND + ldsw + lane * 16); \
        GLDS1(SB(b, 0), pB2, voffB, 0); SCHED; \
        LDAH(At, SA(b, 1), 0); SCHED; GLDS1(SB(b, 0), pB2, voffB, 1); SCHED; \
        LDAH(At, SA(b, 1), 2); SCHED; \
        asm volatile("s_waitcnt lgkmcnt(8)" : "+v"(fv) :: "memory"); \
        if (UNLIKELY((t) >= 6 && !__all(fv >= seq))) SPIN_FLAGS(pF2 + vfl); \
        GLDS1(SA(b, 0), pA2, voffA, 0); \
        if (UNLIKELY(sd == 3)) { ++sidx; if (sidx < 7) { sslot = ((sidx * 256 + bid) * PACE_NUM) / PACE_DEN - SEND_SHIFT; SEND_SETUP(); } else TPF_SETUP(); } \
          \
        const char* const pA2o = pA2; \
        pA1 = pA2; pA2 = pA3; pB2 = pB3; pF2 = pF3; \
        e3 += (e3 < 8u) ? 1u : 8u; e3 -= (e3 >= 64u) ? 55u : 0u; \
        { const unsigned c = (r8 + ((SRC_PERM && !(e3 & 7u)) ? ((sord >> (e3 >> 1)) & 7u) << 3 : e3)) & 63u; \
          pA3 = (e3 < 8u ? aLocB : aRem) + (((c >> 3) << 21) | ((c & 7u) << 8)); \
          pB3 = wCol + ((size_t)c << 21); pF3 = fRem + (c << 9); } \
        { const int r = rgR; rgR = rgN; rgN = rgW; rgW = r; } \
        asm volatile("" : "+s"(pA3), "+s"(pB3), "+s"(pF3), "+s"(e3), "+s"(rgR), "+s"(rgN), "+s"(rgW)); \
        SCHED; GLDS1(SA(b, 0), pA2o, voffA, 1); \
        if (UNLIKELY(sd == 1)) asm volatile("global_store_dwordx4 %0, %1, off sc0 sc1\n\ts_nop 1" :: "v"(sDst + vsend), "v"(pv) : "memory"); \
        if (PFI) { const unsigned c = (r8 + eP) & 63u; PREFETCH(wCol + ((size_t)c << 21)); \
            PREFETCHA((eP < 8u ? aLocB : aRem) - h * HSTEP_A + (((c >> 3) << 21) | ((c & 7u) << 8))); \
            eP += (eP < 8u) ? 1u : 8u; eP -= (eP >= 64u) ? 55u : 0u; } \
        TS(ts5); \
        if (TOUCH_T && UNLIKELY(td == 0)) __builtin_amdgcn_global_load_lds((const unsigned*)((TOUCH_M & 1) ? myws + W_SLAB + (size_t)bid * SLAB_B + (size_t)wid * 8192 + lane * 128 : pF3 + vfl), (LAS unsigned*)(lds + L_JUNK), 4, 0, 0); \
        if (TOUCH_T && UNLIKELY(td == 2)) __builtin_amdgcn_global_load_lds((const unsigned*)((TOUCH_M & 2) ? (const char*)out + (size_t)(pm * 256 + h * 128 + wid * 16 + (lane >> 2)) * (NN * 2) + pn * 512 + (lane & 3) * 128 : pF3 + vfl), (LAS unsigned*)(lds + L_JUNK), 4, 0, 0); \
        if (UNLIKELY(sd == 0 || sd == 1 || (sd == 3 && wid == 0) || (TOUCH_T && (unsigned)td < 4u))) wait_vm<10 + 2 * PFN>(); else wait_vm<9 + 2 * PFN>(); \
        WAIT_L(0); DIAG2_ACC; BAR; SCHED; MMA32_BAR(1); \
    } while (0)

    const unsigned fadr = (unsigned)(uintptr_t)lds + FSLOT(0) + lane * 4;
    static_assert(NF % 3 == 2, "the ring of second w halves must end in its home position");
    int rgR = SB(0, 1), rgN = SB(1, 1), rgW = L_X;
    for (int t = 0; t < NF - 2; t += 2) {
        K_TILE(0, t);
        K_TILE(1, t + 1);
    }

#define PF_PTR(T) (wCol + ((size_t)((r8 + (unsigned)e_of((T) + PF_DIST)) & 63u) << 21))
#define PFA_C(T) ((r8 + (unsigned)e_of((T) + PF_DIST)) & 63u)
#define PFA_PTR(T) (aRem - h * HSTEP_A + (((PFA_C(T) >> 3) << 21) | ((PFA_C(T) & 7u) << 8)))
#define TAIL_C(u) ((r8 + (unsigned)e_of(NF + (u) / 2)) & 63u)
#define TAIL_A(u) (aRemT + ((((TAIL_C(u) >> 3) << 21) | ((TAIL_C(u) & 7u) << 8)) + ((u) & 1) * 128))
#define TAIL_B(u) (wT + (((size_t)TAIL_C(u) << 21) + ((size_t)((u) & 1) << 20)))
    const char* tfp;
    { int k = lane >> 3; k = k < SH / 2 ? k : SH / 2 - 1; const int u = NF + k - 8, a = u / 7, j = u - a * 7;
      tfp = fRem + (((r8 + (unsigned)((j + 1) * 8 + a)) & 63u) << 9) + vfl; }
    {
        LDB(B0, SB(0, 0)); LDA(At, SA(0, 0)); LDB(B1, SB(0, 1));
        __builtin_amdgcn_global_load_lds((const unsigned*)tfp, (LAS unsigned*)(lds + FSLOT(0)), 4, 0, 17);
        STAGE(SA(1, 1), pA1 + dA, voffA);
        wait_vm<9 + PFN>(); WAIT_L(0); BAR; SCHED; MMA32_BAR(0);
        fv = *(const volatile LAS unsigned*)(lds + FSLOT(1) + lane * 4);
        LDA(At, SA(0, 1));
        STAGE(slotB0(0), TAIL_B(0), voffB); STAGE(slotB1(0), TAIL_B(0) + HSTEP_B, voffB);
        if (UNLIKELY(!__all(fv >= seq))) SPIN_FLAGS(pF2 + vfl);
        STAGE(slotA(0), TAIL_A(0), voffA);
        if constexpr (pf_is(NF - 2)) { PREFETCH(PF_PTR(NF - 2)); PREFETCHA(PFA_PTR(NF - 2)); }
        wait_vm<9 + PFN + pf_at(NF - 2)>(); WAIT_L(0); BAR; SCHED; MMA32_BAR(1);
        LDB(B0, SB(1, 0)); LDA(At, SA(1, 0)); LDB(B1, SB(1, 1));
        STAGE(slotB0(2), TAIL_B(2), voffB);
        wait_vm<8 + pf_at(NF - 2)>(); WAIT_L(0); BAR; SCHED; MMA32_BAR(0);
        fv = *(const volatile LAS unsigned*)(lds + FSLOT(0) + lane * 4);
        LDA(At, SA(1, 1));
        STAGE(slotB0(1), TAIL_B(1), voffB);
        if (UNLIKELY(!__all(fv >= seq))) SPIN_FLAGS(tfp);
        STAGE(slotA(1), TAIL_A(1), voffA); STAGE(slotB1(1), TAIL_B(1) + HSTEP_B, voffB);
        if constexpr (pf_is(NF - 1)) { PREFETCH(PF_PTR(NF - 1)); PREFETCHA(PFA_PTR(NF - 1)); }
        wait_vm<8 + pf_at(NF - 2) + pf_at(NF - 1)>(); WAIT_L(0); BAR; SCHED; MMA32_BAR(1);
    }
    STAMP(2);
    char* const slabOut = myws + W_SLAB + (size_t)bid * SLAB_B + (size_t)wid * 8192 + lane * 16;
    const char* const slabIn = myws + W_SLAB + (size_t)partner * SLAB_B + (size_t)wid * 8192 + lane * 16;
    const char* const pflag = myws + W_EFLAG + (size_t)(partner * 8 + wid) * 128;
    u32x4 in[8];
    bool early = false;
    static_for([&](auto uc) {
        constexpr int u = decltype(uc)::value;
        constexpr bool st = u + 2 < SH;
#pragma unroll
        for (int jj = 0; jj < n_store(u); ++jj) {
            const int j = u * XS_PER + jj, bj = j >> 2, m = j & 3;
            const f32x4 v0 = acc[1][bj][m][0], v1 = acc[1][bj][m][1];
            u32x4 w; w.x = cvt_pk_bf16(v0[0], v0[1]); w.y = cvt_pk_bf16(v0[2], v0[3]); w.z = cvt_pk_bf16(v1[0], v1[1]); w.w = cvt_pk_bf16(v1[2], v1[3]);
            asm volatile("global_store_dwordx4 %0, %1, off sc1\n\ts_nop 1" :: "v"(slabOut + j * 1024), "v"(w) : "memory");
        }
        if constexpr (u == S_FLAG) asm volatile("global_store_dword %0, %1, off sc1" :: "v"(myws + W_EFLAG + (size_t)(bid * 8 + wid) * 128), "v"(seq) : "memory");
        if constexpr (u == S_SAMP) __builtin_amdgcn_global_load_lds((const unsigned*)pflag, (LAS unsigned*)(lds + FSLOT(0)), 4, 0, 17);
        if constexpr (u == S_LOAD) asm volatile("ds_read_b32 %0, %1" : "=v"(fv) : "v"(fadr) : "memory");
        SCHED;
        if constexpr (st && u > 0) { GLDS1(slotB0(u + 2), TAIL_B(u + 2), voffB, 0); SCHED; }
        LDB(B0, slotB0(u)); SCHED;
        if constexpr (st && u > 0) { GLDS1(slotB0(u + 2), TAIL_B(u + 2), voffB, 1); SCHED; }
        LDAH(At, slotA(u), 0); SCHED;
        if constexpr (st) { GLDS1(slotA(u + 2), TAIL_A(u + 2), voffA, 0); SCHED; }
        LDAH(At, slotA(u), 2); SCHED;
        if constexpr (st) { GLDS1(slotA(u + 2), TAIL_A(u + 2), voffA, 1); SCHED; }
        LDB(B1, slotB1(u)); SCHED;
        if constexpr (st) { GLDS1(slotB1(u + 2), TAIL_B(u + 2) + HSTEP_B, voffB, 0); GLDS1(slotB1(u + 2), TAIL_B(u + 2) + HSTEP_B, voffB, 1); }
        asm volatile("" ::: "memory");
        if constexpr (u == S_LOAD) {
            asm volatile("s_waitcnt lgkmcnt(15)" : "+v"(fv) :: "memory");
            early = (unsigned)__builtin_amdgcn_readfirstlane(fv) >= seq;
            asm volatile("" ::: "memory");
#pragma unroll
            for (int i = 0; i < 8; ++i) asm volatile("global_load_dwordx4 %0, %1, off offset:%2 sc1" : "=v"(in[i]) : "v"(slabIn + (i >> 2) * 4096), "i"((i & 3) * 1024) : "memory");
        }
        if constexpr ((u & 1) && pf_is(NF + u / 2)) { PREFETCH(PF_PTR(NF + u / 2)); PREFETCHA(PFA_PTR(NF + u / 2)); }
        if constexpr (u + 1 < SH) wait_vm<tail_wait(u)>();
        WAIT_L(0); BAR; SCHED; MMA32_BAR(0);
    }, std::make_integer_sequence<int, SH>{});
    if (wr == 0) BAR;
    STAMP(3);

    {
        asm volatile("s_waitcnt vmcnt(0)" : "+v"(in[0]), "+v"(in[1]), "+v"(in[2]), "+v"(in[3]), "+v"(in[4]), "+v"(in[5]), "+v"(in[6]), "+v"(in[7]) :: "memory");
        if (UNLIKELY(!early)) {
            const GAS unsigned* fp = (const GAS unsigned*)pflag;
            const unsigned long long t0 = __builtin_amdgcn_s_memrealtime();
            while ((unsigned)__builtin_amdgcn_readfirstlane(__hip_atomic_load(fp, __ATOMIC_RELAXED, __HIP_MEMORY_SCOPE_AGENT)) < seq) {
                if (__builtin_amdgcn_readfirstlane(__hip_atomic_load(tmo, __ATOMIC_RELAXED, __HIP_MEMORY_SCOPE_AGENT)) == seq || __builtin_amdgcn_s_memrealtime() - t0 > DEADLINE_TICKS) {
                    __hip_atomic_store(tmo, seq, __ATOMIC_RELAXED, __HIP_MEMORY_SCOPE_AGENT); break; }
                __builtin_amdgcn_s_sleep(1);
            }
            asm volatile("" ::: "memory");
#pragma unroll
            for (int i = 0; i < 8; ++i) asm volatile("global_load_dwordx4 %0, %1, off offset:%2 sc1" : "=v"(in[i]) : "v"(slabIn + (i >> 2) * 4096), "i"((i & 3) * 1024) : "memory");
            asm volatile("s_waitcnt vmcnt(0)" : "+v"(in[0]), "+v"(in[1]), "+v"(in[2]), "+v"(in[3]), "+v"(in[4]), "+v"(in[5]), "+v"(in[6]), "+v"(in[7]) :: "memory");
            DIAG_SLOW;
        }
        const bool bad = __builtin_amdgcn_readfirstlane(__hip_atomic_load(tmo, __ATOMIC_RELAXED, __HIP_MEMORY_SCOPE_AGENT)) == seq;
        STAMP(4);
        unsigned short* orow = out + (size_t)(pm * 256 + h * 128 + wr * 64 + fr) * NN + pn * 256 + wc * 32 + (fq & 1) * 16 + (fq >> 1) * 8;
#pragma unroll
        for (int bj = 0; bj < 2; ++bj)
#pragma unroll
            for (int m = 0; m < 4; ++m) {
                f32x4 v0 = acc[0][bj][m][0], v1 = acc[0][bj][m][1];
                const u32x4 p = in[bj * 4 + m];
                v0[0] += __uint_as_float(p.x << 16); v0[1] += __uint_as_float(p.x & 0xffff0000u); v0[2] += __uint_as_float(p.y << 16); v0[3] += __uint_as_float(p.y & 0xffff0000u);
                v1[0] += __uint_as_float(p.z << 16); v1[1] += __uint_as_float(p.z & 0xffff0000u); v1[2] += __uint_as_float(p.w << 16); v1[3] += __uint_as_float(p.w & 0xffff0000u);
                unsigned a0 = cvt_pk_bf16(gelu_tanh(v0[0]), gelu_tanh(v0[1])), a1 = cvt_pk_bf16(gelu_tanh(v0[2]), gelu_tanh(v0[3]));
                unsigned c0 = cvt_pk_bf16(gelu_tanh(v1[0]), gelu_tanh(v1[1])), c1 = cvt_pk_bf16(gelu_tanh(v1[2]), gelu_tanh(v1[3]));
                { auto r = __builtin_amdgcn_permlane16_swap(a0, c0, false, false); a0 = r[0]; c0 = r[1]; }
                { auto r = __builtin_amdgcn_permlane16_swap(a1, c1, false, false); a1 = r[0]; c1 = r[1]; }
                u32x4 w = {a0, a1, c0, c1};
                if (bad) w = (u32x4){0x7fc07fc0u, 0x7fc07fc0u, 0x7fc07fc0u, 0x7fc07fc0u};
                *(u32x4*)(orow + (size_t)m * 16 * NN + bj * 128) = w;
            }
    }
}

extern "C" void kernel_launch(int rank, int nranks, ncclComm_t comm,
                              hipStream_t stream,
                              void* const* d_in, const int* in_sizes, int n_in,
                              void* d_out, int out_size,
                              const int* in_dtypes, int out_dtype,
                              void* const* d_out_all,
                              void* const* d_ws_all, size_t ws_bytes) {
    (void)comm; (void)in_sizes; (void)n_in; (void)out_size; (void)in_dtypes; (void)out_dtype; (void)d_out_all;
    if (nranks != NR || (unsigned)rank >= (unsigned)NR || ws_bytes < WS_NEED) { fprintf(stderr, "a2a_gemm: unsupported configuration\n"); return; }
    static bool attr_set = false;
    if (!attr_set) {
        HIP_CHECK(hipFuncSetAttribute((const void*)a2a_gemm_gelu_kernel, hipFuncAttributeMaxDynamicSharedMemorySize, LDS_BYTES));
        attr_set = true;
    }
    hipLaunchKernelGGL(a2a_gemm_gelu_kernel, dim3(NWG), dim3(512), LDS_BYTES, stream, rank,
                       (const unsigned short*)d_in[0], (const unsigned short*)d_in[1], (unsigned short*)d_out, d_ws_all);
    HIP_CHECK(hipGetLastError());
}
```

```cpp
#include <hip/hip_runtime.h>
#include <rccl/rccl.h>
#include <cstdio>
#include <cstdint>
#include <utility>

#define HIP_CHECK(expr) do { hipError_t _e = (expr); if (_e != hipSuccess) { \
    fprintf(stderr, "HIP error %s at %s:%d\n", hipGetErrorString(_e), __FILE__, __LINE__); \
    return; } } while (0)

#define LAS __attribute__((address_space(3)))
#define GAS __attribute__((address_space(1)))
typedef short bf16x8 __attribute__((ext_vector_type(8)));
typedef short s16x4 __attribute__((ext_vector_type(4)));
typedef float f32x4 __attribute__((ext_vector_type(4)));
typedef unsigned u32x4 __attribute__((ext_vector_type(4)));

#define PACE_NUM 9
#define PACE_DEN 448
#define BAL_S 6
#define BAL_LAG 4
#define XS_SAMP 11
#define XS_ST 4
#define XS_LD 4

namespace {
constexpr int NR = 8;
constexpr int MS = 1024;
constexpr int KB = 1024;
constexpr int NN = 8192;
constexpr int SH = 16;
constexpr int NF = 64 - SH / 2;
constexpr int NWG = 256;
constexpr int HTB = 16384;
constexpr int slot_of(int q) { return (q * PACE_NUM) / PACE_DEN - 1; }
static_assert(BAL_S >= 0 && BAL_S <= 6 && slot_of(1791) + 3 + BAL_LAG + 3 < NF - 6 - 4, "all events must finish inside the generic K-loop");
static_assert(slot_of(256) - slot_of(0) >= 5 || (256 * PACE_NUM) / PACE_DEN >= 4, "one piece at a time");
constexpr int L_X = 8 * HTB;
constexpr int L_SEND = 9 * HTB;
constexpr int L_FLAG = L_SEND + 8192;
constexpr int LDS_BYTES = L_FLAG + 4096;
constexpr size_t W_CTRL = (size_t)128 << 20;
constexpr size_t W_TMO = W_CTRL + 256;
constexpr size_t W_XFLAG = W_CTRL + 4096;
constexpr size_t W_EFLAG = W_CTRL + 65536;
constexpr size_t W_SEQ = W_EFLAG + 262144;
constexpr size_t W_REC = W_SEQ + 32768;
constexpr size_t W_DIAG = W_CTRL + 524288;
constexpr size_t RECV_BLK = (size_t)MS * KB * 2;
constexpr size_t W_RECV0 = (size_t)80 << 20;
constexpr size_t W_RECV1 = (size_t)96 << 20;
constexpr size_t W_SLAB = (size_t)112 << 20;
constexpr size_t SLAB_B = 65536;
constexpr size_t WS_NEED = (size_t)129 << 20;
constexpr unsigned long long DEADLINE_TICKS = 40000000ull;

constexpr int S_FLAG = 8 / XS_ST + 1;
constexpr int S_SAMP = XS_SAMP;
constexpr int S_LOAD = S_SAMP + 2;
static_assert(S_SAMP > S_FLAG && S_LOAD + 8 / XS_LD <= SH, "tail too short for the hand-over schedule");
constexpr int n_stage(int u) { return u + 2 < SH ? (u == 0 ? 4 : 6) : 0; }
constexpr int n_post(int u) { return (u >= 0 && u < 8 / XS_ST ? XS_ST : 0) + (u >= S_LOAD && u < S_LOAD + 8 / XS_LD ? XS_LD : 0) + (u == S_FLAG) + (u == S_SAMP); }
constexpr int tail_wait(int u) { return n_stage(u) + n_post(u - 1); }
constexpr int e_of(int T) { return T < 8 ? T : ((T - 8) % 7 + 1) * 8 + (T - 8) / 7; }
static_assert(e_of(NF) == 62 && e_of(NF - 6) == 14, "take-over targets");

__device__ __forceinline__ int lds_byteA(int r, int c) { const int st = (r >> 4) * 2 + (c >> 5), rr = r & 15, cc = c & 31, ob = rr * 64 + cc * 2; return st * 1024 + (ob ^ (((ob >> 9) & 1) << 5)); }
__device__ __forceinline__ void stage_rcA(int b, int& R, int& C) { const int st = b / 1024, sb = b % 1024, swz = sb ^ (((sb >> 9) & 1) << 5); R = (st >> 1) * 16 + swz / 64; C = (st & 1) * 32 + (swz % 64) / 2; }
__device__ __forceinline__ unsigned cvt_pk_bf16(float lo, float hi) { unsigned r; asm volatile("v_cvt_pk_bf16_f32 %0, %1, %2" : "=v"(r) : "v"(lo), "v"(hi)); return r; }
__device__ __forceinline__ float gelu_tanh(float y) {
    const float u = y * y * 0.044715f + 1.0f;
    const float e = __builtin_amdgcn_exp2f(y * u * (-2.0f * 0.7978845608028654f * 1.4426950408889634f));
    return y * __builtin_amdgcn_rcpf(1.0f + e);
}
template <int N> __device__ __forceinline__ void wait_vm_l0() { if constexpr (N < 63) asm volatile("s_waitcnt vmcnt(%0) lgkmcnt(0)" :: "n"(N) : "memory"); else asm volatile("s_waitcnt lgkmcnt(0)" ::: "memory"); }
template <class F, int... I> __device__ __forceinline__ void static_for(F&& f, std::integer_sequence<int, I...>) { (f(std::integral_constant<int, I>{}), ...); }
__device__ __forceinline__ const char* mk_ptr(unsigned lo, unsigned hi) { return (const char*)(((unsigned long long)hi << 32) | lo); }
}

__global__ __launch_bounds__(512, 2)
void a2a_gemm_gelu_kernel(int rank, const unsigned short* __restrict__ xloc, const unsigned short* __restrict__ wmat,
                          unsigned short* __restrict__ out, void* const* __restrict__ ws_all) {
    extern __shared__ __attribute__((aligned(16))) unsigned char shm[];
    LAS unsigned char* lds = (LAS unsigned char*)shm;
    const int tid = threadIdx.x, wid = __builtin_amdgcn_readfirstlane(tid >> 6), lane = tid & 63, wr = wid >> 2, wc = wid & 3, fr = lane & 15, fq = lane >> 4;
    const int bid = blockIdx.x, xcd = bid & 7, idx = bid >> 3;
    const int h = xcd & 1, pn = (xcd >> 1) * 8 + (idx & 7), pm = idx >> 3;
    const int partner = bid ^ 1, bpair = bid & ~1;
    char* const myws = (char*)ws_all[rank];
#define STAMP(i)
    STAMP(0);
    unsigned seqv;
    asm volatile("global_load_dword %0, %1, off sc1" : "=v"(seqv) : "v"(myws + W_SEQ + (size_t)bid * 128) : "memory");

    constexpr unsigned HSTEP_A = 128u * KB * 2;
    constexpr int HSTEP_B = 256;
    unsigned voffA[2], voffP[2], voffB[2];
#pragma unroll
    for (int i = 0; i < 2; ++i) {
        int R, C; stage_rcA(tid * 16 + i * 8192, R, C);
        const unsigned o = (unsigned)(R * KB + C) * 2u;
        voffA[i] = o + (unsigned)h * HSTEP_A; voffP[i] = o + (unsigned)(h ^ 1) * HSTEP_A;
        const int b = tid * 16 + i * 8192, k = b >> 8, cp = (b >> 4) & 15, f = (k & 3) | (((k >> 3) & 1) << 2), c = cp ^ (f << 1);
        voffB[i] = (unsigned)k * (NN * 2) + (unsigned)c * 16u;
    }
    const unsigned lds0 = (unsigned)(uintptr_t)lds;
    const unsigned ldsw = lds0 + (unsigned)wid * 1024u, ldsf = lds0 + L_FLAG + (unsigned)wid * 256u;
    unsigned aoff = lds0 + lds_byteA(wr * 64 + fr, fq * 8);
    asm volatile("" : "+v"(aoff));
    unsigned boff[2];
    { const int g = lane >> 4, q = (lane & 15) >> 2, p = lane & 3, f = q | ((g & 1) << 2);
#pragma unroll
      for (int nf = 0; nf < 2; ++nf) boff[nf] = lds0 + (8 * g + q) * 256 + (((wc * 4 + nf * 2 + (p >> 1)) ^ (f << 1)) * 16) + (p & 1) * 8; }
    const unsigned boffS[2] = {boff[0] + 4 * HTB, boff[1] + 4 * HTB};

#define SA(b, hh) (((b) * 2 + (hh)) * HTB)
#define SB(b, hh) ((4 + (b) * 2 + (hh)) * HTB)
#define SETM0(sreg, imm) asm volatile("s_add_u32 m0, %0, %1" :: "s"(sreg), "i"(imm) : "scc")
#define GLDS(voff, sbase) asm volatile("global_load_lds_dwordx4 %0, %1" :: "v"(voff), "s"(sbase) : "memory")
#define GLDSH(voff, sbase) asm volatile("global_load_lds_dwordx4 %0, %1 offset:256" :: "v"(voff), "s"(sbase) : "memory")
#define GLDS_N(sreg, imm, voff, sbase) asm volatile("s_add_u32 m0, %2, %3\n\ts_nop 0\n\tglobal_load_lds_dwordx4 %0, %1" :: "v"(voff), "s"(sbase), "s"(sreg), "i"(imm) : "scc", "memory")
#define GLDSH_N(sreg, imm, voff, sbase) asm volatile("s_add_u32 m0, %2, %3\n\ts_nop 0\n\tglobal_load_lds_dwordx4 %0, %1 offset:256" :: "v"(voff), "s"(sbase), "s"(sreg), "i"((imm) - 256) : "scc", "memory")
#define STAGE(sreg, imm, voff, sbase) do { GLDS_N(sreg, imm, (voff)[0], sbase); GLDS_N(sreg, (imm) + 8192, (voff)[1], sbase); } while (0)
#define STAGEH(sreg, imm, voff, sbase) do { GLDSH_N(sreg, imm, (voff)[0], sbase); GLDSH_N(sreg, (imm) + 8192, (voff)[1], sbase); } while (0)
#define GFLAG(voff, sbase) asm volatile("global_load_lds_dword %0, %1 sc0 sc1" :: "v"(voff), "s"(sbase) : "memory")
#define LDAH(va, off, m0) do { _Pragma("unroll") for (int m = (m0); m < (m0) + 2; ++m) _Pragma("unroll") for (int k = 0; k < 2; ++k) At[m][k] = *(const LAS bf16x8*)(uintptr_t)((va) + (off) + m * 2048 + k * 1024); } while (0)
#define LDB(dst, va0, va1, off) do { s16x4 _r0, _r1, _r2, _r3, _r4, _r5, _r6, _r7; \
        asm volatile("ds_read_b64_tr_b16 %0, %8 offset:%10\n\tds_read_b64_tr_b16 %1, %8 offset:%11\n\tds_read_b64_tr_b16 %2, %8 offset:%12\n\tds_read_b64_tr_b16 %3, %8 offset:%13\n\t" \
                     "ds_read_b64_tr_b16 %4, %9 offset:%10\n\tds_read_b64_tr_b16 %5, %9 offset:%11\n\tds_read_b64_tr_b16 %6, %9 offset:%12\n\tds_read_b64_tr_b16 %7, %9 offset:%13" \
                     : "=&v"(_r0), "=&v"(_r1), "=&v"(_r2), "=&v"(_r3), "=&v"(_r4), "=&v"(_r5), "=&v"(_r6), "=&v"(_r7) \
                     : "v"(va0), "v"(va1), "i"(off), "i"((off) + 1024), "i"((off) + 8192), "i"((off) + 9216) : "memory"); \
        dst[0][0] = __builtin_shufflevector(_r0, _r1, 0, 1, 2, 3, 4, 5, 6, 7); dst[0][1] = __builtin_shufflevector(_r2, _r3, 0, 1, 2, 3, 4, 5, 6, 7); \
        dst[1][0] = __builtin_shufflevector(_r4, _r5, 0, 1, 2, 3, 4, 5, 6, 7); dst[1][1] = __builtin_shufflevector(_r6, _r7, 0, 1, 2, 3, 4, 5, 6, 7); } while (0)
#define BAR __builtin_amdgcn_s_barrier()
#define SCHED __builtin_amdgcn_sched_barrier(0)
#define UNLIKELY(x) __builtin_expect(!!(x), 0)
#define PIN_S(...) asm volatile("" : __VA_ARGS__)
#define MM1(ai, I, ...) { constexpr int i = (I); \
        acc[ai][i >> 4][(i >> 1) & 3][i & 1] = __builtin_amdgcn_mfma_f32_16x16x32_bf16((i >> 4) ? B1[i & 1][(i >> 3) & 1] : B0[i & 1][(i >> 3) & 1], At[(i >> 1) & 3][(i >> 3) & 1], acc[ai][i >> 4][(i >> 1) & 3][i & 1], 0, 0, 0); \
        SCHED; { __VA_ARGS__ } SCHED; }
#define MM4(ai, I, ...) MM1(ai, I, __VA_ARGS__) MM1(ai, (I) + 1, __VA_ARGS__) MM1(ai, (I) + 2, __VA_ARGS__) MM1(ai, (I) + 3, __VA_ARGS__)
#define MM16(ai, I, ...) MM4(ai, I, __VA_ARGS__) MM4(ai, (I) + 4, __VA_ARGS__) MM4(ai, (I) + 8, __VA_ARGS__) MM4(ai, (I) + 12, __VA_ARGS__)
#define MMA32(ai, ...) do { BAR; SCHED; __builtin_amdgcn_s_setprio(1); MM16(ai, 0, __VA_ARGS__) MM16(ai, 16, __VA_ARGS__) __builtin_amdgcn_s_setprio(0); BAR; SCHED; } while (0)
#define HOOK
#define AT(n) if constexpr (i == (n))

    f32x4 acc[2][2][4][2];
#pragma unroll
    for (int a = 0; a < 2; ++a)
#pragma unroll
        for (int b = 0; b < 2; ++b)
#pragma unroll
            for (int m = 0; m < 4; ++m)
#pragma unroll
                for (int n = 0; n < 2; ++n) acc[a][b][m][n] = (f32x4){0.f, 0.f, 0.f, 0.f};
    bf16x8 At[4][2], B0[2][2], B1[2][2];

    const size_t rowOff = (size_t)pm * 256 * KB * 2;
    const char* const aLocT = (const char*)xloc + rowOff;
    const char* const wT = (const char*)wmat + (size_t)pn * 512;
    const char* const fRem = myws + W_XFLAG + (size_t)pm * 128;
    const unsigned g0 = (unsigned)rank * 16u + (unsigned)h;
#define OFF_A(g) ((((g) & 0x70u) << 17) | (((g) & 15u) << 7))
    const char* const aLoc = aLocT + OFF_A(g0);
    const char* const bLoc = wT + ((size_t)g0 << 20);

    STAGE(ldsw, SB(0, 0), voffB, bLoc); STAGE(ldsw, SA(0, 0), voffA, aLoc); STAGEH(ldsw, SB(0, 1), voffB, bLoc); STAGE(ldsw, SA(0, 1), voffP, aLoc);
    const int ek = lane >> 3, ej = lane & 7;
    const int eq = (ek < 7 ? ek : 6) * 256 + bid, ea = eq / 224, er = eq - ea * 224, erg = er / 7, ed = (rank + 1 + er - erg * 7) & 7;
    unsigned long long epw;
    asm volatile("global_load_dwordx2 %0, %1, off" : "=v"(epw) : "v"(ws_all + ed) : "memory");
    unsigned vsend = (unsigned)(wid * 4 + (lane >> 4)) * (KB * 2) + (unsigned)(lane & 15) * 16u;
    int sslot = slot_of(bpair);
    if (sslot < 0) {
        const int rg = bid / 7, dj = bid - rg * 7, d = (rank + 1 + dj) & 7;
        GLDS_N(ldsw, L_SEND, vsend, (const char*)xloc + (size_t)d * RECV_BLK + (size_t)rg * 32 * (KB * 2));
    }
    STAGE(ldsw, SB(1, 0), voffB, bLoc + ((size_t)1 << 21)); STAGE(ldsw, SA(1, 0), voffA, aLoc + 256); STAGEH(ldsw, SB(1, 1), voffB, bLoc + ((size_t)1 << 21));
    const unsigned vfl = (unsigned)(lane & 7) * 4u;
    SETM0(ldsf, 0); asm volatile("s_nop 0"); GFLAG(vfl, fRem);
    unsigned vq;
    { const int u = lane - 8, a = u / 7, j = u - a * 7; vq = (((unsigned)(rank * 8 + (lane < 8 ? lane : (j + 1) * 8 + a)) & 63u) << 1) | (unsigned)h | (lane < NF ? 0u : 128u); }
    asm volatile("s_waitcnt vmcnt(7)" : "+v"(seqv), "+v"(epw) :: "memory");
    BAR; asm volatile("" ::: "memory");
    const unsigned seq = (unsigned)__builtin_amdgcn_readfirstlane(seqv) + 1u;
    if (tid == 0) asm volatile("global_store_dword %0, %1, off sc1" :: "v"(myws + W_SEQ + (size_t)bid * 128), "v"(seq) : "memory");
    const size_t parOff = (seq & 1u) ? W_RECV1 : W_RECV0;
    const char* const aRemT = myws + parOff + rowOff;
    GAS unsigned* const tmo = (GAS unsigned*)(myws + W_TMO);
    unsigned vev;
    { const size_t o = (size_t)erg * 32 * (KB * 2) + (size_t)ea * 256;
      const unsigned long long src = ek < 7 ? (unsigned long long)xloc + (size_t)ed * RECV_BLK + o : (unsigned long long)myws + W_REC + (size_t)bpair * 128;
      const unsigned long long dst = epw + parOff + (size_t)rank * RECV_BLK + o;
      const unsigned long long flg = wid == 1 ? (unsigned long long)myws + W_REC + (size_t)bid * 128 + (size_t)ek * 4
                                              : epw + W_XFLAG + (size_t)((rank * 8 + ea) * 4 + (erg >> 3)) * 128 + (size_t)(erg & 7) * 4;
      const unsigned long long p = ej < 2 ? src : (ej < 4 ? dst : flg);
      vev = ej == 6 ? (unsigned)(slot_of(eq & ~1) + (ek < 7 ? 0 : 3 + BAL_LAG)) : (unsigned)((ej & 1) ? p >> 32 : p); }
    unsigned lut = 0x04021813u | (wid < 2 ? 0x13000000u : 0u);
    int sidx = 0;
    const char* sSrc; const char* sDst; const char* sFlag;
#define RL(v, i) ((unsigned)__builtin_amdgcn_readlane((int)(v), (i)))
#define EV_SETUP() do { const int e8 = sidx * 8; sSrc = mk_ptr(RL(vev, e8), RL(vev, e8 + 1)); sDst = mk_ptr(RL(vev, e8 + 2), RL(vev, e8 + 3)); \
        sFlag = mk_ptr(RL(vev, e8 + 4), RL(vev, e8 + 5)); sslot = (int)RL(vev, e8 + 6); } while (0)
    EV_SETUP();
    const char* pA1 = aLoc + 256; const char* pA2 = aLoc + 512; const char* pB2 = bLoc + ((size_t)2 << 21);
    unsigned gq = g0 + 6u, gn;
    unsigned rgR = SB(0, 1), rgN = SB(1, 1), rgW = L_X, mW = ldsw + L_X - 256;
    unsigned vr0 = boff[0] + rgR, vr1 = boff[1] + rgR;
    unsigned evb = (unsigned)(0 - sslot) < 4u ? lut >> (8 * (0 - sslot)) : 0u;
    unsigned fv = 0xffffffffu;
    const unsigned vzero = 0, fadr = ldsf + lane * 4, padr = ldsw + L_SEND + lane * 16;
    int t = 0;
    if (wr == 1) BAR;
    STAMP(1);

#define DIAG_SLOW
#define SPIN_UNTIL(v, ptr, cond) do { const GAS unsigned* fp = (const GAS unsigned*)(ptr); const unsigned long long t0 = __builtin_amdgcn_s_memrealtime(); \
            while (v = __hip_atomic_load(fp, __ATOMIC_RELAXED, __HIP_MEMORY_SCOPE_SYSTEM), !__all(cond)) { \
                if (__builtin_amdgcn_readfirstlane(__hip_atomic_load(tmo, __ATOMIC_RELAXED, __HIP_MEMORY_SCOPE_AGENT)) == seq || __builtin_amdgcn_s_memrealtime() - t0 > DEADLINE_TICKS) { \
                    __hip_atomic_store(tmo, seq, __ATOMIC_RELAXED, __HIP_MEMORY_SCOPE_AGENT); break; } \
                __builtin_amdgcn_s_sleep(2); } \
            asm volatile("s_waitcnt lgkmcnt(0)" ::: "memory"); DIAG_SLOW; } while (0)
#define SPIN_FLAGS(ptr) SPIN_UNTIL(fv, ptr, fv >= seq)
#define SPLIT() do { unsigned rv = pv.z; \
        if (UNLIKELY(!__all((rv >> 24) == (seq & 255u)))) SPIN_UNTIL(rv, sSrc + vsend + 8, (rv >> 24) == (seq & 255u)); \
        const unsigned a0 = RL(rv, 0), b0 = RL(rv, 1), a1 = RL(rv, 2), b1 = RL(rv, 3); \
        const int d2 = (int)((a1 - a0) << 8) >> 8, d6 = (int)((b1 - b0) << 8) >> 8, dur = (int)(((b0 - a0) & 0xffffffu) + ((b1 - a1) & 0xffffffu)); \
        const int t2 = (int)RL(vev, 2 * 8 + 6) + 3, t6 = (int)RL(vev, 6 * 8 + 6) + 3, num = 2 * (d6 * (t6 - t2) + (d6 - d2) * (64 - t6)); \
        int s = 0; _Pragma("unroll") for (int z = 1; z <= BAL_S; ++z) s += (num > (2 * z - 1) * dur) - (-num > (2 * z - 1) * dur); \
        const unsigned gl = RL(vq, 0), ln = (fadr - ldsf) >> 2; \
        s = (gl & 1u) ? -s : s; \
        const int left = NF + s - (int)ln; \
        const unsigned gt = RL(vq, NF + (int)(((bid >> 6) << 1) | (gl & 1u))); \
        unsigned nv = ((gl + 124u - 16u * (unsigned)(left > 0 ? left : 0)) & 127u) ^ (left > 0 ? 1u : 128u); PIN_S("+v"(nv)); \
        vq = (int)ln >= NF + (s < 0 ? s : 0) ? nv : vq; \
        GLDS_N(ldsw, L_SEND, ((unsigned)wid * 16u + (ln >> 2)) * (NN * 2) + (ln & 3u) * 128u, wT + ((size_t)(gt & 0x7eu) << 20)); } while (0)
#define WAIT2(bit, n) asm volatile("s_bitcmp1_b32 %0, %1\n\ts_cbranch_scc1 1f\n\ts_waitcnt vmcnt(%2)\n1:\n\ts_waitcnt vmcnt(%3) lgkmcnt(0)" :: "s"(evb), "n"(bit), "n"(n), "n"((n) + 1) : "scc", "memory")
#define CHECK_HOOK(fo) HOOK { \
            AT(0) asm volatile("ds_read_b32 %0, %1 offset:%2" : "=v"(fv) : "v"(fadr), "i"(fo) : "memory"); \
            AT(12) { asm volatile("s_waitcnt lgkmcnt(0)" : "+v"(fv) :: "memory"); \
                if (UNLIKELY(t >= 6 && !__all(fv >= seq))) SPIN_FLAGS(fRem + ((RL(vq, t + 2) & 0x7eu) << 8) + vfl); } }
#define K_TILE(b) do { \
          \
        if (UNLIKELY(evb & 1u)) { PIN_S("+s"(evb)); \
            if (!(evb & 4u)) { SETM0(ldsw, L_SEND); \
                if (sidx < 7) asm volatile("s_nop 0\n\tglobal_load_lds_dwordx4 %0, %1" :: "v"(vsend), "s"(sSrc) : "memory"); \
                else asm volatile("s_nop 0\n\tglobal_load_lds_dwordx4 %0, %1 sc0 sc1" :: "v"(vsend), "s"(sSrc) : "memory"); } \
            else { const unsigned val = wid ? (seq << 24) | ((unsigned)__builtin_amdgcn_s_memrealtime() & 0xffffffu) : seq; \
                asm volatile("global_store_dword %0, %1, %2 sc0 sc1" :: "v"(vzero), "v"(val), "s"(sFlag) : "memory"); } } \
        SETM0(ldsw, SA((b) ^ 1, 1)); LDB(B0, boffS[0], boffS[1], (b) * 2 * HTB); SCHED; GLDS(voffP[0], pA1); \
        SETM0(ldsw, SA((b) ^ 1, 1) + 8192); SCHED; LDAH(aoff, SA(b, 0), 0); SCHED; GLDS(voffP[1], pA1); \
        SETM0(mW, 0); SCHED; LDAH(aoff, SA(b, 0), 2); SCHED; GLDSH(voffB[0], pB2); \
        SETM0(mW, 8192); LDB(B1, vr0, vr1, 0); SCHED; GLDSH(voffB[1], pB2); \
        WAIT2(1, 11); \
        MMA32(0, CHECK_HOOK(((b) ^ 1) * 2048)); \
          \
        PIN_S("+s"(evb)); \
        if (UNLIKELY(evb & 4u)) { ++sidx; EV_SETUP(); \
            if (sidx == 7) { lut = 0x00121813u; const unsigned ln = (fadr - ldsf) >> 2; vsend = (ln & 2u) * 64u + (ln & 1u) * 16u; } } \
        SETM0(ldsw, SB(b, 0)); SCHED; LDAH(aoff, SA(b, 1), 0); SCHED; GLDS(voffB[0], pB2); \
        SETM0(ldsw, SB(b, 0) + 8192); SCHED; LDAH(aoff, SA(b, 1), 2); SCHED; GLDS(voffB[1], pB2); \
        GLDS_N(ldsw, SA(b, 0), voffA[0], pA2); GLDS_N(ldsw, SA(b, 0) + 8192, voffA[1], pA2); \
        PIN_S("+s"(evb)); \
        if (UNLIKELY(evb & 8u)) { u32x4 pv; asm volatile("ds_read_b128 %0, %1\n\ts_waitcnt lgkmcnt(0)" : "=v"(pv) : "v"(padr) : "memory"); \
            if (sidx < 7) asm volatile("global_store_dwordx4 %0, %1, %2 sc0 sc1\n\ts_nop 1" :: "v"(vsend), "v"(pv), "s"(sDst) : "memory"); else SPLIT(); } \
        WAIT2(4, 9); \
        MMA32(1, HOOK { \
            AT(0) { gn = RL(vq, t + 4); pA1 = pA2; PIN_S("+s"(gn), "+s"(pA1)); } \
            AT(1) { pB2 = wT + ((size_t)gq << 20); PIN_S("+s"(pB2)); } \
            AT(2) { o1 = (gq & 0x70u) << 17; PIN_S("+s"(o1)); } \
            AT(3) { o1 |= (gq & 15u) << 7; PIN_S("+s"(o1)); } \
            AT(4) { pA2 = t < 5 ? aLocT : aRemT; PIN_S("+s"(pA2)); } \
            AT(5) { pA2 += o1; gq = gn; PIN_S("+s"(pA2), "+s"(gq)); } \
            AT(6) { o1 = (gn & 0x7eu) << 8; PIN_S("+s"(o1)); } \
            AT(7) { pF = fRem + o1; PIN_S("+s"(pF)); } \
            AT(8) { SETM0(ldsf, ((b) ^ 1) * 2048); } \
            AT(9) { GFLAG(vfl, pF); } \
            AT(10) { const unsigned r = rgR; rgR = rgN; rgN = rgW; rgW = r; PIN_S("+s"(rgR), "+s"(rgN), "+s"(rgW)); } \
            AT(11) { mW = ldsw + rgW - 256u; PIN_S("+s"(mW)); } \
            AT(12) { vr0 = boff[0] + rgR; vr1 = boff[1] + rgR; PIN_S("+v"(vr0), "+v"(vr1)); } \
            AT(13) { ++t; o1 = (unsigned)(t - sslot); PIN_S("+s"(t), "+s"(o1)); } \
            AT(14) { evb = lut >> (o1 * 8u); PIN_S("+s"(evb)); } \
            AT(15) { evb = o1 < 4u ? evb & 31u : 0u; PIN_S("+s"(evb)); } \
            if constexpr (b) { AT(17) { o1 = gn & 128u; PIN_S("+s"(o1)); } AT(18) { evb |= o1; PIN_S("+s"(evb)); } } }); \
    } while (0)

    unsigned o1; const char* pF;
    unsigned pb = 0;
    do {
        K_TILE(0);
        K_TILE(1);
    } while (!(evb & 128u));
    if (!(RL(vq, t + 2) & 128u)) { K_TILE(0); pb = 2 * HTB; }

    const unsigned qb = 2 * HTB - pb, fo = pb >> 4;
    const unsigned r16 = (unsigned)rank * 16u;
#define TAIL_G(u) ((r16 + (unsigned)(e_of(NF + (u) / 2) * 2 + ((u) & 1))) & 127u)
#define TAIL_A(u) (aRemT + OFF_A(TAIL_G(u)))
#define TAIL_B(u) (wT + ((size_t)TAIL_G(u) << 20))
    const unsigned sB0[3] = {4 * HTB + pb, 4 * HTB + qb, rgW}, sA[3] = {pb, qb, pb + HTB}, sB1[3] = {rgR, rgN, qb + HTB};
    unsigned mB0[3], mA[3], mB1[3], vA[3], vB0[3][2], vB1[3][2];
    unsigned tfo;
    { const int u = NF + (lane >> 3) - 8, a = u / 7, j = u - a * 7;
      tfo = ((((unsigned)rank * 8u + (unsigned)((j + 1) * 8 + a)) & 63u) << 9) + vfl; }
    const char *tA, *tB, *tA1, *tB1, *tB2;
    {
        const unsigned fa0 = fadr + fo, fa1 = fadr + (2048 - fo);
        vA[0] = aoff + sA[0]; vB0[0][0] = boff[0] + sB0[0]; vB0[0][1] = boff[1] + sB0[0]; mB1[2] = ldsw + sB1[2];
        SETM0(mB1[2], 0); LDB(B0, vB0[0][0], vB0[0][1], 0); SCHED; GLDS(voffP[0], pA1);
        SETM0(mB1[2], 8192); SCHED; LDAH(vA[0], 0, 0); SCHED; GLDS(voffP[1], pA1);
        LDAH(vA[0], 0, 2); SCHED; LDB(B1, vr0, vr1, 0);
        wait_vm_l0<9>();
        MMA32(0, HOOK {
            AT(0) asm volatile("ds_read_b32 %0, %1" : "=v"(fv) : "v"(fa1) : "memory");
            AT(1) { o1 = ldsf + fo; PIN_S("+s"(o1)); SETM0(o1, 0); }
            AT(2) GFLAG(tfo, fRem);
            AT(3) { tA = TAIL_A(0); PIN_S("+s"(tA)); }
            AT(6) { tB = TAIL_B(0); PIN_S("+s"(tB)); }
            AT(8) { mB0[0] = ldsw + sB0[0]; mA[0] = ldsw + sA[0]; mB1[0] = ldsw + sB1[0]; PIN_S("+s"(mB0[0]), "+s"(mA[0]), "+s"(mB1[0])); }
            AT(12) { asm volatile("s_waitcnt lgkmcnt(0)" : "+v"(fv) :: "memory");
                if (UNLIKELY(!__all(fv >= seq))) SPIN_FLAGS(fRem + ((TAIL_G(0) & ~1u) << 8) + vfl); } });
        SETM0(mB0[0], 0); SCHED; LDAH(vA[0], HTB, 0); SCHED; GLDS(voffB[0], tB);
        SETM0(mB0[0], 8192); SCHED; LDAH(vA[0], HTB, 2); SCHED; GLDS(voffB[1], tB);
        STAGEH(mB1[0], 0, voffB, tB); STAGE(mA[0], 0, voffA, tA);
        wait_vm_l0<10>();
        MMA32(1, HOOK {
            AT(0) { vA[1] = aoff + sA[1]; vB0[1][0] = boff[0] + sB0[1]; PIN_S("+v"(vA[1]), "+v"(vB0[1][0])); }
            AT(1) { vB0[1][1] = boff[1] + sB0[1]; vB1[1][0] = boff[0] + sB1[1]; PIN_S("+v"(vB0[1][1]), "+v"(vB1[1][0])); }
            AT(2) { vB1[1][1] = boff[1] + sB1[1]; mB0[2] = ldsw + sB0[2]; PIN_S("+v"(vB1[1][1]), "+s"(mB0[2])); }
            AT(3) { tB2 = TAIL_B(2); PIN_S("+s"(tB2)); } });
        SETM0(mB0[2], 0); LDB(B0, vB0[1][0], vB0[1][1], 0); SCHED; GLDS(voffB[0], tB2);
        SETM0(mB0[2], 8192); SCHED; LDAH(vA[1], 0, 0); SCHED; GLDS(voffB[1], tB2);
        LDAH(vA[1], 0, 2); SCHED; LDB(B1, vB1[1][0], vB1[1][1], 0);
        wait_vm_l0<8>();
        MMA32(0, HOOK {
            AT(0) asm volatile("ds_read_b32 %0, %1" : "=v"(fv) : "v"(fa0) : "memory");
            AT(1) { tA1 = tA + 128; PIN_S("+s"(tA1)); }
            AT(2) { tB1 = tB + ((size_t)1 << 20); PIN_S("+s"(tB1)); }
            AT(3) { mB0[1] = ldsw + sB0[1]; mA[1] = ldsw + sA[1]; mB1[1] = ldsw + sB1[1]; PIN_S("+s"(mB0[1]), "+s"(mA[1]), "+s"(mB1[1])); }
            AT(12) { asm volatile("s_waitcnt lgkmcnt(0)" : "+v"(fv) :: "memory");
                if (UNLIKELY(!__all(fv >= seq))) SPIN_FLAGS(fRem + tfo); } });
        SETM0(mB0[1], 0); SCHED; LDAH(vA[1], HTB, 0); SCHED; GLDS(voffB[0], tB1);
        SETM0(mB0[1], 8192); SCHED; LDAH(vA[1], HTB, 2); SCHED; GLDS(voffB[1], tB1);
        STAGE(mA[1], 0, voffA, tA1); STAGEH(mB1[1], 0, voffB, tB1);
        wait_vm_l0<8>();
        MMA32(1, HOOK {
            AT(0) { tA = TAIL_A(2); PIN_S("+s"(tA)); }
            AT(4) { tB = tB2; mA[2] = ldsw + sA[2]; PIN_S("+s"(tB), "+s"(mA[2])); } });
        vB1[0][0] = vr0; vB1[0][1] = vr1;
    }
    STAMP(2);
    const char* const slabOut = myws + W_SLAB + (size_t)bid * SLAB_B + (size_t)wid * 8192;
    const char* const slabIn = myws + W_SLAB + (size_t)partner * SLAB_B + (size_t)wid * 8192;
    const char* const pflag = myws + W_EFLAG + (size_t)(partner * 8 + wid) * 128;
    const unsigned vl16 = (unsigned)lane * 16u, vl16b = vl16 + 4096u;
    u32x4 in[8], w;
    bool early = false;
    static_for([&](auto uc) {
        constexpr int u = decltype(uc)::value, c = u % 3, n = (u + 2) % 3;
        constexpr bool st = u + 2 < SH;
        if constexpr (st && u > 0) { SETM0(mB0[n], 0); }
        LDB(B0, vB0[c][0], vB0[c][1], 0); SCHED;
        if constexpr (st && u > 0) { GLDS(voffB[0], tB); SETM0(mB0[n], 8192); SCHED; }
        LDAH(vA[c], 0, 0); SCHED;
        if constexpr (st && u > 0) { GLDS(voffB[1], tB); }
        if constexpr (st) { SETM0(mA[n], 0); SCHED; }
        LDAH(vA[c], 0, 2); SCHED;
        if constexpr (st) { GLDS(voffA[0], tA); SETM0(mA[n], 8192); }
        LDB(B1, vB1[c][0], vB1[c][1], 0); SCHED;
        if constexpr (st) { GLDS(voffA[1], tA); STAGEH(mB1[n], 0, voffB, tB); }
        if constexpr (u + 1 < SH) wait_vm_l0<tail_wait(u)>(); else wait_vm_l0<63>();
        MMA32(0, HOOK {
            if constexpr (u < 8 / XS_ST) {
                constexpr int j = u * XS_ST + i / (32 / XS_ST), bj = j >> 2, m = j & 3, z = i % (32 / XS_ST);
                if constexpr (z == 0) { w.x = cvt_pk_bf16(acc[1][bj][m][0][0], acc[1][bj][m][0][1]); w.y = cvt_pk_bf16(acc[1][bj][m][0][2], acc[1][bj][m][0][3]); }
                if constexpr (z == 1) { w.z = cvt_pk_bf16(acc[1][bj][m][1][0], acc[1][bj][m][1][1]); w.w = cvt_pk_bf16(acc[1][bj][m][1][2], acc[1][bj][m][1][3]); }
                if constexpr (z == 2) asm volatile("global_store_dwordx4 %0, %1, %2 offset:%3 sc1\n\ts_nop 1" :: "v"(bj ? vl16b : vl16), "v"(w), "s"(slabOut), "i"(m * 1024) : "memory");
            }
            if constexpr (u == 1) {
                AT(8) { vA[2] = aoff + sA[2]; vB0[2][0] = boff[0] + sB0[2]; PIN_S("+v"(vA[2]), "+v"(vB0[2][0])); }
                AT(9) { vB0[2][1] = boff[1] + sB0[2]; vB1[2][0] = boff[0] + sB1[2]; PIN_S("+v"(vB0[2][1]), "+v"(vB1[2][0])); }
                AT(10) { vB1[2][1] = boff[1] + sB1[2]; PIN_S("+v"(vB1[2][1])); }
            }
            if constexpr (u + 3 < SH) {
                AT(3) { if constexpr (u & 1) tA = TAIL_A(u + 3); else tA += 128; PIN_S("+s"(tA)); }
                AT(7) { if constexpr (u & 1) tB = TAIL_B(u + 3); else tB += (size_t)1 << 20; PIN_S("+s"(tB)); }
            }
            if constexpr (u == S_FLAG) { AT(0) asm volatile("global_store_dword %0, %1, %2 sc1" :: "v"(vzero), "v"(seq), "s"(myws + W_EFLAG + (size_t)(bid * 8 + wid) * 128) : "memory"); }
            if constexpr (u == S_SAMP) { AT(0) SETM0(ldsf, 0); AT(1) GFLAG(vzero, pflag); }
            if constexpr (u == S_LOAD) {
                AT(0) asm volatile("ds_read_b32 %0, %1" : "=v"(fv) : "v"(fadr) : "memory");
                AT(12) { asm volatile("s_waitcnt lgkmcnt(0)" : "+v"(fv) :: "memory"); early = (unsigned)__builtin_amdgcn_readfirstlane(fv) >= seq; asm volatile("" ::: "memory"); }
            }
            if constexpr (u >= S_LOAD && u < S_LOAD + 8 / XS_LD && i >= 16 && (i - 16) % (16 / XS_LD) == 0) {
                constexpr int j = ((u - S_LOAD) * XS_LD + (i - 16) / (16 / XS_LD)) & 7;
                asm volatile("global_load_dwordx4 %0, %1, %2 offset:%3 sc1" : "=v"(in[j]) : "v"((j & 4) ? vl16b : vl16), "s"(slabIn), "i"((j & 3) * 1024) : "memory");
            } });
    }, std::make_integer_sequence<int, SH>{});
    if (wr == 0) BAR;
    STAMP(3);

    {
        asm volatile("s_waitcnt vmcnt(0)" : "+v"(in[0]), "+v"(in[1]), "+v"(in[2]), "+v"(in[3]), "+v"(in[4]), "+v"(in[5]), "+v"(in[6]), "+v"(in[7]) :: "memory");
        if (UNLIKELY(!early)) {
            const GAS unsigned* fp = (const GAS unsigned*)pflag;
            const unsigned long long t0 = __builtin_amdgcn_s_memrealtime();
            while ((unsigned)__builtin_amdgcn_readfirstlane(__hip_atomic_load(fp, __ATOMIC_RELAXED, __HIP_MEMORY_SCOPE_AGENT)) < seq) {
                if (__builtin_amdgcn_readfirstlane(__hip_atomic_load(tmo, __ATOMIC_RELAXED, __HIP_MEMORY_SCOPE_AGENT)) == seq || __builtin_amdgcn_s_memrealtime() - t0 > DEADLINE_TICKS) {
                    __hip_atomic_store(tmo, seq, __ATOMIC_RELAXED, __HIP_MEMORY_SCOPE_AGENT); break; }
                __builtin_amdgcn_s_sleep(1);
            }
            asm volatile("" ::: "memory");
#pragma unroll
            for (int i = 0; i < 8; ++i) asm volatile("global_load_dwordx4 %0, %1, %2 offset:%3 sc1" : "=v"(in[i]) : "v"((i >> 2) ? vl16b : vl16), "s"(slabIn), "i"((i & 3) * 1024) : "memory");
            asm volatile("s_waitcnt vmcnt(0)" : "+v"(in[0]), "+v"(in[1]), "+v"(in[2]), "+v"(in[3]), "+v"(in[4]), "+v"(in[5]), "+v"(in[6]), "+v"(in[7]) :: "memory");
            DIAG_SLOW;
        }
        const bool bad = __builtin_amdgcn_readfirstlane(__hip_atomic_load(tmo, __ATOMIC_RELAXED, __HIP_MEMORY_SCOPE_AGENT)) == seq;
        STAMP(4);
        unsigned short* orow = out + (size_t)(pm * 256 + h * 128 + wr * 64 + fr) * NN + pn * 256 + wc * 32 + (fq & 1) * 16 + (fq >> 1) * 8;
#pragma unroll
        for (int bj = 0; bj < 2; ++bj)
#pragma unroll
            for (int m = 0; m < 4; ++m) {
                f32x4 v0 = acc[0][bj][m][0], v1 = acc[0][bj][m][1];
                const u32x4 p = in[bj * 4 + m];
                v0[0] += __uint_as_float(p.x << 16); v0[1] += __uint_as_float(p.x & 0xffff0000u); v0[2] += __uint_as_float(p.y << 16); v0[3] += __uint_as_float(p.y & 0xffff0000u);
                v1[0] += __uint_as_float(p.z << 16); v1[1] += __uint_as_float(p.z & 0xffff0000u); v1[2] += __uint_as_float(p.w << 16); v1[3] += __uint_as_float(p.w & 0xffff0000u);
                unsigned a0 = cvt_pk_bf16(gelu_tanh(v0[0]), gelu_tanh(v0[1])), a1 = cvt_pk_bf16(gelu_tanh(v0[2]), gelu_tanh(v0[3]));
                unsigned c0 = cvt_pk_bf16(gelu_tanh(v1[0]), gelu_tanh(v1[1])), c1 = cvt_pk_bf16(gelu_tanh(v1[2]), gelu_tanh(v1[3]));
                { auto r = __builtin_amdgcn_permlane16_swap(a0, c0, false, false); a0 = r[0]; c0 = r[1]; }
                { auto r = __builtin_amdgcn_permlane16_swap(a1, c1, false, false); a1 = r[0]; c1 = r[1]; }
                u32x4 o = {a0, a1, c0, c1};
                if (bad) o = (u32x4){0x7fc07fc0u, 0x7fc07fc0u, 0x7fc07fc0u, 0x7fc07fc0u};
                *(u32x4*)(orow + (size_t)m * 16 * NN + bj * 128) = o;
            }
    }
}

extern "C" void kernel_launch(int rank, int nranks, ncclComm_t comm,
                              hipStream_t stream,
                              void* const* d_in, const int* in_sizes, int n_in,
                              void* d_out, int out_size,
                              const int* in_dtypes, int out_dtype,
                              void* const* d_out_all,
                              void* const* d_ws_all, size_t ws_bytes) {
    (void)comm; (void)in_sizes; (void)n_in; (void)out_size; (void)in_dtypes; (void)out_dtype; (void)d_out_all;
    if (nranks != NR || (unsigned)rank >= (unsigned)NR || ws_bytes < WS_NEED) { fprintf(stderr, "a2a_gemm: unsupported configuration\n"); return; }
    static bool attr_set = false;
    if (!attr_set) {
        HIP_CHECK(hipFuncSetAttribute((const void*)a2a_gemm_gelu_kernel, hipFuncAttributeMaxDynamicSharedMemorySize, LDS_BYTES));
        attr_set = true;
    }
    hipLaunchKernelGGL(a2a_gemm_gelu_kernel, dim3(NWG), dim3(512), LDS_BYTES, stream, rank,
                       (const unsigned short*)d_in[0], (const unsigned short*)d_in[1], (unsigned short*)d_out, d_ws_all);
    HIP_CHECK(hipGetLastError());
}
```

```cpp
#include <hip/hip_runtime.h>
#include <rccl/rccl.h>
#include <cstdio>
#include <cstdint>
#include <utility>

#define HIP_CHECK(expr) do { hipError_t _e = (expr); if (_e != hipSuccess) { \
    fprintf(stderr, "HIP error %s at %s:%d\n", hipGetErrorString(_e), __FILE__, __LINE__); \
    return; } } while (0)

#define LAS __attribute__((address_space(3)))
#define GAS __attribute__((address_space(1)))
typedef short bf16x8 __attribute__((ext_vector_type(8)));
typedef short s16x4 __attribute__((ext_vector_type(4)));
typedef float f32x4 __attribute__((ext_vector_type(4)));
typedef unsigned u32x4 __attribute__((ext_vector_type(4)));

#define PACE_NUM 9
#define PACE_DEN 448
#define BAL_S 6
#define BAL_LAG 4
#define XS_SAMP 10
#define XS_ST 4
#define G_WAIT 59
#define G_BAR 61
#define XS_LD 4

namespace {
constexpr int NR = 8;
constexpr int MS = 1024;
constexpr int KB = 1024;
constexpr int NN = 8192;
constexpr int SH = 16;
constexpr int NF = 64 - SH / 2;
constexpr int NWG = 256;
constexpr int HTB = 16384;
constexpr int slot_of(int q) { return (q * PACE_NUM) / PACE_DEN - 1; }
static_assert(BAL_S >= 0 && BAL_S <= 6 && slot_of(1791) + 2 + BAL_LAG + 3 < NF - 6 - 4, "all events must finish inside the generic K-loop");
static_assert((256 * PACE_NUM) / PACE_DEN >= 4, "one piece at a time");
constexpr int L_X = 8 * HTB;
constexpr int L_SEND = 9 * HTB;
constexpr int L_FLAG = L_SEND + 8192;
constexpr int LDS_BYTES = L_FLAG + 2048;
constexpr size_t W_CTRL = (size_t)128 << 20;
constexpr size_t W_TMO = W_CTRL + 256;
constexpr size_t W_XFLAG = W_CTRL + 4096;
constexpr size_t W_EFLAG = W_CTRL + 65536;
constexpr size_t W_SEQ = W_EFLAG + 262144;
constexpr size_t W_REC = W_SEQ + 32768;
constexpr size_t W_DIAG = W_CTRL + 524288;
constexpr size_t RECV_BLK = (size_t)MS * KB * 2;
constexpr size_t W_RECV0 = (size_t)80 << 20;
constexpr size_t W_RECV1 = (size_t)96 << 20;
constexpr size_t W_SLAB = (size_t)112 << 20;
constexpr size_t SLAB_B = 65536;
constexpr size_t WS_NEED = (size_t)129 << 20;
constexpr unsigned long long DEADLINE_TICKS = 40000000ull;

constexpr int S_FLAG = 16 / XS_ST + 1;
constexpr int S_SAMP = XS_SAMP;
constexpr int S_LOAD = S_SAMP + 2;
static_assert(S_SAMP > S_FLAG && S_LOAD + 16 / XS_LD <= SH, "tail too short for the hand-over schedule");
constexpr int n_stage(int u) { return (u + 3 < SH ? 8 : 0) + (u + 2 < SH ? 4 : 0); }
constexpr int n_extra(int u) { return (u < 16 / XS_ST ? XS_ST : 0) + (u >= S_LOAD && u < S_LOAD + 16 / XS_LD ? XS_LD : 0) + (u == S_FLAG) + (u == S_SAMP); }
constexpr int tail_wait(int u) { return n_stage(u) + n_extra(u); }
constexpr int e_of(int T) { return T < 8 ? T : ((T - 8) % 7 + 1) * 8 + (T - 8) / 7; }
static_assert(e_of(NF) == 62 && e_of(NF - 6) == 14, "take-over targets");

__device__ __forceinline__ int lds_byteA(int r, int c) { const int st = (r >> 4) * 2 + (c >> 5), rr = r & 15, cc = c & 31, ob = rr * 64 + cc * 2; return st * 1024 + (ob ^ (((ob >> 9) & 1) << 5)); }
__device__ __forceinline__ void stage_rcA(int b, int& R, int& C) { const int st = b / 1024, sb = b % 1024, swz = sb ^ (((sb >> 9) & 1) << 5); R = (st >> 1) * 16 + swz / 64; C = (st & 1) * 32 + (swz % 64) / 2; }
__device__ __forceinline__ unsigned cvt_pk_bf16(float lo, float hi) { unsigned r; asm volatile("v_cvt_pk_bf16_f32 %0, %1, %2" : "=v"(r) : "v"(lo), "v"(hi)); return r; }
__device__ __forceinline__ float gelu_tanh(float y) {
    const float u = y * y * 0.044715f + 1.0f;
    const float e = __builtin_amdgcn_exp2f(y * u * (-2.0f * 0.7978845608028654f * 1.4426950408889634f));
    return y * __builtin_amdgcn_rcpf(1.0f + e);
}
template <int N> __device__ __forceinline__ void wait_vm_l0() { if constexpr (N < 63) asm volatile("s_waitcnt vmcnt(%0) lgkmcnt(0)" :: "n"(N) : "memory"); else asm volatile("s_waitcnt lgkmcnt(0)" ::: "memory"); }
template <class F, int... I> __device__ __forceinline__ void static_for(F&& f, std::integer_sequence<int, I...>) { (f(std::integral_constant<int, I>{}), ...); }
__device__ __forceinline__ const char* mk_ptr(unsigned lo, unsigned hi) { return (const char*)(((unsigned long long)hi << 32) | lo); }
}

__global__ __launch_bounds__(256, 1)
void a2a_gemm_gelu_kernel(int rank, const unsigned short* __restrict__ xloc, const unsigned short* __restrict__ wmat,
                          unsigned short* __restrict__ out, void* const* __restrict__ ws_all) {
    extern __shared__ __attribute__((aligned(16))) unsigned char shm[];
    LAS unsigned char* lds = (LAS unsigned char*)shm;
    const int tid = threadIdx.x, wid = __builtin_amdgcn_readfirstlane(tid >> 6), lane = tid & 63, wr = wid >> 1, wc = wid & 1, fr = lane & 15, fq = lane >> 4;
    const int bid = blockIdx.x, xcd = bid & 7, idx = bid >> 3;
    const int h = xcd & 1, pn = (xcd >> 1) * 8 + (idx & 7), pm = idx >> 3;
    const int partner = bid ^ 1, bpair = bid & ~1;
    char* const myws = (char*)ws_all[rank];
#define STAMP(i)
    STAMP(0);
    unsigned seqv;
    asm volatile("global_load_dword %0, %1, off sc1" : "=v"(seqv) : "v"(myws + W_SEQ + (size_t)bid * 128) : "memory");

    constexpr unsigned HSTEP_A = 128u * KB * 2;
    unsigned voffA[4], voffP[4], voffB[4];
#pragma unroll
    for (int i = 0; i < 4; ++i) {
        int R, C; stage_rcA(tid * 16 + i * 4096, R, C);
        const unsigned o = (unsigned)(R * KB + C) * 2u;
        voffA[i] = o + (unsigned)h * HSTEP_A; voffP[i] = o + (unsigned)(h ^ 1) * HSTEP_A;
        const int b = tid * 16 + i * 4096, k = b >> 8, cp = (b >> 4) & 15, f = (k & 3) | (((k >> 3) & 1) << 2), c = cp ^ (f << 1);
        voffB[i] = (unsigned)k * (NN * 2) + (unsigned)c * 16u;
    }
    const unsigned lds0 = (unsigned)(uintptr_t)lds;
    const unsigned ldsw = lds0 + (unsigned)wid * 1024u, ldsf = lds0 + L_FLAG + (unsigned)wid * 256u, ldss = lds0 + L_SEND + (unsigned)wid * 2048u;
    unsigned va = lds0 + lds_byteA(wr * 64 + fr, fq * 8);
    unsigned vb[8];
    { const int g = lane >> 4, q = (lane & 15) >> 2, p = lane & 3, f = q | ((g & 1) << 2);
#pragma unroll
      for (int j = 0; j < 8; ++j) { vb[j] = lds0 + (4 + wc) * HTB + (8 * g + q) * 256 + ((j ^ f) << 5) + p * 8; asm volatile("" : "+v"(vb[j])); } }
    asm volatile("" : "+v"(va));

#define SA(b, hh) (((b) * 2 + (hh)) * HTB)
#define SB(b, hh) ((4 + (b) * 2 + (hh)) * HTB)
#define SETM0(sreg, imm) asm volatile("s_add_u32 m0, %0, %1" :: "s"(sreg), "i"(imm) : "scc")
#define GLDS(voff, sbase) asm volatile("global_load_lds_dwordx4 %0, %1" :: "v"(voff), "s"(sbase) : "memory")
#define GLDSH(voff, sbase) asm volatile("global_load_lds_dwordx4 %0, %1 offset:256" :: "v"(voff), "s"(sbase) : "memory")
#define GLDS_N(sreg, imm, voff, sbase) asm volatile("s_add_u32 m0, %2, %3\n\ts_nop 0\n\tglobal_load_lds_dwordx4 %0, %1" :: "v"(voff), "s"(sbase), "s"(sreg), "i"(imm) : "scc", "memory")
#define GLDSH_N(sreg, imm, voff, sbase) asm volatile("s_add_u32 m0, %2, %3\n\ts_nop 0\n\tglobal_load_lds_dwordx4 %0, %1 offset:256" :: "v"(voff), "s"(sbase), "s"(sreg), "i"((imm) - 256) : "scc", "memory")
#define GFLAG(voff, sbase) asm volatile("global_load_lds_dword %0, %1 sc0 sc1" :: "v"(voff), "s"(sbase) : "memory")
#define STAGE_A(slot, voff, sbase) do { _Pragma("unroll") for (int _i = 0; _i < 4; ++_i) GLDS_N(ldsw, (slot) + _i * 4096, (voff)[_i], sbase); } while (0)
#define STAGE_B(b, kk, sbase) do { _Pragma("unroll") for (int _i = 2 * (kk); _i < 2 * (kk) + 2; ++_i) { GLDS_N(ldsw, SB(b, 0) + _i * 4096, voffB[_i], sbase); GLDSH_N(ldsw, SB(b, 1) + _i * 4096, voffB[_i], sbase); } } while (0)
#define LDA1(AR, z, addr, off) asm volatile("ds_read_b128 %0, %1 offset:%2" : "=v"(AR[(z) >> 1][(z) & 1]) : "v"(addr), "i"((off) + (z) * 1024) : "memory")
#define LDB1(j, kk, off) do { s16x4 _r0, _r1; \
        asm volatile("ds_read_b64_tr_b16 %0, %2 offset:%3\n\tds_read_b64_tr_b16 %1, %2 offset:%4" : "=&v"(_r0), "=&v"(_r1) : "v"(vb[j]), "i"((off) + (kk) * 8192), "i"((off) + (kk) * 8192 + 1024) : "memory"); \
        Bf[j][kk] = __builtin_shufflevector(_r0, _r1, 0, 1, 2, 3, 4, 5, 6, 7); } while (0)
#define BAR __builtin_amdgcn_s_barrier()
#define SCHED __builtin_amdgcn_sched_barrier(0)
#define UNLIKELY(x) __builtin_expect(!!(x), 0)
#define PIN(...) asm volatile("" : __VA_ARGS__)
#define MM1(ai, AR, I, ...) { constexpr int i = (I); \
        asm volatile("v_mfma_f32_16x16x32_bf16 %0, %1, %2, %0" : "+a"(acc[ai][(i >> 3) & 3][i & 7]) : "v"(Bf[i & 7][i >> 5]), "v"(AR[(i >> 3) & 3][i >> 5])); \
        SCHED; { __VA_ARGS__ } SCHED; }
#define MM4(ai, AR, I, ...) MM1(ai, AR, I, __VA_ARGS__) MM1(ai, AR, (I) + 1, __VA_ARGS__) MM1(ai, AR, (I) + 2, __VA_ARGS__) MM1(ai, AR, (I) + 3, __VA_ARGS__)
#define MM16(ai, AR, I, ...) MM4(ai, AR, I, __VA_ARGS__) MM4(ai, AR, (I) + 4, __VA_ARGS__) MM4(ai, AR, (I) + 8, __VA_ARGS__) MM4(ai, AR, (I) + 12, __VA_ARGS__)
#define MMA64(ai, AR, ...) do { SCHED; MM16(ai, AR, 0, __VA_ARGS__) MM16(ai, AR, 16, __VA_ARGS__) MM16(ai, AR, 32, __VA_ARGS__) MM16(ai, AR, 48, __VA_ARGS__) } while (0)
#define AT(n) if constexpr (i == (n))
#define EVERY(i0, step, cnt) if constexpr (i >= (i0) && i < (i0) + (step) * (cnt) && (i - (i0)) % (step) == 0)
#define ZOF(i0, step) (((i - (i0)) / (step)) & 15)
#define WAIT_B1(n) asm volatile("s_waitcnt lgkmcnt(%8)" : "+v"(Bf[0][1]), "+v"(Bf[1][1]), "+v"(Bf[2][1]), "+v"(Bf[3][1]), "+v"(Bf[4][1]), "+v"(Bf[5][1]), "+v"(Bf[6][1]), "+v"(Bf[7][1]) : "n"(n) : "memory")

    f32x4 acc[2][4][8];
#pragma unroll
    for (int a = 0; a < 2; ++a)
#pragma unroll
        for (int m = 0; m < 4; ++m)
#pragma unroll
            for (int n = 0; n < 8; ++n) acc[a][m][n] = (f32x4){0.f, 0.f, 0.f, 0.f};
    bf16x8 Ao[4][2], Ap[4][2], Bf[8][2];

    const size_t rowOff = (size_t)pm * 256 * KB * 2;
    const char* const aLocT = (const char*)xloc + rowOff;
    const char* const wT = (const char*)wmat + (size_t)pn * 512;
    const char* const fRem = myws + W_XFLAG + (size_t)pm * 128;
    const unsigned g0 = (unsigned)rank * 16u + (unsigned)h;
#define OFF_A(g) ((((g) & 0x70u) << 17) | (((g) & 15u) << 7))
    const char* const aLoc = aLocT + OFF_A(g0);
    const char* const bLoc = wT + ((size_t)g0 << 20);

    STAGE_B(0, 0, bLoc); STAGE_A(SA(0, 0), voffA, aLoc); STAGE_B(0, 1, bLoc); STAGE_A(SA(0, 1), voffP, aLoc);
    const int ek = lane >> 3, ej = lane & 7;
    const int eq = (ek < 7 ? ek : 6) * 256 + bid, ea = eq / 224, er = eq - ea * 224, erg = er / 7, ed = (rank + 1 + er - erg * 7) & 7;
    unsigned long long epw;
    asm volatile("global_load_dwordx2 %0, %1, off" : "=v"(epw) : "v"(ws_all + ed) : "memory");
    unsigned vs0 = (unsigned)(wid * 8 + (lane >> 4)) * (KB * 2) + (unsigned)(lane & 15) * 16u, vs1 = vs0 + 4 * KB * 2;
    int sslot = slot_of(bpair);
    if (sslot < 0) {
        const int rg = bid / 7, dj = bid - rg * 7, d = (rank + 1 + dj) & 7;
        const char* const p = (const char*)xloc + (size_t)d * RECV_BLK + (size_t)rg * 32 * (KB * 2);
        GLDS_N(ldss, 0, vs0, p); GLDS_N(ldss, 1024, vs1, p);
    }
    STAGE_B(1, 0, bLoc + ((size_t)1 << 21)); STAGE_A(SA(1, 0), voffA, aLoc + 256); STAGE_B(1, 1, bLoc + ((size_t)1 << 21)); STAGE_A(SA(1, 1), voffP, aLoc + 256);
    const unsigned vfl = (unsigned)(lane & 7) * 4u;
    unsigned vq;
    { const int u = lane - 8, a = u / 7, j = u - a * 7; vq = (((unsigned)(rank * 8 + (lane < 8 ? lane : (j + 1) * 8 + a)) & 63u) << 1) | (unsigned)h | (lane < NF ? 0u : 128u); }
    asm volatile("s_waitcnt vmcnt(16)" : "+v"(seqv), "+v"(epw) :: "memory");
    BAR; asm volatile("" ::: "memory");
#pragma unroll
    for (int j = 0; j < 8; ++j) { LDB1(j, 0, 0); LDA1(Ao, j, va, SA(0, 0)); }
    const unsigned seq = (unsigned)__builtin_amdgcn_readfirstlane(seqv) + 1u;
    if (tid == 0) asm volatile("global_store_dword %0, %1, off sc1" :: "v"(myws + W_SEQ + (size_t)bid * 128), "v"(seq) : "memory");
    const size_t parOff = (seq & 1u) ? W_RECV1 : W_RECV0;
    const char* const aRemT = myws + parOff + rowOff;
    GAS unsigned* const tmo = (GAS unsigned*)(myws + W_TMO);
    const char* const recP = myws + W_REC + (size_t)bid * 128;
    unsigned vev;
    { const size_t o = (size_t)erg * 32 * (KB * 2) + (size_t)ea * 256;
      const unsigned long long src = ek < 7 ? (unsigned long long)xloc + (size_t)ed * RECV_BLK + o : (unsigned long long)myws + W_REC + (size_t)bpair * 128;
      const unsigned long long dst = epw + parOff + (size_t)rank * RECV_BLK + o;
      const unsigned long long flg = epw + W_XFLAG + (size_t)((rank * 8 + ea) * 4 + (erg >> 3)) * 128 + (size_t)(erg & 7) * 4;
      const unsigned long long p = ej < 2 ? src : (ej < 4 ? dst : flg);
      vev = ej == 6 ? (unsigned)(slot_of(eq & ~1) + (ek < 7 ? 0 : 2 + BAL_LAG)) : (unsigned)((ej & 1) ? p >> 32 : p); }
    unsigned lut = 0x00090703u | (wid == 0 ? 0x00020000u : 0u);
    int sidx = 0;
    const char* sSrc; const char* sDst; const char* sFlag;
#define RL(v, i) ((unsigned)__builtin_amdgcn_readlane((int)(v), (i)))
#define EV_SETUP() do { const int e8 = sidx * 8; sSrc = mk_ptr(RL(vev, e8), RL(vev, e8 + 1)); sDst = mk_ptr(RL(vev, e8 + 2), RL(vev, e8 + 3)); \
        sFlag = mk_ptr(RL(vev, e8 + 4), RL(vev, e8 + 5)); sslot = (int)RL(vev, e8 + 6); } while (0)
    EV_SETUP();
    const char* pA = aLoc + 512; const char* pB = bLoc + ((size_t)2 << 21); const char* pF = fRem;
    unsigned gq = g0 + 6u, gn, o1;
    unsigned evb = (unsigned)(0 - sslot) < 4u ? lut >> (8 * (0 - sslot)) : 0u;
    unsigned fv = 0xffffffffu, thr = 0; unsigned long long fm;
    const unsigned vzero = 0, fadr = ldsf + lane * 4, padr = ldss + lane * 16;
    int t = 0;
    asm volatile("s_waitcnt lgkmcnt(0)" ::: "memory");
    BAR;
    STAMP(1);

#define DIAG_SLOW
#define SPIN_UNTIL(v, ptr, cond) do { const GAS unsigned* fp = (const GAS unsigned*)(ptr); const unsigned long long t0 = __builtin_amdgcn_s_memrealtime(); \
            while (v = __hip_atomic_load(fp, __ATOMIC_RELAXED, __HIP_MEMORY_SCOPE_SYSTEM), !__all(cond)) { \
                if (__builtin_amdgcn_readfirstlane(__hip_atomic_load(tmo, __ATOMIC_RELAXED, __HIP_MEMORY_SCOPE_AGENT)) == seq || __builtin_amdgcn_s_memrealtime() - t0 > DEADLINE_TICKS) { \
                    __hip_atomic_store(tmo, seq, __ATOMIC_RELAXED, __HIP_MEMORY_SCOPE_AGENT); break; } \
                __builtin_amdgcn_s_sleep(2); } \
            DIAG_SLOW; __builtin_amdgcn_s_waitcnt(0xC07F); asm volatile("" ::: "memory"); } while (0)
#define SPIN_FLAGS(ptr) SPIN_UNTIL(fv, ptr, fv >= seq)
#define SPLIT() do { unsigned rv = pv0.z; \
        if (UNLIKELY(!__all((rv >> 24) == (seq & 255u)))) SPIN_UNTIL(rv, sSrc + vs0 + 8, (rv >> 24) == (seq & 255u)); \
        const unsigned a0 = RL(rv, 0), b0 = RL(rv, 1), a1 = RL(rv, 2), b1 = RL(rv, 3); \
        const int d2 = (int)((a1 - a0) << 8) >> 8, d6 = (int)((b1 - b0) << 8) >> 8, dur = (int)(((b0 - a0) & 0xffffffu) + ((b1 - a1) & 0xffffffu)); \
        const int t2 = (int)RL(vev, 2 * 8 + 6) + 2, t6 = (int)RL(vev, 6 * 8 + 6) + 2, num = 2 * (d6 * (t6 - t2) + (d6 - d2) * (64 - t6)); \
        int s = 0; _Pragma("unroll") for (int z = 1; z <= BAL_S; ++z) s += (num > (2 * z - 1) * dur) - (-num > (2 * z - 1) * dur); \
        const unsigned gl = RL(vq, 0), ln = (fadr - ldsf) >> 2; \
        s = (gl & 1u) ? -s : s; \
        const int left = NF + s - (int)ln; \
        const unsigned gt = RL(vq, NF + (int)(((bid >> 6) << 1) | (gl & 1u))); \
        unsigned nv = ((gl + 124u - 16u * (unsigned)(left > 0 ? left : 0)) & 127u) ^ (left > 0 ? 1u : 128u); PIN("+v"(nv)); \
        vq = (int)ln >= NF + (s < 0 ? s : 0) ? nv : vq; \
        const unsigned po = ((unsigned)wid * 32u + (ln >> 2)) * (NN * 2) + (ln & 3u) * 128u; const char* const pp = wT + ((size_t)(gt & 0x7eu) << 20); \
        GLDS_N(ldss, 0, po, pp); GLDS_N(ldss, 1024, po + 16u * NN * 2, pp); } while (0)
#define EVENT() do { PIN("+s"(evb)); \
        if (!(evb & 12u)) { \
            if (sidx < 7) { GLDS_N(ldss, 0, vs0, sSrc); GLDS_N(ldss, 1024, vs1, sSrc); } \
            else { SETM0(ldss, 0); asm volatile("s_nop 0\n\tglobal_load_lds_dwordx4 %0, %1 sc0 sc1\n\tglobal_load_lds_dwordx4 %0, %1 sc0 sc1" :: "v"(vs0), "s"(sSrc) : "memory"); } \
        } \
        PIN("+s"(evb)); \
        if (evb & 4u) { u32x4 pv0, pv1; \
            asm volatile("ds_read_b128 %0, %2\n\tds_read_b128 %1, %2 offset:1024\n\ts_waitcnt lgkmcnt(0)" : "=&v"(pv0), "=&v"(pv1) : "v"(padr) : "memory"); \
            if (sidx < 7) asm volatile("global_store_dwordx4 %0, %2, %4 sc0 sc1\n\tglobal_store_dwordx4 %1, %3, %4 sc0 sc1\n\ts_nop 1" :: "v"(vs0), "v"(vs1), "v"(pv0), "v"(pv1), "s"(sDst) : "memory"); \
            else SPLIT(); \
        } \
        PIN("+s"(evb)); \
        if ((evb & 10u) == 10u) { const unsigned val = (seq << 24) | ((unsigned)__builtin_amdgcn_s_memrealtime() & 0xffffffu); \
                asm volatile("global_store_dword %0, %1, %2 sc0 sc1\n\tglobal_store_dword %3, %4, %5 sc0 sc1" :: "v"(vzero), "v"(seq), "s"(sFlag), "v"((unsigned)sidx * 4u), "v"(val), "s"(recP) : "memory"); } \
        PIN("+s"(evb)); \
        if (evb & 8u) { \
            ++sidx; EV_SETUP(); \
            if (sidx == 7) { lut = 0x00000703u; const unsigned ln = (fadr - ldsf) >> 2; vs0 = (ln & 2u) * 64u + (ln & 1u) * 16u; } \
        } } while (0)
#define WAIT2(bit, n) asm volatile("s_bitcmp1_b32 %0, %1\n\ts_cbranch_scc1 1f\n\ts_waitcnt vmcnt(%2)\n1:\n\ts_waitcnt vmcnt(%3) lgkmcnt(0)" :: "s"(evb), "n"(bit), "n"(n), "n"((n) + 2) : "scc", "memory")
#define ST_B(i0, step, m0a, o0, m0b, o1_, kk, sbase) ST_BL(1, i0, step, m0a, o0, m0b, o1_, kk, sbase)
#define ST_BL(ld, i0, step, m0a, o0, m0b, o1_, kk, sbase) \
            EVERY((i0) - (ld), step, 4) { constexpr int z = ZOF((i0) - (ld), step); if constexpr (z & 1) SETM0(m0b, (o1_) + (2 * (kk) + (z >> 1)) * 4096 - 256); else SETM0(m0a, (o0) + (2 * (kk) + (z >> 1)) * 4096); } \
            EVERY(i0, step, 4) { constexpr int z = ZOF(i0, step); if constexpr (z & 1) GLDSH(voffB[2 * (kk) + ((z >> 1) & 1)], sbase); else GLDS(voffB[2 * (kk) + ((z >> 1) & 1)], sbase); }
#define ST_A(i0, step, m0a, o0, voff, sbase) \
            EVERY((i0) - 1, step, 4) { SETM0(m0a, (o0) + ZOF((i0) - 1, step) * 4096); } \
            EVERY(i0, step, 4) { GLDS((voff)[ZOF(i0, step) & 3], sbase); }
#define CHECK(ptr) \
            AT(31) { WAIT_B1(8); PIN("+v"(fv)); fm = __builtin_amdgcn_ballot_w64(fv >= thr); PIN("+s"(fm)); } \
            AT(32) { if (UNLIKELY(fm != ~0ull)) SPIN_FLAGS(ptr); }
#define K_TILE(b) do { \
          \
        if (UNLIKELY(evb & 1u)) EVENT(); \
        MMA64(0, Ao, \
            AT(1) { asm volatile("ds_read_b32 %0, %1 offset:%2" : "=v"(fv) : "v"(fadr), "i"(((b) ^ 1) * 1024) : "memory"); SETM0(ldsf, (b) * 1024); } \
            AT(3) GFLAG(vfl, pF); \
            EVERY(0, 2, 8) LDB1(ZOF(0, 2) & 7, 1, (b) * 2 * HTB); \
            EVERY(16, 2, 8) LDA1(Ap, ZOF(16, 2) & 7, va, SA(b, 1)); \
            ST_BL(2, 7, 6, ldsw, SB(b, 0), ldsw, SB(b, 1), 0, pB) \
            CHECK(fRem + ((RL(vq, t + 2) & 0x7eu) << 8) + vfl) \
            ST_A(34, 4, ldsw, SA(b, 0), voffA, pA) \
            AT(G_WAIT) WAIT2(1, 17); \
            AT(G_BAR) BAR; ); \
          \
        MMA64(1, Ap, \
            ST_B(1, 4, ldsw, SB(b, 0), ldsw, SB(b, 1), 1, pB) \
            ST_A(17, 4, ldsw, SA(b, 1), voffP, pA) \
            EVERY(2, 4, 8) LDA1(Ao, ZOF(2, 4) & 7, va, SA((b) ^ 1, 0)); \
            EVERY(32, 2, 8) LDB1(ZOF(32, 2) & 7, 0, ((b) ^ 1) * 2 * HTB); \
            AT(33) { gn = RL(vq, t + 4); PIN("+s"(gn)); } \
            AT(35) { pB = wT + ((size_t)gq << 20); PIN("+s"(pB)); } \
            AT(37) { o1 = (gq & 0x70u) << 17; PIN("+s"(o1)); } \
            AT(39) { o1 |= (gq & 15u) << 7; PIN("+s"(o1)); } \
            AT(41) { pA = t < 5 ? aLocT : aRemT; PIN("+s"(pA)); } \
            AT(43) { pA += o1; gq = gn; PIN("+s"(pA), "+s"(gq)); } \
            AT(45) { o1 = (gn & 0x7eu) << 8; PIN("+s"(o1)); } \
            AT(47) { pF = fRem + o1; PIN("+s"(pF)); } \
            AT(48) { thr = t < 5 ? 0u : seq; PIN("+s"(thr)); } \
            AT(49) { ++t; o1 = (unsigned)(t - sslot); PIN("+s"(t), "+s"(o1)); } \
            AT(50) { evb = lut >> (o1 * 8u); PIN("+s"(evb)); } \
            AT(51) { evb = o1 < 4u ? evb & 31u : 0u; PIN("+s"(evb)); } \
            if constexpr (b) { AT(53) { o1 = gn & 128u; PIN("+s"(o1)); } AT(54) { evb |= o1; PIN("+s"(evb)); } } \
            AT(G_WAIT) wait_vm_l0<16>(); \
            AT(G_BAR) BAR; ); \
    } while (0)

    unsigned pb = 0;
    do {
        K_TILE(0);
        K_TILE(1);
    } while (!(evb & 128u));
    if (!(RL(vq, t + 2) & 128u)) { K_TILE(0); pb = 2 * HTB; }

    const unsigned qb = 2 * HTB - pb, fo = pb >> 5;
    const unsigned r16 = (unsigned)rank * 16u;
#define TAIL_G(u) ((r16 + (unsigned)(e_of(NF + (u) / 2) * 2 + ((u) & 1))) & 127u)
#define TAIL_A(u) (aRemT + OFF_A(TAIL_G(u)))
#define TAIL_B(u) (wT + ((size_t)TAIL_G(u) << 20))
    const unsigned mA[3] = {ldsw + pb, ldsw + qb, ldsw + qb + HTB};
    const unsigned mB0[3] = {ldsw + 4 * HTB + pb, ldsw + 4 * HTB + qb, ldsw + pb + HTB}, mB1[3] = {ldsw + 5 * HTB + pb, ldsw + 5 * HTB + qb, ldsw + L_X};
    const unsigned rB2 = (wc ? L_X : pb + HTB) - (4 + wc) * HTB;
    const unsigned dB[3] = {pb - rB2, qb - pb, rB2 - qb};
    const unsigned vA3[3] = {va + pb, va + qb, va + qb + HTB};
    unsigned tfo;
    { const int u = NF + (lane >> 3) - 8, a = u / 7, j = u - a * 7;
      tfo = ((((unsigned)rank * 8u + (unsigned)((j + 1) * 8 + a)) & 63u) << 9) + vfl; }
    const char *tA = TAIL_A(0), *tB = TAIL_B(0), *tB2, *tBo;
#define MOVE_B(i0, d) EVERY(i0, 1, 4) { constexpr int z = ZOF(i0, 1) & 3; vb[2 * z] += (d); vb[2 * z + 1] += (d); PIN("+v"(vb[2 * z]), "+v"(vb[2 * z + 1])); }
    {
#pragma unroll
        for (int j = 0; j < 8; ++j) vb[j] += pb;
        MMA64(0, Ao,
            AT(1) { asm volatile("ds_read_b32 %0, %1" : "=v"(fv) : "v"(fadr + (1024 - fo)) : "memory"); SETM0(ldsf + fo, 0); }
            AT(3) GFLAG(tfo, fRem);
            EVERY(0, 2, 8) LDB1(ZOF(0, 2) & 7, 1, 0);
            EVERY(16, 2, 8) LDA1(Ap, ZOF(16, 2) & 7, vA3[0], HTB);
            ST_BL(2, 7, 6, mB0[0], 0, mB1[0], 0, 0, tB)
            CHECK(fRem + ((TAIL_G(0) & ~1u) << 8) + vfl)
            ST_A(34, 4, mA[0], 0, voffA, tA)
            MOVE_B(48, dB[1])
            AT(53) { tB2 = TAIL_B(2); PIN("+s"(tB2)); }
            AT(G_WAIT) wait_vm_l0<17>();
            AT(G_BAR) BAR; );
        MMA64(1, Ap,
            ST_B(1, 4, mB0[0], 0, mB1[0], 0, 1, tB)
            ST_B(17, 4, mB0[2], 0, mB1[2], 0, 0, tB2)
            EVERY(2, 4, 8) LDA1(Ao, ZOF(2, 4) & 7, vA3[1], 0);
            EVERY(32, 2, 8) LDB1(ZOF(32, 2) & 7, 0, 0);
            AT(33) { tA += 128; PIN("+s"(tA)); }
            AT(35) { tB += (size_t)1 << 20; PIN("+s"(tB)); }
            AT(G_WAIT) wait_vm_l0<16>();
            AT(G_BAR) BAR; );
        MMA64(0, Ao,
            AT(1) asm volatile("ds_read_b32 %0, %1" : "=v"(fv) : "v"(fadr + fo) : "memory");
            EVERY(0, 2, 8) LDB1(ZOF(0, 2) & 7, 1, 0);
            EVERY(16, 2, 8) LDA1(Ap, ZOF(16, 2) & 7, vA3[1], HTB);
            ST_BL(2, 7, 6, mB0[1], 0, mB1[1], 0, 0, tB)
            CHECK(fRem + tfo)
            ST_A(34, 4, mA[1], 0, voffA, tA)
            MOVE_B(48, pb - qb)
            AT(53) { tA = TAIL_A(2); PIN("+s"(tA)); }
            AT(G_WAIT) wait_vm_l0<16>();
            AT(G_BAR) BAR; );
        MMA64(1, Ap,
            ST_B(1, 4, mB0[1], 0, mB1[1], 0, 1, tB)
            ST_A(17, 4, mA[2], 0, voffA, tA)
            EVERY(2, 4, 8) LDA1(Ao, ZOF(2, 4) & 7, vA3[0], 0);
            EVERY(32, 2, 8) LDB1(ZOF(32, 2) & 7, 0, 0);
            AT(33) { tA += 128; PIN("+s"(tA)); }
            AT(35) { tBo = tB2; tB = tB2 + ((size_t)1 << 20); PIN("+s"(tB), "+s"(tBo)); }
            AT(G_WAIT) wait_vm_l0<8>();
            AT(G_BAR) BAR; );
    }
    STAMP(2);
    const char* const slabOut = myws + W_SLAB + (size_t)bid * SLAB_B + (size_t)wid * 16384;
    const char* const slabIn = myws + W_SLAB + (size_t)partner * SLAB_B + (size_t)wid * 16384;
    const char* const pflag = myws + W_EFLAG + (size_t)(partner * 4 + wid) * 128;
    const unsigned vl[4] = {(unsigned)lane * 16u, (unsigned)lane * 16u + 4096u, (unsigned)lane * 16u + 8192u, (unsigned)lane * 16u + 12288u};
    u32x4 in[16], w;
    bool early = false;
#define TAIL_PHASE(AC, AN) \
        MMA64(0, AC, \
            EVERY(0, 2, 8) LDB1(ZOF(0, 2) & 7, 1, 0); \
            if constexpr (u + 1 < SH) { \
                MOVE_B(17, dB[(u + 1) % 3]) \
                EVERY(16, 2, 8) LDA1(AN, ZOF(16, 2) & 7, vA3[(u + 1) % 3], 0); \
                EVERY(32, 2, 8) LDB1(ZOF(32, 2) & 7, 0, 0); } \
            AT(31) WAIT_B1(u + 1 < SH ? 8 : 0); \
            if constexpr (u + 2 < SH) { ST_B(1, 4, mB0[(u + 2) % 3], 0, mB1[(u + 2) % 3], 0, 1, tBo) } \
            if constexpr (u + 3 < SH) { ST_B(17, 4, mB0[u % 3], 0, mB1[u % 3], 0, 0, tB) ST_A(33, 4, mA[u % 3], 0, voffA, tA) } \
            if constexpr (u < 16 / XS_ST) { EVERY(0, 32 / XS_ST, XS_ST) { constexpr int p = (u * XS_ST + ZOF(0, 32 / XS_ST)) & 15; \
                    w.x = cvt_pk_bf16(acc[1][p >> 2][(p & 3) * 2][0], acc[1][p >> 2][(p & 3) * 2][1]); } \
                EVERY(1, 32 / XS_ST, XS_ST) { constexpr int p = (u * XS_ST + ZOF(1, 32 / XS_ST)) & 15; \
                    w.y = cvt_pk_bf16(acc[1][p >> 2][(p & 3) * 2][2], acc[1][p >> 2][(p & 3) * 2][3]); } \
                EVERY(2, 32 / XS_ST, XS_ST) { constexpr int p = (u * XS_ST + ZOF(2, 32 / XS_ST)) & 15; \
                    w.z = cvt_pk_bf16(acc[1][p >> 2][(p & 3) * 2 + 1][0], acc[1][p >> 2][(p & 3) * 2 + 1][1]); } \
                EVERY(3, 32 / XS_ST, XS_ST) { constexpr int p = (u * XS_ST + ZOF(3, 32 / XS_ST)) & 15; \
                    w.w = cvt_pk_bf16(acc[1][p >> 2][(p & 3) * 2 + 1][2], acc[1][p >> 2][(p & 3) * 2 + 1][3]); \
                    asm volatile("global_store_dwordx4 %0, %1, %2 offset:%3 sc1\n\ts_nop 1" :: "v"(vl[p >> 2]), "v"(w), "s"(slabOut), "i"((p & 3) * 1024) : "memory"); } } \
            if constexpr (u == S_FLAG) { AT(3) asm volatile("global_store_dword %0, %1, %2 sc1" :: "v"(vzero), "v"(seq), "s"(myws + W_EFLAG + (size_t)(bid * 4 + wid) * 128) : "memory"); } \
            if constexpr (u == S_SAMP) { AT(2) SETM0(ldsf, 0); AT(3) GFLAG(vzero, pflag); } \
            if constexpr (u == S_LOAD) { AT(31) { PIN("+v"(fv)); early = (unsigned)__builtin_amdgcn_readfirstlane(fv) >= seq; asm volatile("" ::: "memory"); } } \
            if constexpr (u >= S_LOAD && u < S_LOAD + 16 / XS_LD) { EVERY(35, 24 / XS_LD, XS_LD) { constexpr int p = ((u - S_LOAD) * XS_LD + ZOF(35, 24 / XS_LD)) & 15; \
                    asm volatile("global_load_dwordx4 %0, %1, %2 offset:%3 sc1" : "=v"(in[p]) : "v"(vl[p >> 2]), "s"(slabIn), "i"((p & 3) * 1024) : "memory"); } } \
            if constexpr (u + 4 < SH) { \
                AT(49) { if constexpr (u & 1) tA += 128; else tA = TAIL_A(u + 4); PIN("+s"(tA)); } \
                AT(53) { tBo = tB; if constexpr (u & 1) tB += (size_t)1 << 20; else tB = TAIL_B(u + 4); PIN("+s"(tB), "+s"(tBo)); } } \
            if constexpr (u + 4 == SH) { AT(53) { tBo = tB; PIN("+s"(tBo)); } } \
            if constexpr (u + 1 < SH) { AT(G_WAIT) wait_vm_l0<tail_wait(u)>(); AT(G_BAR) BAR; } )
    static_for([&](auto uc) {
        constexpr int u = decltype(uc)::value;
        if constexpr (u == S_LOAD) asm volatile("ds_read_b32 %0, %1" : "=v"(fv) : "v"(fadr) : "memory");
        if constexpr (u & 1) { TAIL_PHASE(Ap, Ao); } else { TAIL_PHASE(Ao, Ap); }
    }, std::make_integer_sequence<int, SH>{});
    asm volatile("s_nop 15\n\ts_nop 15" ::: "memory"); SCHED;
    STAMP(3);

    {
#define WAIT_IN() asm volatile("s_waitcnt vmcnt(0)" : "+v"(in[0]), "+v"(in[1]), "+v"(in[2]), "+v"(in[3]), "+v"(in[4]), "+v"(in[5]), "+v"(in[6]), "+v"(in[7]), \
                               "+v"(in[8]), "+v"(in[9]), "+v"(in[10]), "+v"(in[11]), "+v"(in[12]), "+v"(in[13]), "+v"(in[14]), "+v"(in[15]) :: "memory")
        WAIT_IN();
        if (UNLIKELY(!early)) {
            const GAS unsigned* fp = (const GAS unsigned*)pflag;
            const unsigned long long t0 = __builtin_amdgcn_s_memrealtime();
            while ((unsigned)__builtin_amdgcn_readfirstlane(__hip_atomic_load(fp, __ATOMIC_RELAXED, __HIP_MEMORY_SCOPE_AGENT)) < seq) {
                if (__builtin_amdgcn_readfirstlane(__hip_atomic_load(tmo, __ATOMIC_RELAXED, __HIP_MEMORY_SCOPE_AGENT)) == seq || __builtin_amdgcn_s_memrealtime() - t0 > DEADLINE_TICKS) {
                    __hip_atomic_store(tmo, seq, __ATOMIC_RELAXED, __HIP_MEMORY_SCOPE_AGENT); break; }
                __builtin_amdgcn_s_sleep(1);
            }
            asm volatile("" ::: "memory");
#pragma unroll
            for (int p = 0; p < 16; ++p) asm volatile("global_load_dwordx4 %0, %1, %2 offset:%3 sc1" : "=v"(in[p]) : "v"(vl[p >> 2]), "s"(slabIn), "i"((p & 3) * 1024) : "memory");
            WAIT_IN();
            DIAG_SLOW;
        }
        const bool bad = __builtin_amdgcn_readfirstlane(__hip_atomic_load(tmo, __ATOMIC_RELAXED, __HIP_MEMORY_SCOPE_AGENT)) == seq;
        STAMP(4);
        unsigned short* orow = out + (size_t)(pm * 256 + h * 128 + wr * 64 + fr) * NN + pn * 256 + wc * 128 + (fq & 1) * 16 + (fq >> 1) * 8;
#pragma unroll
        for (int m = 0; m < 4; ++m)
#pragma unroll
            for (int j = 0; j < 4; ++j) {
                f32x4 v0 = acc[0][m][2 * j], v1 = acc[0][m][2 * j + 1];
                const u32x4 p = in[m * 4 + j];
                v0[0] += __uint_as_float(p.x << 16); v0[1] += __uint_as_float(p.x & 0xffff0000u); v0[2] += __uint_as_float(p.y << 16); v0[3] += __uint_as_float(p.y & 0xffff0000u);
                v1[0] += __uint_as_float(p.z << 16); v1[1] += __uint_as_float(p.z & 0xffff0000u); v1[2] += __uint_as_float(p.w << 16); v1[3] += __uint_as_float(p.w & 0xffff0000u);
                unsigned a0 = cvt_pk_bf16(gelu_tanh(v0[0]), gelu_tanh(v0[1])), a1 = cvt_pk_bf16(gelu_tanh(v0[2]), gelu_tanh(v0[3]));
                unsigned c0 = cvt_pk_bf16(gelu_tanh(v1[0]), gelu_tanh(v1[1])), c1 = cvt_pk_bf16(gelu_tanh(v1[2]), gelu_tanh(v1[3]));
                { auto r = __builtin_amdgcn_permlane16_swap(a0, c0, false, false); a0 = r[0]; c0 = r[1]; }
                { auto r = __builtin_amdgcn_permlane16_swap(a1, c1, false, false); a1 = r[0]; c1 = r[1]; }
                u32x4 o = {a0, a1, c0, c1};
                if (bad) o = (u32x4){0x7fc07fc0u, 0x7fc07fc0u, 0x7fc07fc0u, 0x7fc07fc0u};
                *(u32x4*)(orow + (size_t)m * 16 * NN + j * 32) = o;
            }
    }
}

extern "C" void kernel_launch(int rank, int nranks, ncclComm_t comm,
                              hipStream_t stream,
                              void* const* d_in, const int* in_sizes, int n_in,
                              void* d_out, int out_size,
                              const int* in_dtypes, int out_dtype,
                              void* const* d_out_all,
                              void* const* d_ws_all, size_t ws_bytes) {
    (void)comm; (void)in_sizes; (void)n_in; (void)out_size; (void)in_dtypes; (void)out_dtype; (void)d_out_all;
    if (nranks != NR || (unsigned)rank >= (unsigned)NR || ws_bytes < WS_NEED) { fprintf(stderr, "a2a_gemm: unsupported configuration\n"); return; }
    static bool attr_set = false;
    if (!attr_set) {
        HIP_CHECK(hipFuncSetAttribute((const void*)a2a_gemm_gelu_kernel, hipFuncAttributeMaxDynamicSharedMemorySize, LDS_BYTES));
        attr_set = true;
    }
    hipLaunchKernelGGL(a2a_gemm_gelu_kernel, dim3(NWG), dim3(256), LDS_BYTES, stream, rank,
                       (const unsigned short*)d_in[0], (const unsigned short*)d_in[1], (unsigned short*)d_out, d_ws_all);
    HIP_CHECK(hipGetLastError());
}
```

```cpp
#include <hip/hip_runtime.h>
#include <rccl/rccl.h>
#include <cstdio>
#include <cstdint>

#define HIP_CHECK(expr) do { hipError_t _e = (expr); if (_e != hipSuccess) { \
    fprintf(stderr, "HIP error %s at %s:%d\n", hipGetErrorString(_e), __FILE__, __LINE__); \
    return; } } while (0)

#define LAS __attribute__((address_space(3)))
#define GAS __attribute__((address_space(1)))
typedef short bf16x8 __attribute__((ext_vector_type(8)));
typedef short s16x4 __attribute__((ext_vector_type(4)));
typedef float f32x2 __attribute__((ext_vector_type(2)));
typedef float f32x4 __attribute__((ext_vector_type(4)));
typedef unsigned u32x4 __attribute__((ext_vector_type(4)));
typedef __bf16 bf2 __attribute__((ext_vector_type(2)));

#define AV 0
#define EO 1
#define PF0 0
#define PFD 7
#define PFT 0
#define PFL 4
#define TSO 0
#define GPOS 0
#define PFK 0
#define RLK 0
#define DYN 0
#define RLS1 0
#define PACE0 64
#define PACE1 88
#define BAL_S 6
#define T_SEND 43
#define SEND2 1
#define SPOS 2
#define RECV_TILED 1
#define OUTST 5
#define OUTMOD "sc0 sc1 nt"
#define PNMAP 2
#define LDPAT 0
#define PFS 1u
#define PF_R0 22
#define NPF 14u
#define PFQ 3
#define VN 3
#define XD 1
#define XI 0
#define CM 3
#define CMG (CM != 0 && CM != 1 && CM != 2)
#define ROT 0
#define ORD 0
#define SKEW 0
#define BAL_I1 6
#define BAL_H 64
#define SW 0
#define SWMOD "sc1"
#define BAL_I2 18

namespace {
constexpr int NR = 8;
constexpr int MS = 1024;
constexpr int KB = 1024;
constexpr int NN = 8192;
constexpr int SH = 16;
constexpr int NF = 64 - SH / 2;
constexpr int NWG = 256;
constexpr int HTB = 16384;
__host__ __device__ constexpr int slot_of(int q) { return (q < 448 ? q * PACE0 : 448 * PACE0 + (q - 448) * PACE1) / (448 * 16); }
static_assert(BAL_S >= 0 && BAL_S <= 6 && slot_of(3583) + 3 <= (SEND2 ? NF - 6 - 3 : T_SEND) && T_SEND % 2 == 1 && T_SEND + 4 <= NF - 6 && 2 * BAL_I2 + 5 <= T_SEND && BAL_I1 < BAL_I2, "schedule");
static_assert(256 * PACE0 >= 448 * 16 && 256 * PACE1 >= 448 * 16 && PF_R0 >= 2 && PF_R0 + 14 + 14 * PFS <= NF - 6 && (SKEW == 0 || SKEW == 2), "one piece per slot");
constexpr unsigned IDLE = 0xf0000000u;
constexpr int L_FLAG = 9 * HTB;
constexpr int LDS_BYTES = L_FLAG + 4096;
constexpr size_t W_CTRL = (size_t)128 << 20;
constexpr size_t W_TMO = W_CTRL + 256;
constexpr size_t W_XFLAG = W_CTRL + 4096;
constexpr size_t W_EFLAG = W_CTRL + 65536;
constexpr size_t W_SEQ = W_EFLAG + 262144;
constexpr size_t W_REC = W_SEQ + 32768;
#define SCRS 16
constexpr size_t W_SCR = SCRS == 16 ? W_CTRL + 40960 : W_EFLAG + 131072;
constexpr size_t W_DIAG = W_CTRL + 524288;
constexpr size_t RECV_BLK = (size_t)MS * KB * 2;
constexpr size_t W_RECV0 = (size_t)80 << 20;
constexpr size_t W_RECV1 = (size_t)96 << 20;
constexpr size_t W_SLAB = (size_t)112 << 20;
constexpr size_t SLAB_B = 65536;
constexpr size_t WS_NEED = (size_t)129 << 20;
constexpr unsigned long long DEADLINE_TICKS = 40000000ull;
__host__ __device__ constexpr int e_of(int T, int sk = 0) { return T < 8 ? T : ((T - 8 + sk) % 7 + 1) * 8 + (((T - 8) / 7 + ((T - 8 + sk) % 7 + 1) * ROT) & 7); }

__host__ __device__ constexpr unsigned g_of(int T, int h) { return T < 8 ? 2 * T + h : ORD == 1 || (ORD == 3 && T < 15) ? ((2 * T - 16 + h) % 7 + 1) * 16 + (((2 * T - 16 + h) / 7 + ((2 * T - 16 + h) % 7 + 1) * 2 * ROT) & 15) : e_of(T) * 2 + h; }
__device__ __forceinline__ int lds_byteA(int r, int c) { const int st = (r >> 4) * 2 + (c >> 5), rr = r & 15, cc = c & 31, ob = rr * 64 + cc * 2; return st * 1024 + (ob ^ (((ob >> 9) & 1) << 5)); }
__device__ __forceinline__ unsigned cvt_pk_bf16(float lo, float hi) { const f32x2 v = {lo, hi}; return __builtin_bit_cast(unsigned, __builtin_convertvector(v, bf2)); }
__device__ __forceinline__ const char* mk_ptr(unsigned lo, unsigned hi) { return (const char*)(((unsigned long long)hi << 32) | lo); }
}

__global__ __launch_bounds__(256, 1)
void a2a_gemm_gelu_kernel(int rank, const unsigned short* __restrict__ xloc, const unsigned short* __restrict__ wmat,
                          unsigned short* __restrict__ out, void* const* __restrict__ ws_all) {
    extern __shared__ __attribute__((aligned(16))) unsigned char shm[];
    LAS unsigned char* lds = (LAS unsigned char*)shm;
    const int tid = threadIdx.x, wid = __builtin_amdgcn_readfirstlane(tid >> 6), lane = tid & 63, wr = wid >> 1, wc = wid & 1, fr = lane & 15, fq = lane >> 4;
    const int bx = blockIdx.x, bid = (bx & 1) ? ((((bx >> 3) ^ XI) << 3) | ((bx - 2 * XD) & 7)) : bx, xcd = bid & 7, idx = bid >> 3;
    const int h = xcd & 1, pn = (idx & 6) * 4 + (xcd & 6) + (idx & 1), pm = idx >> 3;
    const int partner = bid ^ 1, bpair = bid & ~1, sk = SKEW * pm;
    char* const myws = (char*)ws_all[rank];
#define STAMP(i)
#define STAMPP(i)
#define TSTAMP(u)
#define TS4A
#define TS5A
#define TS4B(v)
#define TS6A
#define TS6B
#define TS6C
    STAMP(0);
#define CPF 0
#define S __builtin_amdgcn_sched_barrier(0);
#define BAR __builtin_amdgcn_s_barrier()
#define BW0 6
#define BWN 16
#define BSL "s_sleep 1"
#define BUB do { unsigned _b; asm volatile("s_sub_u32 %0, %1, %2\n\ts_cmp_lt_u32 %0, %3\n\ts_cbranch_scc0 1f\n\t" BSL "\n1:" : "=s"(_b) : "s"(t), "n"(BW0), "n"(BWN) : "scc"); } while (0)
#define UNLIKELY(x) __builtin_expect(!!(x), 0)
#define PIN(...) asm volatile("" : __VA_ARGS__)
#define RL(v, i) ((unsigned)__builtin_amdgcn_readlane((int)(v), (i)))
#define RLS(d, v, i) do { d = RL(v, i); PIN("+s"(d)); } while (0)
#define RLM(d, s) asm volatile("v_readlane_b32 %0, %1, m0" : "=s"(d) : "v"(s))
#define M0(sreg) asm volatile("s_mov_b32 m0, %0" :: "s"(sreg))
#define M0P1 asm volatile("s_add_i32 m0, %0, 1" :: "s"(t) : "scc")
#define GLP(voff, sbase, off) asm volatile("global_load_lds_dwordx4 %0, %1 offset:%2" :: "v"(voff), "s"(sbase), "i"(off) : "memory")
#define NTM 0
#define MOD_pB ""
#define MOD_pA ""
#define MOD_qB ""
#define MOD_qC ""
#define MOD_qA ""
#define MOD_S ""
#define GL(voff, p, off) asm volatile("global_load_lds_dwordx4 %0, %1 offset:%2" MOD_##p :: "v"(voff), "s"(mk_ptr(p##l, p##h)), "i"(off) : "memory")
#define GL4(sreg, voff, sbase) asm volatile("s_mov_b32 m0, %5\n\ts_nop 0\n\tglobal_load_lds_dwordx4 %0, %4\n\tglobal_load_lds_dwordx4 %1, %4 offset:1024\n\t" \
        "global_load_lds_dwordx4 %2, %4 offset:2048\n\tglobal_load_lds_dwordx4 %3, %4 offset:3072" :: "v"((voff)[0]), "v"((voff)[1]), "v"((voff)[2]), "v"((voff)[3]), "s"(sbase), "s"(sreg) : "memory")
#define XQ 0
#define GFLAG(voff, sbase) asm volatile("global_load_lds_dword %0, %1 sc0 sc1" :: "v"(voff), "s"(sbase) : "memory")
#define pF mk_ptr(pFl, pFh)
#define CAo "v"
#define CAp "v"
#define CF "v"
#define LDA(n, m, k, addr, off) asm volatile("ds_read_b128 %0, %1 offset:%2" : "=" C##n(n[m][k]) : "v"(addr), "i"(off) : "memory")
#define LB0(c, addr, off) asm volatile("ds_read_b64_tr_b16 %0, %1 offset:%2" : "=" CF(blo[c]) : "v"(addr), "i"(off) : "memory")
#define LB1(c, dst, addr, off) do { s16x4 _r1; asm volatile("ds_read_b64_tr_b16 %0, %1 offset:%2" : "=" CF(_r1) : "v"(addr), "i"(off) : "memory"); \
        dst = __builtin_shufflevector(blo[c], _r1, 0, 1, 2, 3, 4, 5, 6, 7); } while (0)
#define LDF asm volatile("ds_read_b32 %0, %1" : "=v"(fv) : "v"(fadr) : "memory")
#define WL(n) asm volatile("s_waitcnt lgkmcnt(%0)" :: "n"(n) : "memory")
#define WLF asm volatile("s_waitcnt lgkmcnt(0)" : "+v"(fv) :: "memory")
#define WV(n) asm volatile("s_waitcnt vmcnt(%0)" :: "n"(n) : "memory")
#define WVL(n) asm volatile("s_waitcnt vmcnt(%0) lgkmcnt(0)" :: "n"(n) : "memory")
#define MF(cc, cb, ca, c, b, a) asm volatile("v_mfma_f32_16x16x32_bf16 %0, %1, %2, %0" : "+" cc(c) : cb(b), ca(a))
#define MF1(c, i, m, j, k, n) asm volatile("v_mfma_f32_16x16x32_bf16 %0, %1, %2, %0" :: #c(acc[i][m][j]), CF(Bf[j][k]), C##n(n[m][k]))
#define MFZ(c, i, m, j, k, n) asm volatile("v_mfma_f32_16x16x32_bf16 %0, %1, %2, 0" :: #c(acc[i][m][j]), CF(Bf[j][k]), C##n(n[m][k]))
#define MF2(c, i, m, j, k, n) MF(#c, CF, C##n, acc[i][m][j], Bf[j][k], n[m][k])
#define INV(j) (AV && (j) >= AV)
#define AR(d, a) do { float _t; asm volatile("v_accvgpr_read_b32 %0, %1" : "=v"(_t) : "a"(a)); d = _t; } while (0)
#define CVT cvt_pk_bf16

    const unsigned lds0 = (unsigned)(uintptr_t)lds, ldsw = lds0 + (unsigned)wid * 4096u;
    const size_t rowOff = (size_t)pm * 256 * KB * 2;
#define CMT (CM == 3 ? 0x74216530u : CM == 4 ? 0x65217430u : CM == 5 ? 0x64317520u : CM == 6 ? 0x54327610u : (unsigned)CM)
#define LINE_OF(u) (CM == 0 ? (pn & 1) * 4 + (u) : CM == 1 ? (pn & 1) * 2 + ((u) & 1) + ((u) >> 1) * 4 : CM == 2 ? (pn & 1) + 2 * (u) : (int)((CMT >> ((pn & 1) * 16 + 4 * (u))) & 7))
    const char* const wT = (const char*)wmat + (size_t)(pn >> 1) * 1024;
    const unsigned g0 = (unsigned)rank * 16u + (unsigned)h;
    const char* const bLoc = wT + ((size_t)g0 << 20);
    unsigned mA[4], mB[4];
#pragma unroll
    for (int i = 0; i < 4; ++i) { mA[i] = ldsw + i * HTB; mB[i] = ldsw + (5 + i) * HTB; PIN("+s"(mA[i]), "+s"(mB[i])); }
    unsigned voffA[4], voffP[4], voffB[8];
#pragma unroll
    for (int i = 0; i < 4; ++i) {
        const int k = wid * 16 + i * 4 + fq, f = (k & 3) | (((k >> 3) & 1) << 2), c = fr ^ (f << 1);
        voffB[i] = (unsigned)k * (NN * 2) + (unsigned)(LINE_OF(c >> 3) * 128 + (c & 7) * 16) - (unsigned)i * 1024u;
        voffB[4 + i] = voffB[i] + (unsigned)((LINE_OF(2 + (c >> 3)) - LINE_OF(c >> 3)) * 128);
    }
    GL4(mB[0], voffB, bLoc); S
    constexpr unsigned HSTEP_A = 128u * KB * 2;
    const char* const aLocT = (const char*)xloc + rowOff - 4096;
#define OFF_A(g) ((((g) & 0x70u) << 17) | (((g) & 15u) << 7))
    const char* const aLoc = aLocT + OFF_A(g0);
    const unsigned swz = ((unsigned)lane * 16u) ^ (((unsigned)lane >> 5) << 5);
    { const unsigned o0 = ((unsigned)wid * 32u + (swz >> 6)) * (KB * 2) + (swz & 63u) + 4096u;
#pragma unroll
      for (int i = 0; i < 4; ++i) {
        const unsigned o = o0 + (unsigned)((i >> 1) * 16 * KB * 2 + (i & 1) * 64 - i * 1024);
        voffA[i] = o + (unsigned)h * HSTEP_A; voffP[i] = o + (unsigned)(h ^ 1) * HSTEP_A;
        PIN("+v"(voffA[i]), "+v"(voffP[i]), "+v"(voffB[4 + i]));
      } }
    GL4(mA[0], voffA, aLoc); GL4(mB[1], voffB + 4, bLoc); GL4(mA[1], voffP, aLoc); S
    unsigned seqv;
    asm volatile("global_load_dword %0, %1, off sc1" : "=v"(seqv) : "v"(myws + W_SEQ + (size_t)bid * 128) : "memory");
    S STAMPP(2); S
    const int sq1 = __builtin_amdgcn_ds_permute((lane < 14 ? slot_of(lane * 256 + bid) : 63) * 4, lane < 14 ? lane * 256 + bid + 1 : 0), sq = sq1 - 1;
    const bool sact = sq1 != 0;
    const int sa0 = sq / 448, sr = sq - sa0 * 448, si = sr / 7, sc = sr - si * 7, srg = SKEW ? ((((sc >> 1) - (si >> 4)) & 3) << 4 | (si & 15)) : si, sd = (rank - 1 - sc) & 7, sa = (sa0 + (sc + 1) * ROT) & 7;
    unsigned long long epw;
    asm volatile("global_load_dwordx2 %0, %1, off" : "=v"(epw) : "v"(ws_all + (sact ? sd : rank)) : "memory");
    unsigned mF = lds0 + L_FLAG + (unsigned)wid * (DYN ? 1024u : 512u); PIN("+s"(mF));
    unsigned vs0 = (unsigned)(wid * 4 + fq) * (KB * 2) + (unsigned)fr * 16u;
    unsigned vs1 = (unsigned)(fr >> 3) * 131072u + (unsigned)((fr >> 2) & 1) * 1024u + (((unsigned)(wid * 4 + fq) * 64u + (unsigned)(fr & 3) * 16u) ^ ((unsigned)(wid >> 1) << 5));
    const unsigned min0 = (unsigned)wid * 8192u, min1 = (unsigned)wid * 256u;
    PIN("+v"(vs0), "+v"(vs1));
    const unsigned nrec = (unsigned)W_SCR + (unsigned)(bid * 4 + wid) * SCRS + 16u, idleS = 8u * (unsigned)RECV_BLK - 16u - min0;
    const unsigned sso = sact ? (unsigned)sd * (unsigned)RECV_BLK + (unsigned)srg * ((ORD ? 32u : 16u) * KB * 2) + (unsigned)sa * (ORD ? 128u : 256u) : idleS;
#define SRD(p, n) ((u32x4){(unsigned)(unsigned long long)(p), (unsigned)((unsigned long long)(p) >> 32) & 0xffffu, (unsigned)(n), 0x20000u})
    const u32x4 dX = SRD(xloc, 8u * RECV_BLK);
    u32x4 pay[2];
#define SLD0(b, soff) asm volatile("buffer_load_dwordx4 %0, %1, %2, %3 offen" : "=v"(pay[b]) : "v"(vs0), "s"(dX), "s"(soff) : "memory")
#define SLD(b) asm volatile("buffer_load_dwordx4 %0, %1, %2, %3 offen" MOD_S : "=v"(pay[b]) : "v"(vs0), "s"(dS), "s"(oS) : "memory")
#define SST(b) asm volatile("buffer_store_dwordx4 %0, %1, %2, %3 offen sc0 sc1" :: "v"(pay[b]), "v"(vs1), "s"(dD), "s"(oD) : "memory")
#define SFL asm volatile("buffer_store_dword %0, %1, %2, %3 offen sc0 sc1" :: "v"(vseq), "v"(vone), "s"(dF), "s"(oF) : "memory")
#define SHL1(v, idle) ((unsigned)__builtin_amdgcn_update_dpp((int)(idle), (int)(v), 0x130, 0xf, 0xf, false))
#define SHR1(v, idle) ((unsigned)__builtin_amdgcn_update_dpp((int)(idle), (int)(v), 0x138, 0xf, 0xf, false))
    unsigned oS, oD, oF;
    RLS(oS, sso, 0); RLS(oD, sso, 1);
    asm volatile("s_nop 4");
    SLD0(0, oS);
    SLD0(1, oD);
    GL4(mB[2], voffB, bLoc + ((size_t)1 << 21)); GL4(mA[2], voffA, aLoc + 256); GL4(mB[3], voffB + 4, bLoc + ((size_t)1 << 21)); GL4(mA[3], voffP, aLoc + 256); S
#define PFA0 3
#define PFA asm volatile("buffer_load_dword %0, %1, %2, %3 offen" : "+v"(vdump) : "v"((lane & 7) >= PFA0 ? (unsigned)(wid * 8 + (lane >> 3)) * (KB * 2) + (unsigned)(lane & 7) * 256u : IDLE), "s"(dX), \
        "s"((unsigned)(rank * MS + pm * 256 + (idx & 7) * 32) * (KB * 2) + (unsigned)h * 128u) : "memory")

    f32x4 acc[2][4][8];
#pragma unroll
    for (int i = 0; i < 32; ++i) { if (INV(i & 7)) asm volatile("" : "=v"(acc[0][i >> 3][i & 7])); else asm volatile("" : "=a"(acc[0][i >> 3][i & 7])); asm volatile("" : "=a"(acc[1][i >> 3][i & 7])); }
    bf16x8 Ao[4][2], Ap[4][2], Bf[8][2]; s16x4 blo[2];
    unsigned va = lds0 + lds_byteA(wr * 64 + fr, fq * 8);
    unsigned vb[4];
    { const int q = fr >> 2, p = lane & 3, f = q | ((fq & 1) << 2);
#pragma unroll
      for (int j = 0; j < 4; ++j) { vb[j] = lds0 + 5 * HTB + (8 * fq + q) * 256 + (((wc * 4 + j) ^ f) << 5) + p * 8; PIN("+v"(vb[j])); } }
    PIN("+v"(va));
    const char* const fRem = myws + W_XFLAG + (size_t)pm * 128;
    const unsigned vfl = (unsigned)(lane & (ORD ? 7 : 15)) * 4u;
#define OFF_F(g) ((((g) & 0x7eu) << 8) | (ORD ? ((g) & 1u) << 5 : 0u))
#define G_OF(T) (((unsigned)rank * 16u + g_of((T) < NF ? (T) : 2 * NF - 1 - (T), (T) < NF ? h : h ^ 1)) & 127u)
#define G_SH(u) (((unsigned)rank * 16u + g_of(NF + ((u) >> 1), (u) & 1)) & 127u)
    const unsigned g3 = G_OF(lane + (DYN ? 0 : 3)), g4 = G_OF(lane + 4);
    unsigned tAl, tAh, tBl, tBh, tFl, tFh, tTh, tTl, tTh2;
#define SPLIT64(lo, hi, v) do { const unsigned long long _v = (unsigned long long)(v); lo = (unsigned)_v; hi = (unsigned)(_v >> 32); PIN("+v"(lo), "+v"(hi)); } while (0)
    SPLIT64(tBl, tBh, wT + ((size_t)g3 << 20));
    SPLIT64(tFl, tFh, fRem + OFF_F(g4));
    const char* const aRem0 = myws + W_RECV0 + (size_t)pm * 32768;
    const unsigned long long tA0 = (unsigned long long)(lane < (DYN ? 8 : 5) ? aLocT + OFF_A(g3) : aRem0 + (((XQ & 64) ? 16u + (unsigned)h : g3) << 17));
    const unsigned gt = G_SH(lane & 15);
    const unsigned long long tT0 = (unsigned long long)((lane & 16) ? wT + ((size_t)gt << 20) : aRem0 + (gt << 17));
    unsigned tSo = sso, tSl, tSh;
    { const unsigned long long im = __builtin_amdgcn_ballot_w64(!sact);
      const unsigned r0 = __builtin_amdgcn_mbcnt_hi((unsigned)(im >> 32), __builtin_amdgcn_mbcnt_lo((unsigned)im, 0u)) - PF_R0 - (PFS > 1 ? (unsigned)idx % PFS : 0u), r = r0 / PFS;
      const bool pf = !sact && r0 < NPF * PFS && r0 % PFS == 0;
      const unsigned long long bs = pf ? (unsigned long long)wmat + ((size_t)((((unsigned)rank + 1u + (r >> 1)) & 7u) * 8u + ((7u + (1u + (r >> 1)) * ROT) & 7u)) << 21) : (unsigned long long)xloc;
      tSo = pf ? (ORD ? (unsigned)idx * 65536u + ((unsigned)xcd * 2u + (r & 1u)) * 128u : ((r & 1u) * 32u + (unsigned)idx) * 32768u + (unsigned)xcd * 256u) : tSo; tSl = (unsigned)bs; tSh = (unsigned)(bs >> 32) & 0xffffu; }
    tSo = SHL1(tSo, idleS); tSl = SHL1(tSl, dX[0]); tSh = SHL1(tSh, dX[1]);
    tSo = SHL1(tSo, idleS); tSl = SHL1(tSl, dX[0]); tSh = SHL1(tSh, dX[1]);
    unsigned tDo = sact ? (unsigned)(W_RECV0 + (size_t)rank * RECV_BLK) + (unsigned)sa * 262144u + (unsigned)srg * 2048u : nrec - 16u - min1;
    unsigned tFo = sact && wid == 0 ? (unsigned)W_XFLAG + (unsigned)((rank * 8 + sa) * 4 + (srg >> 4)) * 128u + (unsigned)(srg & 15) * 4u : nrec - 4u;
    tFo = SHR1(tFo, nrec - 4u);
    unsigned vone = lane ? IDLE : 0u, vl4 = (unsigned)lane * 4u, vm24 = 0xffffffu;
    PIN("+v"(tSo), "+v"(tDo), "+v"(tFo), "+v"(vone), "+v"(vl4), "+v"(vm24), "+v"(tSl), "+v"(tSh));
    const unsigned vzero = 0, fadr = mF + lane * 4;
    const u32x4 dR = SRD(myws + W_REC + (size_t)wid * 32768 + (size_t)bid * 128, 128u);
    const char* const recPair = myws + W_REC + (size_t)bpair * 128;
    const unsigned mwl = (unsigned)(unsigned long long)myws, mwh = (unsigned)((unsigned long long)myws >> 32) & 0xffffu;
    unsigned vdump = 0;
    const u32x4 dW = SRD(wT, (unsigned)(NN * NN * 2 - (pn >> 1) * 1024));
#define G_PF(T) ((T) < NF ? G_OF(T) : G_SH(((T) - NF) & 15))
#define CA 6
#define VS 4
    static_assert(PFQ == 3, "idle offset");
    const unsigned pfIdle = dW[2] - 4u - (unsigned)(((pm * 2 + (pn & 1)) * 4 + wid) * 2) * (NN * 2);
    const int sl = lane + 1, dn = sl < CA ? 0 : min((sl - CA) / VS + 1, VN);
    unsigned tPf = sl >= CA && (sl - CA) % VS == 0 && (sl - CA) / VS < VN ? pfIdle : G_PF(sl + PFD + (PF0 ? 3 : 0) - dn) << 20, oP = (g0 + 2u * PFD) << 20;
    static_assert(PFD >= 4 && PFD <= 7, "local K-tiles");
    const int j8 = pm * 2 + (pn & 1), ncc = 8 >> PFQ, rpw = 16 >> PFQ;
    unsigned vpf = lane < 16 ? (unsigned)(((j8 / ncc) * 4 + wid) * rpw + (lane >> PFQ)) * (NN * 2) + (unsigned)(((j8 % ncc) << PFQ) + (lane & ((1 << PFQ) - 1))) * 128u : IDLE;
    PIN("+v"(tPf), "+v"(vpf), "+s"(oP));
#define PFW asm volatile("buffer_load_dword %0, %1, %2, %3 offen" : "+v"(vdump) : "v"(vpf), "s"(dW), "s"(oP) : "memory")
    const unsigned vpq = (unsigned)((j8 * 4 + wid) * 2 + ((lane >> 3) & 1)) * (NN * 2) + (unsigned)(lane & 7) * 128u;
    unsigned vpf0 = vpq + (G_OF(PFD + (lane >> 4)) << 20); PIN("+v"(vpf0));
#define PFW0 asm volatile("buffer_load_dword %0, %1, %2, 0 offen" : "+v"(vdump) : "v"(vpf0), "s"(dW) : "memory")
    asm volatile("buffer_load_dword %0, %1, %2, %3 offen" : "+v"(vdump) : "v"(vpq + ((unsigned)(lane >> 4) << 21)), "s"(dW), "s"((g0 + 2u * (PFD - 4)) << 20) : "memory");
    asm volatile("buffer_load_dword %0, %1, %2, %3 offen" : "+v"(vdump) : "v"(vs0), "s"(dX), "s"(idleS + 12u) : "memory");
    asm volatile("buffer_load_dword %0, %1, %2, %3 offen" : "+v"(vdump) : "v"(vs0), "s"(dX), "s"(idleS + 12u) : "memory");
    S
    bf16x8 sel[2]; unsigned vo[2];
    { const unsigned e = (unsigned)(fr - 4 * fq), one = 0x3f80u << ((e & 1u) * 16u), w0 = (e >> 1) == 0 ? one : 0u, w1 = (e >> 1) == 1 ? one : 0u;
      sel[0] = __builtin_bit_cast(bf16x8, (u32x4){w0, w1, 0u, 0u}); sel[1] = __builtin_bit_cast(bf16x8, (u32x4){0u, 0u, w0, w1});
      vo[0] = (unsigned)(fr * NN * 2 + LINE_OF(wc) * 128 + (fq & 1) * 32 + (fq >> 1) * 16); vo[1] = vo[0] + (unsigned)((LINE_OF(2 + wc) - LINE_OF(wc)) * 128);
      PIN("+" CAo(sel[0]), "+" CAo(sel[1]), "+v"(vo[0]), "+v"(vo[1])); }
    S
    asm volatile("s_waitcnt vmcnt(%2)" : "+v"(seqv), "+v"(epw) : "n"(16 + 3) : "memory");
    S STAMPP(4); S
    BAR; asm volatile("" ::: "memory");
#pragma unroll
    for (int j = 0; j < 8; ++j) { LB0(0, vb[j >> 1], (j & 1) * 8192); LB1(0, Bf[j >> 1][j & 1], vb[j >> 1], (j & 1) * 8192 + 1024); LDA(Ao, j >> 1, j & 1, va, j * 1024); }
    S
    const unsigned seq = (unsigned)__builtin_amdgcn_readfirstlane(seqv) + 1u;
    if (tid == 0) asm volatile("global_store_dword %0, %1, off sc1" :: "v"(myws + W_SEQ + (size_t)bid * 128), "v"(seq) : "memory");
    const unsigned parAdd = (seq & 1u) ? (unsigned)(W_RECV1 - W_RECV0) : 0u;
    SPLIT64(tAl, tAh, tA0 + (lane < (DYN ? 8 : 5) ? 0u : parAdd));
    const unsigned tgt = DYN ? seq * 8u : seq;
    SPLIT64(tTl, tTh2, tT0 + ((lane & 16) ? 0u : parAdd));
    GAS unsigned* const tmo = (GAS unsigned*)(myws + W_TMO);
    unsigned tPl = (unsigned)epw, tPh = (unsigned)(epw >> 32) & 0xffffu, vseq = seq, vtag = seq << 24, vrec = 0, vst, oR = 0;
    unsigned tQl = SHR1(tPl, mwl), tQh = SHR1(tPh, mwh);
    tQl = wid ? mwl : tQl; tQh = wid ? mwh : tQh;
    tDo += sact ? parAdd : 0u;
    tTh = lane < 5 ? 0u : (lane == 5 || lane == T_SEND - 1 ? 0xffffffffu : tgt);
    PIN("+v"(tPl), "+v"(tPh), "+v"(tQl), "+v"(tQh), "+v"(tDo), "+v"(vseq), "+v"(vtag), "+v"(tTh));
    unsigned long long rt;
    u32x4 dD = {RL(tPl, 0), RL(tPh, 0), nrec, 0x20000u}, dF = {RL(tQl, 0), RL(tQh, 0), nrec, 0x20000u}, dS = {RL(tSl, 0), RL(tSh, 0), dX[2], 0x20000u};
#define RLM4(d, e, s) do { unsigned _s; RLM(_s, s); d[e] = _s; } while (0)
    RLS(oD, tDo, 0); RLS(oS, tSo, 0);
    RLS(oF, tFo, 0); PIN("+s"(dD), "+s"(dF), "+s"(dS), "+s"(oF), "+s"(oR));
    int exitT;
#define RT asm volatile("s_memrealtime %0" : "=s"(rt))
#define RTV asm volatile("v_and_or_b32 %0, %1, %2, %3" : "=v"(vst) : "s"((unsigned)rt), "v"(vm24), "v"(vtag))
#define RST asm volatile("buffer_store_dword %0, %1, %2, %3 offen sc0 sc1" :: "v"(vst), "v"(vone), "s"(dR), "s"(oR) : "memory")
#define RLD asm volatile("global_load_dword %0, %1, %2 sc0 sc1" : "=v"(vrec) : "v"(vl4), "s"(recPair) : "memory")
    unsigned pAl, pAh, pBl, pBh, pFl, pFh, thr;
    { const unsigned long long a = (unsigned long long)(aLoc + 512), b = (unsigned long long)(bLoc + ((size_t)2 << 21)), f = (unsigned long long)fRem;
      pAl = (unsigned)a; pAh = (unsigned)(a >> 32); pBl = (unsigned)b; pBh = (unsigned)(b >> 32); pFl = (unsigned)f; pFh = (unsigned)(f >> 32); }
    unsigned fv, fvN[2] = {0u, 0u};
#define GV(b) asm volatile("global_load_dword %0, %1, %2 sc0 sc1" : "=v"(fvN[b]) : "v"(vfl), "s"(pF) : "memory")
    unsigned long long fm;
    int t = 0;
    asm volatile("s_waitcnt lgkmcnt(0)" ::: "memory");
    BAR;
    STAMP(1);

#define DIAG_SLOW
#define WKIND(k)
#define SPIN_UNTIL(v, ptr, cond) do { const GAS unsigned* fp = (const GAS unsigned*)(ptr); const unsigned long long t0 = __builtin_amdgcn_s_memrealtime(); \
            while (v = __hip_atomic_load(fp, __ATOMIC_RELAXED, __HIP_MEMORY_SCOPE_SYSTEM), !__all(cond)) { \
                if (__builtin_amdgcn_readfirstlane(__hip_atomic_load(tmo, __ATOMIC_RELAXED, __HIP_MEMORY_SCOPE_AGENT)) == seq || __builtin_amdgcn_s_memrealtime() - t0 > DEADLINE_TICKS) { \
                    __hip_atomic_store(tmo, seq, __ATOMIC_RELAXED, __HIP_MEMORY_SCOPE_AGENT); break; } \
                __builtin_amdgcn_s_sleep(2); } \
            DIAG_SLOW; __builtin_amdgcn_s_waitcnt(0xC07F); asm volatile("" ::: "memory"); } while (0)
#define SPIN_FLAGS(ptr) SPIN_UNTIL(fv, ptr, fv >= tgt)
#define FLAGS_ONLY(b) do { if (UNLIKELY(!__all(fv >= seq))) SPIN_FLAGS(mk_ptr(RL(tFl, t - 2), RL(tFh, t - 2)) + vfl); } while (0)
#define SLOW (fm != 0)
        MFZ(a, 0, 0, 0, 0, Ao); S M0(mF); LB0(0, vb[0], 16384); S
        MF2(a, 0, 0, 0, 1, Ao); S GFLAG(vfl, pF); S
        MFZ(a, 0, 1, 0, 1, Ao); S M0(mB[0]); LB1(0, Bf[4][0], vb[0], 17408); S
        MF2(a, 0, 1, 0, 0, Ao); S GL(voffB[0], pB, 0); S
        MFZ(a, 0, 2, 0, 0, Ao); S LB0(0, vb[0], 24576); S
        MF2(a, 0, 2, 0, 1, Ao); S
        MFZ(a, 0, 3, 0, 1, Ao); S LB1(0, Bf[4][1], vb[0], 25600); S
        MF2(a, 0, 3, 0, 0, Ao); S TS4A; GL(voffB[1], pB, 1024); S
        MFZ(a, 0, 3, 1, 0, Ao); S LB0(0, vb[1], 16384); S
        MF2(a, 0, 3, 1, 1, Ao); S
        MFZ(a, 0, 2, 1, 1, Ao); S LB1(0, Bf[5][0], vb[1], 17408); S
        MF2(a, 0, 2, 1, 0, Ao); S GL(voffB[2], pB, 2048); S
        MFZ(a, 0, 1, 1, 0, Ao); S LB0(0, vb[1], 24576); S
        MF2(a, 0, 1, 1, 1, Ao); S
        MFZ(a, 0, 0, 1, 1, Ao); S LB1(0, Bf[5][1], vb[1], 25600); S
        MF2(a, 0, 0, 1, 0, Ao); S GL(voffB[3], pB, 3072); S
        MFZ(a, 0, 0, 2, 0, Ao); S LB0(0, vb[2], 16384); S
        MF2(a, 0, 0, 2, 1, Ao); S
        MFZ(a, 0, 1, 2, 1, Ao); S M0(mA[0]); LB1(0, Bf[6][0], vb[2], 17408); S
        MF2(a, 0, 1, 2, 0, Ao); S GL(voffA[0], pA, 0); S
        MFZ(a, 0, 2, 2, 0, Ao); S LB0(0, vb[2], 24576); S
        MF2(a, 0, 2, 2, 1, Ao); S
        MFZ(a, 0, 3, 2, 1, Ao); S LB1(0, Bf[6][1], vb[2], 25600); S
        MF2(a, 0, 3, 2, 0, Ao); S GL(voffA[1], pA, 1024); S
        MFZ(a, 0, 3, 3, 0, Ao); S LB0(0, vb[3], 16384); S
        MF2(a, 0, 3, 3, 1, Ao); S
        MFZ(a, 0, 2, 3, 1, Ao); S LB1(0, Bf[7][0], vb[3], 17408); S
        MF2(a, 0, 2, 3, 0, Ao); S GL(voffA[2], pA, 2048); S
        MFZ(a, 0, 1, 3, 0, Ao); S LB0(0, vb[3], 24576); S
        MF2(a, 0, 1, 3, 1, Ao); S
        MFZ(a, 0, 0, 3, 1, Ao); S LB1(0, Bf[7][1], vb[3], 25600); S
        MF2(a, 0, 0, 3, 0, Ao); S WL(12); GL(voffA[3], pA, 3072); S
        MFZ(a, 0, 0, 4, 0, Ao); S
        MF2(a, 0, 0, 4, 1, Ao); S LDA(Ap, 0, 0, va, 16384); S
        MFZ(a, 0, 1, 4, 1, Ao); S
        MF2(a, 0, 1, 4, 0, Ao); S PFW; S
        MFZ(a, 0, 2, 4, 0, Ao); S LDA(Ap, 0, 1, va, 17408); S
        MF2(a, 0, 2, 4, 1, Ao); S
        MFZ(a, 0, 3, 4, 1, Ao); S
        MF2(a, 0, 3, 4, 0, Ao); S WL(10); LDA(Ap, 1, 0, va, 18432); SST(0); S
        MFZ(a, 0, 3, 5, 0, Ao); S
        MF2(a, 0, 3, 5, 1, Ao); S
        MFZ(a, 0, 2, 5, 1, Ao); S LDA(Ap, 1, 1, va, 19456); S
        MF2(a, 0, 2, 5, 0, Ao); S
        MFZ(a, 0, 1, 5, 0, Ao); S SLD(0); S
        MF2(a, 0, 1, 5, 1, Ao); S LDA(Ap, 2, 0, va, 20480); S
        MFZ(a, 0, 0, 5, 1, Ao); S
        MF2(a, 0, 0, 5, 0, Ao); S WL(9); S
        MFZ(a, 0, 0, 6, 0, Ao); S LDA(Ap, 2, 1, va, 21504); S
        MF2(a, 0, 0, 6, 1, Ao); S
        MFZ(a, 0, 1, 6, 1, Ao); S
        MF2(a, 0, 1, 6, 0, Ao); S LDA(Ap, 3, 0, va, 22528); S
        MFZ(a, 0, 2, 6, 0, Ao); S
        MF2(a, 0, 2, 6, 1, Ao); S
        MFZ(a, 0, 3, 6, 1, Ao); S LDA(Ap, 3, 1, va, 23552); S
        MF2(a, 0, 3, 6, 0, Ao); S WL(8); S
        MFZ(a, 0, 3, 7, 0, Ao); S
        MF2(a, 0, 3, 7, 1, Ao); S
        MFZ(a, 0, 2, 7, 1, Ao); S
        MF2(a, 0, 2, 7, 0, Ao); S WVL(23); S
        MFZ(a, 0, 1, 7, 0, Ao); S
        MF2(a, 0, 1, 7, 1, Ao); S BAR; S
        MFZ(a, 0, 0, 7, 1, Ao); S TS4B(tsX); S
        MF2(a, 0, 0, 7, 0, Ao); S
        MFZ(a, 1, 0, 0, 0, Ap); S M0(mB[1]); SFL; S
        MF2(a, 1, 0, 0, 1, Ap); S GL(voffB[4], pB, 0); S
        MFZ(a, 1, 1, 0, 1, Ap); S
        MF2(a, 1, 1, 0, 0, Ap); S RT; LDA(Ao, 0, 0, va, 32768); S
        MFZ(a, 1, 2, 0, 0, Ap); S
        MF2(a, 1, 2, 0, 1, Ap); S GL(voffB[5], pB, 1024); LDA(Ao, 0, 1, va, 33792); S
        MFZ(a, 1, 3, 0, 1, Ap); S
        MF2(a, 1, 3, 0, 0, Ap); S TS5A; LDA(Ao, 1, 0, va, 34816); S
        MFZ(a, 1, 3, 1, 0, Ap); S
        MF2(a, 1, 3, 1, 1, Ap); S GL(voffB[6], pB, 2048); LDA(Ao, 1, 1, va, 35840); S
        MFZ(a, 1, 2, 1, 1, Ap); S
        MF2(a, 1, 2, 1, 0, Ap); S LB0(0, vb[0], 32768); S
        MFZ(a, 1, 1, 1, 0, Ap); S LDA(Ao, 2, 0, va, 36864); S
        MF2(a, 1, 1, 1, 1, Ap); S GL(voffB[7], pB, 3072); LB1(0, Bf[0][0], vb[0], 33792); S
        MFZ(a, 1, 0, 1, 1, Ap); S LDA(Ao, 2, 1, va, 37888); S
        MF2(a, 1, 0, 1, 0, Ap); S LB0(0, vb[0], 40960); S
        MFZ(a, 1, 0, 2, 0, Ap); S M0(mA[1]); LDA(Ao, 3, 0, va, 38912); S
        MF2(a, 1, 0, 2, 1, Ap); S GL(voffP[0], pA, 0); LB1(0, Bf[0][1], vb[0], 41984); S
        MFZ(a, 1, 1, 2, 1, Ap); S LDA(Ao, 3, 1, va, 39936); S
        MF2(a, 1, 1, 2, 0, Ap); S LB0(0, vb[1], 32768); S
        MFZ(a, 1, 2, 2, 0, Ap); S
        MF2(a, 1, 2, 2, 1, Ap); S GL(voffP[1], pA, 1024); LB1(0, Bf[1][0], vb[1], 33792); S
        MFZ(a, 1, 3, 2, 1, Ap); S
        MF2(a, 1, 3, 2, 0, Ap); S LB0(0, vb[1], 40960); S
        MFZ(a, 1, 3, 3, 0, Ap); S
        MF2(a, 1, 3, 3, 1, Ap); S GL(voffP[2], pA, 2048); LB1(0, Bf[1][1], vb[1], 41984); S
        MFZ(a, 1, 2, 3, 1, Ap); S
        MF2(a, 1, 2, 3, 0, Ap); S LB0(0, vb[2], 32768); S
        MFZ(a, 1, 1, 3, 0, Ap); S
        MF2(a, 1, 1, 3, 1, Ap); S GL(voffP[3], pA, 3072); LB1(0, Bf[2][0], vb[2], 33792); S
        MFZ(a, 1, 0, 3, 1, Ap); S M0(t); S
        MF2(a, 1, 0, 3, 0, Ap); S ++t; PIN("+s"(t)); LB0(0, vb[2], 40960); RLM(pAl, tAl); S
        MFZ(a, 1, 0, 4, 0, Ap); S RLM(pAh, tAh); RLM(pBl, tBl); S
        MF2(a, 1, 0, 4, 1, Ap); S LB1(0, Bf[2][1], vb[2], 41984); RLM(pBh, tBh); S
        MFZ(a, 1, 1, 4, 1, Ap); S TS6A; RLM(pFl, tFl); RLM(pFh, tFh); S
        MF2(a, 1, 1, 4, 0, Ap); S LB0(0, vb[3], 32768); RLM(thr, tTh); S
        MFZ(a, 1, 2, 4, 0, Ap); S RLM(oP, tPf); S
        MF2(a, 1, 2, 4, 1, Ap); S M0(t); LB1(0, Bf[3][0], vb[3], 33792); S
        MFZ(a, 1, 3, 4, 1, Ap); S RLM4(dD, 0, tPl); RLM(oD, tDo); S
        MF2(a, 1, 3, 4, 0, Ap); S LB0(0, vb[3], 40960); RLM4(dD, 1, tPh); S
        MFZ(a, 1, 3, 5, 0, Ap); S RLM4(dF, 0, tQl); RLM4(dS, 0, tSl); S
        MF2(a, 1, 3, 5, 1, Ap); S LB1(0, Bf[3][1], vb[3], 41984); RLM4(dF, 1, tQh); S
        MFZ(a, 1, 2, 5, 1, Ap); S RLM4(dS, 1, tSh); RLM(oF, tFo); S
        MF2(a, 1, 2, 5, 0, Ap); S RLM(oS, tSo); S
        MFZ(a, 1, 1, 5, 0, Ap); S
        MF2(a, 1, 1, 5, 1, Ap); S
        MFZ(a, 1, 0, 5, 1, Ap); S
        MF2(a, 1, 0, 5, 0, Ap); S
        MFZ(a, 1, 0, 6, 0, Ap); S
        MF2(a, 1, 0, 6, 1, Ap); S TS6B; S
        MFZ(a, 1, 1, 6, 1, Ap); S WV(20); S
        MF2(a, 1, 1, 6, 0, Ap); S LDF; S
        MFZ(a, 1, 2, 6, 0, Ap); S
        MF2(a, 1, 2, 6, 1, Ap); S
        MFZ(a, 1, 3, 6, 1, Ap); S
        MF2(a, 1, 3, 6, 0, Ap); S
        MFZ(a, 1, 3, 7, 0, Ap); S
        MF2(a, 1, 3, 7, 1, Ap); S
        MFZ(a, 1, 2, 7, 1, Ap); S
        MF2(a, 1, 2, 7, 0, Ap); S WLF; S
        MFZ(a, 1, 1, 7, 0, Ap); S fm = __builtin_amdgcn_ballot_w64(fv < thr); PIN("+s"(fm)); S
        MF2(a, 1, 1, 7, 1, Ap); S BAR; S
        MFZ(a, 1, 0, 7, 1, Ap); S TS6C; TS4B(tsY); S
        MF2(a, 1, 0, 7, 0, Ap); S

    fv = 0xffffffffu; fvN[1] = 0xffffffffu;
    do {
        FLAGS_ONLY(1);
        do {
        MF1(a, 0, 0, 0, 0, Ao); S LB0(0, vb[0], 49152); S
        MF2(a, 0, 0, 0, 1, Ao); S RLD; S
        MF1(a, 0, 1, 0, 1, Ao); S M0(mF); LB1(0, Bf[4][0], vb[0], 50176); S
        MF2(a, 0, 1, 0, 0, Ao); S GFLAG(vfl, pF); S
        MF1(a, 0, 2, 0, 0, Ao); S M0(mB[2]); LB0(0, vb[0], 57344); S
        MF2(a, 0, 2, 0, 1, Ao); S GL(voffB[0], pB, 0); S
        MF1(a, 0, 3, 0, 1, Ao); S LB1(0, Bf[4][1], vb[0], 58368); S
        MF2(a, 0, 3, 0, 0, Ao); S TS4A; S
        MF1(a, 0, 3, 1, 0, Ao); S LB0(0, vb[1], 49152); S
        MF2(a, 0, 3, 1, 1, Ao); S GL(voffB[1], pB, 1024); S
        MF1(a, 0, 2, 1, 1, Ao); S LB1(0, Bf[5][0], vb[1], 50176); S
        MF2(a, 0, 2, 1, 0, Ao); S
        MF1(a, 0, 1, 1, 0, Ao); S LB0(0, vb[1], 57344); S
        MF2(a, 0, 1, 1, 1, Ao); S GL(voffB[2], pB, 2048); S
        MF1(a, 0, 0, 1, 1, Ao); S LB1(0, Bf[5][1], vb[1], 58368); S
        MF2(a, 0, 0, 1, 0, Ao); S
        MF1(a, 0, 0, 2, 0, Ao); S LB0(0, vb[2], 49152); S
        MF2(a, 0, 0, 2, 1, Ao); S GL(voffB[3], pB, 3072); S
        MF1(a, 0, 1, 2, 1, Ao); S LB1(0, Bf[6][0], vb[2], 50176); S
        MF2(a, 0, 1, 2, 0, Ao); S
        MF1(a, 0, 2, 2, 0, Ao); S M0(mA[2]); LB0(0, vb[2], 57344); S
        MF2(a, 0, 2, 2, 1, Ao); S GL(voffA[0], pA, 0); S
        MF1(a, 0, 3, 2, 1, Ao); S LB1(0, Bf[6][1], vb[2], 58368); S
        MF2(a, 0, 3, 2, 0, Ao); S
        MF1(a, 0, 3, 3, 0, Ao); S LB0(0, vb[3], 49152); S
        MF2(a, 0, 3, 3, 1, Ao); S GL(voffA[1], pA, 1024); S
        MF1(a, 0, 2, 3, 1, Ao); S LB1(0, Bf[7][0], vb[3], 50176); S
        MF2(a, 0, 2, 3, 0, Ao); S
        MF1(a, 0, 1, 3, 0, Ao); S LB0(0, vb[3], 57344); S
        MF2(a, 0, 1, 3, 1, Ao); S GL(voffA[2], pA, 2048); S
        MF1(a, 0, 0, 3, 1, Ao); S LB1(0, Bf[7][1], vb[3], 58368); S
        MF2(a, 0, 0, 3, 0, Ao); S WL(12); S
        MF1(a, 0, 0, 4, 0, Ao); S
        MF2(a, 0, 0, 4, 1, Ao); S GL(voffA[3], pA, 3072); LDA(Ap, 0, 0, va, 49152); S
        MF1(a, 0, 1, 4, 1, Ao); S
        MF2(a, 0, 1, 4, 0, Ao); S
        MF1(a, 0, 2, 4, 0, Ao); S LDA(Ap, 0, 1, va, 50176); S
        MF2(a, 0, 2, 4, 1, Ao); S PFW; S
        MF1(a, 0, 3, 4, 1, Ao); S
        MF2(a, 0, 3, 4, 0, Ao); S WL(10); LDA(Ap, 1, 0, va, 51200); S
        MF1(a, 0, 3, 5, 0, Ao); S
        MF2(a, 0, 3, 5, 1, Ao); S SST(1); S
        MF1(a, 0, 2, 5, 1, Ao); S LDA(Ap, 1, 1, va, 52224); S
        MF2(a, 0, 2, 5, 0, Ao); S
        MF1(a, 0, 1, 5, 0, Ao); S
        MF2(a, 0, 1, 5, 1, Ao); S LDA(Ap, 2, 0, va, 53248); SLD(1); S
        MF1(a, 0, 0, 5, 1, Ao); S
        MF2(a, 0, 0, 5, 0, Ao); S WL(9); S
        MF1(a, 0, 0, 6, 0, Ao); S LDA(Ap, 2, 1, va, 54272); S
        MF2(a, 0, 0, 6, 1, Ao); S
        MF1(a, 0, 1, 6, 1, Ao); S
        MF2(a, 0, 1, 6, 0, Ao); S LDA(Ap, 3, 0, va, 55296); S
        MF1(a, 0, 2, 6, 0, Ao); S
        MF2(a, 0, 2, 6, 1, Ao); S
        MF1(a, 0, 3, 6, 1, Ao); S LDA(Ap, 3, 1, va, 56320); S
        MF2(a, 0, 3, 6, 0, Ao); S WL(8); S
        MF1(a, 0, 3, 7, 0, Ao); S
        MF2(a, 0, 3, 7, 1, Ao); S
        MF1(a, 0, 2, 7, 1, Ao); S
        MF2(a, 0, 2, 7, 0, Ao); S WVL(23); S
        MF1(a, 0, 1, 7, 0, Ao); S
        MF2(a, 0, 1, 7, 1, Ao); S BAR; S
        MF1(a, 0, 0, 7, 1, Ao); S TS4B(tsX); S
        MF2(a, 0, 0, 7, 0, Ao); S
        MF1(a, 1, 0, 0, 0, Ap); S M0(mB[3]); SFL; S
        MF2(a, 1, 0, 0, 1, Ap); S GL(voffB[4], pB, 0); S
        MF1(a, 1, 1, 0, 1, Ap); S
        MF2(a, 1, 1, 0, 0, Ap); S LDA(Ao, 0, 0, va, 0); S
        MF1(a, 1, 2, 0, 0, Ap); S
        MF2(a, 1, 2, 0, 1, Ap); S GL(voffB[5], pB, 1024); LDA(Ao, 0, 1, va, 1024); S
        MF1(a, 1, 3, 0, 1, Ap); S
        MF2(a, 1, 3, 0, 0, Ap); S TS5A; LDA(Ao, 1, 0, va, 2048); S
        MF1(a, 1, 3, 1, 0, Ap); S
        MF2(a, 1, 3, 1, 1, Ap); S GL(voffB[6], pB, 2048); LDA(Ao, 1, 1, va, 3072); S
        MF1(a, 1, 2, 1, 1, Ap); S
        MF2(a, 1, 2, 1, 0, Ap); S LB0(0, vb[0], 0); S
        MF1(a, 1, 1, 1, 0, Ap); S LDA(Ao, 2, 0, va, 4096); S
        MF2(a, 1, 1, 1, 1, Ap); S GL(voffB[7], pB, 3072); LB1(0, Bf[0][0], vb[0], 1024); S
        MF1(a, 1, 0, 1, 1, Ap); S LDA(Ao, 2, 1, va, 5120); S
        MF2(a, 1, 0, 1, 0, Ap); S LB0(0, vb[0], 8192); S
        MF1(a, 1, 0, 2, 0, Ap); S M0(mA[3]); LDA(Ao, 3, 0, va, 6144); S
        MF2(a, 1, 0, 2, 1, Ap); S GL(voffP[0], pA, 0); LB1(0, Bf[0][1], vb[0], 9216); S
        MF1(a, 1, 1, 2, 1, Ap); S LDA(Ao, 3, 1, va, 7168); S
        MF2(a, 1, 1, 2, 0, Ap); S LB0(0, vb[1], 0); S
        MF1(a, 1, 2, 2, 0, Ap); S
        MF2(a, 1, 2, 2, 1, Ap); S GL(voffP[1], pA, 1024); LB1(0, Bf[1][0], vb[1], 1024); S
        MF1(a, 1, 3, 2, 1, Ap); S
        MF2(a, 1, 3, 2, 0, Ap); S LB0(0, vb[1], 8192); S
        MF1(a, 1, 3, 3, 0, Ap); S
        MF2(a, 1, 3, 3, 1, Ap); S GL(voffP[2], pA, 2048); LB1(0, Bf[1][1], vb[1], 9216); S
        MF1(a, 1, 2, 3, 1, Ap); S
        MF2(a, 1, 2, 3, 0, Ap); S LB0(0, vb[2], 0); S
        MF1(a, 1, 1, 3, 0, Ap); S
        MF2(a, 1, 1, 3, 1, Ap); S GL(voffP[3], pA, 3072); LB1(0, Bf[2][0], vb[2], 1024); S
        MF1(a, 1, 0, 3, 1, Ap); S RTV; S
        MF2(a, 1, 0, 3, 0, Ap); S LB0(0, vb[2], 8192); S
        MF1(a, 1, 0, 4, 0, Ap); S RST; S
        MF2(a, 1, 0, 4, 1, Ap); S oR += 4; PIN("+s"(oR)); M0(t); LB1(0, Bf[2][1], vb[2], 9216); S
        MF1(a, 1, 1, 4, 1, Ap); S TS6A; ++t; PIN("+s"(t)); RLM(pAl, tAl); RLM(pAh, tAh); S
        MF2(a, 1, 1, 4, 0, Ap); S LB0(0, vb[3], 0); RLM(pBl, tBl); S
        MF1(a, 1, 2, 4, 0, Ap); S RLM(pBh, tBh); RLM(pFl, tFl); S
        MF2(a, 1, 2, 4, 1, Ap); S LB1(0, Bf[3][0], vb[3], 1024); RLM(pFh, tFh); S
        MF1(a, 1, 3, 4, 1, Ap); S RLM(thr, tTh); RLM(oP, tPf); S
        MF2(a, 1, 3, 4, 0, Ap); S M0(t); LB0(0, vb[3], 8192); S
        MF1(a, 1, 3, 5, 0, Ap); S RLM4(dD, 0, tPl); RLM(oD, tDo); S
        MF2(a, 1, 3, 5, 1, Ap); S LB1(0, Bf[3][1], vb[3], 9216); RLM4(dD, 1, tPh); S
        MF1(a, 1, 2, 5, 1, Ap); S RLM4(dF, 0, tQl); RLM4(dS, 0, tSl); S
        MF2(a, 1, 2, 5, 0, Ap); S RLM4(dF, 1, tQh); RLM4(dS, 1, tSh); S
        MF1(a, 1, 1, 5, 0, Ap); S RLM(oF, tFo); RLM(oS, tSo); S
        MF2(a, 1, 1, 5, 1, Ap); S
        MF1(a, 1, 0, 5, 1, Ap); S
        MF2(a, 1, 0, 5, 0, Ap); S
        MF1(a, 1, 0, 6, 0, Ap); S
        MF2(a, 1, 0, 6, 1, Ap); S TS6B; S
        MF1(a, 1, 1, 6, 1, Ap); S WV(21); S
        MF2(a, 1, 1, 6, 0, Ap); S LDF; S
        MF1(a, 1, 2, 6, 0, Ap); S
        MF2(a, 1, 2, 6, 1, Ap); S
        MF1(a, 1, 3, 6, 1, Ap); S
        MF2(a, 1, 3, 6, 0, Ap); S
        MF1(a, 1, 3, 7, 0, Ap); S
        MF2(a, 1, 3, 7, 1, Ap); S
        MF1(a, 1, 2, 7, 1, Ap); S
        MF2(a, 1, 2, 7, 0, Ap); S WLF; S
        MF1(a, 1, 1, 7, 0, Ap); S fm = __builtin_amdgcn_ballot_w64(fv < thr); PIN("+s"(fm)); S
        MF2(a, 1, 1, 7, 1, Ap); S BAR; S
        MF1(a, 1, 0, 7, 1, Ap); S TS6C; TS4B(tsY); S
        MF2(a, 1, 0, 7, 0, Ap); S

            if (UNLIKELY(SLOW)) {
#pragma unroll
                for (int i = 0; i < 4; ++i) { voffA[i] = vl4 * 4u + (unsigned)(wid * 4096 + h * HTB); voffP[i] = vl4 * 4u + (unsigned)(wid * 4096 + (h ^ 1) * HTB); PIN("+v"(voffA[i]), "+v"(voffP[i])); }
                FLAGS_ONLY(0);
            }
        MF1(a, 0, 0, 0, 0, Ao); S M0(mF); LB0(0, vb[0], 16384); S
        MF2(a, 0, 0, 0, 1, Ao); S GFLAG(vfl, pF); S
        MF1(a, 0, 1, 0, 1, Ao); S M0(mB[0]); LB1(0, Bf[4][0], vb[0], 17408); S
        MF2(a, 0, 1, 0, 0, Ao); S GL(voffB[0], pB, 0); S
        MF1(a, 0, 2, 0, 0, Ao); S LB0(0, vb[0], 24576); S
        MF2(a, 0, 2, 0, 1, Ao); S
        MF1(a, 0, 3, 0, 1, Ao); S LB1(0, Bf[4][1], vb[0], 25600); S
        MF2(a, 0, 3, 0, 0, Ao); S TS4A; GL(voffB[1], pB, 1024); S
        MF1(a, 0, 3, 1, 0, Ao); S LB0(0, vb[1], 16384); S
        MF2(a, 0, 3, 1, 1, Ao); S
        MF1(a, 0, 2, 1, 1, Ao); S LB1(0, Bf[5][0], vb[1], 17408); S
        MF2(a, 0, 2, 1, 0, Ao); S GL(voffB[2], pB, 2048); S
        MF1(a, 0, 1, 1, 0, Ao); S LB0(0, vb[1], 24576); S
        MF2(a, 0, 1, 1, 1, Ao); S
        MF1(a, 0, 0, 1, 1, Ao); S LB1(0, Bf[5][1], vb[1], 25600); S
        MF2(a, 0, 0, 1, 0, Ao); S GL(voffB[3], pB, 3072); S
        MF1(a, 0, 0, 2, 0, Ao); S LB0(0, vb[2], 16384); S
        MF2(a, 0, 0, 2, 1, Ao); S
        MF1(a, 0, 1, 2, 1, Ao); S M0(mA[0]); LB1(0, Bf[6][0], vb[2], 17408); S
        MF2(a, 0, 1, 2, 0, Ao); S GL(voffA[0], pA, 0); S
        MF1(a, 0, 2, 2, 0, Ao); S LB0(0, vb[2], 24576); S
        MF2(a, 0, 2, 2, 1, Ao); S
        MF1(a, 0, 3, 2, 1, Ao); S LB1(0, Bf[6][1], vb[2], 25600); S
        MF2(a, 0, 3, 2, 0, Ao); S GL(voffA[1], pA, 1024); S
        MF1(a, 0, 3, 3, 0, Ao); S LB0(0, vb[3], 16384); S
        MF2(a, 0, 3, 3, 1, Ao); S
        MF1(a, 0, 2, 3, 1, Ao); S LB1(0, Bf[7][0], vb[3], 17408); S
        MF2(a, 0, 2, 3, 0, Ao); S GL(voffA[2], pA, 2048); S
        MF1(a, 0, 1, 3, 0, Ao); S LB0(0, vb[3], 24576); S
        MF2(a, 0, 1, 3, 1, Ao); S
        MF1(a, 0, 0, 3, 1, Ao); S LB1(0, Bf[7][1], vb[3], 25600); S
        MF2(a, 0, 0, 3, 0, Ao); S WL(12); GL(voffA[3], pA, 3072); S
        MF1(a, 0, 0, 4, 0, Ao); S
        MF2(a, 0, 0, 4, 1, Ao); S LDA(Ap, 0, 0, va, 16384); S
        MF1(a, 0, 1, 4, 1, Ao); S
        MF2(a, 0, 1, 4, 0, Ao); S PFW; S
        MF1(a, 0, 2, 4, 0, Ao); S LDA(Ap, 0, 1, va, 17408); S
        MF2(a, 0, 2, 4, 1, Ao); S
        MF1(a, 0, 3, 4, 1, Ao); S
        MF2(a, 0, 3, 4, 0, Ao); S WL(10); LDA(Ap, 1, 0, va, 18432); SST(0); S
        MF1(a, 0, 3, 5, 0, Ao); S
        MF2(a, 0, 3, 5, 1, Ao); S
        MF1(a, 0, 2, 5, 1, Ao); S LDA(Ap, 1, 1, va, 19456); S
        MF2(a, 0, 2, 5, 0, Ao); S
        MF1(a, 0, 1, 5, 0, Ao); S SLD(0); S
        MF2(a, 0, 1, 5, 1, Ao); S LDA(Ap, 2, 0, va, 20480); S
        MF1(a, 0, 0, 5, 1, Ao); S
        MF2(a, 0, 0, 5, 0, Ao); S WL(9); S
        MF1(a, 0, 0, 6, 0, Ao); S LDA(Ap, 2, 1, va, 21504); S
        MF2(a, 0, 0, 6, 1, Ao); S
        MF1(a, 0, 1, 6, 1, Ao); S
        MF2(a, 0, 1, 6, 0, Ao); S LDA(Ap, 3, 0, va, 22528); S
        MF1(a, 0, 2, 6, 0, Ao); S
        MF2(a, 0, 2, 6, 1, Ao); S
        MF1(a, 0, 3, 6, 1, Ao); S LDA(Ap, 3, 1, va, 23552); S
        MF2(a, 0, 3, 6, 0, Ao); S WL(8); S
        MF1(a, 0, 3, 7, 0, Ao); S
        MF2(a, 0, 3, 7, 1, Ao); S
        MF1(a, 0, 2, 7, 1, Ao); S
        MF2(a, 0, 2, 7, 0, Ao); S WVL(23); S
        MF1(a, 0, 1, 7, 0, Ao); S
        MF2(a, 0, 1, 7, 1, Ao); S BAR; S
        MF1(a, 0, 0, 7, 1, Ao); S TS4B(tsX); S
        MF2(a, 0, 0, 7, 0, Ao); S
        MF1(a, 1, 0, 0, 0, Ap); S M0(mB[1]); SFL; S
        MF2(a, 1, 0, 0, 1, Ap); S GL(voffB[4], pB, 0); S
        MF1(a, 1, 1, 0, 1, Ap); S
        MF2(a, 1, 1, 0, 0, Ap); S RT; LDA(Ao, 0, 0, va, 32768); S
        MF1(a, 1, 2, 0, 0, Ap); S
        MF2(a, 1, 2, 0, 1, Ap); S GL(voffB[5], pB, 1024); LDA(Ao, 0, 1, va, 33792); S
        MF1(a, 1, 3, 0, 1, Ap); S
        MF2(a, 1, 3, 0, 0, Ap); S TS5A; LDA(Ao, 1, 0, va, 34816); S
        MF1(a, 1, 3, 1, 0, Ap); S
        MF2(a, 1, 3, 1, 1, Ap); S GL(voffB[6], pB, 2048); LDA(Ao, 1, 1, va, 35840); S
        MF1(a, 1, 2, 1, 1, Ap); S
        MF2(a, 1, 2, 1, 0, Ap); S LB0(0, vb[0], 32768); S
        MF1(a, 1, 1, 1, 0, Ap); S LDA(Ao, 2, 0, va, 36864); S
        MF2(a, 1, 1, 1, 1, Ap); S GL(voffB[7], pB, 3072); LB1(0, Bf[0][0], vb[0], 33792); S
        MF1(a, 1, 0, 1, 1, Ap); S LDA(Ao, 2, 1, va, 37888); S
        MF2(a, 1, 0, 1, 0, Ap); S LB0(0, vb[0], 40960); S
        MF1(a, 1, 0, 2, 0, Ap); S M0(mA[1]); LDA(Ao, 3, 0, va, 38912); S
        MF2(a, 1, 0, 2, 1, Ap); S GL(voffP[0], pA, 0); LB1(0, Bf[0][1], vb[0], 41984); S
        MF1(a, 1, 1, 2, 1, Ap); S LDA(Ao, 3, 1, va, 39936); S
        MF2(a, 1, 1, 2, 0, Ap); S LB0(0, vb[1], 32768); S
        MF1(a, 1, 2, 2, 0, Ap); S
        MF2(a, 1, 2, 2, 1, Ap); S GL(voffP[1], pA, 1024); LB1(0, Bf[1][0], vb[1], 33792); S
        MF1(a, 1, 3, 2, 1, Ap); S
        MF2(a, 1, 3, 2, 0, Ap); S LB0(0, vb[1], 40960); S
        MF1(a, 1, 3, 3, 0, Ap); S
        MF2(a, 1, 3, 3, 1, Ap); S GL(voffP[2], pA, 2048); LB1(0, Bf[1][1], vb[1], 41984); S
        MF1(a, 1, 2, 3, 1, Ap); S
        MF2(a, 1, 2, 3, 0, Ap); S LB0(0, vb[2], 32768); S
        MF1(a, 1, 1, 3, 0, Ap); S
        MF2(a, 1, 1, 3, 1, Ap); S GL(voffP[3], pA, 3072); LB1(0, Bf[2][0], vb[2], 33792); S
        MF1(a, 1, 0, 3, 1, Ap); S M0(t); S
        MF2(a, 1, 0, 3, 0, Ap); S ++t; PIN("+s"(t)); LB0(0, vb[2], 40960); RLM(pAl, tAl); S
        MF1(a, 1, 0, 4, 0, Ap); S RLM(pAh, tAh); RLM(pBl, tBl); S
        MF2(a, 1, 0, 4, 1, Ap); S LB1(0, Bf[2][1], vb[2], 41984); RLM(pBh, tBh); S
        MF1(a, 1, 1, 4, 1, Ap); S TS6A; RLM(pFl, tFl); RLM(pFh, tFh); S
        MF2(a, 1, 1, 4, 0, Ap); S LB0(0, vb[3], 32768); RLM(thr, tTh); S
        MF1(a, 1, 2, 4, 0, Ap); S RLM(oP, tPf); S
        MF2(a, 1, 2, 4, 1, Ap); S M0(t); LB1(0, Bf[3][0], vb[3], 33792); S
        MF1(a, 1, 3, 4, 1, Ap); S RLM4(dD, 0, tPl); RLM(oD, tDo); S
        MF2(a, 1, 3, 4, 0, Ap); S LB0(0, vb[3], 40960); RLM4(dD, 1, tPh); S
        MF1(a, 1, 3, 5, 0, Ap); S RLM4(dF, 0, tQl); RLM4(dS, 0, tSl); S
        MF2(a, 1, 3, 5, 1, Ap); S LB1(0, Bf[3][1], vb[3], 41984); RLM4(dF, 1, tQh); S
        MF1(a, 1, 2, 5, 1, Ap); S RLM4(dS, 1, tSh); RLM(oF, tFo); S
        MF2(a, 1, 2, 5, 0, Ap); S RLM(oS, tSo); S
        MF1(a, 1, 1, 5, 0, Ap); S
        MF2(a, 1, 1, 5, 1, Ap); S
        MF1(a, 1, 0, 5, 1, Ap); S
        MF2(a, 1, 0, 5, 0, Ap); S
        MF1(a, 1, 0, 6, 0, Ap); S
        MF2(a, 1, 0, 6, 1, Ap); S TS6B; S
        MF1(a, 1, 1, 6, 1, Ap); S WV(20); S
        MF2(a, 1, 1, 6, 0, Ap); S LDF; S
        MF1(a, 1, 2, 6, 0, Ap); S
        MF2(a, 1, 2, 6, 1, Ap); S
        MF1(a, 1, 3, 6, 1, Ap); S
        MF2(a, 1, 3, 6, 0, Ap); S
        MF1(a, 1, 3, 7, 0, Ap); S
        MF2(a, 1, 3, 7, 1, Ap); S
        MF1(a, 1, 2, 7, 1, Ap); S
        MF2(a, 1, 2, 7, 0, Ap); S WLF; S
        MF1(a, 1, 1, 7, 0, Ap); S fm = __builtin_amdgcn_ballot_w64(fv < thr); PIN("+s"(fm)); S
        MF2(a, 1, 1, 7, 1, Ap); S BAR; S
        MF1(a, 1, 0, 7, 1, Ap); S TS6C; TS4B(tsY); S
        MF2(a, 1, 0, 7, 0, Ap); S

        } while (!SLOW);
    } while (t < T_SEND);
    {
        PIN("+v"(vrec)); WKIND(2);
        constexpr unsigned long long need = (1ull << BAL_I1) | (1ull << BAL_I2) | (1ull << (32 + BAL_I1)) | (1ull << (32 + BAL_I2));
        if (UNLIKELY((__builtin_amdgcn_ballot_w64((vrec >> 24) != (seq & 255u)) & need) != 0))
            SPIN_UNTIL(vrec, recPair + vl4, (vrec >> 24) == (seq & 255u) || !((need >> lane) & 1));
        const unsigned a0 = RL(vrec, BAL_I1), b0 = RL(vrec, BAL_I2), a1 = RL(vrec, 32 + BAL_I1), b1 = RL(vrec, 32 + BAL_I2);
        const int d2 = (int)((a1 - a0) << 8) >> 8, d6 = (int)((b1 - b0) << 8) >> 8, dur = (int)(((b0 - a0) & 0xffffffu) + ((b1 - a1) & 0xffffffu));
        constexpr int t2 = 2 * BAL_I1 + 1 + TSO, t6 = 2 * BAL_I2 + 1 + TSO;
        const int num = 2 * (d6 * (t6 - t2) + (d6 - d2) * (BAL_H - t6));
        const int lim = (int)((vl4 >> 1) | 1u) * dur;
        int s = __builtin_popcount((unsigned)__builtin_amdgcn_ballot_w64(num > lim) & ((1u << BAL_S) - 1u)) - __builtin_popcount((unsigned)__builtin_amdgcn_ballot_w64(-num > lim) & ((1u << BAL_S) - 1u));
        s = h ? -s : s;
        exitT = NF + s - 2;
        tTh = vl4 == (unsigned)(((exitT - 1) | 1) - 1) * 4u ? 0xffffffffu : tTh;
        const unsigned long long f0 = (unsigned long long)(fRem + OFF_F(G_SH(0)));
        tFl = vl4 == (unsigned)(exitT - 2) * 4u ? (unsigned)f0 : tFl; tFh = vl4 == (unsigned)(exitT - 2) * 4u ? (unsigned)(f0 >> 32) : tFh;
        PIN("+v"(tTh), "+v"(tFl), "+v"(tFh)); WKIND(0);
        S
    }
    do {
        FLAGS_ONLY(1);
        do {
        MF1(a, 0, 0, 0, 0, Ao); S M0(mF); LB0(0, vb[0], 49152); S
        MF2(a, 0, 0, 0, 1, Ao); S GFLAG(vfl, pF); S
        MF1(a, 0, 1, 0, 1, Ao); S M0(mB[2]); LB1(0, Bf[4][0], vb[0], 50176); S
        MF2(a, 0, 1, 0, 0, Ao); S GL(voffB[0], pB, 0); S
        MF1(a, 0, 2, 0, 0, Ao); S LB0(0, vb[0], 57344); S
        MF2(a, 0, 2, 0, 1, Ao); S
        MF1(a, 0, 3, 0, 1, Ao); S LB1(0, Bf[4][1], vb[0], 58368); S
        MF2(a, 0, 3, 0, 0, Ao); S TS4A; GL(voffB[1], pB, 1024); S
        MF1(a, 0, 3, 1, 0, Ao); S LB0(0, vb[1], 49152); S
        MF2(a, 0, 3, 1, 1, Ao); S
        MF1(a, 0, 2, 1, 1, Ao); S LB1(0, Bf[5][0], vb[1], 50176); S
        MF2(a, 0, 2, 1, 0, Ao); S GL(voffB[2], pB, 2048); S
        MF1(a, 0, 1, 1, 0, Ao); S LB0(0, vb[1], 57344); S
        MF2(a, 0, 1, 1, 1, Ao); S
        MF1(a, 0, 0, 1, 1, Ao); S LB1(0, Bf[5][1], vb[1], 58368); S
        MF2(a, 0, 0, 1, 0, Ao); S GL(voffB[3], pB, 3072); S
        MF1(a, 0, 0, 2, 0, Ao); S LB0(0, vb[2], 49152); S
        MF2(a, 0, 0, 2, 1, Ao); S
        MF1(a, 0, 1, 2, 1, Ao); S M0(mA[2]); LB1(0, Bf[6][0], vb[2], 50176); S
        MF2(a, 0, 1, 2, 0, Ao); S GL(voffA[0], pA, 0); S
        MF1(a, 0, 2, 2, 0, Ao); S LB0(0, vb[2], 57344); S
        MF2(a, 0, 2, 2, 1, Ao); S
        MF1(a, 0, 3, 2, 1, Ao); S LB1(0, Bf[6][1], vb[2], 58368); S
        MF2(a, 0, 3, 2, 0, Ao); S GL(voffA[1], pA, 1024); S
        MF1(a, 0, 3, 3, 0, Ao); S LB0(0, vb[3], 49152); S
        MF2(a, 0, 3, 3, 1, Ao); S
        MF1(a, 0, 2, 3, 1, Ao); S LB1(0, Bf[7][0], vb[3], 50176); S
        MF2(a, 0, 2, 3, 0, Ao); S GL(voffA[2], pA, 2048); S
        MF1(a, 0, 1, 3, 0, Ao); S LB0(0, vb[3], 57344); S
        MF2(a, 0, 1, 3, 1, Ao); S
        MF1(a, 0, 0, 3, 1, Ao); S LB1(0, Bf[7][1], vb[3], 58368); S
        MF2(a, 0, 0, 3, 0, Ao); S WL(12); GL(voffA[3], pA, 3072); S
        MF1(a, 0, 0, 4, 0, Ao); S
        MF2(a, 0, 0, 4, 1, Ao); S LDA(Ap, 0, 0, va, 49152); S
        MF1(a, 0, 1, 4, 1, Ao); S
        MF2(a, 0, 1, 4, 0, Ao); S PFW; S
        MF1(a, 0, 2, 4, 0, Ao); S LDA(Ap, 0, 1, va, 50176); S
        MF2(a, 0, 2, 4, 1, Ao); S
        MF1(a, 0, 3, 4, 1, Ao); S
        MF2(a, 0, 3, 4, 0, Ao); S WL(10); LDA(Ap, 1, 0, va, 51200); SST(1); S
        MF1(a, 0, 3, 5, 0, Ao); S
        MF2(a, 0, 3, 5, 1, Ao); S
        MF1(a, 0, 2, 5, 1, Ao); S LDA(Ap, 1, 1, va, 52224); S
        MF2(a, 0, 2, 5, 0, Ao); S
        MF1(a, 0, 1, 5, 0, Ao); S SLD(1); S
        MF2(a, 0, 1, 5, 1, Ao); S LDA(Ap, 2, 0, va, 53248); S
        MF1(a, 0, 0, 5, 1, Ao); S
        MF2(a, 0, 0, 5, 0, Ao); S WL(9); S
        MF1(a, 0, 0, 6, 0, Ao); S LDA(Ap, 2, 1, va, 54272); S
        MF2(a, 0, 0, 6, 1, Ao); S
        MF1(a, 0, 1, 6, 1, Ao); S
        MF2(a, 0, 1, 6, 0, Ao); S LDA(Ap, 3, 0, va, 55296); S
        MF1(a, 0, 2, 6, 0, Ao); S
        MF2(a, 0, 2, 6, 1, Ao); S
        MF1(a, 0, 3, 6, 1, Ao); S LDA(Ap, 3, 1, va, 56320); S
        MF2(a, 0, 3, 6, 0, Ao); S WL(8); S
        MF1(a, 0, 3, 7, 0, Ao); S
        MF2(a, 0, 3, 7, 1, Ao); S
        MF1(a, 0, 2, 7, 1, Ao); S
        MF2(a, 0, 2, 7, 0, Ao); S WVL(22); S
        MF1(a, 0, 1, 7, 0, Ao); S
        MF2(a, 0, 1, 7, 1, Ao); S BAR; S
        MF1(a, 0, 0, 7, 1, Ao); S TS4B(tsX); S
        MF2(a, 0, 0, 7, 0, Ao); S
        MF1(a, 1, 0, 0, 0, Ap); S M0(mB[3]); SFL; S
        MF2(a, 1, 0, 0, 1, Ap); S GL(voffB[4], pB, 0); S
        MF1(a, 1, 1, 0, 1, Ap); S
        MF2(a, 1, 1, 0, 0, Ap); S LDA(Ao, 0, 0, va, 0); S
        MF1(a, 1, 2, 0, 0, Ap); S
        MF2(a, 1, 2, 0, 1, Ap); S GL(voffB[5], pB, 1024); LDA(Ao, 0, 1, va, 1024); S
        MF1(a, 1, 3, 0, 1, Ap); S
        MF2(a, 1, 3, 0, 0, Ap); S TS5A; LDA(Ao, 1, 0, va, 2048); S
        MF1(a, 1, 3, 1, 0, Ap); S
        MF2(a, 1, 3, 1, 1, Ap); S GL(voffB[6], pB, 2048); LDA(Ao, 1, 1, va, 3072); S
        MF1(a, 1, 2, 1, 1, Ap); S
        MF2(a, 1, 2, 1, 0, Ap); S LB0(0, vb[0], 0); S
        MF1(a, 1, 1, 1, 0, Ap); S LDA(Ao, 2, 0, va, 4096); S
        MF2(a, 1, 1, 1, 1, Ap); S GL(voffB[7], pB, 3072); LB1(0, Bf[0][0], vb[0], 1024); S
        MF1(a, 1, 0, 1, 1, Ap); S LDA(Ao, 2, 1, va, 5120); S
        MF2(a, 1, 0, 1, 0, Ap); S LB0(0, vb[0], 8192); S
        MF1(a, 1, 0, 2, 0, Ap); S M0(mA[3]); LDA(Ao, 3, 0, va, 6144); S
        MF2(a, 1, 0, 2, 1, Ap); S GL(voffP[0], pA, 0); LB1(0, Bf[0][1], vb[0], 9216); S
        MF1(a, 1, 1, 2, 1, Ap); S LDA(Ao, 3, 1, va, 7168); S
        MF2(a, 1, 1, 2, 0, Ap); S LB0(0, vb[1], 0); S
        MF1(a, 1, 2, 2, 0, Ap); S
        MF2(a, 1, 2, 2, 1, Ap); S GL(voffP[1], pA, 1024); LB1(0, Bf[1][0], vb[1], 1024); S
        MF1(a, 1, 3, 2, 1, Ap); S
        MF2(a, 1, 3, 2, 0, Ap); S LB0(0, vb[1], 8192); S
        MF1(a, 1, 3, 3, 0, Ap); S
        MF2(a, 1, 3, 3, 1, Ap); S GL(voffP[2], pA, 2048); LB1(0, Bf[1][1], vb[1], 9216); S
        MF1(a, 1, 2, 3, 1, Ap); S
        MF2(a, 1, 2, 3, 0, Ap); S LB0(0, vb[2], 0); S
        MF1(a, 1, 1, 3, 0, Ap); S
        MF2(a, 1, 1, 3, 1, Ap); S GL(voffP[3], pA, 3072); LB1(0, Bf[2][0], vb[2], 1024); S
        MF1(a, 1, 0, 3, 1, Ap); S M0(t); S
        MF2(a, 1, 0, 3, 0, Ap); S ++t; PIN("+s"(t)); LB0(0, vb[2], 8192); RLM(pAl, tAl); S
        MF1(a, 1, 0, 4, 0, Ap); S RLM(pAh, tAh); RLM(pBl, tBl); S
        MF2(a, 1, 0, 4, 1, Ap); S LB1(0, Bf[2][1], vb[2], 9216); RLM(pBh, tBh); S
        MF1(a, 1, 1, 4, 1, Ap); S TS6A; RLM(pFl, tFl); RLM(pFh, tFh); S
        MF2(a, 1, 1, 4, 0, Ap); S LB0(0, vb[3], 0); RLM(thr, tTh); S
        MF1(a, 1, 2, 4, 0, Ap); S RLM(oP, tPf); S
        MF2(a, 1, 2, 4, 1, Ap); S M0(t); LB1(0, Bf[3][0], vb[3], 1024); S
        MF1(a, 1, 3, 4, 1, Ap); S RLM4(dD, 0, tPl); RLM(oD, tDo); S
        MF2(a, 1, 3, 4, 0, Ap); S LB0(0, vb[3], 8192); RLM4(dD, 1, tPh); S
        MF1(a, 1, 3, 5, 0, Ap); S RLM4(dF, 0, tQl); RLM4(dS, 0, tSl); S
        MF2(a, 1, 3, 5, 1, Ap); S LB1(0, Bf[3][1], vb[3], 9216); RLM4(dF, 1, tQh); S
        MF1(a, 1, 2, 5, 1, Ap); S RLM4(dS, 1, tSh); RLM(oF, tFo); S
        MF2(a, 1, 2, 5, 0, Ap); S RLM(oS, tSo); S
        MF1(a, 1, 1, 5, 0, Ap); S
        MF2(a, 1, 1, 5, 1, Ap); S
        MF1(a, 1, 0, 5, 1, Ap); S
        MF2(a, 1, 0, 5, 0, Ap); S
        MF1(a, 1, 0, 6, 0, Ap); S
        MF2(a, 1, 0, 6, 1, Ap); S TS6B; S
        MF1(a, 1, 1, 6, 1, Ap); S WV(20); S
        MF2(a, 1, 1, 6, 0, Ap); S LDF; S
        MF1(a, 1, 2, 6, 0, Ap); S
        MF2(a, 1, 2, 6, 1, Ap); S
        MF1(a, 1, 3, 6, 1, Ap); S
        MF2(a, 1, 3, 6, 0, Ap); S
        MF1(a, 1, 3, 7, 0, Ap); S
        MF2(a, 1, 3, 7, 1, Ap); S
        MF1(a, 1, 2, 7, 1, Ap); S
        MF2(a, 1, 2, 7, 0, Ap); S WLF; S
        MF1(a, 1, 1, 7, 0, Ap); S fm = __builtin_amdgcn_ballot_w64(fv < thr); PIN("+s"(fm)); S
        MF2(a, 1, 1, 7, 1, Ap); S BAR; S
        MF1(a, 1, 0, 7, 1, Ap); S TS6C; TS4B(tsY); S
        MF2(a, 1, 0, 7, 0, Ap); S

            if (UNLIKELY(SLOW)) FLAGS_ONLY(0);
        MF1(a, 0, 0, 0, 0, Ao); S M0(mF); LB0(0, vb[0], 16384); S
        MF2(a, 0, 0, 0, 1, Ao); S GFLAG(vfl, pF); S
        MF1(a, 0, 1, 0, 1, Ao); S M0(mB[0]); LB1(0, Bf[4][0], vb[0], 17408); S
        MF2(a, 0, 1, 0, 0, Ao); S GL(voffB[0], pB, 0); S
        MF1(a, 0, 2, 0, 0, Ao); S LB0(0, vb[0], 24576); S
        MF2(a, 0, 2, 0, 1, Ao); S
        MF1(a, 0, 3, 0, 1, Ao); S LB1(0, Bf[4][1], vb[0], 25600); S
        MF2(a, 0, 3, 0, 0, Ao); S TS4A; GL(voffB[1], pB, 1024); S
        MF1(a, 0, 3, 1, 0, Ao); S LB0(0, vb[1], 16384); S
        MF2(a, 0, 3, 1, 1, Ao); S
        MF1(a, 0, 2, 1, 1, Ao); S LB1(0, Bf[5][0], vb[1], 17408); S
        MF2(a, 0, 2, 1, 0, Ao); S GL(voffB[2], pB, 2048); S
        MF1(a, 0, 1, 1, 0, Ao); S LB0(0, vb[1], 24576); S
        MF2(a, 0, 1, 1, 1, Ao); S
        MF1(a, 0, 0, 1, 1, Ao); S LB1(0, Bf[5][1], vb[1], 25600); S
        MF2(a, 0, 0, 1, 0, Ao); S GL(voffB[3], pB, 3072); S
        MF1(a, 0, 0, 2, 0, Ao); S LB0(0, vb[2], 16384); S
        MF2(a, 0, 0, 2, 1, Ao); S
        MF1(a, 0, 1, 2, 1, Ao); S M0(mA[0]); LB1(0, Bf[6][0], vb[2], 17408); S
        MF2(a, 0, 1, 2, 0, Ao); S GL(voffA[0], pA, 0); S
        MF1(a, 0, 2, 2, 0, Ao); S LB0(0, vb[2], 24576); S
        MF2(a, 0, 2, 2, 1, Ao); S
        MF1(a, 0, 3, 2, 1, Ao); S LB1(0, Bf[6][1], vb[2], 25600); S
        MF2(a, 0, 3, 2, 0, Ao); S GL(voffA[1], pA, 1024); S
        MF1(a, 0, 3, 3, 0, Ao); S LB0(0, vb[3], 16384); S
        MF2(a, 0, 3, 3, 1, Ao); S
        MF1(a, 0, 2, 3, 1, Ao); S LB1(0, Bf[7][0], vb[3], 17408); S
        MF2(a, 0, 2, 3, 0, Ao); S GL(voffA[2], pA, 2048); S
        MF1(a, 0, 1, 3, 0, Ao); S LB0(0, vb[3], 24576); S
        MF2(a, 0, 1, 3, 1, Ao); S
        MF1(a, 0, 0, 3, 1, Ao); S LB1(0, Bf[7][1], vb[3], 25600); S
        MF2(a, 0, 0, 3, 0, Ao); S WL(12); GL(voffA[3], pA, 3072); S
        MF1(a, 0, 0, 4, 0, Ao); S
        MF2(a, 0, 0, 4, 1, Ao); S LDA(Ap, 0, 0, va, 16384); S
        MF1(a, 0, 1, 4, 1, Ao); S
        MF2(a, 0, 1, 4, 0, Ao); S PFW; S
        MF1(a, 0, 2, 4, 0, Ao); S LDA(Ap, 0, 1, va, 17408); S
        MF2(a, 0, 2, 4, 1, Ao); S
        MF1(a, 0, 3, 4, 1, Ao); S
        MF2(a, 0, 3, 4, 0, Ao); S WL(10); LDA(Ap, 1, 0, va, 18432); SST(0); S
        MF1(a, 0, 3, 5, 0, Ao); S
        MF2(a, 0, 3, 5, 1, Ao); S
        MF1(a, 0, 2, 5, 1, Ao); S LDA(Ap, 1, 1, va, 19456); S
        MF2(a, 0, 2, 5, 0, Ao); S
        MF1(a, 0, 1, 5, 0, Ao); S SLD(0); S
        MF2(a, 0, 1, 5, 1, Ao); S LDA(Ap, 2, 0, va, 20480); S
        MF1(a, 0, 0, 5, 1, Ao); S
        MF2(a, 0, 0, 5, 0, Ao); S WL(9); S
        MF1(a, 0, 0, 6, 0, Ao); S LDA(Ap, 2, 1, va, 21504); S
        MF2(a, 0, 0, 6, 1, Ao); S
        MF1(a, 0, 1, 6, 1, Ao); S
        MF2(a, 0, 1, 6, 0, Ao); S LDA(Ap, 3, 0, va, 22528); S
        MF1(a, 0, 2, 6, 0, Ao); S
        MF2(a, 0, 2, 6, 1, Ao); S
        MF1(a, 0, 3, 6, 1, Ao); S LDA(Ap, 3, 1, va, 23552); S
        MF2(a, 0, 3, 6, 0, Ao); S WL(8); S
        MF1(a, 0, 3, 7, 0, Ao); S
        MF2(a, 0, 3, 7, 1, Ao); S
        MF1(a, 0, 2, 7, 1, Ao); S
        MF2(a, 0, 2, 7, 0, Ao); S WVL(22); S
        MF1(a, 0, 1, 7, 0, Ao); S
        MF2(a, 0, 1, 7, 1, Ao); S BAR; S
        MF1(a, 0, 0, 7, 1, Ao); S TS4B(tsX); S
        MF2(a, 0, 0, 7, 0, Ao); S
        MF1(a, 1, 0, 0, 0, Ap); S M0(mB[1]); SFL; S
        MF2(a, 1, 0, 0, 1, Ap); S GL(voffB[4], pB, 0); S
        MF1(a, 1, 1, 0, 1, Ap); S
        MF2(a, 1, 1, 0, 0, Ap); S LDA(Ao, 0, 0, va, 32768); S
        MF1(a, 1, 2, 0, 0, Ap); S
        MF2(a, 1, 2, 0, 1, Ap); S GL(voffB[5], pB, 1024); LDA(Ao, 0, 1, va, 33792); S
        MF1(a, 1, 3, 0, 1, Ap); S
        MF2(a, 1, 3, 0, 0, Ap); S TS5A; LDA(Ao, 1, 0, va, 34816); S
        MF1(a, 1, 3, 1, 0, Ap); S
        MF2(a, 1, 3, 1, 1, Ap); S GL(voffB[6], pB, 2048); LDA(Ao, 1, 1, va, 35840); S
        MF1(a, 1, 2, 1, 1, Ap); S
        MF2(a, 1, 2, 1, 0, Ap); S LB0(0, vb[0], 32768); S
        MF1(a, 1, 1, 1, 0, Ap); S LDA(Ao, 2, 0, va, 36864); S
        MF2(a, 1, 1, 1, 1, Ap); S GL(voffB[7], pB, 3072); LB1(0, Bf[0][0], vb[0], 33792); S
        MF1(a, 1, 0, 1, 1, Ap); S LDA(Ao, 2, 1, va, 37888); S
        MF2(a, 1, 0, 1, 0, Ap); S LB0(0, vb[0], 40960); S
        MF1(a, 1, 0, 2, 0, Ap); S M0(mA[1]); LDA(Ao, 3, 0, va, 38912); S
        MF2(a, 1, 0, 2, 1, Ap); S GL(voffP[0], pA, 0); LB1(0, Bf[0][1], vb[0], 41984); S
        MF1(a, 1, 1, 2, 1, Ap); S LDA(Ao, 3, 1, va, 39936); S
        MF2(a, 1, 1, 2, 0, Ap); S LB0(0, vb[1], 32768); S
        MF1(a, 1, 2, 2, 0, Ap); S
        MF2(a, 1, 2, 2, 1, Ap); S GL(voffP[1], pA, 1024); LB1(0, Bf[1][0], vb[1], 33792); S
        MF1(a, 1, 3, 2, 1, Ap); S
        MF2(a, 1, 3, 2, 0, Ap); S LB0(0, vb[1], 40960); S
        MF1(a, 1, 3, 3, 0, Ap); S
        MF2(a, 1, 3, 3, 1, Ap); S GL(voffP[2], pA, 2048); LB1(0, Bf[1][1], vb[1], 41984); S
        MF1(a, 1, 2, 3, 1, Ap); S
        MF2(a, 1, 2, 3, 0, Ap); S LB0(0, vb[2], 32768); S
        MF1(a, 1, 1, 3, 0, Ap); S
        MF2(a, 1, 1, 3, 1, Ap); S GL(voffP[3], pA, 3072); LB1(0, Bf[2][0], vb[2], 33792); S
        MF1(a, 1, 0, 3, 1, Ap); S M0(t); S
        MF2(a, 1, 0, 3, 0, Ap); S ++t; PIN("+s"(t)); LB0(0, vb[2], 40960); RLM(pAl, tAl); S
        MF1(a, 1, 0, 4, 0, Ap); S RLM(pAh, tAh); RLM(pBl, tBl); S
        MF2(a, 1, 0, 4, 1, Ap); S LB1(0, Bf[2][1], vb[2], 41984); RLM(pBh, tBh); S
        MF1(a, 1, 1, 4, 1, Ap); S TS6A; RLM(pFl, tFl); RLM(pFh, tFh); S
        MF2(a, 1, 1, 4, 0, Ap); S LB0(0, vb[3], 32768); RLM(thr, tTh); S
        MF1(a, 1, 2, 4, 0, Ap); S RLM(oP, tPf); S
        MF2(a, 1, 2, 4, 1, Ap); S M0(t); LB1(0, Bf[3][0], vb[3], 33792); S
        MF1(a, 1, 3, 4, 1, Ap); S RLM4(dD, 0, tPl); RLM(oD, tDo); S
        MF2(a, 1, 3, 4, 0, Ap); S LB0(0, vb[3], 40960); RLM4(dD, 1, tPh); S
        MF1(a, 1, 3, 5, 0, Ap); S RLM4(dF, 0, tQl); RLM4(dS, 0, tSl); S
        MF2(a, 1, 3, 5, 1, Ap); S LB1(0, Bf[3][1], vb[3], 41984); RLM4(dF, 1, tQh); S
        MF1(a, 1, 2, 5, 1, Ap); S RLM4(dS, 1, tSh); RLM(oF, tFo); S
        MF2(a, 1, 2, 5, 0, Ap); S RLM(oS, tSo); S
        MF1(a, 1, 1, 5, 0, Ap); S
        MF2(a, 1, 1, 5, 1, Ap); S
        MF1(a, 1, 0, 5, 1, Ap); S
        MF2(a, 1, 0, 5, 0, Ap); S
        MF1(a, 1, 0, 6, 0, Ap); S
        MF2(a, 1, 0, 6, 1, Ap); S TS6B; S
        MF1(a, 1, 1, 6, 1, Ap); S WV(20); S
        MF2(a, 1, 1, 6, 0, Ap); S LDF; S
        MF1(a, 1, 2, 6, 0, Ap); S
        MF2(a, 1, 2, 6, 1, Ap); S
        MF1(a, 1, 3, 6, 1, Ap); S
        MF2(a, 1, 3, 6, 0, Ap); S
        MF1(a, 1, 3, 7, 0, Ap); S
        MF2(a, 1, 3, 7, 1, Ap); S
        MF1(a, 1, 2, 7, 1, Ap); S
        MF2(a, 1, 2, 7, 0, Ap); S WLF; S
        MF1(a, 1, 1, 7, 0, Ap); S fm = __builtin_amdgcn_ballot_w64(fv < thr); PIN("+s"(fm)); S
        MF2(a, 1, 1, 7, 1, Ap); S BAR; S
        MF1(a, 1, 0, 7, 1, Ap); S TS6C; TS4B(tsY); S
        MF2(a, 1, 0, 7, 0, Ap); S

        } while (!SLOW);
    } while (t + 2 <= exitT);
    unsigned pb = 2 * HTB;
    if (t < exitT) {
        FLAGS_ONLY(1);
        MF1(a, 0, 0, 0, 0, Ao); S M0(mF); LB0(0, vb[0], 49152); S
        MF2(a, 0, 0, 0, 1, Ao); S GFLAG(vfl, pF); S
        MF1(a, 0, 1, 0, 1, Ao); S M0(mB[2]); LB1(0, Bf[4][0], vb[0], 50176); S
        MF2(a, 0, 1, 0, 0, Ao); S GL(voffB[0], pB, 0); S
        MF1(a, 0, 2, 0, 0, Ao); S LB0(0, vb[0], 57344); S
        MF2(a, 0, 2, 0, 1, Ao); S
        MF1(a, 0, 3, 0, 1, Ao); S LB1(0, Bf[4][1], vb[0], 58368); S
        MF2(a, 0, 3, 0, 0, Ao); S TS4A; GL(voffB[1], pB, 1024); S
        MF1(a, 0, 3, 1, 0, Ao); S LB0(0, vb[1], 49152); S
        MF2(a, 0, 3, 1, 1, Ao); S
        MF1(a, 0, 2, 1, 1, Ao); S LB1(0, Bf[5][0], vb[1], 50176); S
        MF2(a, 0, 2, 1, 0, Ao); S GL(voffB[2], pB, 2048); S
        MF1(a, 0, 1, 1, 0, Ao); S LB0(0, vb[1], 57344); S
        MF2(a, 0, 1, 1, 1, Ao); S
        MF1(a, 0, 0, 1, 1, Ao); S LB1(0, Bf[5][1], vb[1], 58368); S
        MF2(a, 0, 0, 1, 0, Ao); S GL(voffB[3], pB, 3072); S
        MF1(a, 0, 0, 2, 0, Ao); S LB0(0, vb[2], 49152); S
        MF2(a, 0, 0, 2, 1, Ao); S
        MF1(a, 0, 1, 2, 1, Ao); S M0(mA[2]); LB1(0, Bf[6][0], vb[2], 50176); S
        MF2(a, 0, 1, 2, 0, Ao); S GL(voffA[0], pA, 0); S
        MF1(a, 0, 2, 2, 0, Ao); S LB0(0, vb[2], 57344); S
        MF2(a, 0, 2, 2, 1, Ao); S
        MF1(a, 0, 3, 2, 1, Ao); S LB1(0, Bf[6][1], vb[2], 58368); S
        MF2(a, 0, 3, 2, 0, Ao); S GL(voffA[1], pA, 1024); S
        MF1(a, 0, 3, 3, 0, Ao); S LB0(0, vb[3], 49152); S
        MF2(a, 0, 3, 3, 1, Ao); S
        MF1(a, 0, 2, 3, 1, Ao); S LB1(0, Bf[7][0], vb[3], 50176); S
        MF2(a, 0, 2, 3, 0, Ao); S GL(voffA[2], pA, 2048); S
        MF1(a, 0, 1, 3, 0, Ao); S LB0(0, vb[3], 57344); S
        MF2(a, 0, 1, 3, 1, Ao); S
        MF1(a, 0, 0, 3, 1, Ao); S LB1(0, Bf[7][1], vb[3], 58368); S
        MF2(a, 0, 0, 3, 0, Ao); S WL(12); GL(voffA[3], pA, 3072); S
        MF1(a, 0, 0, 4, 0, Ao); S
        MF2(a, 0, 0, 4, 1, Ao); S LDA(Ap, 0, 0, va, 49152); S
        MF1(a, 0, 1, 4, 1, Ao); S
        MF2(a, 0, 1, 4, 0, Ao); S
        MF1(a, 0, 2, 4, 0, Ao); S LDA(Ap, 0, 1, va, 50176); S
        MF2(a, 0, 2, 4, 1, Ao); S
        MF1(a, 0, 3, 4, 1, Ao); S
        MF2(a, 0, 3, 4, 0, Ao); S WL(10); LDA(Ap, 1, 0, va, 51200); S
        MF1(a, 0, 3, 5, 0, Ao); S
        MF2(a, 0, 3, 5, 1, Ao); S
        MF1(a, 0, 2, 5, 1, Ao); S LDA(Ap, 1, 1, va, 52224); S
        MF2(a, 0, 2, 5, 0, Ao); S
        MF1(a, 0, 1, 5, 0, Ao); S
        MF2(a, 0, 1, 5, 1, Ao); S LDA(Ap, 2, 0, va, 53248); S
        MF1(a, 0, 0, 5, 1, Ao); S
        MF2(a, 0, 0, 5, 0, Ao); S WL(9); S
        MF1(a, 0, 0, 6, 0, Ao); S LDA(Ap, 2, 1, va, 54272); S
        MF2(a, 0, 0, 6, 1, Ao); S
        MF1(a, 0, 1, 6, 1, Ao); S
        MF2(a, 0, 1, 6, 0, Ao); S LDA(Ap, 3, 0, va, 55296); S
        MF1(a, 0, 2, 6, 0, Ao); S
        MF2(a, 0, 2, 6, 1, Ao); S
        MF1(a, 0, 3, 6, 1, Ao); S LDA(Ap, 3, 1, va, 56320); S
        MF2(a, 0, 3, 6, 0, Ao); S WL(8); S
        MF1(a, 0, 3, 7, 0, Ao); S
        MF2(a, 0, 3, 7, 1, Ao); S
        MF1(a, 0, 2, 7, 1, Ao); S
        MF2(a, 0, 2, 7, 0, Ao); S WVL(17); S
        MF1(a, 0, 1, 7, 0, Ao); S
        MF2(a, 0, 1, 7, 1, Ao); S BAR; S
        MF1(a, 0, 0, 7, 1, Ao); S TS4B(tsX); S
        MF2(a, 0, 0, 7, 0, Ao); S
        MF1(a, 1, 0, 0, 0, Ap); S M0(mB[3]); S
        MF2(a, 1, 0, 0, 1, Ap); S GL(voffB[4], pB, 0); S
        MF1(a, 1, 1, 0, 1, Ap); S
        MF2(a, 1, 1, 0, 0, Ap); S LDA(Ao, 0, 0, va, 0); S
        MF1(a, 1, 2, 0, 0, Ap); S
        MF2(a, 1, 2, 0, 1, Ap); S GL(voffB[5], pB, 1024); LDA(Ao, 0, 1, va, 1024); S
        MF1(a, 1, 3, 0, 1, Ap); S
        MF2(a, 1, 3, 0, 0, Ap); S TS5A; LDA(Ao, 1, 0, va, 2048); S
        MF1(a, 1, 3, 1, 0, Ap); S
        MF2(a, 1, 3, 1, 1, Ap); S GL(voffB[6], pB, 2048); LDA(Ao, 1, 1, va, 3072); S
        MF1(a, 1, 2, 1, 1, Ap); S
        MF2(a, 1, 2, 1, 0, Ap); S LB0(0, vb[0], 0); S
        MF1(a, 1, 1, 1, 0, Ap); S LDA(Ao, 2, 0, va, 4096); S
        MF2(a, 1, 1, 1, 1, Ap); S GL(voffB[7], pB, 3072); LB1(0, Bf[0][0], vb[0], 1024); S
        MF1(a, 1, 0, 1, 1, Ap); S LDA(Ao, 2, 1, va, 5120); S
        MF2(a, 1, 0, 1, 0, Ap); S LB0(0, vb[0], 8192); S
        MF1(a, 1, 0, 2, 0, Ap); S M0(mA[3]); LDA(Ao, 3, 0, va, 6144); S
        MF2(a, 1, 0, 2, 1, Ap); S GL(voffP[0], pA, 0); LB1(0, Bf[0][1], vb[0], 9216); S
        MF1(a, 1, 1, 2, 1, Ap); S LDA(Ao, 3, 1, va, 7168); S
        MF2(a, 1, 1, 2, 0, Ap); S LB0(0, vb[1], 0); S
        MF1(a, 1, 2, 2, 0, Ap); S
        MF2(a, 1, 2, 2, 1, Ap); S GL(voffP[1], pA, 1024); LB1(0, Bf[1][0], vb[1], 1024); S
        MF1(a, 1, 3, 2, 1, Ap); S
        MF2(a, 1, 3, 2, 0, Ap); S LB0(0, vb[1], 8192); S
        MF1(a, 1, 3, 3, 0, Ap); S
        MF2(a, 1, 3, 3, 1, Ap); S GL(voffP[2], pA, 2048); LB1(0, Bf[1][1], vb[1], 9216); S
        MF1(a, 1, 2, 3, 1, Ap); S
        MF2(a, 1, 2, 3, 0, Ap); S LB0(0, vb[2], 0); S
        MF1(a, 1, 1, 3, 0, Ap); S
        MF2(a, 1, 1, 3, 1, Ap); S GL(voffP[3], pA, 3072); LB1(0, Bf[2][0], vb[2], 1024); S
        MF1(a, 1, 0, 3, 1, Ap); S M0(t); S
        MF2(a, 1, 0, 3, 0, Ap); S ++t; PIN("+s"(t)); LB0(0, vb[2], 8192); RLM(pAl, tAl); S
        MF1(a, 1, 0, 4, 0, Ap); S RLM(pAh, tAh); RLM(pBl, tBl); S
        MF2(a, 1, 0, 4, 1, Ap); S LB1(0, Bf[2][1], vb[2], 9216); RLM(pBh, tBh); S
        MF1(a, 1, 1, 4, 1, Ap); S TS6A; RLM(pFl, tFl); RLM(pFh, tFh); S
        MF2(a, 1, 1, 4, 0, Ap); S LB0(0, vb[3], 0); RLM(thr, tTh); S
        MF1(a, 1, 2, 4, 0, Ap); S
        MF2(a, 1, 2, 4, 1, Ap); S LB1(0, Bf[3][0], vb[3], 1024); S
        MF1(a, 1, 3, 4, 1, Ap); S
        MF2(a, 1, 3, 4, 0, Ap); S LB0(0, vb[3], 8192); S
        MF1(a, 1, 3, 5, 0, Ap); S
        MF2(a, 1, 3, 5, 1, Ap); S LB1(0, Bf[3][1], vb[3], 9216); S
        MF1(a, 1, 2, 5, 1, Ap); S
        MF2(a, 1, 2, 5, 0, Ap); S
        MF1(a, 1, 1, 5, 0, Ap); S
        MF2(a, 1, 1, 5, 1, Ap); S
        MF1(a, 1, 0, 5, 1, Ap); S
        MF2(a, 1, 0, 5, 0, Ap); S
        MF1(a, 1, 0, 6, 0, Ap); S
        MF2(a, 1, 0, 6, 1, Ap); S TS6B; S
        MF1(a, 1, 1, 6, 1, Ap); S WV(16); S
        MF2(a, 1, 1, 6, 0, Ap); S LDF; S
        MF1(a, 1, 2, 6, 0, Ap); S
        MF2(a, 1, 2, 6, 1, Ap); S
        MF1(a, 1, 3, 6, 1, Ap); S
        MF2(a, 1, 3, 6, 0, Ap); S
        MF1(a, 1, 3, 7, 0, Ap); S
        MF2(a, 1, 3, 7, 1, Ap); S
        MF1(a, 1, 2, 7, 1, Ap); S
        MF2(a, 1, 2, 7, 0, Ap); S WLF; S
        MF1(a, 1, 1, 7, 0, Ap); S fm = __builtin_amdgcn_ballot_w64(fv < thr); PIN("+s"(fm)); S
        MF2(a, 1, 1, 7, 1, Ap); S BAR; S
        MF1(a, 1, 0, 7, 1, Ap); S TS6C; TS4B(tsY); S
        MF2(a, 1, 0, 7, 0, Ap); S

        pb = 0; fvN[1] = fvN[0];
    }
    FLAGS_ONLY(1);

    WKIND(3);
    const unsigned qb = 2 * HTB - pb;
    unsigned nA[3] = {ldsw + pb, ldsw + qb, ldsw + HTB}, nB0[3] = {ldsw + 5 * HTB + pb, ldsw + 5 * HTB + qb, ldsw + 4 * HTB}, nB1[3] = {ldsw + 6 * HTB + pb, ldsw + 6 * HTB + qb, ldsw + 3 * HTB};
    unsigned vaP = va + pb, vaQ = va + qb, vbP[4], vbQ[4], vb2[4];
#pragma unroll
    for (int i = 0; i < 3; ++i) PIN("+s"(nA[i]), "+s"(nB0[i]), "+s"(nB1[i]));
#pragma unroll
    for (int j = 0; j < 4; ++j) { vbP[j] = vb[j] + pb; vbQ[j] = vb[j] + qb; vb2[j] = vb[j] - 4 * HTB; PIN("+v"(vbP[j]), "+v"(vbQ[j]), "+v"(vb2[j])); }
    PIN("+v"(vaP), "+v"(vaQ));
    unsigned tfo = OFF_F(G_SH(ORD ? lane >> 3 : (lane >> 4) * 2)) + vfl, tfo2 = OFF_F(G_SH(8 + (ORD ? lane >> 3 : (lane >> 4) * 2))) + vfl, fv2;
#define GFLAG2 asm volatile("global_load_lds_dword %0, %1 offset:256 sc0 sc1" :: "v"(tfo2), "s"(fRem - 256) : "memory")
#define LDF2 asm volatile("ds_read_b32 %0, %1 offset:256" : "=v"(fv2) : "v"(fadr) : "memory")
    unsigned qAl, qAh, qBl, qBh, qCl, qCh;
    RLS(qAl, tTl, 0); RLS(qAh, tTh2, 0); RLS(qBl, tTl, 16); RLS(qBh, tTh2, 16);
#define SLS 1024
#define SLB 63
#define XSM 0
    const char* slabOut = myws + (SLS ? (size_t)SLB << 20 : W_SLAB) + (size_t)(bid * 4 + wid) * (16384 + (SLS & ~1));
    const char* slabIn = myws + (SLS ? (size_t)SLB << 20 : W_SLAB) + (size_t)(partner * 4 + wid) * (16384 + (SLS & ~1));
    const char* pflag = myws + W_EFLAG + (size_t)(partner * 4 + wid) * 128;
    const char* oflag = myws + W_EFLAG + (size_t)(bid * 4 + wid) * 128;
    unsigned vl[4] = {(unsigned)lane * 16u, (unsigned)lane * 16u + 4096u, (unsigned)lane * 16u + 8192u, (unsigned)lane * 16u + 12288u};
    PIN("+s"(slabOut), "+s"(slabIn), "+s"(pflag), "+s"(oflag), "+v"(vl[0]), "+v"(vl[1]), "+v"(vl[2]), "+v"(vl[3]), "+v"(tfo), "+v"(tfo2));
    bf16x8 in[16]; u32x4 w[4]; float tf[8];
    unsigned early = 0;
#define XSS "sc1"
#define XST(b, voff, off) asm volatile("global_store_dwordx4 %0, %1, %2 offset:%3 " XSS :: "v"(voff), "v"(w[b]), "s"(slabOut), "i"(off) : "memory")
#define XLD(d, voff, off) asm volatile("global_load_dwordx4 %0, %1, %2 offset:%3 sc1" : "=a"(d) : "v"(voff), "s"(slabIn), "i"(off) : "memory")
#define XFLAG asm volatile("global_store_dword %0, %1, %2 sc1" :: "v"(vzero), "v"(vseq), "s"(oflag) : "memory")
#define LATE_PARTNER \
        if (UNLIKELY(early < seq)) { \
            const GAS unsigned* fp = (const GAS unsigned*)pflag; \
            const unsigned long long t0 = __builtin_amdgcn_s_memrealtime(); \
            while ((unsigned)__builtin_amdgcn_readfirstlane(__hip_atomic_load(fp, __ATOMIC_RELAXED, __HIP_MEMORY_SCOPE_AGENT)) < seq) { \
                if (__builtin_amdgcn_readfirstlane(__hip_atomic_load(tmo, __ATOMIC_RELAXED, __HIP_MEMORY_SCOPE_AGENT)) == seq || __builtin_amdgcn_s_memrealtime() - t0 > DEADLINE_TICKS) { \
                    __hip_atomic_store(tmo, seq, __ATOMIC_RELAXED, __HIP_MEMORY_SCOPE_AGENT); break; } \
                __builtin_amdgcn_s_sleep(1); \
            } \
            asm volatile("" ::: "memory"); \
            _Pragma("unroll") for (int p = 0; p < 16; ++p) XLD(in[p], vl[p >> 2], (p & 3) * 1024); \
            WV(0); \
            DIAG_SLOW; \
        }
    unsigned tm;
#define LDTM asm volatile("global_load_dword %0, %1, %2 sc1" : "=v"(tm) : "v"(vzero), "s"(tmo) : "memory")
    constexpr float G1 = -2.0f * 0.7978845608028654f * 1.4426950408889634f, G3 = G1 * 0.044715f;
    const char* sO[4]; f32x2 ey[4], ez[4]; unsigned eo[4];
#define E_SO(m) do { sO[m] = (const char*)(out + (size_t)(pm * 256 + h * 128 + wr * 64 + (m) * 16) * NN + (pn >> 1) * 512); PIN("+s"(sO[m])); } while (0)
#define ADDP(c, m, j, p) MF(#c, CAo, "a", acc[0][m][j], sel[(j) & 1], in[p])
#define E_AR(e, m, j) AR(ey[(e) >> 1][(e) & 1], acc[0][m][(j) + ((e) >> 2)][(e) & 3])
#define E_CP(i, m, j) ey[i] = (f32x2){acc[0][m][(j) + ((i) >> 1)][((i) & 1) * 2], acc[0][m][(j) + ((i) >> 1)][((i) & 1) * 2 + 1]}
#define E_SQ(i) do { ez[i] = ey[i] * ey[i]; S } while (0)
#define E_FM(i) do { ez[i] = __builtin_elementwise_fma(ez[i], (f32x2){G3, G3}, (f32x2){G1, G1}); S } while (0)
#define E_MU(i) do { ez[i] *= ey[i]; S } while (0)
#define E_EX(i, c) do { ez[i][c] = __builtin_amdgcn_exp2f(ez[i][c]); S } while (0)
#define E_AD(i) do { ez[i] += 1.0f; S } while (0)
#define E_RC(i, c) do { ez[i][c] = __builtin_amdgcn_rcpf(ez[i][c]); S } while (0)
#define E_MY(i) do { ez[i] *= ey[i]; S } while (0)
#define E_CV(i) do { eo[i] = CVT(ez[i][0], ez[i][1]); S } while (0)
#define E_PL(i) do { auto r = __builtin_amdgcn_permlane16_swap(eo[i], eo[(i) + 2], false, false); eo[i] = r[0]; eo[(i) + 2] = r[1]; S } while (0)
#define E_ST(m, r, off) asm volatile("global_store_dwordx4 %0, %1, %2 offset:%3 " OUTMOD :: "v"(vo[r]), "v"((u32x4){eo[0], eo[1], eo[2], eo[3]}), "s"(sO[m]), "i"(off) : "memory")
#define E_KEEP asm volatile("" :: "v"(eo[0]), "v"(eo[1]), "v"(eo[2]), "v"(eo[3]))
    S
    {
        MF1(a, 0, 0, 0, 0, Ao); S M0(mF); LB0(0, vbP[0], 16384); S
        MF2(a, 0, 0, 0, 1, Ao); S GFLAG(tfo, fRem); S
        MF1(a, 0, 1, 0, 1, Ao); S LB1(0, Bf[4][0], vbP[0], 17408); S
        MF2(a, 0, 1, 0, 0, Ao); S GFLAG2; S
        MF1(a, 0, 2, 0, 0, Ao); S M0(nB0[0]); LB0(0, vbP[0], 24576); S
        MF2(a, 0, 2, 0, 1, Ao); S GL(voffB[0], qB, 0); S
        MF1(a, 0, 3, 0, 1, Ao); S LB1(0, Bf[4][1], vbP[0], 25600); S
        MF2(a, 0, 3, 0, 0, Ao); S
        MF1(a, 0, 3, 1, 0, Ao); S LB0(0, vbP[1], 16384); S
        MF2(a, 0, 3, 1, 1, Ao); S GL(voffB[1], qB, 1024); S
        MF1(a, 0, 2, 1, 1, Ao); S LB1(0, Bf[5][0], vbP[1], 17408); S
        MF2(a, 0, 2, 1, 0, Ao); S
        MF1(a, 0, 1, 1, 0, Ao); S LB0(0, vbP[1], 24576); S
        MF2(a, 0, 1, 1, 1, Ao); S GL(voffB[2], qB, 2048); S
        MF1(a, 0, 0, 1, 1, Ao); S LB1(0, Bf[5][1], vbP[1], 25600); S
        MF2(a, 0, 0, 1, 0, Ao); S
        MF1(a, 0, 0, 2, 0, Ao); S LB0(0, vbP[2], 16384); S
        MF2(a, 0, 0, 2, 1, Ao); S GL(voffB[3], qB, 3072); S
        MF1(a, 0, 1, 2, 1, Ao); S LB1(0, Bf[6][0], vbP[2], 17408); S
        MF2(a, 0, 1, 2, 0, Ao); S
        MF1(a, 0, 2, 2, 0, Ao); S M0(nA[0]); LB0(0, vbP[2], 24576); S
        MF2(a, 0, 2, 2, 1, Ao); S GL(voffA[0], qA, 0); S
        MF1(a, 0, 3, 2, 1, Ao); S LB1(0, Bf[6][1], vbP[2], 25600); S
        MF2(a, 0, 3, 2, 0, Ao); S
        MF1(a, 0, 3, 3, 0, Ao); S LB0(0, vbP[3], 16384); S
        MF2(a, 0, 3, 3, 1, Ao); S GL(voffA[1], qA, 1024); S
        MF1(a, 0, 2, 3, 1, Ao); S LB1(0, Bf[7][0], vbP[3], 17408); S
        MF2(a, 0, 2, 3, 0, Ao); S
        MF1(a, 0, 1, 3, 0, Ao); S LB0(0, vbP[3], 24576); S
        MF2(a, 0, 1, 3, 1, Ao); S GL(voffA[2], qA, 2048); S
        MF1(a, 0, 0, 3, 1, Ao); S LB1(0, Bf[7][1], vbP[3], 25600); S
        MF2(a, 0, 0, 3, 0, Ao); S WL(12); S
        MF1(a, 0, 0, 4, 0, Ao); S
        MF2(a, 0, 0, 4, 1, Ao); S GL(voffA[3], qA, 3072); LDA(Ap, 0, 0, vaP, 16384); S
        MF1(a, 0, 1, 4, 1, Ao); S RLS(qCl, tTl, 18); RLS(qCh, tTh2, 18); S
        MF2(a, 0, 1, 4, 0, Ao); S RLS(qAl, tTl, 1); RLS(qAh, tTh2, 1); S
        MF1(a, 0, 2, 4, 0, Ao); S LDA(Ap, 0, 1, vaP, 17408); S
        MF2(a, 0, 2, 4, 1, Ao); S
        MF1(a, 0, 3, 4, 1, Ao); S
        MF2(a, 0, 3, 4, 0, Ao); S WL(10); LDA(Ap, 1, 0, vaP, 18432); S
        MF1(a, 0, 3, 5, 0, Ao); S
        MF2(a, 0, 3, 5, 1, Ao); S
        MF1(a, 0, 2, 5, 1, Ao); S LDA(Ap, 1, 1, vaP, 19456); S
        MF2(a, 0, 2, 5, 0, Ao); S
        MF1(a, 0, 1, 5, 0, Ao); S
        MF2(a, 0, 1, 5, 1, Ao); S LDA(Ap, 2, 0, vaP, 20480); S
        MF1(a, 0, 0, 5, 1, Ao); S
        MF2(a, 0, 0, 5, 0, Ao); S WL(9); S
        MF1(a, 0, 0, 6, 0, Ao); S LDA(Ap, 2, 1, vaP, 21504); S
        MF2(a, 0, 0, 6, 1, Ao); S
        MF1(a, 0, 1, 6, 1, Ao); S
        MF2(a, 0, 1, 6, 0, Ao); S LDA(Ap, 3, 0, vaP, 22528); S
        MF1(a, 0, 2, 6, 0, Ao); S
        MF2(a, 0, 2, 6, 1, Ao); S
        MF1(a, 0, 3, 6, 1, Ao); S LDA(Ap, 3, 1, vaP, 23552); S
        MF2(a, 0, 3, 6, 0, Ao); S WL(8); S
        MF1(a, 0, 3, 7, 0, Ao); S
        MF2(a, 0, 3, 7, 1, Ao); S
        MF1(a, 0, 2, 7, 1, Ao); S
        MF2(a, 0, 2, 7, 0, Ao); S WVL(18); S
        MF1(a, 0, 1, 7, 0, Ao); S
        MF2(a, 0, 1, 7, 1, Ao); S BAR; S
        MF1(a, 0, 0, 7, 1, Ao); S
        MF2(a, 0, 0, 7, 0, Ao); S
        MF1(a, 1, 0, 0, 0, Ap); S M0(nB1[0]); S
        MF2(a, 1, 0, 0, 1, Ap); S GL(voffB[4], qB, 0); S
        MF1(a, 1, 1, 0, 1, Ap); S
        MF2(a, 1, 1, 0, 0, Ap); S LDA(Ao, 0, 0, vaQ, 0); S
        MF1(a, 1, 2, 0, 0, Ap); S
        MF2(a, 1, 2, 0, 1, Ap); S GL(voffB[5], qB, 1024); LDA(Ao, 0, 1, vaQ, 1024); S
        MF1(a, 1, 3, 0, 1, Ap); S
        MF2(a, 1, 3, 0, 0, Ap); S LDA(Ao, 1, 0, vaQ, 2048); S
        MF1(a, 1, 3, 1, 0, Ap); S
        MF2(a, 1, 3, 1, 1, Ap); S GL(voffB[6], qB, 2048); LDA(Ao, 1, 1, vaQ, 3072); S
        MF1(a, 1, 2, 1, 1, Ap); S
        MF2(a, 1, 2, 1, 0, Ap); S LB0(0, vbQ[0], 0); S
        MF1(a, 1, 1, 1, 0, Ap); S LDA(Ao, 2, 0, vaQ, 4096); S
        MF2(a, 1, 1, 1, 1, Ap); S GL(voffB[7], qB, 3072); LB1(0, Bf[0][0], vbQ[0], 1024); S
        MF1(a, 1, 0, 1, 1, Ap); S LDA(Ao, 2, 1, vaQ, 5120); S
        MF2(a, 1, 0, 1, 0, Ap); S LB0(0, vbQ[0], 8192); S
        MF1(a, 1, 0, 2, 0, Ap); S M0(nB0[2]); LDA(Ao, 3, 0, vaQ, 6144); S
        MF2(a, 1, 0, 2, 1, Ap); S GL(voffB[0], qC, 0); LB1(0, Bf[0][1], vbQ[0], 9216); S
        MF1(a, 1, 1, 2, 1, Ap); S LDA(Ao, 3, 1, vaQ, 7168); S
        MF2(a, 1, 1, 2, 0, Ap); S LB0(0, vbQ[1], 0); S
        MF1(a, 1, 2, 2, 0, Ap); S
        MF2(a, 1, 2, 2, 1, Ap); S GL(voffB[1], qC, 1024); LB1(0, Bf[1][0], vbQ[1], 1024); S
        MF1(a, 1, 3, 2, 1, Ap); S
        MF2(a, 1, 3, 2, 0, Ap); S LB0(0, vbQ[1], 8192); S
        MF1(a, 1, 3, 3, 0, Ap); S
        MF2(a, 1, 3, 3, 1, Ap); S GL(voffB[2], qC, 2048); LB1(0, Bf[1][1], vbQ[1], 9216); S
        MF1(a, 1, 2, 3, 1, Ap); S
        MF2(a, 1, 2, 3, 0, Ap); S LB0(0, vbQ[2], 0); S
        MF1(a, 1, 1, 3, 0, Ap); S
        MF2(a, 1, 1, 3, 1, Ap); S GL(voffB[3], qC, 3072); LB1(0, Bf[2][0], vbQ[2], 1024); S
        MF1(a, 1, 0, 3, 1, Ap); S RLS(qBl, tTl, 17); RLS(qBh, tTh2, 17); S
        MF2(a, 1, 0, 3, 0, Ap); S LB0(0, vbQ[2], 8192); S
        MF1(a, 1, 0, 4, 0, Ap); S
        MF2(a, 1, 0, 4, 1, Ap); S LB1(0, Bf[2][1], vbQ[2], 9216); S
        MF1(a, 1, 1, 4, 1, Ap); S
        MF2(a, 1, 1, 4, 0, Ap); S LB0(0, vbQ[3], 0); S
        MF1(a, 1, 2, 4, 0, Ap); S
        MF2(a, 1, 2, 4, 1, Ap); S LB1(0, Bf[3][0], vbQ[3], 1024); S
        MF1(a, 1, 3, 4, 1, Ap); S
        MF2(a, 1, 3, 4, 0, Ap); S LB0(0, vbQ[3], 8192); S
        MF1(a, 1, 3, 5, 0, Ap); S
        MF2(a, 1, 3, 5, 1, Ap); S LB1(0, Bf[3][1], vbQ[3], 9216); S
        MF1(a, 1, 2, 5, 1, Ap); S
        MF2(a, 1, 2, 5, 0, Ap); S
        MF1(a, 1, 1, 5, 0, Ap); S
        MF2(a, 1, 1, 5, 1, Ap); S
        MF1(a, 1, 0, 5, 1, Ap); S
        MF2(a, 1, 0, 5, 0, Ap); S
        MF1(a, 1, 0, 6, 0, Ap); S
        MF2(a, 1, 0, 6, 1, Ap); S
        MF1(a, 1, 1, 6, 1, Ap); S WV(16); S
        MF2(a, 1, 1, 6, 0, Ap); S LDF; S
        MF1(a, 1, 2, 6, 0, Ap); S LDF2; S
        MF2(a, 1, 2, 6, 1, Ap); S
        MF1(a, 1, 3, 6, 1, Ap); S
        MF2(a, 1, 3, 6, 0, Ap); S
        MF1(a, 1, 3, 7, 0, Ap); S
        MF2(a, 1, 3, 7, 1, Ap); S
        MF1(a, 1, 2, 7, 1, Ap); S
        MF2(a, 1, 2, 7, 0, Ap); S WLF; S
        MF1(a, 1, 1, 7, 0, Ap); S PIN("+v"(fv2)); fv = fv < fv2 ? fv : fv2; fm = __builtin_amdgcn_ballot_w64(fv < tgt); PIN("+s"(fm)); S
        MF2(a, 1, 1, 7, 1, Ap); S BAR; S
        MF1(a, 1, 0, 7, 1, Ap); S
        MF2(a, 1, 0, 7, 0, Ap); S
        PIN("+v"(vdump), "+v"(pay[0]), "+v"(pay[1]), "+v"(fvN[0]));
        if (UNLIKELY(fm != 0)) { SPIN_FLAGS(fRem + tfo); SPIN_FLAGS(fRem + tfo2); }
        MF1(a, 0, 0, 0, 0, Ao); S LB0(0, vbQ[0], 16384); S
        MF2(a, 0, 0, 0, 1, Ao); S M0(nB0[1]); S
        MF1(a, 0, 1, 0, 1, Ao); S LB1(0, Bf[4][0], vbQ[0], 17408); GL(voffB[0], qB, 0); S
        MF2(a, 0, 1, 0, 0, Ao); S
        MF1(a, 0, 2, 0, 0, Ao); S LB0(0, vbQ[0], 24576); S
        MF2(a, 0, 2, 0, 1, Ao); S
        MF1(a, 0, 3, 0, 1, Ao); S LB1(0, Bf[4][1], vbQ[0], 25600); GL(voffB[1], qB, 1024); S
        MF2(a, 0, 3, 0, 0, Ao); S
        MF1(a, 0, 3, 1, 0, Ao); S LB0(0, vbQ[1], 16384); S
        MF2(a, 0, 3, 1, 1, Ao); S
        MF1(a, 0, 2, 1, 1, Ao); S LB1(0, Bf[5][0], vbQ[1], 17408); GL(voffB[2], qB, 2048); S
        MF2(a, 0, 2, 1, 0, Ao); S
        MF1(a, 0, 1, 1, 0, Ao); S LB0(0, vbQ[1], 24576); S
        MF2(a, 0, 1, 1, 1, Ao); S
        MF1(a, 0, 0, 1, 1, Ao); S LB1(0, Bf[5][1], vbQ[1], 25600); GL(voffB[3], qB, 3072); S
        MF2(a, 0, 0, 1, 0, Ao); S
        MF1(a, 0, 0, 2, 0, Ao); S LB0(0, vbQ[2], 16384); S
        MF2(a, 0, 0, 2, 1, Ao); S M0(nA[1]); S
        MF1(a, 0, 1, 2, 1, Ao); S LB1(0, Bf[6][0], vbQ[2], 17408); GL(voffA[0], qA, 0); S
        MF2(a, 0, 1, 2, 0, Ao); S
        MF1(a, 0, 2, 2, 0, Ao); S LB0(0, vbQ[2], 24576); S
        MF2(a, 0, 2, 2, 1, Ao); S
        MF1(a, 0, 3, 2, 1, Ao); S LB1(0, Bf[6][1], vbQ[2], 25600); GL(voffA[1], qA, 1024); S
        MF2(a, 0, 3, 2, 0, Ao); S
        MF1(a, 0, 3, 3, 0, Ao); S LB0(0, vbQ[3], 16384); S
        MF2(a, 0, 3, 3, 1, Ao); S
        MF1(a, 0, 2, 3, 1, Ao); S LB1(0, Bf[7][0], vbQ[3], 17408); GL(voffA[2], qA, 2048); S
        MF2(a, 0, 2, 3, 0, Ao); S
        MF1(a, 0, 1, 3, 0, Ao); S LB0(0, vbQ[3], 24576); S
        MF2(a, 0, 1, 3, 1, Ao); S
        MF1(a, 0, 0, 3, 1, Ao); S LB1(0, Bf[7][1], vbQ[3], 25600); GL(voffA[3], qA, 3072); S
        MF2(a, 0, 0, 3, 0, Ao); S WL(12); RLS(qAl, tTl, 2); RLS(qAh, tTh2, 2); S
        MF1(a, 0, 0, 4, 0, Ao); S
        MF2(a, 0, 0, 4, 1, Ao); S LDA(Ap, 0, 0, vaQ, 16384); S
        MF1(a, 0, 1, 4, 1, Ao); S
        MF2(a, 0, 1, 4, 0, Ao); S
        MF1(a, 0, 2, 4, 0, Ao); S LDA(Ap, 0, 1, vaQ, 17408); S
        MF2(a, 0, 2, 4, 1, Ao); S
        MF1(a, 0, 3, 4, 1, Ao); S
        MF2(a, 0, 3, 4, 0, Ao); S WL(10); LDA(Ap, 1, 0, vaQ, 18432); S
        MF1(a, 0, 3, 5, 0, Ao); S
        MF2(a, 0, 3, 5, 1, Ao); S
        MF1(a, 0, 2, 5, 1, Ao); S LDA(Ap, 1, 1, vaQ, 19456); S
        MF2(a, 0, 2, 5, 0, Ao); S
        MF1(a, 0, 1, 5, 0, Ao); S
        MF2(a, 0, 1, 5, 1, Ao); S LDA(Ap, 2, 0, vaQ, 20480); S
        MF1(a, 0, 0, 5, 1, Ao); S
        MF2(a, 0, 0, 5, 0, Ao); S WL(9); S
        MF1(a, 0, 0, 6, 0, Ao); S LDA(Ap, 2, 1, vaQ, 21504); S
        MF2(a, 0, 0, 6, 1, Ao); S
        MF1(a, 0, 1, 6, 1, Ao); S
        MF2(a, 0, 1, 6, 0, Ao); S LDA(Ap, 3, 0, vaQ, 22528); S
        MF1(a, 0, 2, 6, 0, Ao); S
        MF2(a, 0, 2, 6, 1, Ao); S
        MF1(a, 0, 3, 6, 1, Ao); S LDA(Ap, 3, 1, vaQ, 23552); S
        MF2(a, 0, 3, 6, 0, Ao); S WL(8); S
        MF1(a, 0, 3, 7, 0, Ao); S
        MF2(a, 0, 3, 7, 1, Ao); S
        MF1(a, 0, 2, 7, 1, Ao); S
        MF2(a, 0, 2, 7, 0, Ao); S WVL(16); S
        MF1(a, 0, 1, 7, 0, Ao); S
        MF2(a, 0, 1, 7, 1, Ao); S BAR; S
        MF1(a, 0, 0, 7, 1, Ao); S
        MF2(a, 0, 0, 7, 0, Ao); S
        MF1(a, 1, 0, 0, 0, Ap); S M0(nB1[1]); S
        MF2(a, 1, 0, 0, 1, Ap); S GL(voffB[4], qB, 0); S
        MF1(a, 1, 1, 0, 1, Ap); S
        MF2(a, 1, 1, 0, 0, Ap); S LDA(Ao, 0, 0, vaP, 0); S
        MF1(a, 1, 2, 0, 0, Ap); S
        MF2(a, 1, 2, 0, 1, Ap); S GL(voffB[5], qB, 1024); LDA(Ao, 0, 1, vaP, 1024); S
        MF1(a, 1, 3, 0, 1, Ap); S
        MF2(a, 1, 3, 0, 0, Ap); S LDA(Ao, 1, 0, vaP, 2048); S
        MF1(a, 1, 3, 1, 0, Ap); S
        MF2(a, 1, 3, 1, 1, Ap); S GL(voffB[6], qB, 2048); LDA(Ao, 1, 1, vaP, 3072); S
        MF1(a, 1, 2, 1, 1, Ap); S
        MF2(a, 1, 2, 1, 0, Ap); S LB0(0, vbP[0], 0); S
        MF1(a, 1, 1, 1, 0, Ap); S LDA(Ao, 2, 0, vaP, 4096); S
        MF2(a, 1, 1, 1, 1, Ap); S GL(voffB[7], qB, 3072); LB1(0, Bf[0][0], vbP[0], 1024); S
        MF1(a, 1, 0, 1, 1, Ap); S LDA(Ao, 2, 1, vaP, 5120); S
        MF2(a, 1, 0, 1, 0, Ap); S LB0(0, vbP[0], 8192); S
        MF1(a, 1, 0, 2, 0, Ap); S M0(nA[2]); LDA(Ao, 3, 0, vaP, 6144); AR(tf[0], acc[1][3][0][0]); S
        MF2(a, 1, 0, 2, 1, Ap); S GL(voffA[0], qA, 0); LB1(0, Bf[0][1], vbP[0], 9216); S
        MF1(a, 1, 1, 2, 1, Ap); S LDA(Ao, 3, 1, vaP, 7168); AR(tf[1], acc[1][3][0][1]); S
        MF2(a, 1, 1, 2, 0, Ap); S LB0(0, vbP[1], 0); AR(tf[2], acc[1][3][0][2]); S
        MF1(a, 1, 2, 2, 0, Ap); S AR(tf[3], acc[1][3][0][3]); AR(tf[4], acc[1][3][1][0]); S
        MF2(a, 1, 2, 2, 1, Ap); S GL(voffA[1], qA, 1024); LB1(0, Bf[1][0], vbP[1], 1024); S
        MF1(a, 1, 3, 2, 1, Ap); S AR(tf[5], acc[1][3][1][1]); AR(tf[6], acc[1][3][1][2]); S
        MF2(a, 1, 3, 2, 0, Ap); S LB0(0, vbP[1], 8192); AR(tf[7], acc[1][3][1][3]); S
        MF1(a, 1, 3, 3, 0, Ap); S w[0].x = CVT(tf[0], tf[1]); w[0].y = CVT(tf[2], tf[3]); S
        MF2(a, 1, 3, 3, 1, Ap); S GL(voffA[2], qA, 2048); LB1(0, Bf[1][1], vbP[1], 9216); S
        MF1(a, 1, 2, 3, 1, Ap); S w[0].z = CVT(tf[4], tf[5]); w[0].w = CVT(tf[6], tf[7]); S
        MF2(a, 1, 2, 3, 0, Ap); S LB0(0, vbP[2], 0); XST(0, vl[3], 0); S
        MF1(a, 1, 1, 3, 0, Ap); S AR(tf[0], acc[1][2][0][0]); AR(tf[1], acc[1][2][0][1]); S
        MF2(a, 1, 1, 3, 1, Ap); S GL(voffA[3], qA, 3072); LB1(0, Bf[2][0], vbP[2], 1024); S
        MF1(a, 1, 0, 3, 1, Ap); S RLS(qAl, tTl, 3); RLS(qAh, tTh2, 3); S
        MF2(a, 1, 0, 3, 0, Ap); S LB0(0, vbP[2], 8192); RLS(qBl, tTl, 19); S
        MF1(a, 1, 0, 4, 0, Ap); S RLS(qBh, tTh2, 19); AR(tf[2], acc[1][2][0][2]); S
        MF2(a, 1, 0, 4, 1, Ap); S LB1(0, Bf[2][1], vbP[2], 9216); AR(tf[3], acc[1][2][0][3]); S
        MF1(a, 1, 1, 4, 1, Ap); S AR(tf[4], acc[1][2][1][0]); AR(tf[5], acc[1][2][1][1]); S
        MF2(a, 1, 1, 4, 0, Ap); S LB0(0, vbP[3], 0); AR(tf[6], acc[1][2][1][2]); S
        MF1(a, 1, 2, 4, 0, Ap); S AR(tf[7], acc[1][2][1][3]); S
        MF2(a, 1, 2, 4, 1, Ap); S LB1(0, Bf[3][0], vbP[3], 1024); w[0].x = CVT(tf[0], tf[1]); S
        MF1(a, 1, 3, 4, 1, Ap); S w[0].y = CVT(tf[2], tf[3]); w[0].z = CVT(tf[4], tf[5]); S
        MF2(a, 1, 3, 4, 0, Ap); S LB0(0, vbP[3], 8192); w[0].w = CVT(tf[6], tf[7]); S
        MF1(a, 1, 3, 5, 0, Ap); S XST(0, vl[2], 0); S
        MF2(a, 1, 3, 5, 1, Ap); S LB1(0, Bf[3][1], vbP[3], 9216); AR(tf[0], acc[1][1][0][0]); S
        MF1(a, 1, 2, 5, 1, Ap); S AR(tf[1], acc[1][1][0][1]); AR(tf[2], acc[1][1][0][2]); S
        MF2(a, 1, 2, 5, 0, Ap); S AR(tf[3], acc[1][1][0][3]); AR(tf[4], acc[1][1][1][0]); S
        MF1(a, 1, 1, 5, 0, Ap); S AR(tf[5], acc[1][1][1][1]); AR(tf[6], acc[1][1][1][2]); S
        MF2(a, 1, 1, 5, 1, Ap); S AR(tf[7], acc[1][1][1][3]); S
        MF1(a, 1, 0, 5, 1, Ap); S w[0].x = CVT(tf[0], tf[1]); w[0].y = CVT(tf[2], tf[3]); S
        MF2(a, 1, 0, 5, 0, Ap); S w[0].z = CVT(tf[4], tf[5]); w[0].w = CVT(tf[6], tf[7]); S
        MF1(a, 1, 0, 6, 0, Ap); S XST(0, vl[1], 0); S
        MF2(a, 1, 0, 6, 1, Ap); S AR(tf[0], acc[1][0][0][0]); AR(tf[1], acc[1][0][0][1]); S
        MF1(a, 1, 1, 6, 1, Ap); S AR(tf[2], acc[1][0][0][2]); AR(tf[3], acc[1][0][0][3]); S
        MF2(a, 1, 1, 6, 0, Ap); S AR(tf[4], acc[1][0][1][0]); AR(tf[5], acc[1][0][1][1]); S
        MF1(a, 1, 2, 6, 0, Ap); S AR(tf[6], acc[1][0][1][2]); AR(tf[7], acc[1][0][1][3]); S
        MF2(a, 1, 2, 6, 1, Ap); S w[0].x = CVT(tf[0], tf[1]); w[0].y = CVT(tf[2], tf[3]); S
        MF1(a, 1, 3, 6, 1, Ap); S w[0].z = CVT(tf[4], tf[5]); w[0].w = CVT(tf[6], tf[7]); S
        MF2(a, 1, 3, 6, 0, Ap); S XST(0, vl[0], 0); S
        MF1(a, 1, 3, 7, 0, Ap); S AR(tf[0], acc[1][3][2][0]); AR(tf[1], acc[1][3][2][1]); S
        MF2(a, 1, 3, 7, 1, Ap); S AR(tf[2], acc[1][3][2][2]); AR(tf[3], acc[1][3][2][3]); S
        MF1(a, 1, 2, 7, 1, Ap); S AR(tf[4], acc[1][3][3][0]); AR(tf[5], acc[1][3][3][1]); S
        MF2(a, 1, 2, 7, 0, Ap); S WVL(12); AR(tf[6], acc[1][3][3][2]); AR(tf[7], acc[1][3][3][3]); S
        MF1(a, 1, 1, 7, 0, Ap); S w[0].x = CVT(tf[0], tf[1]); w[0].y = CVT(tf[2], tf[3]); S
        MF2(a, 1, 1, 7, 1, Ap); S BAR; w[0].z = CVT(tf[4], tf[5]); w[0].w = CVT(tf[6], tf[7]); S
        MF1(a, 1, 0, 7, 1, Ap); S XST(0, vl[3], 1024); S
        MF2(a, 1, 0, 7, 0, Ap); S AR(tf[0], acc[1][2][2][0]); AR(tf[1], acc[1][2][2][1]); S
        STAMP(2);
        TSTAMP(0);
        MF1(a, 0, 0, 0, 0, Ao); S LB0(0, vbP[0], 16384); AR(tf[2], acc[1][2][2][2]); S
        MF2(a, 0, 0, 0, 1, Ao); S M0(nB1[2]); LB1(0, Bf[4][0], vbP[0], 17408); AR(tf[3], acc[1][2][2][3]); S
        MF1(a, 0, 1, 0, 1, Ao); S LB0(0, vbP[0], 24576); GL(voffB[4], qC, 0); S
        MF2(a, 0, 1, 0, 0, Ao); S LB1(0, Bf[4][1], vbP[0], 25600); AR(tf[4], acc[1][2][3][0]); S
        MF1(a, 0, 2, 0, 0, Ao); S LB0(0, vbP[1], 16384); AR(tf[5], acc[1][2][3][1]); S
        MF2(a, 0, 2, 0, 1, Ao); S LB1(0, Bf[5][0], vbP[1], 17408); AR(tf[6], acc[1][2][3][2]); S
        MF1(a, 0, 3, 0, 1, Ao); S LB0(0, vbP[1], 24576); GL(voffB[5], qC, 1024); S
        MF2(a, 0, 3, 0, 0, Ao); S LB1(0, Bf[5][1], vbP[1], 25600); AR(tf[7], acc[1][2][3][3]); S
        MF1(a, 0, 3, 1, 0, Ao); S LB0(0, vbP[2], 16384); w[0].x = CVT(tf[0], tf[1]); S
        MF2(a, 0, 3, 1, 1, Ao); S LB1(0, Bf[6][0], vbP[2], 17408); w[0].y = CVT(tf[2], tf[3]); S
        MF1(a, 0, 2, 1, 1, Ao); S LB0(0, vbP[2], 24576); GL(voffB[6], qC, 2048); S
        MF2(a, 0, 2, 1, 0, Ao); S LB1(0, Bf[6][1], vbP[2], 25600); w[0].z = CVT(tf[4], tf[5]); S
        MF1(a, 0, 1, 1, 0, Ao); S LB0(0, vbP[3], 16384); w[0].w = CVT(tf[6], tf[7]); S
        MF2(a, 0, 1, 1, 1, Ao); S LB1(0, Bf[7][0], vbP[3], 17408); S
        MF1(a, 0, 0, 1, 1, Ao); S LB0(0, vbP[3], 24576); GL(voffB[7], qC, 3072); S
        MF2(a, 0, 0, 1, 0, Ao); S LB1(0, Bf[7][1], vbP[3], 25600); S
        MF1(a, 0, 0, 2, 0, Ao); S XST(0, vl[2], 1024); S
        MF2(a, 0, 0, 2, 1, Ao); S M0(nB0[0]); LDA(Ap, 0, 0, vaQ, 0); AR(tf[0], acc[1][1][2][0]); S
        MF1(a, 0, 1, 2, 1, Ao); S GL(voffB[0], qB, 0); LB0(1, vbQ[0], 0); S
        MF2(a, 0, 1, 2, 0, Ao); S LB1(1, Bf[0][0], vbQ[0], 1024); AR(tf[1], acc[1][1][2][1]); S
        MF1(a, 0, 2, 2, 0, Ao); S AR(tf[2], acc[1][1][2][2]); AR(tf[3], acc[1][1][2][3]); S
        MF2(a, 0, 2, 2, 1, Ao); S LDA(Ap, 0, 1, vaQ, 1024); AR(tf[4], acc[1][1][3][0]); S
        MF1(a, 0, 3, 2, 1, Ao); S GL(voffB[1], qB, 1024); LB0(1, vbQ[0], 8192); S
        MF2(a, 0, 3, 2, 0, Ao); S LB1(1, Bf[0][1], vbQ[0], 9216); AR(tf[5], acc[1][1][3][1]); S
        MF1(a, 0, 3, 3, 0, Ao); S AR(tf[6], acc[1][1][3][2]); AR(tf[7], acc[1][1][3][3]); S
        MF2(a, 0, 3, 3, 1, Ao); S LDA(Ap, 1, 0, vaQ, 2048); w[0].x = CVT(tf[0], tf[1]); S
        MF1(a, 0, 2, 3, 1, Ao); S GL(voffB[2], qB, 2048); LB0(1, vbQ[1], 0); S
        MF2(a, 0, 2, 3, 0, Ao); S LB1(1, Bf[1][0], vbQ[1], 1024); w[0].y = CVT(tf[2], tf[3]); S
        MF1(a, 0, 1, 3, 0, Ao); S w[0].z = CVT(tf[4], tf[5]); w[0].w = CVT(tf[6], tf[7]); S
        MF2(a, 0, 1, 3, 1, Ao); S LDA(Ap, 1, 1, vaQ, 3072); S
        MF1(a, 0, 0, 3, 1, Ao); S GL(voffB[3], qB, 3072); LB0(1, vbQ[1], 8192); S
        MF2(a, 0, 0, 3, 0, Ao); S WL(15); LB1(1, Bf[1][1], vbQ[1], 9216); S
        MF1(a, 0, 0, 4, 0, Ao); S XST(0, vl[1], 1024); S
        MF2(a, 0, 0, 4, 1, Ao); S M0(nA[0]); LDA(Ap, 2, 0, vaQ, 4096); AR(tf[0], acc[1][0][2][0]); S
        MF1(a, 0, 1, 4, 1, Ao); S GL(voffA[0], qA, 0); LB0(1, vbQ[2], 0); S
        MF2(a, 0, 1, 4, 0, Ao); S LB1(1, Bf[2][0], vbQ[2], 1024); AR(tf[1], acc[1][0][2][1]); S
        MF1(a, 0, 2, 4, 0, Ao); S AR(tf[2], acc[1][0][2][2]); AR(tf[3], acc[1][0][2][3]); S
        MF2(a, 0, 2, 4, 1, Ao); S LDA(Ap, 2, 1, vaQ, 5120); AR(tf[4], acc[1][0][3][0]); S
        MF1(a, 0, 3, 4, 1, Ao); S GL(voffA[1], qA, 1024); LB0(1, vbQ[2], 8192); S
        MF2(a, 0, 3, 4, 0, Ao); S LB1(1, Bf[2][1], vbQ[2], 9216); AR(tf[5], acc[1][0][3][1]); S
        MF1(a, 0, 3, 5, 0, Ao); S AR(tf[6], acc[1][0][3][2]); AR(tf[7], acc[1][0][3][3]); S
        MF2(a, 0, 3, 5, 1, Ao); S LDA(Ap, 3, 0, vaQ, 6144); w[0].x = CVT(tf[0], tf[1]); S
        MF1(a, 0, 2, 5, 1, Ao); S GL(voffA[2], qA, 2048); LB0(1, vbQ[3], 0); S
        MF2(a, 0, 2, 5, 0, Ao); S LB1(1, Bf[3][0], vbQ[3], 1024); w[0].y = CVT(tf[2], tf[3]); S
        MF1(a, 0, 1, 5, 0, Ao); S w[0].z = CVT(tf[4], tf[5]); w[0].w = CVT(tf[6], tf[7]); S
        MF2(a, 0, 1, 5, 1, Ao); S LDA(Ap, 3, 1, vaQ, 7168); S
        MF1(a, 0, 0, 5, 1, Ao); S GL(voffA[3], qA, 3072); LB0(1, vbQ[3], 8192); S
        MF2(a, 0, 0, 5, 0, Ao); S RLS(qAl, tTl, 4); RLS(qAh, tTh2, 4); S
        MF1(a, 0, 0, 6, 0, Ao); S RLS(qCl, tTl, 20); RLS(qCh, tTh2, 20); S
        MF2(a, 0, 0, 6, 1, Ao); S LB1(1, Bf[3][1], vbQ[3], 9216); XST(0, vl[0], 1024); S
        MF1(a, 0, 1, 6, 1, Ao); S AR(tf[0], acc[1][3][4][0]); AR(tf[1], acc[1][3][4][1]); S
        MF2(a, 0, 1, 6, 0, Ao); S AR(tf[2], acc[1][3][4][2]); AR(tf[3], acc[1][3][4][3]); S
        MF1(a, 0, 2, 6, 0, Ao); S AR(tf[4], acc[1][3][5][0]); AR(tf[5], acc[1][3][5][1]); S
        MF2(a, 0, 2, 6, 1, Ao); S AR(tf[6], acc[1][3][5][2]); AR(tf[7], acc[1][3][5][3]); S
        MF1(a, 0, 3, 6, 1, Ao); S w[0].x = CVT(tf[0], tf[1]); w[0].y = CVT(tf[2], tf[3]); S
        MF2(a, 0, 3, 6, 0, Ao); S WL(15); w[0].z = CVT(tf[4], tf[5]); w[0].w = CVT(tf[6], tf[7]); S
        MF1(a, 0, 3, 7, 0, Ao); S XST(0, vl[3], 2048); S
        MF2(a, 0, 3, 7, 1, Ao); S AR(tf[0], acc[1][2][4][0]); AR(tf[1], acc[1][2][4][1]); S
        MF1(a, 0, 2, 7, 1, Ao); S AR(tf[2], acc[1][2][4][2]); AR(tf[3], acc[1][2][4][3]); S
        MF2(a, 0, 2, 7, 0, Ao); S WVL(17); AR(tf[4], acc[1][2][5][0]); AR(tf[5], acc[1][2][5][1]); S
        MF1(a, 0, 1, 7, 0, Ao); S AR(tf[6], acc[1][2][5][2]); AR(tf[7], acc[1][2][5][3]); S
        MF2(a, 0, 1, 7, 1, Ao); S BAR; w[0].x = CVT(tf[0], tf[1]); w[0].y = CVT(tf[2], tf[3]); S
        MF1(a, 0, 0, 7, 1, Ao); S w[0].z = CVT(tf[4], tf[5]); w[0].w = CVT(tf[6], tf[7]); S
        MF2(a, 0, 0, 7, 0, Ao); S XST(0, vl[2], 2048); S
        TSTAMP(1);
        MF1(a, 0, 0, 0, 0, Ap); S LB0(0, vbQ[0], 16384); AR(tf[0], acc[1][1][4][0]); S
        MF2(a, 0, 0, 0, 1, Ap); S M0(nB1[0]); LB1(0, Bf[4][0], vbQ[0], 17408); AR(tf[1], acc[1][1][4][1]); S
        MF1(a, 0, 1, 0, 1, Ap); S LB0(0, vbQ[0], 24576); GL(voffB[4], qB, 0); S
        MF2(a, 0, 1, 0, 0, Ap); S LB1(0, Bf[4][1], vbQ[0], 25600); AR(tf[2], acc[1][1][4][2]); S
        MF1(a, 0, 2, 0, 0, Ap); S LB0(0, vbQ[1], 16384); AR(tf[3], acc[1][1][4][3]); S
        MF2(a, 0, 2, 0, 1, Ap); S LB1(0, Bf[5][0], vbQ[1], 17408); AR(tf[4], acc[1][1][5][0]); S
        MF1(a, 0, 3, 0, 1, Ap); S LB0(0, vbQ[1], 24576); GL(voffB[5], qB, 1024); S
        MF2(a, 0, 3, 0, 0, Ap); S LB1(0, Bf[5][1], vbQ[1], 25600); AR(tf[5], acc[1][1][5][1]); S
        MF1(a, 0, 3, 1, 0, Ap); S LB0(0, vbQ[2], 16384); AR(tf[6], acc[1][1][5][2]); S
        MF2(a, 0, 3, 1, 1, Ap); S LB1(0, Bf[6][0], vbQ[2], 17408); AR(tf[7], acc[1][1][5][3]); S
        MF1(a, 0, 2, 1, 1, Ap); S LB0(0, vbQ[2], 24576); GL(voffB[6], qB, 2048); S
        MF2(a, 0, 2, 1, 0, Ap); S LB1(0, Bf[6][1], vbQ[2], 25600); w[0].x = CVT(tf[0], tf[1]); S
        MF1(a, 0, 1, 1, 0, Ap); S LB0(0, vbQ[3], 16384); w[0].y = CVT(tf[2], tf[3]); S
        MF2(a, 0, 1, 1, 1, Ap); S LB1(0, Bf[7][0], vbQ[3], 17408); w[0].z = CVT(tf[4], tf[5]); S
        MF1(a, 0, 0, 1, 1, Ap); S LB0(0, vbQ[3], 24576); GL(voffB[7], qB, 3072); S
        MF2(a, 0, 0, 1, 0, Ap); S LB1(0, Bf[7][1], vbQ[3], 25600); w[0].w = CVT(tf[6], tf[7]); S
        MF1(a, 0, 0, 2, 0, Ap); S XST(0, vl[1], 2048); S
        MF2(a, 0, 0, 2, 1, Ap); S M0(nB0[1]); LDA(Ao, 0, 0, va, 16384); AR(tf[0], acc[1][0][4][0]); S
        MF1(a, 0, 1, 2, 1, Ap); S GL(voffB[0], qC, 0); LB0(1, vb2[0], 49152); S
        MF2(a, 0, 1, 2, 0, Ap); S LB1(1, Bf[0][0], vb2[0], 50176); AR(tf[1], acc[1][0][4][1]); S
        MF1(a, 0, 2, 2, 0, Ap); S AR(tf[2], acc[1][0][4][2]); AR(tf[3], acc[1][0][4][3]); S
        MF2(a, 0, 2, 2, 1, Ap); S LDA(Ao, 0, 1, va, 17408); AR(tf[4], acc[1][0][5][0]); S
        MF1(a, 0, 3, 2, 1, Ap); S GL(voffB[1], qC, 1024); LB0(1, vb2[0], 57344); S
        MF2(a, 0, 3, 2, 0, Ap); S LB1(1, Bf[0][1], vb2[0], 58368); AR(tf[5], acc[1][0][5][1]); S
        MF1(a, 0, 3, 3, 0, Ap); S AR(tf[6], acc[1][0][5][2]); AR(tf[7], acc[1][0][5][3]); S
        MF2(a, 0, 3, 3, 1, Ap); S LDA(Ao, 1, 0, va, 18432); w[0].x = CVT(tf[0], tf[1]); S
        MF1(a, 0, 2, 3, 1, Ap); S GL(voffB[2], qC, 2048); LB0(1, vb2[1], 49152); S
        MF2(a, 0, 2, 3, 0, Ap); S LB1(1, Bf[1][0], vb2[1], 50176); w[0].y = CVT(tf[2], tf[3]); S
        MF1(a, 0, 1, 3, 0, Ap); S w[0].z = CVT(tf[4], tf[5]); w[0].w = CVT(tf[6], tf[7]); S
        MF2(a, 0, 1, 3, 1, Ap); S LDA(Ao, 1, 1, va, 19456); S
        MF1(a, 0, 0, 3, 1, Ap); S GL(voffB[3], qC, 3072); LB0(1, vb2[1], 57344); S
        MF2(a, 0, 0, 3, 0, Ap); S WL(15); LB1(1, Bf[1][1], vb2[1], 58368); S
        MF1(a, 0, 0, 4, 0, Ap); S XST(0, vl[0], 2048); S
        MF2(a, 0, 0, 4, 1, Ap); S M0(nA[1]); LDA(Ao, 2, 0, va, 20480); AR(tf[0], acc[1][3][6][0]); S
        MF1(a, 0, 1, 4, 1, Ap); S GL(voffA[0], qA, 0); LB0(1, vb2[2], 49152); S
        MF2(a, 0, 1, 4, 0, Ap); S LB1(1, Bf[2][0], vb2[2], 50176); AR(tf[1], acc[1][3][6][1]); S
        MF1(a, 0, 2, 4, 0, Ap); S AR(tf[2], acc[1][3][6][2]); AR(tf[3], acc[1][3][6][3]); S
        MF2(a, 0, 2, 4, 1, Ap); S LDA(Ao, 2, 1, va, 21504); AR(tf[4], acc[1][3][7][0]); S
        MF1(a, 0, 3, 4, 1, Ap); S GL(voffA[1], qA, 1024); LB0(1, vb2[2], 57344); S
        MF2(a, 0, 3, 4, 0, Ap); S LB1(1, Bf[2][1], vb2[2], 58368); AR(tf[5], acc[1][3][7][1]); S
        MF1(a, 0, 3, 5, 0, Ap); S AR(tf[6], acc[1][3][7][2]); AR(tf[7], acc[1][3][7][3]); S
        MF2(a, 0, 3, 5, 1, Ap); S LDA(Ao, 3, 0, va, 22528); w[0].x = CVT(tf[0], tf[1]); S
        MF1(a, 0, 2, 5, 1, Ap); S GL(voffA[2], qA, 2048); LB0(1, vb2[3], 49152); S
        MF2(a, 0, 2, 5, 0, Ap); S LB1(1, Bf[3][0], vb2[3], 50176); w[0].y = CVT(tf[2], tf[3]); S
        MF1(a, 0, 1, 5, 0, Ap); S w[0].z = CVT(tf[4], tf[5]); w[0].w = CVT(tf[6], tf[7]); S
        MF2(a, 0, 1, 5, 1, Ap); S LDA(Ao, 3, 1, va, 23552); S
        MF1(a, 0, 0, 5, 1, Ap); S GL(voffA[3], qA, 3072); LB0(1, vb2[3], 57344); S
        MF2(a, 0, 0, 5, 0, Ap); S RLS(qAl, tTl, 5); RLS(qAh, tTh2, 5); S
        MF1(a, 0, 0, 6, 0, Ap); S RLS(qBl, tTl, 21); RLS(qBh, tTh2, 21); S
        MF2(a, 0, 0, 6, 1, Ap); S LB1(1, Bf[3][1], vb2[3], 58368); XST(0, vl[3], 3072); S
        MF1(a, 0, 1, 6, 1, Ap); S AR(tf[0], acc[1][2][6][0]); AR(tf[1], acc[1][2][6][1]); S
        MF2(a, 0, 1, 6, 0, Ap); S AR(tf[2], acc[1][2][6][2]); AR(tf[3], acc[1][2][6][3]); S
        MF1(a, 0, 2, 6, 0, Ap); S AR(tf[4], acc[1][2][7][0]); AR(tf[5], acc[1][2][7][1]); S
        MF2(a, 0, 2, 6, 1, Ap); S AR(tf[6], acc[1][2][7][2]); AR(tf[7], acc[1][2][7][3]); S
        MF1(a, 0, 3, 6, 1, Ap); S w[0].x = CVT(tf[0], tf[1]); w[0].y = CVT(tf[2], tf[3]); S
        MF2(a, 0, 3, 6, 0, Ap); S WL(15); w[0].z = CVT(tf[4], tf[5]); w[0].w = CVT(tf[6], tf[7]); S
        MF1(a, 0, 3, 7, 0, Ap); S XST(0, vl[2], 3072); S
        MF2(a, 0, 3, 7, 1, Ap); S AR(tf[0], acc[1][1][6][0]); AR(tf[1], acc[1][1][6][1]); S
        MF1(a, 0, 2, 7, 1, Ap); S AR(tf[2], acc[1][1][6][2]); AR(tf[3], acc[1][1][6][3]); S
        MF2(a, 0, 2, 7, 0, Ap); S WVL(17); AR(tf[4], acc[1][1][7][0]); AR(tf[5], acc[1][1][7][1]); S
        MF1(a, 0, 1, 7, 0, Ap); S AR(tf[6], acc[1][1][7][2]); AR(tf[7], acc[1][1][7][3]); S
        MF2(a, 0, 1, 7, 1, Ap); S BAR; w[0].x = CVT(tf[0], tf[1]); w[0].y = CVT(tf[2], tf[3]); S
        MF1(a, 0, 0, 7, 1, Ap); S w[0].z = CVT(tf[4], tf[5]); w[0].w = CVT(tf[6], tf[7]); S
        MF2(a, 0, 0, 7, 0, Ap); S XST(0, vl[1], 3072); S
        TSTAMP(2);
        MF1(a, 0, 0, 0, 0, Ao); S LB0(0, vb2[0], 32768); AR(tf[0], acc[1][0][6][0]); S
        MF2(a, 0, 0, 0, 1, Ao); S M0(nB1[1]); LB1(0, Bf[4][0], vb2[0], 33792); AR(tf[1], acc[1][0][6][1]); S
        MF1(a, 0, 1, 0, 1, Ao); S LB0(0, vb2[0], 40960); GL(voffB[4], qC, 0); S
        MF2(a, 0, 1, 0, 0, Ao); S LB1(0, Bf[4][1], vb2[0], 41984); AR(tf[2], acc[1][0][6][2]); S
        MF1(a, 0, 2, 0, 0, Ao); S LB0(0, vb2[1], 32768); AR(tf[3], acc[1][0][6][3]); S
        MF2(a, 0, 2, 0, 1, Ao); S LB1(0, Bf[5][0], vb2[1], 33792); AR(tf[4], acc[1][0][7][0]); S
        MF1(a, 0, 3, 0, 1, Ao); S LB0(0, vb2[1], 40960); GL(voffB[5], qC, 1024); S
        MF2(a, 0, 3, 0, 0, Ao); S LB1(0, Bf[5][1], vb2[1], 41984); AR(tf[5], acc[1][0][7][1]); S
        MF1(a, 0, 3, 1, 0, Ao); S LB0(0, vb2[2], 32768); AR(tf[6], acc[1][0][7][2]); S
        MF2(a, 0, 3, 1, 1, Ao); S LB1(0, Bf[6][0], vb2[2], 33792); AR(tf[7], acc[1][0][7][3]); S
        MF1(a, 0, 2, 1, 1, Ao); S LB0(0, vb2[2], 40960); GL(voffB[6], qC, 2048); S
        MF2(a, 0, 2, 1, 0, Ao); S LB1(0, Bf[6][1], vb2[2], 41984); w[0].x = CVT(tf[0], tf[1]); S
        MF1(a, 0, 1, 1, 0, Ao); S LB0(0, vb2[3], 32768); w[0].y = CVT(tf[2], tf[3]); S
        MF2(a, 0, 1, 1, 1, Ao); S LB1(0, Bf[7][0], vb2[3], 33792); w[0].z = CVT(tf[4], tf[5]); S
        MF1(a, 0, 0, 1, 1, Ao); S LB0(0, vb2[3], 40960); GL(voffB[7], qC, 3072); S
        MF2(a, 0, 0, 1, 0, Ao); S LB1(0, Bf[7][1], vb2[3], 41984); w[0].w = CVT(tf[6], tf[7]); S
        MF1(a, 0, 0, 2, 0, Ao); S XST(0, vl[0], 3072); S
        MF2(a, 0, 0, 2, 1, Ao); S M0(nB0[2]); LDA(Ap, 0, 0, vaP, 0); S
        MF1(a, 0, 1, 2, 1, Ao); S GL(voffB[0], qB, 0); LB0(1, vbP[0], 0); S
        MF2(a, 0, 1, 2, 0, Ao); S LB1(1, Bf[0][0], vbP[0], 1024); S
        MF1(a, 0, 2, 2, 0, Ao); S
        MF2(a, 0, 2, 2, 1, Ao); S LDA(Ap, 0, 1, vaP, 1024); S
        MF1(a, 0, 3, 2, 1, Ao); S GL(voffB[1], qB, 1024); LB0(1, vbP[0], 8192); S
        MF2(a, 0, 3, 2, 0, Ao); S LB1(1, Bf[0][1], vbP[0], 9216); S
        MF1(a, 0, 3, 3, 0, Ao); S
        MF2(a, 0, 3, 3, 1, Ao); S LDA(Ap, 1, 0, vaP, 2048); S
        MF1(a, 0, 2, 3, 1, Ao); S GL(voffB[2], qB, 2048); LB0(1, vbP[1], 0); S
        MF2(a, 0, 2, 3, 0, Ao); S LB1(1, Bf[1][0], vbP[1], 1024); S
        MF1(a, 0, 1, 3, 0, Ao); S
        MF2(a, 0, 1, 3, 1, Ao); S LDA(Ap, 1, 1, vaP, 3072); S
        MF1(a, 0, 0, 3, 1, Ao); S GL(voffB[3], qB, 3072); LB0(1, vbP[1], 8192); S
        MF2(a, 0, 0, 3, 0, Ao); S WL(15); LB1(1, Bf[1][1], vbP[1], 9216); S
        MF1(a, 0, 0, 4, 0, Ao); S
        MF2(a, 0, 0, 4, 1, Ao); S M0(nA[2]); LDA(Ap, 2, 0, vaP, 4096); S
        MF1(a, 0, 1, 4, 1, Ao); S GL(voffA[0], qA, 0); LB0(1, vbP[2], 0); S
        MF2(a, 0, 1, 4, 0, Ao); S LB1(1, Bf[2][0], vbP[2], 1024); S
        MF1(a, 0, 2, 4, 0, Ao); S
        MF2(a, 0, 2, 4, 1, Ao); S LDA(Ap, 2, 1, vaP, 5120); S
        MF1(a, 0, 3, 4, 1, Ao); S GL(voffA[1], qA, 1024); LB0(1, vbP[2], 8192); S
        MF2(a, 0, 3, 4, 0, Ao); S LB1(1, Bf[2][1], vbP[2], 9216); S
        MF1(a, 0, 3, 5, 0, Ao); S
        MF2(a, 0, 3, 5, 1, Ao); S LDA(Ap, 3, 0, vaP, 6144); S
        MF1(a, 0, 2, 5, 1, Ao); S GL(voffA[2], qA, 2048); LB0(1, vbP[3], 0); S
        MF2(a, 0, 2, 5, 0, Ao); S LB1(1, Bf[3][0], vbP[3], 1024); S
        MF1(a, 0, 1, 5, 0, Ao); S
        MF2(a, 0, 1, 5, 1, Ao); S LDA(Ap, 3, 1, vaP, 7168); S
        MF1(a, 0, 0, 5, 1, Ao); S GL(voffA[3], qA, 3072); LB0(1, vbP[3], 8192); S
        MF2(a, 0, 0, 5, 0, Ao); S RLS(qAl, tTl, 6); RLS(qAh, tTh2, 6); S
        MF1(a, 0, 0, 6, 0, Ao); S RLS(qCl, tTl, 22); RLS(qCh, tTh2, 22); S
        MF2(a, 0, 0, 6, 1, Ao); S LB1(1, Bf[3][1], vbP[3], 9216); S
        MF1(a, 0, 1, 6, 1, Ao); S
        MF2(a, 0, 1, 6, 0, Ao); S
        MF1(a, 0, 2, 6, 0, Ao); S
        MF2(a, 0, 2, 6, 1, Ao); S
        MF1(a, 0, 3, 6, 1, Ao); S
        MF2(a, 0, 3, 6, 0, Ao); S WL(15); S
        MF1(a, 0, 3, 7, 0, Ao); S
        MF2(a, 0, 3, 7, 1, Ao); S
        MF1(a, 0, 2, 7, 1, Ao); S
        MF2(a, 0, 2, 7, 0, Ao); S WVL(14); S
        MF1(a, 0, 1, 7, 0, Ao); S
        MF2(a, 0, 1, 7, 1, Ao); S BAR; S
        MF1(a, 0, 0, 7, 1, Ao); S
        MF2(a, 0, 0, 7, 0, Ao); S
        TSTAMP(3);
        MF1(a, 0, 0, 0, 0, Ap); S LB0(0, vbP[0], 16384); S
        MF2(a, 0, 0, 0, 1, Ap); S M0(nB1[2]); LB1(0, Bf[4][0], vbP[0], 17408); S
        MF1(a, 0, 1, 0, 1, Ap); S LB0(0, vbP[0], 24576); GL(voffB[4], qB, 0); S
        MF2(a, 0, 1, 0, 0, Ap); S LB1(0, Bf[4][1], vbP[0], 25600); S
        MF1(a, 0, 2, 0, 0, Ap); S LB0(0, vbP[1], 16384); S
        MF2(a, 0, 2, 0, 1, Ap); S LB1(0, Bf[5][0], vbP[1], 17408); S
        MF1(a, 0, 3, 0, 1, Ap); S LB0(0, vbP[1], 24576); GL(voffB[5], qB, 1024); S
        MF2(a, 0, 3, 0, 0, Ap); S LB1(0, Bf[5][1], vbP[1], 25600); S
        MF1(a, 0, 3, 1, 0, Ap); S LB0(0, vbP[2], 16384); S
        MF2(a, 0, 3, 1, 1, Ap); S LB1(0, Bf[6][0], vbP[2], 17408); S
        MF1(a, 0, 2, 1, 1, Ap); S LB0(0, vbP[2], 24576); GL(voffB[6], qB, 2048); S
        MF2(a, 0, 2, 1, 0, Ap); S LB1(0, Bf[6][1], vbP[2], 25600); S
        MF1(a, 0, 1, 1, 0, Ap); S LB0(0, vbP[3], 16384); S
        MF2(a, 0, 1, 1, 1, Ap); S LB1(0, Bf[7][0], vbP[3], 17408); S
        MF1(a, 0, 0, 1, 1, Ap); S LB0(0, vbP[3], 24576); GL(voffB[7], qB, 3072); S
        MF2(a, 0, 0, 1, 0, Ap); S LB1(0, Bf[7][1], vbP[3], 25600); S
        MF1(a, 0, 0, 2, 0, Ap); S
        MF2(a, 0, 0, 2, 1, Ap); S M0(nB0[0]); LDA(Ao, 0, 0, vaQ, 0); S
        MF1(a, 0, 1, 2, 1, Ap); S GL(voffB[0], qC, 0); LB0(1, vbQ[0], 0); S
        MF2(a, 0, 1, 2, 0, Ap); S LB1(1, Bf[0][0], vbQ[0], 1024); S
        MF1(a, 0, 2, 2, 0, Ap); S
        MF2(a, 0, 2, 2, 1, Ap); S LDA(Ao, 0, 1, vaQ, 1024); S
        MF1(a, 0, 3, 2, 1, Ap); S GL(voffB[1], qC, 1024); LB0(1, vbQ[0], 8192); S
        MF2(a, 0, 3, 2, 0, Ap); S LB1(1, Bf[0][1], vbQ[0], 9216); S
        MF1(a, 0, 3, 3, 0, Ap); S
        MF2(a, 0, 3, 3, 1, Ap); S LDA(Ao, 1, 0, vaQ, 2048); S
        MF1(a, 0, 2, 3, 1, Ap); S GL(voffB[2], qC, 2048); LB0(1, vbQ[1], 0); S
        MF2(a, 0, 2, 3, 0, Ap); S LB1(1, Bf[1][0], vbQ[1], 1024); S
        MF1(a, 0, 1, 3, 0, Ap); S
        MF2(a, 0, 1, 3, 1, Ap); S LDA(Ao, 1, 1, vaQ, 3072); S
        MF1(a, 0, 0, 3, 1, Ap); S GL(voffB[3], qC, 3072); LB0(1, vbQ[1], 8192); S
        MF2(a, 0, 0, 3, 0, Ap); S WL(15); LB1(1, Bf[1][1], vbQ[1], 9216); S
        MF1(a, 0, 0, 4, 0, Ap); S
        MF2(a, 0, 0, 4, 1, Ap); S M0(nA[0]); LDA(Ao, 2, 0, vaQ, 4096); S
        MF1(a, 0, 1, 4, 1, Ap); S GL(voffA[0], qA, 0); LB0(1, vbQ[2], 0); S
        MF2(a, 0, 1, 4, 0, Ap); S LB1(1, Bf[2][0], vbQ[2], 1024); S
        MF1(a, 0, 2, 4, 0, Ap); S
        MF2(a, 0, 2, 4, 1, Ap); S LDA(Ao, 2, 1, vaQ, 5120); S
        MF1(a, 0, 3, 4, 1, Ap); S GL(voffA[1], qA, 1024); LB0(1, vbQ[2], 8192); S
        MF2(a, 0, 3, 4, 0, Ap); S LB1(1, Bf[2][1], vbQ[2], 9216); S
        MF1(a, 0, 3, 5, 0, Ap); S
        MF2(a, 0, 3, 5, 1, Ap); S LDA(Ao, 3, 0, vaQ, 6144); S
        MF1(a, 0, 2, 5, 1, Ap); S GL(voffA[2], qA, 2048); LB0(1, vbQ[3], 0); S
        MF2(a, 0, 2, 5, 0, Ap); S LB1(1, Bf[3][0], vbQ[3], 1024); S
        MF1(a, 0, 1, 5, 0, Ap); S
        MF2(a, 0, 1, 5, 1, Ap); S LDA(Ao, 3, 1, vaQ, 7168); S
        MF1(a, 0, 0, 5, 1, Ap); S GL(voffA[3], qA, 3072); LB0(1, vbQ[3], 8192); S
        MF2(a, 0, 0, 5, 0, Ap); S RLS(qAl, tTl, 7); RLS(qAh, tTh2, 7); S
        MF1(a, 0, 0, 6, 0, Ap); S RLS(qBl, tTl, 23); RLS(qBh, tTh2, 23); S
        MF2(a, 0, 0, 6, 1, Ap); S LB1(1, Bf[3][1], vbQ[3], 9216); S
        MF1(a, 0, 1, 6, 1, Ap); S
        MF2(a, 0, 1, 6, 0, Ap); S
        MF1(a, 0, 2, 6, 0, Ap); S
        MF2(a, 0, 2, 6, 1, Ap); S
        MF1(a, 0, 3, 6, 1, Ap); S
        MF2(a, 0, 3, 6, 0, Ap); S WL(15); S
        MF1(a, 0, 3, 7, 0, Ap); S
        MF2(a, 0, 3, 7, 1, Ap); S
        MF1(a, 0, 2, 7, 1, Ap); S
        MF2(a, 0, 2, 7, 0, Ap); S WVL(12); S
        MF1(a, 0, 1, 7, 0, Ap); S
        MF2(a, 0, 1, 7, 1, Ap); S BAR; S
        MF1(a, 0, 0, 7, 1, Ap); S
        MF2(a, 0, 0, 7, 0, Ap); S
        TSTAMP(4);
        MF1(a, 0, 0, 0, 0, Ao); S LB0(0, vbQ[0], 16384); S
        MF2(a, 0, 0, 0, 1, Ao); S M0(nB1[0]); LB1(0, Bf[4][0], vbQ[0], 17408); S
        MF1(a, 0, 1, 0, 1, Ao); S LB0(0, vbQ[0], 24576); GL(voffB[4], qC, 0); S
        MF2(a, 0, 1, 0, 0, Ao); S LB1(0, Bf[4][1], vbQ[0], 25600); S
        MF1(a, 0, 2, 0, 0, Ao); S LB0(0, vbQ[1], 16384); S
        MF2(a, 0, 2, 0, 1, Ao); S LB1(0, Bf[5][0], vbQ[1], 17408); S
        MF1(a, 0, 3, 0, 1, Ao); S LB0(0, vbQ[1], 24576); GL(voffB[5], qC, 1024); S
        MF2(a, 0, 3, 0, 0, Ao); S LB1(0, Bf[5][1], vbQ[1], 25600); S
        MF1(a, 0, 3, 1, 0, Ao); S LB0(0, vbQ[2], 16384); S
        MF2(a, 0, 3, 1, 1, Ao); S LB1(0, Bf[6][0], vbQ[2], 17408); S
        MF1(a, 0, 2, 1, 1, Ao); S LB0(0, vbQ[2], 24576); GL(voffB[6], qC, 2048); S
        MF2(a, 0, 2, 1, 0, Ao); S LB1(0, Bf[6][1], vbQ[2], 25600); S
        MF1(a, 0, 1, 1, 0, Ao); S LB0(0, vbQ[3], 16384); S
        MF2(a, 0, 1, 1, 1, Ao); S LB1(0, Bf[7][0], vbQ[3], 17408); S
        MF1(a, 0, 0, 1, 1, Ao); S LB0(0, vbQ[3], 24576); GL(voffB[7], qC, 3072); S
        MF2(a, 0, 0, 1, 0, Ao); S LB1(0, Bf[7][1], vbQ[3], 25600); S
        MF1(a, 0, 0, 2, 0, Ao); S
        MF2(a, 0, 0, 2, 1, Ao); S M0(nB0[1]); LDA(Ap, 0, 0, va, 16384); S
        MF1(a, 0, 1, 2, 1, Ao); S GL(voffB[0], qB, 0); LB0(1, vb2[0], 49152); S
        MF2(a, 0, 1, 2, 0, Ao); S LB1(1, Bf[0][0], vb2[0], 50176); S
        MF1(a, 0, 2, 2, 0, Ao); S
        MF2(a, 0, 2, 2, 1, Ao); S LDA(Ap, 0, 1, va, 17408); S
        MF1(a, 0, 3, 2, 1, Ao); S GL(voffB[1], qB, 1024); LB0(1, vb2[0], 57344); S
        MF2(a, 0, 3, 2, 0, Ao); S LB1(1, Bf[0][1], vb2[0], 58368); S
        MF1(a, 0, 3, 3, 0, Ao); S
        MF2(a, 0, 3, 3, 1, Ao); S LDA(Ap, 1, 0, va, 18432); S
        MF1(a, 0, 2, 3, 1, Ao); S GL(voffB[2], qB, 2048); LB0(1, vb2[1], 49152); S
        MF2(a, 0, 2, 3, 0, Ao); S LB1(1, Bf[1][0], vb2[1], 50176); S
        MF1(a, 0, 1, 3, 0, Ao); S
        MF2(a, 0, 1, 3, 1, Ao); S LDA(Ap, 1, 1, va, 19456); S
        MF1(a, 0, 0, 3, 1, Ao); S GL(voffB[3], qB, 3072); LB0(1, vb2[1], 57344); S
        MF2(a, 0, 0, 3, 0, Ao); S WL(15); LB1(1, Bf[1][1], vb2[1], 58368); S
        MF1(a, 0, 0, 4, 0, Ao); S
        MF2(a, 0, 0, 4, 1, Ao); S M0(nA[1]); LDA(Ap, 2, 0, va, 20480); S
        MF1(a, 0, 1, 4, 1, Ao); S GL(voffA[0], qA, 0); LB0(1, vb2[2], 49152); S
        MF2(a, 0, 1, 4, 0, Ao); S LB1(1, Bf[2][0], vb2[2], 50176); S
        MF1(a, 0, 2, 4, 0, Ao); S
        MF2(a, 0, 2, 4, 1, Ao); S LDA(Ap, 2, 1, va, 21504); S
        MF1(a, 0, 3, 4, 1, Ao); S GL(voffA[1], qA, 1024); LB0(1, vb2[2], 57344); S
        MF2(a, 0, 3, 4, 0, Ao); S LB1(1, Bf[2][1], vb2[2], 58368); S
        MF1(a, 0, 3, 5, 0, Ao); S
        MF2(a, 0, 3, 5, 1, Ao); S LDA(Ap, 3, 0, va, 22528); S
        MF1(a, 0, 2, 5, 1, Ao); S GL(voffA[2], qA, 2048); LB0(1, vb2[3], 49152); S
        MF2(a, 0, 2, 5, 0, Ao); S LB1(1, Bf[3][0], vb2[3], 50176); S
        MF1(a, 0, 1, 5, 0, Ao); S
        MF2(a, 0, 1, 5, 1, Ao); S LDA(Ap, 3, 1, va, 23552); S
        MF1(a, 0, 0, 5, 1, Ao); S GL(voffA[3], qA, 3072); LB0(1, vb2[3], 57344); S
        MF2(a, 0, 0, 5, 0, Ao); S RLS(qAl, tTl, 8); RLS(qAh, tTh2, 8); S
        MF1(a, 0, 0, 6, 0, Ao); S RLS(qCl, tTl, 24); RLS(qCh, tTh2, 24); S
        MF2(a, 0, 0, 6, 1, Ao); S LB1(1, Bf[3][1], vb2[3], 58368); S
        MF1(a, 0, 1, 6, 1, Ao); S
        MF2(a, 0, 1, 6, 0, Ao); S
        MF1(a, 0, 2, 6, 0, Ao); S
        MF2(a, 0, 2, 6, 1, Ao); S
        MF1(a, 0, 3, 6, 1, Ao); S
        MF2(a, 0, 3, 6, 0, Ao); S WL(15); S
        MF1(a, 0, 3, 7, 0, Ao); S
        MF2(a, 0, 3, 7, 1, Ao); S
        MF1(a, 0, 2, 7, 1, Ao); S
        MF2(a, 0, 2, 7, 0, Ao); S WVL(12); S
        MF1(a, 0, 1, 7, 0, Ao); S
        MF2(a, 0, 1, 7, 1, Ao); S BAR; S
        MF1(a, 0, 0, 7, 1, Ao); S
        MF2(a, 0, 0, 7, 0, Ao); S
        TSTAMP(5);
        MF1(a, 0, 0, 0, 0, Ap); S LB0(0, vb2[0], 32768); S
        MF2(a, 0, 0, 0, 1, Ap); S M0(nB1[1]); LB1(0, Bf[4][0], vb2[0], 33792); XFLAG; S
        MF1(a, 0, 1, 0, 1, Ap); S LB0(0, vb2[0], 40960); GL(voffB[4], qB, 0); S
        MF2(a, 0, 1, 0, 0, Ap); S LB1(0, Bf[4][1], vb2[0], 41984); S
        MF1(a, 0, 2, 0, 0, Ap); S LB0(0, vb2[1], 32768); S
        MF2(a, 0, 2, 0, 1, Ap); S LB1(0, Bf[5][0], vb2[1], 33792); S
        MF1(a, 0, 3, 0, 1, Ap); S LB0(0, vb2[1], 40960); GL(voffB[5], qB, 1024); S
        MF2(a, 0, 3, 0, 0, Ap); S LB1(0, Bf[5][1], vb2[1], 41984); S
        MF1(a, 0, 3, 1, 0, Ap); S LB0(0, vb2[2], 32768); S
        MF2(a, 0, 3, 1, 1, Ap); S LB1(0, Bf[6][0], vb2[2], 33792); S
        MF1(a, 0, 2, 1, 1, Ap); S LB0(0, vb2[2], 40960); GL(voffB[6], qB, 2048); S
        MF2(a, 0, 2, 1, 0, Ap); S LB1(0, Bf[6][1], vb2[2], 41984); S
        MF1(a, 0, 1, 1, 0, Ap); S LB0(0, vb2[3], 32768); S
        MF2(a, 0, 1, 1, 1, Ap); S LB1(0, Bf[7][0], vb2[3], 33792); S
        MF1(a, 0, 0, 1, 1, Ap); S LB0(0, vb2[3], 40960); GL(voffB[7], qB, 3072); S
        MF2(a, 0, 0, 1, 0, Ap); S LB1(0, Bf[7][1], vb2[3], 41984); S
        MF1(a, 0, 0, 2, 0, Ap); S
        MF2(a, 0, 0, 2, 1, Ap); S M0(nB0[2]); LDA(Ao, 0, 0, vaP, 0); S
        MF1(a, 0, 1, 2, 1, Ap); S GL(voffB[0], qC, 0); LB0(1, vbP[0], 0); S
        MF2(a, 0, 1, 2, 0, Ap); S LB1(1, Bf[0][0], vbP[0], 1024); S
        MF1(a, 0, 2, 2, 0, Ap); S
        MF2(a, 0, 2, 2, 1, Ap); S LDA(Ao, 0, 1, vaP, 1024); S
        MF1(a, 0, 3, 2, 1, Ap); S GL(voffB[1], qC, 1024); LB0(1, vbP[0], 8192); S
        MF2(a, 0, 3, 2, 0, Ap); S LB1(1, Bf[0][1], vbP[0], 9216); S
        MF1(a, 0, 3, 3, 0, Ap); S
        MF2(a, 0, 3, 3, 1, Ap); S LDA(Ao, 1, 0, vaP, 2048); S
        MF1(a, 0, 2, 3, 1, Ap); S GL(voffB[2], qC, 2048); LB0(1, vbP[1], 0); S
        MF2(a, 0, 2, 3, 0, Ap); S LB1(1, Bf[1][0], vbP[1], 1024); S
        MF1(a, 0, 1, 3, 0, Ap); S
        MF2(a, 0, 1, 3, 1, Ap); S LDA(Ao, 1, 1, vaP, 3072); S
        MF1(a, 0, 0, 3, 1, Ap); S GL(voffB[3], qC, 3072); LB0(1, vbP[1], 8192); S
        MF2(a, 0, 0, 3, 0, Ap); S WL(15); LB1(1, Bf[1][1], vbP[1], 9216); S
        MF1(a, 0, 0, 4, 0, Ap); S
        MF2(a, 0, 0, 4, 1, Ap); S M0(nA[2]); LDA(Ao, 2, 0, vaP, 4096); S
        MF1(a, 0, 1, 4, 1, Ap); S GL(voffA[0], qA, 0); LB0(1, vbP[2], 0); S
        MF2(a, 0, 1, 4, 0, Ap); S LB1(1, Bf[2][0], vbP[2], 1024); S
        MF1(a, 0, 2, 4, 0, Ap); S
        MF2(a, 0, 2, 4, 1, Ap); S LDA(Ao, 2, 1, vaP, 5120); S
        MF1(a, 0, 3, 4, 1, Ap); S GL(voffA[1], qA, 1024); LB0(1, vbP[2], 8192); S
        MF2(a, 0, 3, 4, 0, Ap); S LB1(1, Bf[2][1], vbP[2], 9216); S
        MF1(a, 0, 3, 5, 0, Ap); S
        MF2(a, 0, 3, 5, 1, Ap); S LDA(Ao, 3, 0, vaP, 6144); S
        MF1(a, 0, 2, 5, 1, Ap); S GL(voffA[2], qA, 2048); LB0(1, vbP[3], 0); S
        MF2(a, 0, 2, 5, 0, Ap); S LB1(1, Bf[3][0], vbP[3], 1024); S
        MF1(a, 0, 1, 5, 0, Ap); S
        MF2(a, 0, 1, 5, 1, Ap); S LDA(Ao, 3, 1, vaP, 7168); S
        MF1(a, 0, 0, 5, 1, Ap); S GL(voffA[3], qA, 3072); LB0(1, vbP[3], 8192); S
        MF2(a, 0, 0, 5, 0, Ap); S RLS(qAl, tTl, 9); RLS(qAh, tTh2, 9); S
        MF1(a, 0, 0, 6, 0, Ap); S RLS(qBl, tTl, 25); RLS(qBh, tTh2, 25); S
        MF2(a, 0, 0, 6, 1, Ap); S LB1(1, Bf[3][1], vbP[3], 9216); S
        MF1(a, 0, 1, 6, 1, Ap); S
        MF2(a, 0, 1, 6, 0, Ap); S
        MF1(a, 0, 2, 6, 0, Ap); S
        MF2(a, 0, 2, 6, 1, Ap); S
        MF1(a, 0, 3, 6, 1, Ap); S
        MF2(a, 0, 3, 6, 0, Ap); S WL(15); S
        MF1(a, 0, 3, 7, 0, Ap); S
        MF2(a, 0, 3, 7, 1, Ap); S
        MF1(a, 0, 2, 7, 1, Ap); S
        MF2(a, 0, 2, 7, 0, Ap); S WVL(13); S
        MF1(a, 0, 1, 7, 0, Ap); S
        MF2(a, 0, 1, 7, 1, Ap); S BAR; S
        MF1(a, 0, 0, 7, 1, Ap); S
        MF2(a, 0, 0, 7, 0, Ap); S
        TSTAMP(6);
        MF1(a, 0, 0, 0, 0, Ao); S LB0(0, vbP[0], 16384); S
        MF2(a, 0, 0, 0, 1, Ao); S M0(nB1[2]); LB1(0, Bf[4][0], vbP[0], 17408); S
        MF1(a, 0, 1, 0, 1, Ao); S LB0(0, vbP[0], 24576); GL(voffB[4], qC, 0); S
        MF2(a, 0, 1, 0, 0, Ao); S LB1(0, Bf[4][1], vbP[0], 25600); S
        MF1(a, 0, 2, 0, 0, Ao); S LB0(0, vbP[1], 16384); S
        MF2(a, 0, 2, 0, 1, Ao); S LB1(0, Bf[5][0], vbP[1], 17408); S
        MF1(a, 0, 3, 0, 1, Ao); S LB0(0, vbP[1], 24576); GL(voffB[5], qC, 1024); S
        MF2(a, 0, 3, 0, 0, Ao); S LB1(0, Bf[5][1], vbP[1], 25600); S
        MF1(a, 0, 3, 1, 0, Ao); S LB0(0, vbP[2], 16384); S
        MF2(a, 0, 3, 1, 1, Ao); S LB1(0, Bf[6][0], vbP[2], 17408); S
        MF1(a, 0, 2, 1, 1, Ao); S LB0(0, vbP[2], 24576); GL(voffB[6], qC, 2048); S
        MF2(a, 0, 2, 1, 0, Ao); S LB1(0, Bf[6][1], vbP[2], 25600); S
        MF1(a, 0, 1, 1, 0, Ao); S LB0(0, vbP[3], 16384); S
        MF2(a, 0, 1, 1, 1, Ao); S LB1(0, Bf[7][0], vbP[3], 17408); S
        MF1(a, 0, 0, 1, 1, Ao); S LB0(0, vbP[3], 24576); GL(voffB[7], qC, 3072); S
        MF2(a, 0, 0, 1, 0, Ao); S LB1(0, Bf[7][1], vbP[3], 25600); S
        MF1(a, 0, 0, 2, 0, Ao); S
        MF2(a, 0, 0, 2, 1, Ao); S M0(nB0[0]); LDA(Ap, 0, 0, vaQ, 0); S
        MF1(a, 0, 1, 2, 1, Ao); S GL(voffB[0], qB, 0); LB0(1, vbQ[0], 0); S
        MF2(a, 0, 1, 2, 0, Ao); S LB1(1, Bf[0][0], vbQ[0], 1024); S
        MF1(a, 0, 2, 2, 0, Ao); S
        MF2(a, 0, 2, 2, 1, Ao); S LDA(Ap, 0, 1, vaQ, 1024); S
        MF1(a, 0, 3, 2, 1, Ao); S GL(voffB[1], qB, 1024); LB0(1, vbQ[0], 8192); S
        MF2(a, 0, 3, 2, 0, Ao); S LB1(1, Bf[0][1], vbQ[0], 9216); S
        MF1(a, 0, 3, 3, 0, Ao); S
        MF2(a, 0, 3, 3, 1, Ao); S LDA(Ap, 1, 0, vaQ, 2048); S
        MF1(a, 0, 2, 3, 1, Ao); S GL(voffB[2], qB, 2048); LB0(1, vbQ[1], 0); S
        MF2(a, 0, 2, 3, 0, Ao); S LB1(1, Bf[1][0], vbQ[1], 1024); S
        MF1(a, 0, 1, 3, 0, Ao); S
        MF2(a, 0, 1, 3, 1, Ao); S LDA(Ap, 1, 1, vaQ, 3072); S
        MF1(a, 0, 0, 3, 1, Ao); S GL(voffB[3], qB, 3072); LB0(1, vbQ[1], 8192); S
        MF2(a, 0, 0, 3, 0, Ao); S WL(15); LB1(1, Bf[1][1], vbQ[1], 9216); S
        MF1(a, 0, 0, 4, 0, Ao); S
        MF2(a, 0, 0, 4, 1, Ao); S M0(nA[0]); LDA(Ap, 2, 0, vaQ, 4096); S
        MF1(a, 0, 1, 4, 1, Ao); S GL(voffA[0], qA, 0); LB0(1, vbQ[2], 0); S
        MF2(a, 0, 1, 4, 0, Ao); S LB1(1, Bf[2][0], vbQ[2], 1024); S
        MF1(a, 0, 2, 4, 0, Ao); S
        MF2(a, 0, 2, 4, 1, Ao); S LDA(Ap, 2, 1, vaQ, 5120); S
        MF1(a, 0, 3, 4, 1, Ao); S GL(voffA[1], qA, 1024); LB0(1, vbQ[2], 8192); S
        MF2(a, 0, 3, 4, 0, Ao); S LB1(1, Bf[2][1], vbQ[2], 9216); S
        MF1(a, 0, 3, 5, 0, Ao); S
        MF2(a, 0, 3, 5, 1, Ao); S LDA(Ap, 3, 0, vaQ, 6144); S
        MF1(a, 0, 2, 5, 1, Ao); S GL(voffA[2], qA, 2048); LB0(1, vbQ[3], 0); S
        MF2(a, 0, 2, 5, 0, Ao); S LB1(1, Bf[3][0], vbQ[3], 1024); S
        MF1(a, 0, 1, 5, 0, Ao); S
        MF2(a, 0, 1, 5, 1, Ao); S LDA(Ap, 3, 1, vaQ, 7168); S
        MF1(a, 0, 0, 5, 1, Ao); S GL(voffA[3], qA, 3072); LB0(1, vbQ[3], 8192); S
        MF2(a, 0, 0, 5, 0, Ao); S RLS(qAl, tTl, 10); RLS(qAh, tTh2, 10); S
        MF1(a, 0, 0, 6, 0, Ao); S RLS(qCl, tTl, 26); RLS(qCh, tTh2, 26); S
        MF2(a, 0, 0, 6, 1, Ao); S LB1(1, Bf[3][1], vbQ[3], 9216); S
        MF1(a, 0, 1, 6, 1, Ao); S
        MF2(a, 0, 1, 6, 0, Ao); S
        MF1(a, 0, 2, 6, 0, Ao); S
        MF2(a, 0, 2, 6, 1, Ao); S
        MF1(a, 0, 3, 6, 1, Ao); S
        MF2(a, 0, 3, 6, 0, Ao); S WL(15); S
        MF1(a, 0, 3, 7, 0, Ao); S
        MF2(a, 0, 3, 7, 1, Ao); S
        MF1(a, 0, 2, 7, 1, Ao); S
        MF2(a, 0, 2, 7, 0, Ao); S WVL(12); S
        MF1(a, 0, 1, 7, 0, Ao); S
        MF2(a, 0, 1, 7, 1, Ao); S BAR; S
        MF1(a, 0, 0, 7, 1, Ao); S
        MF2(a, 0, 0, 7, 0, Ao); S
        TSTAMP(7);
        MF1(a, 0, 0, 0, 0, Ap); S LB0(0, vbQ[0], 16384); S
        MF2(a, 0, 0, 0, 1, Ap); S M0(nB1[0]); LB1(0, Bf[4][0], vbQ[0], 17408); S
        MF1(a, 0, 1, 0, 1, Ap); S LB0(0, vbQ[0], 24576); GL(voffB[4], qB, 0); S
        MF2(a, 0, 1, 0, 0, Ap); S LB1(0, Bf[4][1], vbQ[0], 25600); S
        MF1(a, 0, 2, 0, 0, Ap); S LB0(0, vbQ[1], 16384); S
        MF2(a, 0, 2, 0, 1, Ap); S LB1(0, Bf[5][0], vbQ[1], 17408); S
        MF1(a, 0, 3, 0, 1, Ap); S LB0(0, vbQ[1], 24576); GL(voffB[5], qB, 1024); S
        MF2(a, 0, 3, 0, 0, Ap); S LB1(0, Bf[5][1], vbQ[1], 25600); S
        MF1(a, 0, 3, 1, 0, Ap); S LB0(0, vbQ[2], 16384); S
        MF2(a, 0, 3, 1, 1, Ap); S LB1(0, Bf[6][0], vbQ[2], 17408); S
        MF1(a, 0, 2, 1, 1, Ap); S LB0(0, vbQ[2], 24576); GL(voffB[6], qB, 2048); S
        MF2(a, 0, 2, 1, 0, Ap); S LB1(0, Bf[6][1], vbQ[2], 25600); S
        MF1(a, 0, 1, 1, 0, Ap); S LB0(0, vbQ[3], 16384); S
        MF2(a, 0, 1, 1, 1, Ap); S LB1(0, Bf[7][0], vbQ[3], 17408); S
        MF1(a, 0, 0, 1, 1, Ap); S LB0(0, vbQ[3], 24576); GL(voffB[7], qB, 3072); S
        MF2(a, 0, 0, 1, 0, Ap); S LB1(0, Bf[7][1], vbQ[3], 25600); S
        MF1(a, 0, 0, 2, 0, Ap); S
        MF2(a, 0, 0, 2, 1, Ap); S M0(nB0[1]); LDA(Ao, 0, 0, va, 16384); S
        MF1(a, 0, 1, 2, 1, Ap); S GL(voffB[0], qC, 0); LB0(1, vb2[0], 49152); S
        MF2(a, 0, 1, 2, 0, Ap); S LB1(1, Bf[0][0], vb2[0], 50176); S
        MF1(a, 0, 2, 2, 0, Ap); S
        MF2(a, 0, 2, 2, 1, Ap); S LDA(Ao, 0, 1, va, 17408); S
        MF1(a, 0, 3, 2, 1, Ap); S GL(voffB[1], qC, 1024); LB0(1, vb2[0], 57344); S
        MF2(a, 0, 3, 2, 0, Ap); S LB1(1, Bf[0][1], vb2[0], 58368); S
        MF1(a, 0, 3, 3, 0, Ap); S
        MF2(a, 0, 3, 3, 1, Ap); S LDA(Ao, 1, 0, va, 18432); S
        MF1(a, 0, 2, 3, 1, Ap); S GL(voffB[2], qC, 2048); LB0(1, vb2[1], 49152); S
        MF2(a, 0, 2, 3, 0, Ap); S LB1(1, Bf[1][0], vb2[1], 50176); S
        MF1(a, 0, 1, 3, 0, Ap); S
        MF2(a, 0, 1, 3, 1, Ap); S LDA(Ao, 1, 1, va, 19456); S
        MF1(a, 0, 0, 3, 1, Ap); S GL(voffB[3], qC, 3072); LB0(1, vb2[1], 57344); S
        MF2(a, 0, 0, 3, 0, Ap); S WL(15); LB1(1, Bf[1][1], vb2[1], 58368); S
        MF1(a, 0, 0, 4, 0, Ap); S
        MF2(a, 0, 0, 4, 1, Ap); S M0(nA[1]); LDA(Ao, 2, 0, va, 20480); S
        MF1(a, 0, 1, 4, 1, Ap); S GL(voffA[0], qA, 0); LB0(1, vb2[2], 49152); S
        MF2(a, 0, 1, 4, 0, Ap); S LB1(1, Bf[2][0], vb2[2], 50176); S
        MF1(a, 0, 2, 4, 0, Ap); S
        MF2(a, 0, 2, 4, 1, Ap); S LDA(Ao, 2, 1, va, 21504); S
        MF1(a, 0, 3, 4, 1, Ap); S GL(voffA[1], qA, 1024); LB0(1, vb2[2], 57344); S
        MF2(a, 0, 3, 4, 0, Ap); S LB1(1, Bf[2][1], vb2[2], 58368); S
        MF1(a, 0, 3, 5, 0, Ap); S
        MF2(a, 0, 3, 5, 1, Ap); S LDA(Ao, 3, 0, va, 22528); S
        MF1(a, 0, 2, 5, 1, Ap); S GL(voffA[2], qA, 2048); LB0(1, vb2[3], 49152); S
        MF2(a, 0, 2, 5, 0, Ap); S LB1(1, Bf[3][0], vb2[3], 50176); S
        MF1(a, 0, 1, 5, 0, Ap); S
        MF2(a, 0, 1, 5, 1, Ap); S LDA(Ao, 3, 1, va, 23552); S
        MF1(a, 0, 0, 5, 1, Ap); S GL(voffA[3], qA, 3072); LB0(1, vb2[3], 57344); S
        MF2(a, 0, 0, 5, 0, Ap); S RLS(qAl, tTl, 11); RLS(qAh, tTh2, 11); S
        MF1(a, 0, 0, 6, 0, Ap); S RLS(qBl, tTl, 27); RLS(qBh, tTh2, 27); S
        MF2(a, 0, 0, 6, 1, Ap); S LB1(1, Bf[3][1], vb2[3], 58368); S
        MF1(a, 0, 1, 6, 1, Ap); S
        MF2(a, 0, 1, 6, 0, Ap); S
        MF1(a, 0, 2, 6, 0, Ap); S
        MF2(a, 0, 2, 6, 1, Ap); S
        MF1(a, 0, 3, 6, 1, Ap); S
        MF2(a, 0, 3, 6, 0, Ap); S WL(15); S
        MF1(a, 0, 3, 7, 0, Ap); S
        MF2(a, 0, 3, 7, 1, Ap); S
        MF1(a, 0, 2, 7, 1, Ap); S
        MF2(a, 0, 2, 7, 0, Ap); S WVL(12); S
        MF1(a, 0, 1, 7, 0, Ap); S
        MF2(a, 0, 1, 7, 1, Ap); S BAR; S
        MF1(a, 0, 0, 7, 1, Ap); S
        MF2(a, 0, 0, 7, 0, Ap); S
        TSTAMP(8);
        MF1(a, 0, 0, 0, 0, Ao); S LB0(0, vb2[0], 32768); S
        MF2(a, 0, 0, 0, 1, Ao); S M0(nB1[1]); LB1(0, Bf[4][0], vb2[0], 33792); S
        MF1(a, 0, 1, 0, 1, Ao); S LB0(0, vb2[0], 40960); GL(voffB[4], qC, 0); S
        MF2(a, 0, 1, 0, 0, Ao); S LB1(0, Bf[4][1], vb2[0], 41984); S
        MF1(a, 0, 2, 0, 0, Ao); S LB0(0, vb2[1], 32768); S
        MF2(a, 0, 2, 0, 1, Ao); S LB1(0, Bf[5][0], vb2[1], 33792); S
        MF1(a, 0, 3, 0, 1, Ao); S LB0(0, vb2[1], 40960); GL(voffB[5], qC, 1024); S
        MF2(a, 0, 3, 0, 0, Ao); S LB1(0, Bf[5][1], vb2[1], 41984); S
        MF1(a, 0, 3, 1, 0, Ao); S LB0(0, vb2[2], 32768); S
        MF2(a, 0, 3, 1, 1, Ao); S LB1(0, Bf[6][0], vb2[2], 33792); S
        MF1(a, 0, 2, 1, 1, Ao); S LB0(0, vb2[2], 40960); GL(voffB[6], qC, 2048); S
        MF2(a, 0, 2, 1, 0, Ao); S LB1(0, Bf[6][1], vb2[2], 41984); S
        MF1(a, 0, 1, 1, 0, Ao); S LB0(0, vb2[3], 32768); S
        MF2(a, 0, 1, 1, 1, Ao); S LB1(0, Bf[7][0], vb2[3], 33792); S
        MF1(a, 0, 0, 1, 1, Ao); S LB0(0, vb2[3], 40960); GL(voffB[7], qC, 3072); S
        MF2(a, 0, 0, 1, 0, Ao); S LB1(0, Bf[7][1], vb2[3], 41984); S
        MF1(a, 0, 0, 2, 0, Ao); S
        MF2(a, 0, 0, 2, 1, Ao); S M0(nB0[2]); LDA(Ap, 0, 0, vaP, 0); S
        MF1(a, 0, 1, 2, 1, Ao); S GL(voffB[0], qB, 0); LB0(1, vbP[0], 0); S
        MF2(a, 0, 1, 2, 0, Ao); S LB1(1, Bf[0][0], vbP[0], 1024); S
        MF1(a, 0, 2, 2, 0, Ao); S
        MF2(a, 0, 2, 2, 1, Ao); S LDA(Ap, 0, 1, vaP, 1024); S
        MF1(a, 0, 3, 2, 1, Ao); S GL(voffB[1], qB, 1024); LB0(1, vbP[0], 8192); S
        MF2(a, 0, 3, 2, 0, Ao); S LB1(1, Bf[0][1], vbP[0], 9216); S
        MF1(a, 0, 3, 3, 0, Ao); S
        MF2(a, 0, 3, 3, 1, Ao); S LDA(Ap, 1, 0, vaP, 2048); S
        MF1(a, 0, 2, 3, 1, Ao); S GL(voffB[2], qB, 2048); LB0(1, vbP[1], 0); S
        MF2(a, 0, 2, 3, 0, Ao); S LB1(1, Bf[1][0], vbP[1], 1024); S
        MF1(a, 0, 1, 3, 0, Ao); S
        MF2(a, 0, 1, 3, 1, Ao); S LDA(Ap, 1, 1, vaP, 3072); S
        MF1(a, 0, 0, 3, 1, Ao); S GL(voffB[3], qB, 3072); LB0(1, vbP[1], 8192); S
        MF2(a, 0, 0, 3, 0, Ao); S WL(15); LB1(1, Bf[1][1], vbP[1], 9216); S
        MF1(a, 0, 0, 4, 0, Ao); S
        MF2(a, 0, 0, 4, 1, Ao); S M0(nA[2]); LDA(Ap, 2, 0, vaP, 4096); S
        MF1(a, 0, 1, 4, 1, Ao); S GL(voffA[0], qA, 0); LB0(1, vbP[2], 0); S
        MF2(a, 0, 1, 4, 0, Ao); S LB1(1, Bf[2][0], vbP[2], 1024); S
        MF1(a, 0, 2, 4, 0, Ao); S
        MF2(a, 0, 2, 4, 1, Ao); S LDA(Ap, 2, 1, vaP, 5120); S
        MF1(a, 0, 3, 4, 1, Ao); S GL(voffA[1], qA, 1024); LB0(1, vbP[2], 8192); S
        MF2(a, 0, 3, 4, 0, Ao); S LB1(1, Bf[2][1], vbP[2], 9216); S
        MF1(a, 0, 3, 5, 0, Ao); S
        MF2(a, 0, 3, 5, 1, Ao); S LDA(Ap, 3, 0, vaP, 6144); S
        MF1(a, 0, 2, 5, 1, Ao); S GL(voffA[2], qA, 2048); LB0(1, vbP[3], 0); S
        MF2(a, 0, 2, 5, 0, Ao); S LB1(1, Bf[3][0], vbP[3], 1024); S
        MF1(a, 0, 1, 5, 0, Ao); S
        MF2(a, 0, 1, 5, 1, Ao); S LDA(Ap, 3, 1, vaP, 7168); S
        MF1(a, 0, 0, 5, 1, Ao); S GL(voffA[3], qA, 3072); LB0(1, vbP[3], 8192); S
        MF2(a, 0, 0, 5, 0, Ao); S RLS(qAl, tTl, 12); RLS(qAh, tTh2, 12); S
        MF1(a, 0, 0, 6, 0, Ao); S RLS(qCl, tTl, 28); RLS(qCh, tTh2, 28); S
        MF2(a, 0, 0, 6, 1, Ao); S LB1(1, Bf[3][1], vbP[3], 9216); S
        MF1(a, 0, 1, 6, 1, Ao); S
        MF2(a, 0, 1, 6, 0, Ao); S
        MF1(a, 0, 2, 6, 0, Ao); S
        MF2(a, 0, 2, 6, 1, Ao); S
        MF1(a, 0, 3, 6, 1, Ao); S
        MF2(a, 0, 3, 6, 0, Ao); S WL(15); S
        MF1(a, 0, 3, 7, 0, Ao); S
        MF2(a, 0, 3, 7, 1, Ao); S
        MF1(a, 0, 2, 7, 1, Ao); S
        MF2(a, 0, 2, 7, 0, Ao); S WVL(12); S
        MF1(a, 0, 1, 7, 0, Ao); S
        MF2(a, 0, 1, 7, 1, Ao); S BAR; S
        MF1(a, 0, 0, 7, 1, Ao); S
        MF2(a, 0, 0, 7, 0, Ao); S
        TSTAMP(9);
        MF1(a, 0, 0, 0, 0, Ap); S M0(mF); LB0(0, vbP[0], 16384); S
        MF2(a, 0, 0, 0, 1, Ap); S LB1(0, Bf[4][0], vbP[0], 17408); GFLAG(vzero, pflag); S
        MF1(a, 0, 1, 0, 1, Ap); S M0(nB1[2]); LB0(0, vbP[0], 24576); S
        MF2(a, 0, 1, 0, 0, Ap); S LB1(0, Bf[4][1], vbP[0], 25600); GL(voffB[4], qB, 0); S
        MF1(a, 0, 2, 0, 0, Ap); S LB0(0, vbP[1], 16384); S
        MF2(a, 0, 2, 0, 1, Ap); S LB1(0, Bf[5][0], vbP[1], 17408); S
        MF1(a, 0, 3, 0, 1, Ap); S LB0(0, vbP[1], 24576); S
        MF2(a, 0, 3, 0, 0, Ap); S LB1(0, Bf[5][1], vbP[1], 25600); GL(voffB[5], qB, 1024); S
        MF1(a, 0, 3, 1, 0, Ap); S LB0(0, vbP[2], 16384); S
        MF2(a, 0, 3, 1, 1, Ap); S LB1(0, Bf[6][0], vbP[2], 17408); S
        MF1(a, 0, 2, 1, 1, Ap); S LB0(0, vbP[2], 24576); S
        MF2(a, 0, 2, 1, 0, Ap); S LB1(0, Bf[6][1], vbP[2], 25600); GL(voffB[6], qB, 2048); S
        MF1(a, 0, 1, 1, 0, Ap); S LB0(0, vbP[3], 16384); S
        MF2(a, 0, 1, 1, 1, Ap); S LB1(0, Bf[7][0], vbP[3], 17408); S
        MF1(a, 0, 0, 1, 1, Ap); S LB0(0, vbP[3], 24576); S
        MF2(a, 0, 0, 1, 0, Ap); S LB1(0, Bf[7][1], vbP[3], 25600); GL(voffB[7], qB, 3072); S
        MF1(a, 0, 0, 2, 0, Ap); S
        MF2(a, 0, 0, 2, 1, Ap); S LDA(Ao, 0, 0, vaQ, 0); S
        MF1(a, 0, 1, 2, 1, Ap); S M0(nB0[0]); LB0(1, vbQ[0], 0); S
        MF2(a, 0, 1, 2, 0, Ap); S GL(voffB[0], qC, 0); LB1(1, Bf[0][0], vbQ[0], 1024); S
        MF1(a, 0, 2, 2, 0, Ap); S
        MF2(a, 0, 2, 2, 1, Ap); S LDA(Ao, 0, 1, vaQ, 1024); S
        MF1(a, 0, 3, 2, 1, Ap); S LB0(1, vbQ[0], 8192); S
        MF2(a, 0, 3, 2, 0, Ap); S GL(voffB[1], qC, 1024); LB1(1, Bf[0][1], vbQ[0], 9216); S
        MF1(a, 0, 3, 3, 0, Ap); S
        MF2(a, 0, 3, 3, 1, Ap); S LDA(Ao, 1, 0, vaQ, 2048); S
        MF1(a, 0, 2, 3, 1, Ap); S LB0(1, vbQ[1], 0); S
        MF2(a, 0, 2, 3, 0, Ap); S GL(voffB[2], qC, 2048); LB1(1, Bf[1][0], vbQ[1], 1024); S
        MF1(a, 0, 1, 3, 0, Ap); S
        MF2(a, 0, 1, 3, 1, Ap); S LDA(Ao, 1, 1, vaQ, 3072); S
        MF1(a, 0, 0, 3, 1, Ap); S LB0(1, vbQ[1], 8192); S
        MF2(a, 0, 0, 3, 0, Ap); S WL(15); GL(voffB[3], qC, 3072); LB1(1, Bf[1][1], vbQ[1], 9216); S
        MF1(a, 0, 0, 4, 0, Ap); S
        MF2(a, 0, 0, 4, 1, Ap); S LDA(Ao, 2, 0, vaQ, 4096); S
        MF1(a, 0, 1, 4, 1, Ap); S M0(nA[0]); LB0(1, vbQ[2], 0); S
        MF2(a, 0, 1, 4, 0, Ap); S GL(voffA[0], qA, 0); LB1(1, Bf[2][0], vbQ[2], 1024); S
        MF1(a, 0, 2, 4, 0, Ap); S
        MF2(a, 0, 2, 4, 1, Ap); S LDA(Ao, 2, 1, vaQ, 5120); S
        MF1(a, 0, 3, 4, 1, Ap); S LB0(1, vbQ[2], 8192); S
        MF2(a, 0, 3, 4, 0, Ap); S GL(voffA[1], qA, 1024); LB1(1, Bf[2][1], vbQ[2], 9216); S
        MF1(a, 0, 3, 5, 0, Ap); S
        MF2(a, 0, 3, 5, 1, Ap); S LDA(Ao, 3, 0, vaQ, 6144); S
        MF1(a, 0, 2, 5, 1, Ap); S LB0(1, vbQ[3], 0); S
        MF2(a, 0, 2, 5, 0, Ap); S GL(voffA[2], qA, 2048); LB1(1, Bf[3][0], vbQ[3], 1024); S
        MF1(a, 0, 1, 5, 0, Ap); S
        MF2(a, 0, 1, 5, 1, Ap); S LDA(Ao, 3, 1, vaQ, 7168); S
        MF1(a, 0, 0, 5, 1, Ap); S LB0(1, vbQ[3], 8192); S
        MF2(a, 0, 0, 5, 0, Ap); S GL(voffA[3], qA, 3072); LB1(1, Bf[3][1], vbQ[3], 9216); S
        MF1(a, 0, 0, 6, 0, Ap); S RLS(qAl, tTl, 13); RLS(qAh, tTh2, 13); S
        MF2(a, 0, 0, 6, 1, Ap); S RLS(qBl, tTl, 29); RLS(qBh, tTh2, 29); S
        MF1(a, 0, 1, 6, 1, Ap); S
        MF2(a, 0, 1, 6, 0, Ap); S
        MF1(a, 0, 2, 6, 0, Ap); S
        MF2(a, 0, 2, 6, 1, Ap); S
        MF1(a, 0, 3, 6, 1, Ap); S
        MF2(a, 0, 3, 6, 0, Ap); S WL(15); S
        MF1(a, 0, 3, 7, 0, Ap); S
        MF2(a, 0, 3, 7, 1, Ap); S
        MF1(a, 0, 2, 7, 1, Ap); S
        MF2(a, 0, 2, 7, 0, Ap); S WVL(13); S
        MF1(a, 0, 1, 7, 0, Ap); S
        MF2(a, 0, 1, 7, 1, Ap); S BAR; S
        MF1(a, 0, 0, 7, 1, Ap); S
        MF2(a, 0, 0, 7, 0, Ap); S
        TSTAMP(10);
        MF1(a, 0, 0, 0, 0, Ao); S LB0(0, vbQ[0], 16384); S
        MF2(a, 0, 0, 0, 1, Ao); S M0(nB1[0]); LB1(0, Bf[4][0], vbQ[0], 17408); S
        MF1(a, 0, 1, 0, 1, Ao); S LB0(0, vbQ[0], 24576); GL(voffB[4], qC, 0); S
        MF2(a, 0, 1, 0, 0, Ao); S LB1(0, Bf[4][1], vbQ[0], 25600); S
        MF1(a, 0, 2, 0, 0, Ao); S LB0(0, vbQ[1], 16384); S
        MF2(a, 0, 2, 0, 1, Ao); S LB1(0, Bf[5][0], vbQ[1], 17408); S
        MF1(a, 0, 3, 0, 1, Ao); S LB0(0, vbQ[1], 24576); GL(voffB[5], qC, 1024); S
        MF2(a, 0, 3, 0, 0, Ao); S LB1(0, Bf[5][1], vbQ[1], 25600); S
        MF1(a, 0, 3, 1, 0, Ao); S LB0(0, vbQ[2], 16384); S
        MF2(a, 0, 3, 1, 1, Ao); S LB1(0, Bf[6][0], vbQ[2], 17408); S
        MF1(a, 0, 2, 1, 1, Ao); S LB0(0, vbQ[2], 24576); GL(voffB[6], qC, 2048); S
        MF2(a, 0, 2, 1, 0, Ao); S LB1(0, Bf[6][1], vbQ[2], 25600); S
        MF1(a, 0, 1, 1, 0, Ao); S LB0(0, vbQ[3], 16384); S
        MF2(a, 0, 1, 1, 1, Ao); S LB1(0, Bf[7][0], vbQ[3], 17408); S
        MF1(a, 0, 0, 1, 1, Ao); S LB0(0, vbQ[3], 24576); GL(voffB[7], qC, 3072); S
        MF2(a, 0, 0, 1, 0, Ao); S LB1(0, Bf[7][1], vbQ[3], 25600); S
        MF1(a, 0, 0, 2, 0, Ao); S
        MF2(a, 0, 0, 2, 1, Ao); S M0(nB0[1]); LDA(Ap, 0, 0, va, 16384); S
        MF1(a, 0, 1, 2, 1, Ao); S GL(voffB[0], qB, 0); LB0(1, vb2[0], 49152); S
        MF2(a, 0, 1, 2, 0, Ao); S LB1(1, Bf[0][0], vb2[0], 50176); S
        MF1(a, 0, 2, 2, 0, Ao); S
        MF2(a, 0, 2, 2, 1, Ao); S LDA(Ap, 0, 1, va, 17408); S
        MF1(a, 0, 3, 2, 1, Ao); S GL(voffB[1], qB, 1024); LB0(1, vb2[0], 57344); S
        MF2(a, 0, 3, 2, 0, Ao); S LB1(1, Bf[0][1], vb2[0], 58368); S
        MF1(a, 0, 3, 3, 0, Ao); S
        MF2(a, 0, 3, 3, 1, Ao); S LDA(Ap, 1, 0, va, 18432); S
        MF1(a, 0, 2, 3, 1, Ao); S GL(voffB[2], qB, 2048); LB0(1, vb2[1], 49152); S
        MF2(a, 0, 2, 3, 0, Ao); S LB1(1, Bf[1][0], vb2[1], 50176); S
        MF1(a, 0, 1, 3, 0, Ao); S
        MF2(a, 0, 1, 3, 1, Ao); S LDA(Ap, 1, 1, va, 19456); S
        MF1(a, 0, 0, 3, 1, Ao); S GL(voffB[3], qB, 3072); LB0(1, vb2[1], 57344); S
        MF2(a, 0, 0, 3, 0, Ao); S WL(15); LB1(1, Bf[1][1], vb2[1], 58368); S
        MF1(a, 0, 0, 4, 0, Ao); S
        MF2(a, 0, 0, 4, 1, Ao); S M0(nA[1]); LDA(Ap, 2, 0, va, 20480); S
        MF1(a, 0, 1, 4, 1, Ao); S GL(voffA[0], qA, 0); LB0(1, vb2[2], 49152); S
        MF2(a, 0, 1, 4, 0, Ao); S LB1(1, Bf[2][0], vb2[2], 50176); S
        MF1(a, 0, 2, 4, 0, Ao); S
        MF2(a, 0, 2, 4, 1, Ao); S LDA(Ap, 2, 1, va, 21504); S
        MF1(a, 0, 3, 4, 1, Ao); S GL(voffA[1], qA, 1024); LB0(1, vb2[2], 57344); S
        MF2(a, 0, 3, 4, 0, Ao); S LB1(1, Bf[2][1], vb2[2], 58368); S
        MF1(a, 0, 3, 5, 0, Ao); S
        MF2(a, 0, 3, 5, 1, Ao); S LDA(Ap, 3, 0, va, 22528); S
        MF1(a, 0, 2, 5, 1, Ao); S GL(voffA[2], qA, 2048); LB0(1, vb2[3], 49152); S
        MF2(a, 0, 2, 5, 0, Ao); S LB1(1, Bf[3][0], vb2[3], 50176); S
        MF1(a, 0, 1, 5, 0, Ao); S
        MF2(a, 0, 1, 5, 1, Ao); S LDA(Ap, 3, 1, va, 23552); S
        MF1(a, 0, 0, 5, 1, Ao); S GL(voffA[3], qA, 3072); LB0(1, vb2[3], 57344); S
        MF2(a, 0, 0, 5, 0, Ao); S RLS(qAl, tTl, 14); RLS(qAh, tTh2, 14); S
        MF1(a, 0, 0, 6, 0, Ao); S RLS(qCl, tTl, 30); RLS(qCh, tTh2, 30); S
        MF2(a, 0, 0, 6, 1, Ao); S LB1(1, Bf[3][1], vb2[3], 58368); S
        MF1(a, 0, 1, 6, 1, Ao); S
        MF2(a, 0, 1, 6, 0, Ao); S
        MF1(a, 0, 2, 6, 0, Ao); S
        MF2(a, 0, 2, 6, 1, Ao); S
        MF1(a, 0, 3, 6, 1, Ao); S
        MF2(a, 0, 3, 6, 0, Ao); S WL(15); S
        MF1(a, 0, 3, 7, 0, Ao); S
        MF2(a, 0, 3, 7, 1, Ao); S
        MF1(a, 0, 2, 7, 1, Ao); S
        MF2(a, 0, 2, 7, 0, Ao); S WVL(12); S
        MF1(a, 0, 1, 7, 0, Ao); S
        MF2(a, 0, 1, 7, 1, Ao); S BAR; S
        MF1(a, 0, 0, 7, 1, Ao); S
        MF2(a, 0, 0, 7, 0, Ao); S
        TSTAMP(11);
        LDF;
        MF1(a, 0, 0, 0, 0, Ap); S LB0(0, vb2[0], 32768); S
        MF2(a, 0, 0, 0, 1, Ap); S M0(nB1[1]); LB1(0, Bf[4][0], vb2[0], 33792); S
        MF1(a, 0, 1, 0, 1, Ap); S LB0(0, vb2[0], 40960); GL(voffB[4], qB, 0); S
        MF2(a, 0, 1, 0, 0, Ap); S LB1(0, Bf[4][1], vb2[0], 41984); S
        MF1(a, 0, 2, 0, 0, Ap); S LB0(0, vb2[1], 32768); S
        MF2(a, 0, 2, 0, 1, Ap); S LB1(0, Bf[5][0], vb2[1], 33792); S
        MF1(a, 0, 3, 0, 1, Ap); S LB0(0, vb2[1], 40960); GL(voffB[5], qB, 1024); S
        MF2(a, 0, 3, 0, 0, Ap); S LB1(0, Bf[5][1], vb2[1], 41984); S
        MF1(a, 0, 3, 1, 0, Ap); S LB0(0, vb2[2], 32768); S
        MF2(a, 0, 3, 1, 1, Ap); S LB1(0, Bf[6][0], vb2[2], 33792); S
        MF1(a, 0, 2, 1, 1, Ap); S LB0(0, vb2[2], 40960); GL(voffB[6], qB, 2048); S
        MF2(a, 0, 2, 1, 0, Ap); S LB1(0, Bf[6][1], vb2[2], 41984); S
        MF1(a, 0, 1, 1, 0, Ap); S LB0(0, vb2[3], 32768); S
        MF2(a, 0, 1, 1, 1, Ap); S LB1(0, Bf[7][0], vb2[3], 33792); S
        MF1(a, 0, 0, 1, 1, Ap); S LB0(0, vb2[3], 40960); GL(voffB[7], qB, 3072); S
        MF2(a, 0, 0, 1, 0, Ap); S LB1(0, Bf[7][1], vb2[3], 41984); S
        MF1(a, 0, 0, 2, 0, Ap); S
        MF2(a, 0, 0, 2, 1, Ap); S M0(nB0[2]); LDA(Ao, 0, 0, vaP, 0); S
        MF1(a, 0, 1, 2, 1, Ap); S GL(voffB[0], qC, 0); LB0(1, vbP[0], 0); S
        MF2(a, 0, 1, 2, 0, Ap); S LB1(1, Bf[0][0], vbP[0], 1024); S
        MF1(a, 0, 2, 2, 0, Ap); S
        MF2(a, 0, 2, 2, 1, Ap); S LDA(Ao, 0, 1, vaP, 1024); S
        MF1(a, 0, 3, 2, 1, Ap); S GL(voffB[1], qC, 1024); LB0(1, vbP[0], 8192); S
        MF2(a, 0, 3, 2, 0, Ap); S LB1(1, Bf[0][1], vbP[0], 9216); S
        MF1(a, 0, 3, 3, 0, Ap); S
        MF2(a, 0, 3, 3, 1, Ap); S LDA(Ao, 1, 0, vaP, 2048); S
        MF1(a, 0, 2, 3, 1, Ap); S GL(voffB[2], qC, 2048); LB0(1, vbP[1], 0); S
        MF2(a, 0, 2, 3, 0, Ap); S LB1(1, Bf[1][0], vbP[1], 1024); S
        MF1(a, 0, 1, 3, 0, Ap); S
        MF2(a, 0, 1, 3, 1, Ap); S LDA(Ao, 1, 1, vaP, 3072); S
        MF1(a, 0, 0, 3, 1, Ap); S GL(voffB[3], qC, 3072); LB0(1, vbP[1], 8192); S
        MF2(a, 0, 0, 3, 0, Ap); S WL(15); LB1(1, Bf[1][1], vbP[1], 9216); S
        MF1(a, 0, 0, 4, 0, Ap); S PIN("+v"(fv)); early = (unsigned)__builtin_amdgcn_readfirstlane(fv); PIN("+s"(early)); S
        MF2(a, 0, 0, 4, 1, Ap); S M0(nA[2]); LDA(Ao, 2, 0, vaP, 4096); S
        MF1(a, 0, 1, 4, 1, Ap); S GL(voffA[0], qA, 0); LB0(1, vbP[2], 0); S
        MF2(a, 0, 1, 4, 0, Ap); S LB1(1, Bf[2][0], vbP[2], 1024); S
        MF1(a, 0, 2, 4, 0, Ap); S XLD(in[12], vl[3], 0); S
        MF2(a, 0, 2, 4, 1, Ap); S LDA(Ao, 2, 1, vaP, 5120); S
        MF1(a, 0, 3, 4, 1, Ap); S GL(voffA[1], qA, 1024); LB0(1, vbP[2], 8192); S
        MF2(a, 0, 3, 4, 0, Ap); S LB1(1, Bf[2][1], vbP[2], 9216); S
        MF1(a, 0, 3, 5, 0, Ap); S XLD(in[8], vl[2], 0); S
        MF2(a, 0, 3, 5, 1, Ap); S LDA(Ao, 3, 0, vaP, 6144); S
        MF1(a, 0, 2, 5, 1, Ap); S GL(voffA[2], qA, 2048); LB0(1, vbP[3], 0); S
        MF2(a, 0, 2, 5, 0, Ap); S LB1(1, Bf[3][0], vbP[3], 1024); S
        MF1(a, 0, 1, 5, 0, Ap); S XLD(in[4], vl[1], 0); S
        MF2(a, 0, 1, 5, 1, Ap); S LDA(Ao, 3, 1, vaP, 7168); S
        MF1(a, 0, 0, 5, 1, Ap); S GL(voffA[3], qA, 3072); LB0(1, vbP[3], 8192); S
        MF2(a, 0, 0, 5, 0, Ap); S RLS(qAl, tTl, 15); RLS(qAh, tTh2, 15); S
        MF1(a, 0, 0, 6, 0, Ap); S RLS(qBl, tTl, 31); RLS(qBh, tTh2, 31); S
        MF2(a, 0, 0, 6, 1, Ap); S LB1(1, Bf[3][1], vbP[3], 9216); XLD(in[0], vl[0], 0); S
        MF1(a, 0, 1, 6, 1, Ap); S
        MF2(a, 0, 1, 6, 0, Ap); S
        MF1(a, 0, 2, 6, 0, Ap); S
        MF2(a, 0, 2, 6, 1, Ap); S
        MF1(a, 0, 3, 6, 1, Ap); S
        MF2(a, 0, 3, 6, 0, Ap); S WL(15); S
        MF1(a, 0, 3, 7, 0, Ap); S
        MF2(a, 0, 3, 7, 1, Ap); S
        MF1(a, 0, 2, 7, 1, Ap); S
        MF2(a, 0, 2, 7, 0, Ap); S WVL(16); S
        MF1(a, 0, 1, 7, 0, Ap); S
        MF2(a, 0, 1, 7, 1, Ap); S BAR; S
        MF1(a, 0, 0, 7, 1, Ap); S
        MF2(a, 0, 0, 7, 0, Ap); S
        TSTAMP(12);
        MF1(a, 0, 0, 0, 0, Ao); S LB0(0, vbP[0], 16384); S
        MF2(a, 0, 0, 0, 1, Ao); S M0(nB1[2]); LB1(0, Bf[4][0], vbP[0], 17408); S
        MF1(a, 0, 1, 0, 1, Ao); S LB0(0, vbP[0], 24576); GL(voffB[4], qC, 0); S
        MF2(a, 0, 1, 0, 0, Ao); S LB1(0, Bf[4][1], vbP[0], 25600); S
        MF1(a, 0, 2, 0, 0, Ao); S LB0(0, vbP[1], 16384); S
        MF2(a, 0, 2, 0, 1, Ao); S LB1(0, Bf[5][0], vbP[1], 17408); S
        MF1(a, 0, 3, 0, 1, Ao); S LB0(0, vbP[1], 24576); GL(voffB[5], qC, 1024); S
        MF2(a, 0, 3, 0, 0, Ao); S LB1(0, Bf[5][1], vbP[1], 25600); S
        MF1(a, 0, 3, 1, 0, Ao); S LB0(0, vbP[2], 16384); S
        MF2(a, 0, 3, 1, 1, Ao); S LB1(0, Bf[6][0], vbP[2], 17408); S
        MF1(a, 0, 2, 1, 1, Ao); S LB0(0, vbP[2], 24576); GL(voffB[6], qC, 2048); S
        MF2(a, 0, 2, 1, 0, Ao); S LB1(0, Bf[6][1], vbP[2], 25600); S
        MF1(a, 0, 1, 1, 0, Ao); S LB0(0, vbP[3], 16384); S
        MF2(a, 0, 1, 1, 1, Ao); S LB1(0, Bf[7][0], vbP[3], 17408); S
        MF1(a, 0, 0, 1, 1, Ao); S LB0(0, vbP[3], 24576); GL(voffB[7], qC, 3072); S
        MF2(a, 0, 0, 1, 0, Ao); S LB1(0, Bf[7][1], vbP[3], 25600); S
        MF1(a, 0, 0, 2, 0, Ao); S
        MF2(a, 0, 0, 2, 1, Ao); S M0(nB0[0]); LDA(Ap, 0, 0, vaQ, 0); S
        MF1(a, 0, 1, 2, 1, Ao); S GL(voffB[0], qB, 0); LB0(1, vbQ[0], 0); S
        MF2(a, 0, 1, 2, 0, Ao); S LB1(1, Bf[0][0], vbQ[0], 1024); S
        MF1(a, 0, 2, 2, 0, Ao); S
        MF2(a, 0, 2, 2, 1, Ao); S LDA(Ap, 0, 1, vaQ, 1024); S
        MF1(a, 0, 3, 2, 1, Ao); S GL(voffB[1], qB, 1024); LB0(1, vbQ[0], 8192); S
        MF2(a, 0, 3, 2, 0, Ao); S LB1(1, Bf[0][1], vbQ[0], 9216); S
        MF1(a, 0, 3, 3, 0, Ao); S
        MF2(a, 0, 3, 3, 1, Ao); S LDA(Ap, 1, 0, vaQ, 2048); S
        MF1(a, 0, 2, 3, 1, Ao); S GL(voffB[2], qB, 2048); LB0(1, vbQ[1], 0); S
        MF2(a, 0, 2, 3, 0, Ao); S LB1(1, Bf[1][0], vbQ[1], 1024); S
        MF1(a, 0, 1, 3, 0, Ao); S
        MF2(a, 0, 1, 3, 1, Ao); S LDA(Ap, 1, 1, vaQ, 3072); S
        MF1(a, 0, 0, 3, 1, Ao); S GL(voffB[3], qB, 3072); LB0(1, vbQ[1], 8192); S
        MF2(a, 0, 0, 3, 0, Ao); S WL(15); LB1(1, Bf[1][1], vbQ[1], 9216); S
        MF1(a, 0, 0, 4, 0, Ao); S
        MF2(a, 0, 0, 4, 1, Ao); S M0(nA[0]); LDA(Ap, 2, 0, vaQ, 4096); S
        MF1(a, 0, 1, 4, 1, Ao); S GL(voffA[0], qA, 0); LB0(1, vbQ[2], 0); S
        MF2(a, 0, 1, 4, 0, Ao); S LB1(1, Bf[2][0], vbQ[2], 1024); S
        MF1(a, 0, 2, 4, 0, Ao); S XLD(in[13], vl[3], 1024); S
        MF2(a, 0, 2, 4, 1, Ao); S LDA(Ap, 2, 1, vaQ, 5120); S
        MF1(a, 0, 3, 4, 1, Ao); S GL(voffA[1], qA, 1024); LB0(1, vbQ[2], 8192); S
        MF2(a, 0, 3, 4, 0, Ao); S LB1(1, Bf[2][1], vbQ[2], 9216); S
        MF1(a, 0, 3, 5, 0, Ao); S XLD(in[9], vl[2], 1024); S
        MF2(a, 0, 3, 5, 1, Ao); S LDA(Ap, 3, 0, vaQ, 6144); S
        MF1(a, 0, 2, 5, 1, Ao); S GL(voffA[2], qA, 2048); LB0(1, vbQ[3], 0); S
        MF2(a, 0, 2, 5, 0, Ao); S LB1(1, Bf[3][0], vbQ[3], 1024); S
        MF1(a, 0, 1, 5, 0, Ao); S XLD(in[5], vl[1], 1024); S
        MF2(a, 0, 1, 5, 1, Ao); S LDA(Ap, 3, 1, vaQ, 7168); S
        MF1(a, 0, 0, 5, 1, Ao); S GL(voffA[3], qA, 3072); LB0(1, vbQ[3], 8192); S
        MF2(a, 0, 0, 5, 0, Ao); S LB1(1, Bf[3][1], vbQ[3], 9216); S
        MF1(a, 0, 0, 6, 0, Ao); S XLD(in[1], vl[0], 1024); S
        MF2(a, 0, 0, 6, 1, Ao); S
        MF1(a, 0, 1, 6, 1, Ao); S
        MF2(a, 0, 1, 6, 0, Ao); S
        MF1(a, 0, 2, 6, 0, Ao); S
        MF2(a, 0, 2, 6, 1, Ao); S
        MF1(a, 0, 3, 6, 1, Ao); S
        MF2(a, 0, 3, 6, 0, Ao); S WL(15); S
        MF1(a, 0, 3, 7, 0, Ao); S
        MF2(a, 0, 3, 7, 1, Ao); S
        MF1(a, 0, 2, 7, 1, Ao); S
        MF2(a, 0, 2, 7, 0, Ao); S WVL(16); S
        MF1(a, 0, 1, 7, 0, Ao); S
        MF2(a, 0, 1, 7, 1, Ao); S BAR; S
        MF1(a, 0, 0, 7, 1, Ao); S
        MF2(a, 0, 0, 7, 0, Ao); S
        TSTAMP(13);
        MF1(a, 0, 0, 0, 0, Ap); S LB0(0, vbQ[0], 16384); S
        MF2(a, 0, 0, 0, 1, Ap); S M0(nB1[0]); LB1(0, Bf[4][0], vbQ[0], 17408); S
        MF1(a, 0, 1, 0, 1, Ap); S LB0(0, vbQ[0], 24576); GL(voffB[4], qB, 0); S
        MF2(a, 0, 1, 0, 0, Ap); S LB1(0, Bf[4][1], vbQ[0], 25600); S
        MF1(a, 0, 2, 0, 0, Ap); S LB0(0, vbQ[1], 16384); S
        MF2(a, 0, 2, 0, 1, Ap); S LB1(0, Bf[5][0], vbQ[1], 17408); S
        MF1(a, 0, 3, 0, 1, Ap); S LB0(0, vbQ[1], 24576); GL(voffB[5], qB, 1024); S
        MF2(a, 0, 3, 0, 0, Ap); S LB1(0, Bf[5][1], vbQ[1], 25600); S
        MF1(a, 0, 3, 1, 0, Ap); S LB0(0, vbQ[2], 16384); S
        MF2(a, 0, 3, 1, 1, Ap); S LB1(0, Bf[6][0], vbQ[2], 17408); S
        MF1(a, 0, 2, 1, 1, Ap); S LB0(0, vbQ[2], 24576); GL(voffB[6], qB, 2048); S
        MF2(a, 0, 2, 1, 0, Ap); S LB1(0, Bf[6][1], vbQ[2], 25600); S
        MF1(a, 0, 1, 1, 0, Ap); S LB0(0, vbQ[3], 16384); S
        MF2(a, 0, 1, 1, 1, Ap); S LB1(0, Bf[7][0], vbQ[3], 17408); S
        MF1(a, 0, 0, 1, 1, Ap); S LB0(0, vbQ[3], 24576); GL(voffB[7], qB, 3072); S
        MF2(a, 0, 0, 1, 0, Ap); S LB1(0, Bf[7][1], vbQ[3], 25600); S
        MF1(a, 0, 0, 2, 0, Ap); S
        MF2(a, 0, 0, 2, 1, Ap); S LDA(Ao, 0, 0, va, 16384); S
        MF1(a, 0, 1, 2, 1, Ap); S LB0(1, vb2[0], 49152); S
        MF2(a, 0, 1, 2, 0, Ap); S LB1(1, Bf[0][0], vb2[0], 50176); S
        MF1(a, 0, 2, 2, 0, Ap); S
        MF2(a, 0, 2, 2, 1, Ap); S LDA(Ao, 0, 1, va, 17408); S
        MF1(a, 0, 3, 2, 1, Ap); S LB0(1, vb2[0], 57344); S
        MF2(a, 0, 3, 2, 0, Ap); S LB1(1, Bf[0][1], vb2[0], 58368); S
        MF1(a, 0, 3, 3, 0, Ap); S
        MF2(a, 0, 3, 3, 1, Ap); S LDA(Ao, 1, 0, va, 18432); S
        MF1(a, 0, 2, 3, 1, Ap); S LB0(1, vb2[1], 49152); S
        MF2(a, 0, 2, 3, 0, Ap); S LB1(1, Bf[1][0], vb2[1], 50176); S
        MF1(a, 0, 1, 3, 0, Ap); S
        MF2(a, 0, 1, 3, 1, Ap); S LDA(Ao, 1, 1, va, 19456); S
        MF1(a, 0, 0, 3, 1, Ap); S LB0(1, vb2[1], 57344); S
        MF2(a, 0, 0, 3, 0, Ap); S WL(15); LB1(1, Bf[1][1], vb2[1], 58368); S
        MF1(a, 0, 0, 4, 0, Ap); S
        MF2(a, 0, 0, 4, 1, Ap); S LDA(Ao, 2, 0, va, 20480); S
        MF1(a, 0, 1, 4, 1, Ap); S LB0(1, vb2[2], 49152); XLD(in[14], vl[3], 2048); S
        MF2(a, 0, 1, 4, 0, Ap); S LB1(1, Bf[2][0], vb2[2], 50176); S
        MF1(a, 0, 2, 4, 0, Ap); S
        MF2(a, 0, 2, 4, 1, Ap); S LDA(Ao, 2, 1, va, 21504); S
        MF1(a, 0, 3, 4, 1, Ap); S LB0(1, vb2[2], 57344); XLD(in[10], vl[2], 2048); S
        MF2(a, 0, 3, 4, 0, Ap); S LB1(1, Bf[2][1], vb2[2], 58368); S
        MF1(a, 0, 3, 5, 0, Ap); S
        MF2(a, 0, 3, 5, 1, Ap); S LDA(Ao, 3, 0, va, 22528); S
        MF1(a, 0, 2, 5, 1, Ap); S LB0(1, vb2[3], 49152); XLD(in[6], vl[1], 2048); S
        MF2(a, 0, 2, 5, 0, Ap); S LB1(1, Bf[3][0], vb2[3], 50176); S
        MF1(a, 0, 1, 5, 0, Ap); S
        MF2(a, 0, 1, 5, 1, Ap); S LDA(Ao, 3, 1, va, 23552); S
        MF1(a, 0, 0, 5, 1, Ap); S LB0(1, vb2[3], 57344); XLD(in[2], vl[0], 2048); S
        MF2(a, 0, 0, 5, 0, Ap); S LB1(1, Bf[3][1], vb2[3], 58368); S
        MF1(a, 0, 0, 6, 0, Ap); S
        MF2(a, 0, 0, 6, 1, Ap); S
        MF1(a, 0, 1, 6, 1, Ap); S
        MF2(a, 0, 1, 6, 0, Ap); S
        MF1(a, 0, 2, 6, 0, Ap); S
        MF2(a, 0, 2, 6, 1, Ap); S
        MF1(a, 0, 3, 6, 1, Ap); S
        MF2(a, 0, 3, 6, 0, Ap); S WL(15); S
        MF1(a, 0, 3, 7, 0, Ap); S
        MF2(a, 0, 3, 7, 1, Ap); S
        MF1(a, 0, 2, 7, 1, Ap); S
        MF2(a, 0, 2, 7, 0, Ap); S WVL(8); S
        MF1(a, 0, 1, 7, 0, Ap); S
        MF2(a, 0, 1, 7, 1, Ap); S BAR; S
        MF1(a, 0, 0, 7, 1, Ap); S
        MF2(a, 0, 0, 7, 0, Ap); S
        TSTAMP(14);
        MF1(a, 0, 0, 0, 0, Ao); S LB0(0, vb2[0], 32768); S
        MF2(a, 0, 0, 0, 1, Ao); S LB1(0, Bf[4][0], vb2[0], 33792); S
        MF1(a, 0, 1, 0, 1, Ao); S E_SO(0); LB0(0, vb2[0], 40960); S
        MF2(a, 0, 1, 0, 0, Ao); S LB1(0, Bf[4][1], vb2[0], 41984); S
        MF1(a, 0, 2, 0, 0, Ao); S E_SO(1); LB0(0, vb2[1], 32768); S
        MF2(a, 0, 2, 0, 1, Ao); S LB1(0, Bf[5][0], vb2[1], 33792); S
        MF1(a, 0, 3, 0, 1, Ao); S E_SO(2); LB0(0, vb2[1], 40960); S
        MF2(a, 0, 3, 0, 0, Ao); S LB1(0, Bf[5][1], vb2[1], 41984); S
        MF1(a, 0, 3, 1, 0, Ao); S E_SO(3); LB0(0, vb2[2], 32768); S
        MF2(a, 0, 3, 1, 1, Ao); S LB1(0, Bf[6][0], vb2[2], 33792); S
        MF1(a, 0, 2, 1, 1, Ao); S LB0(0, vb2[2], 40960); S
        MF2(a, 0, 2, 1, 0, Ao); S LB1(0, Bf[6][1], vb2[2], 41984); S
        MF1(a, 0, 1, 1, 0, Ao); S LB0(0, vb2[3], 32768); S
        MF2(a, 0, 1, 1, 1, Ao); S LB1(0, Bf[7][0], vb2[3], 33792); S
        MF1(a, 0, 0, 1, 1, Ao); S LB0(0, vb2[3], 40960); S
        MF2(a, 0, 0, 1, 0, Ao); S LB1(0, Bf[7][1], vb2[3], 41984); S
        MF1(a, 0, 0, 2, 0, Ao); S
        MF2(a, 0, 0, 2, 1, Ao); S LDA(Ap, 0, 0, vaP, 0); S
        MF1(a, 0, 1, 2, 1, Ao); S LB0(1, vbP[0], 0); S
        MF2(a, 0, 1, 2, 0, Ao); S LB1(1, Bf[0][0], vbP[0], 1024); S
        MF1(a, 0, 2, 2, 0, Ao); S
        MF2(a, 0, 2, 2, 1, Ao); S LDA(Ap, 0, 1, vaP, 1024); S
        MF1(a, 0, 3, 2, 1, Ao); S LB0(1, vbP[0], 8192); S
        MF2(a, 0, 3, 2, 0, Ao); S LB1(1, Bf[0][1], vbP[0], 9216); S
        MF1(a, 0, 3, 3, 0, Ao); S
        MF2(a, 0, 3, 3, 1, Ao); S LDA(Ap, 1, 0, vaP, 2048); S
        MF1(a, 0, 2, 3, 1, Ao); S LB0(1, vbP[1], 0); S
        MF2(a, 0, 2, 3, 0, Ao); S LB1(1, Bf[1][0], vbP[1], 1024); S
        MF1(a, 0, 1, 3, 0, Ao); S
        MF2(a, 0, 1, 3, 1, Ao); S LDA(Ap, 1, 1, vaP, 3072); S
        MF1(a, 0, 0, 3, 1, Ao); S LB0(1, vbP[1], 8192); S
        MF2(a, 0, 0, 3, 0, Ao); S WL(15); LB1(1, Bf[1][1], vbP[1], 9216); S
        MF1(a, 0, 0, 4, 0, Ao); S
        MF2(a, 0, 0, 4, 1, Ao); S LDA(Ap, 2, 0, vaP, 4096); S
        MF1(a, 0, 1, 4, 1, Ao); S LB0(1, vbP[2], 0); XLD(in[15], vl[3], 3072); S
        MF2(a, 0, 1, 4, 0, Ao); S LB1(1, Bf[2][0], vbP[2], 1024); S
        MF1(a, 0, 2, 4, 0, Ao); S
        MF2(a, 0, 2, 4, 1, Ao); S LDA(Ap, 2, 1, vaP, 5120); S
        MF1(a, 0, 3, 4, 1, Ao); S LB0(1, vbP[2], 8192); XLD(in[11], vl[2], 3072); S
        MF2(a, 0, 3, 4, 0, Ao); S LB1(1, Bf[2][1], vbP[2], 9216); S
        MF1(a, 0, 3, 5, 0, Ao); S
        MF2(a, 0, 3, 5, 1, Ao); S LDA(Ap, 3, 0, vaP, 6144); S
        MF1(a, 0, 2, 5, 1, Ao); S LB0(1, vbP[3], 0); XLD(in[7], vl[1], 3072); S
        MF2(a, 0, 2, 5, 0, Ao); S LB1(1, Bf[3][0], vbP[3], 1024); S
        MF1(a, 0, 1, 5, 0, Ao); S
        MF2(a, 0, 1, 5, 1, Ao); S LDA(Ap, 3, 1, vaP, 7168); S
        MF1(a, 0, 0, 5, 1, Ao); S LB0(1, vbP[3], 8192); XLD(in[3], vl[0], 3072); S
        MF2(a, 0, 0, 5, 0, Ao); S LB1(1, Bf[3][1], vbP[3], 9216); S
        MF1(a, 0, 0, 6, 0, Ao); S
        MF2(a, 0, 0, 6, 1, Ao); S
        MF1(a, 0, 1, 6, 1, Ao); S
        MF2(a, 0, 1, 6, 0, Ao); S
        MF1(a, 0, 2, 6, 0, Ao); S
        MF2(a, 0, 2, 6, 1, Ao); S
        MF1(a, 0, 3, 6, 1, Ao); S
        MF2(a, 0, 3, 6, 0, Ao); S WL(15); S
        MF1(a, 0, 3, 7, 0, Ao); S
        MF2(a, 0, 3, 7, 1, Ao); S
        MF1(a, 0, 2, 7, 1, Ao); S
        MF2(a, 0, 2, 7, 0, Ao); S WVL(4); S
        MF1(a, 0, 1, 7, 0, Ao); S
        MF2(a, 0, 1, 7, 1, Ao); S BAR; S
        MF1(a, 0, 0, 7, 1, Ao); S
        MF2(a, 0, 0, 7, 0, Ao); S
        TSTAMP(15); STAMP(3); WKIND(4); LATE_PARTNER LDTM; STAMP(4); S
        MF1(a, 0, 0, 0, 0, Ap); S LB0(0, vbP[0], 16384); S
        MF1(a, 0, 0, 0, 1, Ap); S LB1(0, Bf[4][0], vbP[0], 17408); S
        MF1(a, 0, 1, 0, 1, Ap); S LB0(0, vbP[0], 24576); S
        MF1(a, 0, 1, 0, 0, Ap); S LB1(0, Bf[4][1], vbP[0], 25600); S
        MF1(a, 0, 2, 0, 0, Ap); S LB0(0, vbP[1], 16384); S
        MF1(a, 0, 2, 0, 1, Ap); S LB1(0, Bf[5][0], vbP[1], 17408); S
        MF1(a, 0, 3, 0, 1, Ap); S LB0(0, vbP[1], 24576); S
        MF1(a, 0, 3, 0, 0, Ap); S LB1(0, Bf[5][1], vbP[1], 25600); S
        MF1(a, 0, 3, 1, 0, Ap); S LB0(0, vbP[2], 16384); S
        MF1(a, 0, 3, 1, 1, Ap); S LB1(0, Bf[6][0], vbP[2], 17408); S
        ADDP(a, 3, 1, 12); S LB0(0, vbP[2], 24576); S
        ADDP(a, 3, 0, 12); S LB1(0, Bf[6][1], vbP[2], 25600); S
        MF1(a, 0, 2, 1, 1, Ap); S LB0(0, vbP[3], 16384); S
        MF1(a, 0, 2, 1, 0, Ap); S LB1(0, Bf[7][0], vbP[3], 17408); S
        ADDP(a, 2, 1, 8); S LB0(0, vbP[3], 24576); E_AR(0, 3, 0); S
        ADDP(a, 2, 0, 8); S LB1(0, Bf[7][1], vbP[3], 25600); E_AR(1, 3, 0); S
        MF1(a, 0, 1, 1, 0, Ap); S E_AR(2, 3, 0); E_AR(3, 3, 0); S
        MF1(a, 0, 1, 1, 1, Ap); S E_AR(4, 3, 0); E_AR(5, 3, 0); S
        ADDP(a, 1, 1, 4); S E_AR(6, 3, 0); E_AR(7, 3, 0); S
        ADDP(a, 1, 0, 4); S E_SQ(0); E_SQ(1); S
        MF1(a, 0, 0, 1, 1, Ap); S E_SQ(2); E_SQ(3); S
        MF1(a, 0, 0, 1, 0, Ap); S E_FM(0); E_FM(1); S
        ADDP(a, 0, 1, 0); S E_FM(2); E_FM(3); S
        ADDP(a, 0, 0, 0); S E_MU(0); E_MU(1); S
        MF1(a, 0, 0, 2, 0, Ap); S E_MU(2); E_MU(3); S
        MF1(a, 0, 0, 2, 1, Ap); S E_EX(0, 0); S
        MF1(a, 0, 1, 2, 1, Ap); S E_EX(0, 1); S
        MF1(a, 0, 1, 2, 0, Ap); S E_EX(1, 0); S
        MF1(a, 0, 2, 2, 0, Ap); S E_EX(1, 1); S
        MF1(a, 0, 2, 2, 1, Ap); S E_EX(2, 0); S
        MF1(a, 0, 3, 2, 1, Ap); S E_EX(2, 1); S
        MF1(a, 0, 3, 2, 0, Ap); S E_EX(3, 0); S
        MF1(a, 0, 3, 3, 0, Ap); S E_EX(3, 1); S
        MF1(a, 0, 3, 3, 1, Ap); S E_AD(0); E_AD(1); S
        ADDP(a, 3, 3, 13); S E_AD(2); E_AD(3); S
        ADDP(a, 3, 2, 13); S E_RC(0, 0); S
        MF1(a, 0, 2, 3, 1, Ap); S E_RC(0, 1); S
        MF1(a, 0, 2, 3, 0, Ap); S E_RC(1, 0); S
        ADDP(a, 2, 3, 9); S E_RC(1, 1); S
        ADDP(a, 2, 2, 9); S E_RC(2, 0); S
        MF1(a, 0, 1, 3, 0, Ap); S E_RC(2, 1); S
        MF1(a, 0, 1, 3, 1, Ap); S E_RC(3, 0); S
        ADDP(a, 1, 3, 5); S E_RC(3, 1); S
        ADDP(a, 1, 2, 5); S E_MY(0); E_MY(1); S
        MF1(a, 0, 0, 3, 1, Ap); S E_MY(2); E_MY(3); S
        MF1(a, 0, 0, 3, 0, Ap); S E_AR(0, 3, 2); E_AR(1, 3, 2); S
        ADDP(a, 0, 3, 1); S E_AR(2, 3, 2); E_AR(3, 3, 2); S
        ADDP(a, 0, 2, 1); S WL(0); E_AR(4, 3, 2); E_AR(5, 3, 2); S
        MF1(a, 0, 0, 4, 0, Ap); S E_AR(6, 3, 2); E_AR(7, 3, 2); S
        MF1(a, 0, 0, 4, 1, Ap); S E_CV(0); E_CV(2); S
        MF1(a, 0, 1, 4, 1, Ap); S E_CV(1); E_CV(3); S
        MF1(a, 0, 1, 4, 0, Ap); S E_SQ(0); E_SQ(2); S
        MF1(a, 0, 2, 4, 0, Ap); S E_PL(0); E_PL(1); S
        MF1(a, 0, 2, 4, 1, Ap); S E_ST(3, 0, 0); E_SQ(1); S
        MF1(a, 0, 3, 4, 1, Ap); S E_SQ(3); E_FM(0); S
        MF1(a, 0, 3, 4, 0, Ap); S E_FM(1); E_FM(2); S
        MF1(a, 0, 3, 5, 0, Ap); S E_FM(3); E_KEEP; E_MU(0); S
        MF1(a, 0, 3, 5, 1, Ap); S E_MU(1); E_MU(2); S
        ADDP(a, 3, 5, 14); S E_MU(3); S
        ADDP(a, 3, 4, 14); S E_EX(0, 0); S
        MF1(a, 0, 2, 5, 1, Ap); S E_EX(0, 1); S
        MF1(a, 0, 2, 5, 0, Ap); S E_EX(1, 0); S
        ADDP(a, 2, 5, 10); S E_EX(1, 1); S
        ADDP(a, 2, 4, 10); S E_EX(2, 0); S
        MF1(a, 0, 1, 5, 0, Ap); S E_EX(2, 1); S
        MF1(a, 0, 1, 5, 1, Ap); S E_EX(3, 0); S
        ADDP(a, 1, 5, 6); S E_EX(3, 1); S
        ADDP(a, 1, 4, 6); S E_AD(0); E_AD(1); S
        MF1(a, 0, 0, 5, 1, Ap); S E_AD(2); E_AD(3); S
        MF1(a, 0, 0, 5, 0, Ap); S E_RC(0, 0); S
        ADDP(a, 0, 5, 2); S E_RC(0, 1); S
        ADDP(a, 0, 4, 2); S E_RC(1, 0); S
        MF1(a, 0, 0, 6, 0, Ap); S E_RC(1, 1); S
        MF1(a, 0, 0, 6, 1, Ap); S E_RC(2, 0); S
        MF1(a, 0, 1, 6, 1, Ap); S E_RC(2, 1); S
        MF1(a, 0, 1, 6, 0, Ap); S E_RC(3, 0); S
        MF1(a, 0, 2, 6, 0, Ap); S E_RC(3, 1); S
        MF1(a, 0, 2, 6, 1, Ap); S E_MY(0); E_MY(1); S
        MF1(a, 0, 3, 6, 1, Ap); S E_MY(2); E_MY(3); S
        MF1(a, 0, 3, 6, 0, Ap); S E_AR(0, 2, 0); E_AR(1, 2, 0); S
        MF1(a, 0, 3, 7, 0, Ap); S E_AR(2, 2, 0); E_AR(3, 2, 0); S
        MF1(a, 0, 3, 7, 1, Ap); S WV(2); E_AR(4, 2, 0); E_AR(5, 2, 0); S
        ADDP(a, 3, 7, 15); S E_AR(6, 2, 0); E_AR(7, 2, 0); S
        ADDP(a, 3, 6, 15); S E_CV(0); E_CV(2); S
        MF1(a, 0, 2, 7, 1, Ap); S E_CV(1); E_CV(3); S
        MF1(a, 0, 2, 7, 0, Ap); S E_SQ(0); E_SQ(2); S
        ADDP(a, 2, 7, 11); S E_PL(0); E_PL(1); S
        ADDP(a, 2, 6, 11); S E_ST(3, 0, 64); E_SQ(1); S
        MF1(a, 0, 1, 7, 0, Ap); S E_SQ(3); E_FM(0); S
        MF1(a, 0, 1, 7, 1, Ap); S E_FM(1); E_FM(2); S
        ADDP(a, 1, 7, 7); S E_FM(3); E_KEEP; E_MU(0); S
        ADDP(a, 1, 6, 7); S E_MU(1); E_MU(2); S
        MF1(a, 0, 0, 7, 1, Ap); S E_MU(3); S
        MF1(a, 0, 0, 7, 0, Ap); S E_EX(0, 0); S
        ADDP(a, 0, 7, 3); S E_EX(0, 1); S
        ADDP(a, 0, 6, 3); S E_EX(1, 0); S
        E_EX(1, 1); E_EX(2, 0); E_EX(2, 1); E_EX(3, 0); E_EX(3, 1); E_AD(0); E_AD(1); E_AD(2); S
        E_AD(3); E_RC(0, 0); E_RC(0, 1); E_RC(1, 0); E_RC(1, 1); E_RC(2, 0); E_RC(2, 1); E_RC(3, 0); S
        E_RC(3, 1); E_MY(0); E_MY(1); E_MY(2); E_MY(3); E_AR(0, 2, 2); E_AR(1, 2, 2); E_AR(2, 2, 2); S
        E_AR(3, 2, 2); E_AR(4, 2, 2); E_AR(5, 2, 2); E_AR(6, 2, 2); E_AR(7, 2, 2); E_CV(0); E_CV(2); E_CV(1); S
        E_CV(3); E_SQ(0); E_SQ(2); E_PL(0); E_PL(1); E_ST(2, 0, 0); E_SQ(1); E_SQ(3); S
        E_FM(0); E_FM(1); E_FM(2); E_FM(3); E_KEEP; E_MU(0); E_MU(1); E_MU(2); S
        E_MU(3); E_EX(0, 0); E_EX(0, 1); E_EX(1, 0); E_EX(1, 1); E_EX(2, 0); E_EX(2, 1); E_EX(3, 0); S
        E_EX(3, 1); E_AD(0); E_AD(1); E_AD(2); E_AD(3); E_RC(0, 0); E_RC(0, 1); E_RC(1, 0); S
        E_RC(1, 1); E_RC(2, 0); E_RC(2, 1); E_RC(3, 0); E_RC(3, 1); E_MY(0); E_MY(1); E_MY(2); S
        E_MY(3); E_AR(0, 1, 0); E_AR(1, 1, 0); E_AR(2, 1, 0); E_AR(3, 1, 0); E_AR(4, 1, 0); E_AR(5, 1, 0); E_AR(6, 1, 0); S
        E_AR(7, 1, 0); E_CV(0); E_CV(2); E_CV(1); E_CV(3); E_SQ(0); E_SQ(2); E_PL(0); S
        E_PL(1); E_ST(2, 0, 64); E_SQ(1); E_SQ(3); E_FM(0); E_FM(1); E_FM(2); E_FM(3); S
        E_KEEP; E_MU(0); E_MU(1); E_MU(2); E_MU(3); E_EX(0, 0); E_EX(0, 1); E_EX(1, 0); S
        E_EX(1, 1); E_EX(2, 0); E_EX(2, 1); E_EX(3, 0); E_EX(3, 1); E_AD(0); E_AD(1); E_AD(2); S
        E_AD(3); E_RC(0, 0); E_RC(0, 1); E_RC(1, 0); E_RC(1, 1); E_RC(2, 0); E_RC(2, 1); E_RC(3, 0); S
        E_RC(3, 1); E_MY(0); E_MY(1); E_MY(2); E_MY(3); E_AR(0, 1, 2); E_AR(1, 1, 2); E_AR(2, 1, 2); S
        E_AR(3, 1, 2); E_AR(4, 1, 2); E_AR(5, 1, 2); E_AR(6, 1, 2); E_AR(7, 1, 2); E_CV(0); E_CV(2); E_CV(1); S
        E_CV(3); E_SQ(0); E_SQ(2); E_PL(0); E_PL(1); E_ST(1, 0, 0); E_SQ(1); E_SQ(3); S
        E_FM(0); E_FM(1); E_FM(2); E_FM(3); E_KEEP; E_MU(0); E_MU(1); E_MU(2); S
        E_MU(3); E_EX(0, 0); E_EX(0, 1); E_EX(1, 0); E_EX(1, 1); E_EX(2, 0); E_EX(2, 1); E_EX(3, 0); S
        E_EX(3, 1); E_AD(0); E_AD(1); E_AD(2); E_AD(3); E_RC(0, 0); E_RC(0, 1); E_RC(1, 0); S
        E_RC(1, 1); E_RC(2, 0); E_RC(2, 1); E_RC(3, 0); E_RC(3, 1); E_MY(0); E_MY(1); E_MY(2); S
        E_MY(3); E_AR(0, 0, 0); E_AR(1, 0, 0); E_AR(2, 0, 0); E_AR(3, 0, 0); E_AR(4, 0, 0); E_AR(5, 0, 0); E_AR(6, 0, 0); S
        E_AR(7, 0, 0); E_CV(0); E_CV(2); E_CV(1); E_CV(3); E_SQ(0); E_SQ(2); E_PL(0); S
        E_PL(1); E_ST(1, 0, 64); E_SQ(1); E_SQ(3); E_FM(0); E_FM(1); E_FM(2); E_FM(3); S
        E_KEEP; E_MU(0); E_MU(1); E_MU(2); E_MU(3); E_EX(0, 0); E_EX(0, 1); E_EX(1, 0); S
        E_EX(1, 1); E_EX(2, 0); E_EX(2, 1); E_EX(3, 0); E_EX(3, 1); E_AD(0); E_AD(1); E_AD(2); S
        E_AD(3); E_RC(0, 0); E_RC(0, 1); E_RC(1, 0); E_RC(1, 1); E_RC(2, 0); E_RC(2, 1); E_RC(3, 0); S
        E_RC(3, 1); E_MY(0); E_MY(1); E_MY(2); E_MY(3); E_AR(0, 0, 2); E_AR(1, 0, 2); E_AR(2, 0, 2); S
        E_AR(3, 0, 2); E_AR(4, 0, 2); E_AR(5, 0, 2); E_AR(6, 0, 2); E_AR(7, 0, 2); E_CV(0); E_CV(2); E_CV(1); S
        E_CV(3); E_SQ(0); E_SQ(2); E_PL(0); E_PL(1); E_ST(0, 0, 0); E_SQ(1); E_SQ(3); S
        E_FM(0); E_FM(1); E_FM(2); E_FM(3); E_KEEP; E_MU(0); E_MU(1); E_MU(2); S
        E_MU(3); E_EX(0, 0); E_EX(0, 1); E_EX(1, 0); E_EX(1, 1); E_EX(2, 0); E_EX(2, 1); E_EX(3, 0); S
        E_EX(3, 1); E_AD(0); E_AD(1); E_AD(2); E_AD(3); E_RC(0, 0); E_RC(0, 1); E_RC(1, 0); S
        E_RC(1, 1); E_RC(2, 0); E_RC(2, 1); E_RC(3, 0); E_RC(3, 1); E_MY(0); E_MY(1); E_MY(2); S
        E_MY(3); E_AR(0, 3, 4); E_AR(1, 3, 4); E_AR(2, 3, 4); E_AR(3, 3, 4); E_AR(4, 3, 4); E_AR(5, 3, 4); E_AR(6, 3, 4); S
        E_AR(7, 3, 4); E_CV(0); E_CV(2); E_CV(1); E_CV(3); E_SQ(0); E_SQ(2); E_PL(0); S
        E_PL(1); E_ST(0, 0, 64); E_SQ(1); E_SQ(3); E_FM(0); E_FM(1); E_FM(2); E_FM(3); S
        E_KEEP; E_MU(0); E_MU(1); E_MU(2); E_MU(3); E_EX(0, 0); E_EX(0, 1); E_EX(1, 0); S
        E_EX(1, 1); E_EX(2, 0); E_EX(2, 1); E_EX(3, 0); E_EX(3, 1); E_AD(0); E_AD(1); E_AD(2); S
        E_AD(3); E_RC(0, 0); E_RC(0, 1); E_RC(1, 0); E_RC(1, 1); E_RC(2, 0); E_RC(2, 1); E_RC(3, 0); S
        E_RC(3, 1); E_MY(0); E_MY(1); E_MY(2); E_MY(3); E_AR(0, 3, 6); E_AR(1, 3, 6); E_AR(2, 3, 6); S
        E_AR(3, 3, 6); E_AR(4, 3, 6); E_AR(5, 3, 6); E_AR(6, 3, 6); E_AR(7, 3, 6); E_CV(0); E_CV(2); E_CV(1); S
        E_CV(3); E_SQ(0); E_SQ(2); E_PL(0); E_PL(1); E_ST(3, 1, 0); E_SQ(1); E_SQ(3); S
        E_FM(0); E_FM(1); E_FM(2); E_FM(3); E_KEEP; E_MU(0); E_MU(1); E_MU(2); S
        E_MU(3); E_EX(0, 0); E_EX(0, 1); E_EX(1, 0); E_EX(1, 1); E_EX(2, 0); E_EX(2, 1); E_EX(3, 0); S
        E_EX(3, 1); E_AD(0); E_AD(1); E_AD(2); E_AD(3); E_RC(0, 0); E_RC(0, 1); E_RC(1, 0); S
        E_RC(1, 1); E_RC(2, 0); E_RC(2, 1); E_RC(3, 0); E_RC(3, 1); E_MY(0); E_MY(1); E_MY(2); S
        E_MY(3); E_AR(0, 2, 4); E_AR(1, 2, 4); E_AR(2, 2, 4); E_AR(3, 2, 4); E_AR(4, 2, 4); E_AR(5, 2, 4); E_AR(6, 2, 4); S
        E_AR(7, 2, 4); E_CV(0); E_CV(2); E_CV(1); E_CV(3); E_SQ(0); E_SQ(2); E_PL(0); S
        E_PL(1); E_ST(3, 1, 64); E_SQ(1); E_SQ(3); E_FM(0); E_FM(1); E_FM(2); E_FM(3); S
        E_KEEP; E_MU(0); E_MU(1); E_MU(2); E_MU(3); E_EX(0, 0); E_EX(0, 1); E_EX(1, 0); S
        E_EX(1, 1); E_EX(2, 0); E_EX(2, 1); E_EX(3, 0); E_EX(3, 1); E_AD(0); E_AD(1); E_AD(2); S
        E_AD(3); E_RC(0, 0); E_RC(0, 1); E_RC(1, 0); E_RC(1, 1); E_RC(2, 0); E_RC(2, 1); E_RC(3, 0); S
        E_RC(3, 1); E_MY(0); E_MY(1); E_MY(2); E_MY(3); E_AR(0, 2, 6); E_AR(1, 2, 6); E_AR(2, 2, 6); S
        E_AR(3, 2, 6); E_AR(4, 2, 6); E_AR(5, 2, 6); E_AR(6, 2, 6); E_AR(7, 2, 6); E_CV(0); E_CV(2); E_CV(1); S
        E_CV(3); E_SQ(0); E_SQ(2); E_PL(0); E_PL(1); E_ST(2, 1, 0); E_SQ(1); E_SQ(3); S
        E_FM(0); E_FM(1); E_FM(2); E_FM(3); E_KEEP; E_MU(0); E_MU(1); E_MU(2); S
        E_MU(3); E_EX(0, 0); E_EX(0, 1); E_EX(1, 0); E_EX(1, 1); E_EX(2, 0); E_EX(2, 1); E_EX(3, 0); S
        E_EX(3, 1); E_AD(0); E_AD(1); E_AD(2); E_AD(3); E_RC(0, 0); E_RC(0, 1); E_RC(1, 0); S
        E_RC(1, 1); E_RC(2, 0); E_RC(2, 1); E_RC(3, 0); E_RC(3, 1); E_MY(0); E_MY(1); E_MY(2); S
        E_MY(3); E_AR(0, 1, 4); E_AR(1, 1, 4); E_AR(2, 1, 4); E_AR(3, 1, 4); E_AR(4, 1, 4); E_AR(5, 1, 4); E_AR(6, 1, 4); S
        E_AR(7, 1, 4); E_CV(0); E_CV(2); E_CV(1); E_CV(3); E_SQ(0); E_SQ(2); E_PL(0); S
        E_PL(1); E_ST(2, 1, 64); E_SQ(1); E_SQ(3); E_FM(0); E_FM(1); E_FM(2); E_FM(3); S
        E_KEEP; E_MU(0); E_MU(1); E_MU(2); E_MU(3); E_EX(0, 0); E_EX(0, 1); E_EX(1, 0); S
        E_EX(1, 1); E_EX(2, 0); E_EX(2, 1); E_EX(3, 0); E_EX(3, 1); E_AD(0); E_AD(1); E_AD(2); S
        E_AD(3); E_RC(0, 0); E_RC(0, 1); E_RC(1, 0); E_RC(1, 1); E_RC(2, 0); E_RC(2, 1); E_RC(3, 0); S
        E_RC(3, 1); E_MY(0); E_MY(1); E_MY(2); E_MY(3); E_AR(0, 1, 6); E_AR(1, 1, 6); E_AR(2, 1, 6); S
        E_AR(3, 1, 6); E_AR(4, 1, 6); E_AR(5, 1, 6); E_AR(6, 1, 6); E_AR(7, 1, 6); E_CV(0); E_CV(2); E_CV(1); S
        E_CV(3); E_SQ(0); E_SQ(2); E_PL(0); E_PL(1); E_ST(1, 1, 0); E_SQ(1); E_SQ(3); S
        E_FM(0); E_FM(1); E_FM(2); E_FM(3); E_KEEP; E_MU(0); E_MU(1); E_MU(2); S
        E_MU(3); E_EX(0, 0); E_EX(0, 1); E_EX(1, 0); E_EX(1, 1); E_EX(2, 0); E_EX(2, 1); E_EX(3, 0); S
        E_EX(3, 1); E_AD(0); E_AD(1); E_AD(2); E_AD(3); E_RC(0, 0); E_RC(0, 1); E_RC(1, 0); S
        E_RC(1, 1); E_RC(2, 0); E_RC(2, 1); E_RC(3, 0); E_RC(3, 1); E_MY(0); E_MY(1); E_MY(2); S
        E_MY(3); E_AR(0, 0, 4); E_AR(1, 0, 4); E_AR(2, 0, 4); E_AR(3, 0, 4); E_AR(4, 0, 4); E_AR(5, 0, 4); E_AR(6, 0, 4); S
        E_AR(7, 0, 4); E_CV(0); E_CV(2); E_CV(1); E_CV(3); E_SQ(0); E_SQ(2); E_PL(0); S
        E_PL(1); E_ST(1, 1, 64); E_SQ(1); E_SQ(3); E_FM(0); E_FM(1); E_FM(2); E_FM(3); S
        E_KEEP; E_MU(0); E_MU(1); E_MU(2); E_MU(3); E_EX(0, 0); E_EX(0, 1); E_EX(1, 0); S
        E_EX(1, 1); E_EX(2, 0); E_EX(2, 1); E_EX(3, 0); E_EX(3, 1); E_AD(0); E_AD(1); E_AD(2); S
        E_AD(3); E_RC(0, 0); E_RC(0, 1); E_RC(1, 0); E_RC(1, 1); E_RC(2, 0); E_RC(2, 1); E_RC(3, 0); S
        E_RC(3, 1); E_MY(0); E_MY(1); E_MY(2); E_MY(3); E_AR(0, 0, 6); E_AR(1, 0, 6); E_AR(2, 0, 6); S
        E_AR(3, 0, 6); E_AR(4, 0, 6); E_AR(5, 0, 6); E_AR(6, 0, 6); E_AR(7, 0, 6); E_CV(0); E_CV(2); E_CV(1); S
        E_CV(3); E_SQ(0); E_SQ(2); E_PL(0); E_PL(1); E_ST(0, 1, 0); E_SQ(1); E_SQ(3); S
        E_FM(0); E_FM(1); E_FM(2); E_FM(3); E_KEEP; E_MU(0); E_MU(1); E_MU(2); S
        E_MU(3); E_EX(0, 0); E_EX(0, 1); E_EX(1, 0); E_EX(1, 1); E_EX(2, 0); E_EX(2, 1); E_EX(3, 0); S
        E_EX(3, 1); E_AD(0); E_AD(1); E_AD(2); E_AD(3); E_RC(0, 0); E_RC(0, 1); E_RC(1, 0); S
        E_RC(1, 1); E_RC(2, 0); E_RC(2, 1); E_RC(3, 0); E_RC(3, 1); E_MY(0); E_MY(1); E_MY(2); S
        E_MY(3); E_CV(0); E_CV(2); E_CV(1); E_CV(3); E_PL(0); E_PL(1); E_ST(0, 1, 64); S

    }
    {
        unsigned short* orow = out + (size_t)(pm * 256 + h * 128 + wr * 64 + fr) * NN + (pn >> 1) * 512 + LINE_OF(wc) * 64 + (fq & 1) * 16 + (fq >> 1) * 8;
        const int oright = (LINE_OF(2 + wc) - LINE_OF(wc)) * 64;
        asm volatile("s_waitcnt vmcnt(16)" : "+v"(tm) :: "memory");
        if (UNLIKELY((unsigned)__builtin_amdgcn_readfirstlane(tm) == seq)) {
            asm volatile("s_waitcnt vmcnt(0)" ::: "memory");
#pragma unroll
            for (int p = 0; p < 16; ++p) *(u32x4*)(orow + (size_t)(p >> 2) * 16 * NN + (p & 1) * 32 + (p & 2 ? oright : 0)) = (u32x4){0x7fc07fc0u, 0x7fc07fc0u, 0x7fc07fc0u, 0x7fc07fc0u};
        }
    }
}

extern "C" void kernel_launch(int rank, int nranks, ncclComm_t comm,
                              hipStream_t stream,
                              void* const* d_in, const int* in_sizes, int n_in,
                              void* d_out, int out_size,
                              const int* in_dtypes, int out_dtype,
                              void* const* d_out_all,
                              void* const* d_ws_all, size_t ws_bytes) {
    (void)comm; (void)in_sizes; (void)n_in; (void)out_size; (void)in_dtypes; (void)out_dtype; (void)d_out_all;
    if (nranks != NR || (unsigned)rank >= (unsigned)NR || ws_bytes < WS_NEED) { fprintf(stderr, "a2a_gemm: unsupported configuration\n"); return; }
    static bool attr_set = false;
    if (!attr_set) {
        HIP_CHECK(hipFuncSetAttribute((const void*)a2a_gemm_gelu_kernel, hipFuncAttributeMaxDynamicSharedMemorySize, LDS_BYTES));
        attr_set = true;
    }
    hipLaunchKernelGGL(a2a_gemm_gelu_kernel, dim3(NWG), dim3(256), LDS_BYTES, stream, rank,
                       (const unsigned short*)d_in[0], (const unsigned short*)d_in[1], (unsigned short*)d_out, d_ws_all);
    HIP_CHECK(hipGetLastError());
}
```
